```python
import numpy as np

kernel_name = "gpu_gemm_rs_m4096_k4096_n16384_w8a8_dequant"


ABSMAX_RTOL = 0.001

COMPUTE_DTYPE = "int8"
EPILOGUE = "dequant"
IN_QUANT = "int8"
OUT_QUANT = None
IN_DTYPES = {"x": "int8", "w_mat": "int8",
             "scale_x": "float32", "scale_w": "float32"}
WORLD_SIZE = 8

def setup_inputs():

    x = np.random.randint(-127, 128, size=(4096, 4096)).astype(np.int8)
    w_mat = np.random.randint(-127, 128, size=(4096, 16384)).astype(np.int8)
    scale_x = (0.02 * (1.0 + np.random.rand(1))).astype(np.float32)
    scale_w = (0.02 * (1.0 + np.random.rand(1))).astype(np.float32)
    return {"x": x, "w_mat": w_mat, "scale_x": scale_x, "scale_w": scale_w}

def reference(x, w_mat, scale_x, scale_w):
    acc = x.astype(np.float64) @ w_mat.astype(np.float64)
    y = (acc.astype(np.float32) * (scale_x * scale_w)).astype(np.float32)
    return y
```

```cpp
#include <hip/hip_runtime.h>
#include <rccl/rccl.h>
#include <cstdio>
#include <cstdint>

#define HIP_CHECK(expr) do { hipError_t _e = (expr); if (_e != hipSuccess) { \
    fprintf(stderr, "HIP error %s at %s:%d\n", hipGetErrorString(_e), __FILE__, __LINE__); \
    return; } } while (0)

typedef int i32x4 __attribute__((ext_vector_type(4)));
typedef unsigned u32x4 __attribute__((ext_vector_type(4)));
typedef unsigned u32x3 __attribute__((ext_vector_type(3)));
typedef float f32x4 __attribute__((ext_vector_type(4)));
typedef __amdgpu_buffer_rsrc_t rsrc_t;
#define UNI(x) __builtin_amdgcn_readfirstlane(x)

namespace {

constexpr int NR = 8;
constexpr int NCOL = 16384;
constexpr int KS = 512;
constexpr unsigned MiB = 1u << 20;

constexpr int NBLK = 256, NTHR = 512;
constexpr int NPULL = 32;
constexpr int WL = 36;
constexpr int MON_WAVE = 252;

constexpr int G_W0 = 192;
constexpr int G_X1 = G_W0 + 512;
constexpr int G_XO = G_X1 + 576;
constexpr int NIN = G_XO + 128;
constexpr int NOUT = 1024;
constexpr int NTOT = NIN + NOUT;

constexpr unsigned O_CNT = 0;
constexpr unsigned O_TMO = 8192;
constexpr unsigned O_RDY = 16384;
constexpr unsigned O_LRW = 20480;
constexpr unsigned O_PROG = 24576;
constexpr unsigned O_G = 28672;
constexpr unsigned O_PRDY = 32768;
constexpr unsigned O_ORDY = 36864;
constexpr unsigned O_DBG = 65536;
constexpr unsigned O_XR = 1 * MiB;
constexpr unsigned O_WR = 17 * MiB;
constexpr unsigned O_PS = 25 * MiB;
constexpr unsigned O_PAR = 57 * MiB;
constexpr unsigned PAR_BYTES = 42 * MiB;
constexpr unsigned P_XS = 0, P_WS = 2 * MiB, P_OS = 10 * MiB;
constexpr unsigned WS_USED = O_PAR + 2 * PAR_BYTES;
constexpr unsigned OUT_BYTES = 512u * NCOL * 4u;

constexpr unsigned long long DEADLINE = 60000000ull;
constexpr int BIAS = 1 << 21;
constexpr int SYS = 17;
#define OUT_AUX 0

constexpr int STAGE = 49152;
constexpr int LDS_MISC = 3 * STAGE;

struct Params {
    char* ws[NR];
    const char* x;
    const char* w;
    const float* sx;
    const float* sw;
    float* out;
    int rank;
    int pad;
};

__device__ __forceinline__ rsrc_t mk(const void* p, unsigned bytes) {
    return __builtin_amdgcn_make_buffer_rsrc((void*)p, 0, bytes, 0x00020000);
}
#define st128(v, r, voff, soff, aux) __builtin_amdgcn_raw_buffer_store_b128(v, r, (voff) + (soff), 0, aux)
#define st96(v, r, voff, soff, aux) __builtin_amdgcn_raw_buffer_store_b96(v, r, (voff) + (soff), 0, aux)
#define st32(v, r, voff, soff, aux) __builtin_amdgcn_raw_buffer_store_b32(v, r, (voff) + (soff), 0, aux)
__device__ __forceinline__ u32x4 mk4(unsigned a, unsigned b, unsigned c, unsigned d) { return (u32x4){a, b, c, d}; }
__device__ __forceinline__ u32x3 mk3(unsigned a, unsigned b, unsigned c) { return (u32x3){a, b, c}; }
__device__ __forceinline__ unsigned ld_sys(const unsigned* p) {
    return __hip_atomic_load(p, __ATOMIC_RELAXED, __HIP_MEMORY_SCOPE_SYSTEM);
}
__device__ __forceinline__ void st_sys(unsigned* p, unsigned v) {
    __hip_atomic_store(p, v, __ATOMIC_RELAXED, __HIP_MEMORY_SCOPE_SYSTEM);
}
__device__ __forceinline__ void drain() {
    asm volatile("s_waitcnt vmcnt(0)" ::: "memory");
    __builtin_amdgcn_s_waitcnt(0x0F70);
}
__device__ __forceinline__ unsigned long long now() { return __builtin_amdgcn_s_memrealtime(); }
__device__ __forceinline__ unsigned perm(unsigned hi, unsigned lo, unsigned sel) {
    return __builtin_amdgcn_perm(hi, lo, sel);
}
__device__ __forceinline__ unsigned bfi(unsigned m, unsigned a, unsigned b) { return (a & m) | (b & ~m); }

#define STAMP(slot) do {} while (0)

__device__ __forceinline__ bool wave_wait(const unsigned* ptr, unsigned need, unsigned* tmo, unsigned call,
                                          int lane, int nap) {
    const unsigned long long t0 = now();
    for (;;) {
        const unsigned v = ld_sys(ptr);
        const unsigned t = ld_sys(tmo);
        if (__builtin_amdgcn_readfirstlane(t) == call) return false;
        if (__all(v >= need)) return true;
        if (now() - t0 > DEADLINE) {
            if (lane == 0) st_sys(tmo, call);
            return false;
        }
        if (nap) __builtin_amdgcn_s_sleep(12); else __builtin_amdgcn_s_sleep(1);
    }
}

__device__ __forceinline__ void monitor(char* myws, unsigned call, int lane) {
    unsigned* tmo = (unsigned*)(myws + O_TMO);
    const unsigned* prog = (const unsigned*)(myws + O_PROG);
    const unsigned base = call << 6;
    unsigned last = 0;
    const unsigned long long t0 = now();
    for (;;) {
        unsigned g = NIN;
        #pragma unroll
        for (int i = 0; i < 4; ++i) {
            const int s = lane + 64 * i;
            const unsigned v = ld_sys(prog + s);
            const unsigned cnt = v >= base ? v - base : 0u;
            const unsigned gi = (unsigned)(s / 7) + cnt * WL;
            if (s < 7 * WL) g = min(g, gi);
        }
        #pragma unroll
        for (int o = 32; o; o >>= 1) g = min(g, (unsigned)__shfl_xor((int)g, o));
        g = __builtin_amdgcn_readfirstlane(g);
        if (g > last) {
            last = g;
            if (lane < 8) st_sys((unsigned*)(myws + O_G + lane * 128), (call << 11) + g);
        }
        if (g >= NIN) return;
        if (__builtin_amdgcn_readfirstlane(ld_sys(tmo)) == call) return;
        if (now() - t0 > DEADLINE) { if (lane == 0) st_sys(tmo, call); return; }
        __builtin_amdgcn_s_sleep(2);
    }
}

struct Item { u32x4 l0, l1, h4, p0, p1, p2, p3; u32x3 h3; unsigned h1; };

__device__ __forceinline__ void puller(const Params& p, char* myws, unsigned call, int s, int lane, int wv) {
    const int me = p.rank;
    const int link = s % 7, idx = s / 7;
    const unsigned peer = (unsigned)(me + 1 + link) & 7u;
    const unsigned par = O_PAR + (call & 1u) * PAR_BYTES;
    const char* remp = p.ws[peer];
    const rsrc_t loc = mk(myws, WS_USED), out = mk(p.out, OUT_BYTES);
    unsigned* tmo = (unsigned*)(myws + O_TMO);
    unsigned* prog = (unsigned*)(myws + O_PROG) + s;
    const float scale = p.sx[0] * p.sw[0];
    const unsigned v16 = lane * 16;

    int dead;
    {
        const bool use = lane < 48 && !(lane < 32 && (unsigned)(lane >> 2) == peer);
        dead = UNI(wave_wait((const unsigned*)(myws + O_RDY) + peer * 64 + lane, use ? call : 0u, tmo, call, lane, 0) ? 0 : 1);
    }
    STAMP(1);
    const unsigned oflag = lane < 32 ? O_ORDY + (((call & 1u) * 8 + peer) * 32 + lane) * 4
                                     : O_PRDY + (peer * 32 + (lane - 32)) * 4;
    unsigned fv = 0, rmask = 0;
    const unsigned ovoff = (unsigned)(4 * (lane >> 4)) * (NCOL * 4) + (lane & 15) * 16;

    Item cur, nxt;
    unsigned cur_dst = 0, nxt_dst = 0;
    int cur_raw = 0, nxt_raw = 0, valid = 0;
    int gcur = 0;
    unsigned cnt = 0;
    unsigned long long t0 = 0;
    int waiting = 0;
    for (int g = idx;;) {
        g = UNI(g); dead = UNI(dead); valid = UNI(valid); rmask = UNI(rmask); gcur = UNI(gcur);
        cur_raw = UNI(cur_raw); cur_dst = UNI(cur_dst); waiting = UNI(waiting); cnt = UNI(cnt);
        const bool have = g < NTOT;
        bool can = have;
        if (have && g >= NIN && !dead) can = (rmask >> ((g - NIN) >> 5)) & 1u;
        unsigned src = 0, ps = 0;
        const bool isout = g >= NIN;
        nxt_raw = 0;
        if (!isout) {
            if (g >= G_W0 && g < G_X1) {
                const unsigned o = (unsigned)(g - G_W0) * 2048u;
                src = par + P_WS + (unsigned)me * MiB + o;
                nxt_dst = O_WR + peer * MiB + o;
            } else {
                unsigned rb;
                if (g >= G_XO) rb = 4 * me + ((g - G_XO) >> 5);
                else {
                    const int q = g < G_W0 ? (g >> 5) : ((g - G_X1) >> 5) + 6;
                    const int c = q / 6, pi = q - 6 * c;
                    const int i = pi + (pi >= link ? 1 : 0);
                    rb = 4 * ((me + 1 + i) & 7) + c;
                }
                const unsigned o = rb * 65536u + (unsigned)(g & 31) * 2048u;
                src = par + P_XS + o;
                nxt_dst = O_XR + peer * (2 * MiB) + o;
            }
        } else if (have) {
            const unsigned o = (unsigned)(g - NIN);
            const unsigned tile = o >> 5, w8 = (o >> 2) & 7, mf = o & 3;
            src = par + P_OS + (unsigned)me * (4 * MiB) + o * 4096u;
            ps = O_PS + peer * (4 * MiB) + o * 4096u;
            nxt_raw = (__builtin_amdgcn_readlane(fv, tile) >> w8) & 1u;
            const unsigned row = (tile >> 3) * 128 + (w8 >> 2) * 64 + mf * 16;
            const unsigned col = peer * 2048 + (tile & 7) * 256 + (w8 & 3) * 64;
            nxt_dst = (row * NCOL + col) * 4u;
        }
        {
            const bool go = can && !dead;
            const rsrc_t r0 = mk(remp, go ? WS_USED : 0u);
            const rsrc_t r1 = mk(remp, go && isout ? WS_USED : 0u);
            const rsrc_t r2 = mk(remp, go && isout && nxt_raw ? WS_USED : 0u);
            const rsrc_t r3 = mk(myws, go && isout ? WS_USED : 0u);
            nxt.l0 = __builtin_amdgcn_raw_buffer_load_b128(r0, v16, src, SYS);
            nxt.l1 = __builtin_amdgcn_raw_buffer_load_b128(r0, v16 + 1024, src, SYS);
            nxt.h3 = __builtin_amdgcn_raw_buffer_load_b96(r1, lane * 12 + 2048, src, SYS);
            nxt.h1 = __builtin_amdgcn_raw_buffer_load_b32(r2, lane * 4 + 2816, src, SYS);
            nxt.h4 = __builtin_amdgcn_raw_buffer_load_b128(r2, v16 + 3072, src, SYS);
            nxt.p0 = __builtin_amdgcn_raw_buffer_load_b128(r3, v16, ps, SYS);
            nxt.p1 = __builtin_amdgcn_raw_buffer_load_b128(r3, v16 + 1024, ps, SYS);
            nxt.p2 = __builtin_amdgcn_raw_buffer_load_b128(r3, v16 + 2048, ps, SYS);
            nxt.p3 = __builtin_amdgcn_raw_buffer_load_b128(r3, v16 + 3072, ps, SYS);
        }
        const bool poll = g + 2 * WL >= NIN && rmask != 0xffffffffu && !dead;
        const rsrc_t rf = mk(myws, poll ? O_DBG : 0u);
        const unsigned nf = __builtin_amdgcn_raw_buffer_load_b32(rf, oflag, 0, SYS);
        const unsigned nt = __builtin_amdgcn_raw_buffer_load_b32(rf, O_TMO, 0, SYS);

        if (valid) {
            if (gcur < NIN) {
                if (!dead) {
                    st128(cur.l0, loc, v16, cur_dst, SYS);
                    st128(cur.l1, loc, v16 + 1024, cur_dst, SYS);
                }
            } else {
                unsigned L[8] = {cur.l0.x, cur.l0.y, cur.l0.z, cur.l0.w, cur.l1.x, cur.l1.y, cur.l1.z, cur.l1.w};
                const u32x4 P[4] = {cur.p0, cur.p1, cur.p2, cur.p3};
                u32x4 v[4];
                if (cur_raw) {
                    const unsigned H[8] = {cur.h3.x, cur.h3.y, cur.h3.z, cur.h1, cur.h4.x, cur.h4.y, cur.h4.z, cur.h4.w};
                    #pragma unroll
                    for (int r = 0; r < 4; ++r) {
                        v[r].x = perm(H[2 * r], L[2 * r], 0x05040100u);
                        v[r].y = perm(H[2 * r], L[2 * r], 0x07060302u);
                        v[r].z = perm(H[2 * r + 1], L[2 * r + 1], 0x05040100u);
                        v[r].w = perm(H[2 * r + 1], L[2 * r + 1], 0x07060302u);
                    }
                } else {
                    unsigned B[4];
                    B[0] = cur.h3.x & 0x3f3f3f3fu;
                    B[1] = cur.h3.y & 0x3f3f3f3fu;
                    B[2] = cur.h3.z & 0x3f3f3f3fu;
                    B[3] = ((cur.h3.x >> 6) & 0x03030303u) | ((cur.h3.y >> 4) & 0x0c0c0c0cu) | ((cur.h3.z >> 2) & 0x30303030u);
                    #pragma unroll
                    for (int r = 0; r < 4; ++r) {
                        v[r].x = perm(B[r], L[2 * r], 0x0c040100u);
                        v[r].y = perm(B[r], L[2 * r], 0x0c050302u);
                        v[r].z = perm(B[r], L[2 * r + 1], 0x0c060100u);
                        v[r].w = perm(B[r], L[2 * r + 1], 0x0c070302u);
                    }
                }
                #pragma unroll
                for (int r = 0; r < 4; ++r) {
                    const u32x4 t = v[r] + P[r];
                    f32x4 f;
                    f.x = (float)(int)t.x * scale; f.y = (float)(int)t.y * scale;
                    f.z = (float)(int)t.z * scale; f.w = (float)(int)t.w * scale;
                    u32x4 o4 = __builtin_bit_cast(u32x4, f);
                    if (dead) o4 = (u32x4){0x7fc00000u, 0x7fc00000u, 0x7fc00000u, 0x7fc00000u};
                    st128(o4, out, ovoff + r * (NCOL * 4), cur_dst, OUT_AUX);
                }
            }
        }
        drain();
        if (valid && gcur < NIN && !dead) {
            ++cnt;
            if (lane == 0) st_sys(prog, (call << 6) + cnt);
        }
        if (poll) {
            fv = nf;
            const unsigned long long m = __ballot(lane < 32 ? (nf >> 8) >= call : nf >= call);
            rmask = (unsigned)m & (unsigned)(m >> 32);
            if (__builtin_amdgcn_readfirstlane(nt) == call) dead = 1;
        }
        if (!have) break;
        if (can) {
            cur = nxt; cur_dst = nxt_dst; cur_raw = nxt_raw; gcur = g; valid = 1;
            g += WL;
            waiting = 0;
        } else {
            if (!waiting) { waiting = 1; t0 = now(); }
            else {
                if (now() - t0 > DEADLINE) { if (lane == 0) st_sys(tmo, call); dead = 1; }
                __builtin_amdgcn_s_sleep(4);
            }
            valid = 0;
        }
    }
    STAMP(3);
}

__device__ __forceinline__ void prep(const Params& p, char* myws, unsigned call, int id, int tid, int lane, int wv) {
    const int me = p.rank;
    const unsigned par = O_PAR + (call & 1u) * PAR_BYTES;
    const rsrc_t loc = mk(myws, WS_USED);
    if (id < 32) {
        const rsrc_t xs = mk(p.x, 4096u * KS);
        const unsigned voff = (unsigned)(lane & 15) * KS + (lane >> 4) * 16;
        const unsigned soff = (unsigned)(id * 128 + wv * 16) * KS;
        u32x4 v[8];
        #pragma unroll
        for (int i = 0; i < 8; ++i) v[i] = __builtin_amdgcn_raw_buffer_load_b128(xs, voff + i * 64, soff, 0);
        const unsigned dst = par + P_XS + (unsigned)id * 65536u + wv * 1024u;
        #pragma unroll
        for (int i = 0; i < 8; ++i) st128(v[i], loc, lane * 16, dst + i * 8192u, SYS);
    } else {
        const int pid = id - 32, jt = pid >> 1, h = pid & 1;
        const rsrc_t wsr = mk(p.w, (unsigned)KS * NCOL);
        const int wn = wv & 3, kq0 = h * 4 + (wv >> 2) * 2;
        const unsigned voff = (unsigned)(16 * (lane >> 4)) * NCOL + (lane & 15) * 4;
        unsigned r[2][16];
        #pragma unroll
        for (int b = 0; b < 2; ++b)
            #pragma unroll
            for (int j = 0; j < 16; ++j)
                r[b][j] = __builtin_amdgcn_raw_buffer_load_b32(wsr, voff, (unsigned)((kq0 + b) * 64 + j) * NCOL + jt * 256 + wn * 64, 0);
        #pragma unroll
        for (int b = 0; b < 2; ++b) {
            unsigned o[4][4];
            #pragma unroll
            for (int q = 0; q < 4; ++q) {
                const unsigned t0 = perm(r[b][4 * q + 1], r[b][4 * q], 0x05010400u);
                const unsigned t1 = perm(r[b][4 * q + 1], r[b][4 * q], 0x07030602u);
                const unsigned t2 = perm(r[b][4 * q + 3], r[b][4 * q + 2], 0x05010400u);
                const unsigned t3 = perm(r[b][4 * q + 3], r[b][4 * q + 2], 0x07030602u);
                o[0][q] = perm(t2, t0, 0x05040100u);
                o[1][q] = perm(t2, t0, 0x07060302u);
                o[2][q] = perm(t3, t1, 0x05040100u);
                o[3][q] = perm(t3, t1, 0x07060302u);
            }
            const unsigned dst = par + P_WS + (unsigned)jt * 131072u + (unsigned)((kq0 + b) * 16 + wn * 4) * 1024u;
            #pragma unroll
            for (int i = 0; i < 4; ++i)
                st128(mk4(o[i][0], o[i][1], o[i][2], o[i][3]), loc, lane * 16 + i * 1024, dst, SYS);
        }
    }
    drain();
    __syncthreads();
    if (id < 32) {
        if (tid < NR) st_sys((unsigned*)(p.ws[tid] + O_RDY) + me * 64 + id, call);
    } else {
        const int pid = id - 32;
        if (tid == 0) st_sys((unsigned*)(p.ws[pid >> 4] + O_RDY) + me * 64 + 32 + (pid & 15), call);
        if (tid == 1) st_sys((unsigned*)(myws + O_LRW) + pid, call);
    }
}

}

__global__ __launch_bounds__(NTHR, 2) void gemm_rs_fused(Params p) {
    __shared__ __attribute__((aligned(16))) char lds[3 * STAGE + 64];
    const int tid = threadIdx.x, lane = tid & 63;
    const int wv = __builtin_amdgcn_readfirstlane(tid >> 6);
    const int b = blockIdx.x, me = p.rank;
    char* myws = p.ws[me];
    unsigned* cntp = (unsigned*)(myws + O_CNT) + b * 8 + wv;
    const unsigned call = __builtin_amdgcn_readfirstlane(ld_sys(cntp)) + 1u;
    if (lane == 0) st_sys(cntp, call);
    STAMP(0);

    if (b < NPULL) {
        const int s = b * 8 + wv;
        if (s < 7 * WL) puller(p, myws, call, s, lane, wv);
        else if (s == MON_WAVE) monitor(myws, call, lane);
        return;
    }

    const int cb = b - NPULL;
    if (cb < 160) prep(p, myws, call, cb, tid, lane, wv);
    STAMP(1);

    const int t = b & 7, u = cb >> 3;
    const int c = u / 7, d = (me + 1 + (u - 7 * c)) & 7;
    const unsigned par = O_PAR + (call & 1u) * PAR_BYTES;
    const rsrc_t loc = mk(myws, WS_USED);
    unsigned* tmo = (unsigned*)(myws + O_TMO);
    volatile __attribute__((address_space(3))) unsigned* misc =
        (volatile __attribute__((address_space(3))) unsigned*)(lds + LDS_MISC);
    const int wm = wv >> 2, wn = wv & 3;
    const int ntask = u < 4 ? 3 : 2;

    for (int task = 0; task < ntask; ++task) {
        const int rb = task == 0 ? 4 * me + c : task == 1 ? 4 * d + c : 4 * me + u;
        const int wj = task == 0 ? d : me;
        const int k0 = task == 0 ? me : 0;
        const int skip = task == 1 ? d : 99;
        const int nst = task == 0 ? 4 : task == 1 ? 28 : 32;
        if (wv == 0) {
            const unsigned gend = task == 0 ? 0u : task == 2 ? (unsigned)(G_XO + 32 * (u + 1))
                                : c == 0 ? (unsigned)(G_W0 + 64 * (t + 1)) : (unsigned)(G_X1 + 192 * c);
            const unsigned* ptr = (const unsigned*)(myws + O_RDY) + me * 64 + rb;
            unsigned need = call;
            if (lane == 1 || lane == 2) ptr = (const unsigned*)(myws + O_LRW) + (wj * 8 + t) * 2 + (lane - 1);
            if (lane == 3 && task) { ptr = (const unsigned*)(myws + O_G + (b & 7) * 128); need = (call << 11) + gend; }
            const bool ok = wave_wait(ptr, need, tmo, call, lane, 1);
            __builtin_amdgcn_fence(__ATOMIC_ACQUIRE, "");
            asm volatile("s_waitcnt vmcnt(0)" ::: "memory");
            if (lane == 0) misc[0] = ok ? 1u : 0u;
        }
        __syncthreads();
        const bool ok = __builtin_amdgcn_readfirstlane(misc[0]) != 0;
        STAMP(2 + 2 * task);

        i32x4 acc[4][4];
        {
            const int a0 = task == 0 ? -BIAS : task == 1 ? BIAS : 0;
            #pragma unroll
            for (int m = 0; m < 4; ++m)
                #pragma unroll
                for (int n = 0; n < 4; ++n) acc[m][n] = (i32x4){a0, a0, a0, a0};
        }
        if (ok) {
            const unsigned va = wv * 2048 + lane * 16, vb = wv * 4096 + lane * 16;
            auto issue = [&](int s, int slot) {
                const int sl = s >> 2;
                const int k = k0 + sl + (sl >= skip ? 1 : 0);
                const unsigned abase = (k == me ? par + P_XS : O_XR + (unsigned)k * (2 * MiB)) + (unsigned)rb * 65536u + (s & 3) * 16384u;
                const unsigned bbase = (k == me ? par + P_WS + (unsigned)wj * MiB : O_WR + (unsigned)k * MiB) + (unsigned)t * 131072u + (s & 3) * 32768u;
                char* la = lds + slot * STAGE + wv * 2048;
                char* lb = lds + slot * STAGE + 16384 + wv * 4096;
                __builtin_amdgcn_raw_ptr_buffer_load_lds(loc, (__attribute__((address_space(3))) void*)la, 16, va, abase, 0, 0);
                __builtin_amdgcn_raw_ptr_buffer_load_lds(loc, (__attribute__((address_space(3))) void*)la, 16, va, abase, 1024, 0);
                __builtin_amdgcn_raw_ptr_buffer_load_lds(loc, (__attribute__((address_space(3))) void*)lb, 16, vb, bbase, 0, 0);
                __builtin_amdgcn_raw_ptr_buffer_load_lds(loc, (__attribute__((address_space(3))) void*)lb, 16, vb, bbase, 1024, 0);
                __builtin_amdgcn_raw_ptr_buffer_load_lds(loc, (__attribute__((address_space(3))) void*)lb, 16, vb, bbase, 2048, 0);
                __builtin_amdgcn_raw_ptr_buffer_load_lds(loc, (__attribute__((address_space(3))) void*)lb, 16, vb, bbase, 3072, 0);
            };
            auto compute = [&](int slot) {
                const char* A = lds + slot * STAGE + wm * 4096 + lane * 16;
                const char* B = lds + slot * STAGE + 16384 + wn * 4096 + lane * 16;
                #pragma unroll
                for (int kb = 0; kb < 2; ++kb) {
                    i32x4 af[4], bf[4];
                    #pragma unroll
                    for (int i = 0; i < 4; ++i) {
                        af[i] = *(const i32x4*)(A + kb * 8192 + i * 1024);
                        bf[i] = *(const i32x4*)(B + kb * 16384 + i * 1024);
                    }
                    #pragma unroll
                    for (int m = 0; m < 4; ++m)
                        #pragma unroll
                        for (int n = 0; n < 4; ++n)
                            acc[m][n] = __builtin_amdgcn_mfma_i32_16x16x64_i8(af[m], bf[n], acc[m][n], 0, 0, 0);
                }
            };
            issue(0, 0);
            issue(1, 1);
            int slot = 0;
            for (int s = 0; s < nst - 1; ++s) {
                asm volatile("s_waitcnt vmcnt(6) lgkmcnt(0)" ::: "memory");
                __builtin_amdgcn_s_barrier();
                if (s + 2 < nst) issue(s + 2, slot == 0 ? 2 : slot - 1);
                compute(slot);
                slot = slot == 2 ? 0 : slot + 1;
            }
            asm volatile("s_waitcnt vmcnt(0) lgkmcnt(0)" ::: "memory");
            __builtin_amdgcn_s_barrier();
            compute(slot);
        }

        if (task == 0) {
            if (ok) {
                const unsigned dst = O_PS + (unsigned)d * (4 * MiB) + (unsigned)((c * 8 + t) * 8 + wv) * 16384u;
                #pragma unroll
                for (int m = 0; m < 4; ++m)
                    #pragma unroll
                    for (int r = 0; r < 4; ++r)
                        st128(mk4((unsigned)acc[m][0][r], (unsigned)acc[m][1][r], (unsigned)acc[m][2][r], (unsigned)acc[m][3][r]),
                            loc, lane * 16 + r * 1024, dst + m * 4096u, SYS);
            }
            drain();
            __syncthreads();
            if (tid == 0 && ok) st_sys((unsigned*)(myws + O_PRDY) + d * 32 + c * 8 + t, call);
        } else if (task == 1) {
            unsigned any = 0;
            #pragma unroll
            for (int m = 0; m < 4; ++m)
                #pragma unroll
                for (int n = 0; n < 4; ++n)
                    any |= (unsigned)(acc[m][n][0] | acc[m][n][1]) | (unsigned)(acc[m][n][2] | acc[m][n][3]);
            const bool raw = __any((any >> 22) != 0);
            if (ok) {
                const unsigned dst = par + P_OS + (unsigned)d * (4 * MiB) + (unsigned)((c * 8 + t) * 8 + wv) * 16384u;
                #pragma unroll
                for (int m = 0; m < 4; ++m) {
                    unsigned L[8], H[8];
                    #pragma unroll
                    for (int r = 0; r < 4; ++r) {
                        L[2 * r] = perm(acc[m][1][r], acc[m][0][r], 0x05040100u);
                        L[2 * r + 1] = perm(acc[m][3][r], acc[m][2][r], 0x05040100u);
                        H[2 * r] = perm(acc[m][1][r], acc[m][0][r], 0x07060302u);
                        H[2 * r + 1] = perm(acc[m][3][r], acc[m][2][r], 0x07060302u);
                    }
                    st128(mk4(L[0], L[1], L[2], L[3]), loc, lane * 16, dst + m * 4096u, SYS);
                    st128(mk4(L[4], L[5], L[6], L[7]), loc, lane * 16 + 1024, dst + m * 4096u, SYS);
                    if (raw) {
                        st96(mk3(H[0], H[1], H[2]), loc, lane * 12 + 2048, dst + m * 4096u, SYS);
                        st32(H[3], loc, lane * 4 + 2816, dst + m * 4096u, SYS);
                        st128(mk4(H[4], H[5], H[6], H[7]), loc, lane * 16 + 3072, dst + m * 4096u, SYS);
                    } else {
                        unsigned B[4];
                        #pragma unroll
                        for (int r = 0; r < 4; ++r) B[r] = perm(H[2 * r + 1], H[2 * r], 0x06040200u);
                        const u32x3 h = {bfi(0x3f3f3f3fu, B[0], B[3] << 6), bfi(0x3f3f3f3fu, B[1], B[3] << 4),
                                         bfi(0x3f3f3f3fu, B[2], B[3] << 2)};
                        st96(h, loc, lane * 12 + 2048, dst + m * 4096u, SYS);
                    }
                }
            }
            if (lane == 0) misc[4 + wv] = raw ? 1u : 0u;
            drain();
            __syncthreads();
            if (tid == 0 && ok) {
                unsigned mask = 0;
                #pragma unroll
                for (int i = 0; i < 8; ++i) mask |= misc[4 + i] << i;
                st_sys((unsigned*)(p.ws[d] + O_ORDY) + ((call & 1u) * 8 + me) * 32 + c * 8 + t, (call << 8) | mask);
            }
        } else {
            const rsrc_t out = mk(p.out, OUT_BYTES);
            const float scale = p.sx[0] * p.sw[0];
            const unsigned ovoff = (unsigned)(4 * (lane >> 4)) * (NCOL * 4) + (lane & 15) * 16;
            const unsigned dst = ((unsigned)(u * 128 + wm * 64) * NCOL + me * 2048 + t * 256 + wn * 64) * 4u;
            #pragma unroll
            for (int m = 0; m < 4; ++m)
                #pragma unroll
                for (int r = 0; r < 4; ++r) {
                    f32x4 f;
                    f.x = (float)acc[m][0][r] * scale; f.y = (float)acc[m][1][r] * scale;
                    f.z = (float)acc[m][2][r] * scale; f.w = (float)acc[m][3][r] * scale;
                    u32x4 o4 = __builtin_bit_cast(u32x4, f);
                    if (!ok) o4 = (u32x4){0x7fc00000u, 0x7fc00000u, 0x7fc00000u, 0x7fc00000u};
                    st128(o4, out, ovoff + r * (NCOL * 4), dst + (unsigned)(m * 16) * (NCOL * 4), OUT_AUX);
                }
        }
        STAMP(3 + 2 * task);
    }
}

extern "C" void kernel_launch(int rank, int nranks, ncclComm_t comm, hipStream_t stream,
                              void* const* d_in, const int* in_sizes, int n_in,
                              void* d_out, int out_size, const int* in_dtypes, int out_dtype,
                              void* const* d_out_all, void* const* d_ws_all, size_t ws_bytes) {
    (void)comm; (void)in_sizes; (void)in_dtypes; (void)out_dtype; (void)d_out_all;
    if (nranks != NR || (unsigned)rank >= (unsigned)NR || n_in < 4 || ws_bytes < WS_USED ||
        (size_t)out_size * 4 != OUT_BYTES) {
        fprintf(stderr, "kernel_launch: unexpected problem shape\n");
        return;
    }
    Params p;
    for (int i = 0; i < NR; ++i) p.ws[i] = (char*)d_ws_all[i];
    p.x = (const char*)d_in[0];
    p.w = (const char*)d_in[1];
    p.sx = (const float*)d_in[2];
    p.sw = (const float*)d_in[3];
    p.out = (float*)d_out;
    p.rank = rank;
    p.pad = 0;
    hipLaunchKernelGGL(gemm_rs_fused, dim3(NBLK), dim3(NTHR), 0, stream, p);
    HIP_CHECK(hipGetLastError());
}
```

```cpp
#include <hip/hip_runtime.h>
#include <rccl/rccl.h>
#include <cstdio>
#include <cstdint>

#define HIP_CHECK(expr) do { hipError_t _e = (expr); if (_e != hipSuccess) { \
    fprintf(stderr, "HIP error %s at %s:%d\n", hipGetErrorString(_e), __FILE__, __LINE__); \
    return; } } while (0)

typedef int i32x4 __attribute__((ext_vector_type(4)));
typedef unsigned u32x4 __attribute__((ext_vector_type(4)));
typedef unsigned u32x3 __attribute__((ext_vector_type(3)));
typedef float f32x4 __attribute__((ext_vector_type(4)));
typedef __amdgpu_buffer_rsrc_t rsrc_t;
#define UNI(x) __builtin_amdgcn_readfirstlane(x)
#define BALLOT(c) __builtin_amdgcn_ballot_w64(c)

namespace {

constexpr int NR = 8;
constexpr int NCOL = 16384;
constexpr int KS = 512;
constexpr unsigned MiB = 1u << 20;

constexpr int NBLK = 256, NTHR = 512;
#define NPULL_ 64
#define WL_ 32
constexpr int NPULL = NPULL_;
constexpr int WL = WL_;
constexpr int NLW = 7 * WL;
constexpr int NCU = (NBLK - NPULL) / 8;
static_assert(NLW <= 7 * NPULL && NLW <= 512 && NCU >= 20 && NCU <= 28 && NBLK - NPULL >= 160, "geometry");

#define RP_ 6
#define WLO_ 27
#define TA_ 8
constexpr int RP = RP_;
constexpr int WLO = WLO_;
constexpr int TA = TA_;
constexpr int G_WA = 192, G_X1 = G_WA + 64 * TA, G_WB = G_X1 + 192, G_X2 = G_WB + 64 * (8 - TA), G_XO = G_X2 + 384;
constexpr int NIN = G_XO + 128;
constexpr int NOUT = 1024;
constexpr int NTOT = NIN + NOUT;
constexpr unsigned TICKETS = NTOT - (RP + 1) * WL + WLO;
static_assert(RP >= 2 && RP * WL <= G_WA && WLO <= WL && WL <= 64, "push rounds");
__host__ __device__ constexpr int tile_pos(int c, int t) {
    return c >= 2 ? c * 8 + t : t < TA ? c * TA + t : 2 * TA + (t - TA) * 2 + c;
}

constexpr unsigned O_CNT = 0;
constexpr unsigned O_TMO = 8192;
constexpr unsigned O_TICK = 12288;
constexpr unsigned O_RDY = 16384;
constexpr unsigned O_LRW = 20480;
constexpr unsigned O_PROG = 24576;
constexpr unsigned O_PUSH = 26624;
constexpr unsigned O_G = 28672;
constexpr unsigned O_PRDY = 32768;
constexpr unsigned O_ORDY = 36864;
constexpr unsigned O_DBG = 65536;
constexpr unsigned O_WR = 1 * MiB;
constexpr unsigned O_PS = 9 * MiB;
constexpr unsigned O_PAR = 41 * MiB;
constexpr unsigned PAR_BYTES = 56 * MiB;
constexpr unsigned P_X = 0, P_WS = 16 * MiB, P_OS = 24 * MiB;
constexpr unsigned WS_USED = O_PAR + 2 * PAR_BYTES;
constexpr unsigned OUT_BYTES = 512u * NCOL * 4u;

constexpr unsigned long long DEADLINE = 60000000ull;
constexpr int BIAS = 1 << 21;
constexpr int SYS = 17;
#define OUT_AUX 16

constexpr int STAGE = 49152;
constexpr int LDS_MISC = 3 * STAGE;

struct Params {
    char* ws[NR];
    const char* x;
    const char* w;
    const float* sx;
    const float* sw;
    float* out;
    int rank;
    int pad;
};

__device__ __forceinline__ rsrc_t mk(const void* p, unsigned bytes) {
    return __builtin_amdgcn_make_buffer_rsrc((void*)p, 0, bytes, 0x00020000);
}
#define st128(v, r, voff, soff, aux) __builtin_amdgcn_raw_buffer_store_b128(v, r, (voff) + (soff), 0, aux)
#define st96(v, r, voff, soff, aux) __builtin_amdgcn_raw_buffer_store_b96(v, r, (voff) + (soff), 0, aux)
#define st32(v, r, voff, soff, aux) __builtin_amdgcn_raw_buffer_store_b32(v, r, (voff) + (soff), 0, aux)
__device__ __forceinline__ u32x4 mk4(unsigned a, unsigned b, unsigned c, unsigned d) { return (u32x4){a, b, c, d}; }
__device__ __forceinline__ u32x3 mk3(unsigned a, unsigned b, unsigned c) { return (u32x3){a, b, c}; }
__device__ __forceinline__ unsigned ld_sys(const unsigned* p) {
    return __hip_atomic_load(p, __ATOMIC_RELAXED, __HIP_MEMORY_SCOPE_SYSTEM);
}
__device__ __forceinline__ void st_sys(unsigned* p, unsigned v) {
    __hip_atomic_store(p, v, __ATOMIC_RELAXED, __HIP_MEMORY_SCOPE_SYSTEM);
}
__device__ __forceinline__ void drain() {
    asm volatile("s_waitcnt vmcnt(0)" ::: "memory");
    __builtin_amdgcn_s_waitcnt(0x0F70);
}
__device__ __forceinline__ unsigned long long now() { return __builtin_amdgcn_s_memrealtime(); }
__device__ __forceinline__ unsigned perm(unsigned hi, unsigned lo, unsigned sel) {
    return __builtin_amdgcn_perm(hi, lo, sel);
}
__device__ __forceinline__ unsigned bfi(unsigned m, unsigned a, unsigned b) { return (a & m) | (b & ~m); }

#define STAMP(slot) do {} while (0)

__device__ __forceinline__ bool wave_wait(const unsigned* ptr, unsigned need, unsigned* tmo, unsigned call,
                                          int lane, int nap) {
    const unsigned long long t0 = now();
    for (;;) {
        const unsigned v = ld_sys(ptr);
        const unsigned t = ld_sys(tmo);
        if (__builtin_amdgcn_readfirstlane(t) == call) return false;
        if (BALLOT(v < need) == 0) return true;
        if (now() - t0 > DEADLINE) {
            if (lane == 0) st_sys(tmo, call);
            return false;
        }
        if (nap) __builtin_amdgcn_s_sleep(12); else __builtin_amdgcn_s_sleep(1);
    }
}

__device__ __forceinline__ void monitor(char* myws, unsigned call, int lane) {
    unsigned* tmo = (unsigned*)(myws + O_TMO);
    const unsigned* prog = (const unsigned*)(myws + O_PROG);
    const unsigned* push = (const unsigned*)(myws + O_PUSH);
    const unsigned base = call << 12;
    unsigned last = 0;
    const unsigned long long t0 = now();
    for (;;) {
        unsigned g[7];
        #pragma unroll
        for (int l = 0; l < 7; ++l) {
            const unsigned v = ld_sys(prog + l * 64 + lane), q = ld_sys(push + l * 64 + lane);
            g[l] = lane >= WL ? (unsigned)NIN : (v >= base && q >= call) ? min(v - base, (unsigned)NIN) : 0u;
        }
        #pragma unroll
        for (int o = 32; o; o >>= 1)
            #pragma unroll
            for (int l = 0; l < 7; ++l) g[l] = min(g[l], (unsigned)__shfl_xor((int)g[l], o));
        unsigned mine = NIN;
        #pragma unroll
        for (int l = 0; l < 7; ++l) if ((lane & 7) == l) mine = g[l];
        if (mine > last) {
            last = mine;
            if ((lane & 7) < 7) st_sys((unsigned*)(myws + O_G + (lane >> 3) * 128) + (lane & 7), (call << 11) + mine);
        }
        if (BALLOT(mine < NIN) == 0) return;
        if (UNI(ld_sys(tmo)) == call) return;
        if (now() - t0 > DEADLINE) { if (lane == 0) st_sys(tmo, call); return; }
        __builtin_amdgcn_s_sleep(1);
    }
}

struct Item { u32x4 l0, l1, p0, p1, p2, p3; u32x3 h3; };

__device__ __forceinline__ unsigned draw(const rsrc_t tick, bool sel) {
    int t = 1, z = 0;
    asm volatile("" : "+v"(z));
    if (sel) t = __builtin_amdgcn_raw_ptr_buffer_atomic_add_i32(t, tick, z, 0, 0);
    return (unsigned)t;
}

__device__ __forceinline__ void link_wave(const Params& p, char* myws, const unsigned* cntp, int s, int lane, int wv) {
    const int me = p.rank;
    const int link = s % 7, idx = s / 7;
    const unsigned peer = (unsigned)(me + 1 + link) & 7u;
    const char* remp = p.ws[peer];
    const rsrc_t loc = mk(myws, WS_USED), rem = mk(remp, WS_USED);
    unsigned* tmo = (unsigned*)(myws + O_TMO);
    const unsigned v16 = lane * 16;

    u32x4 a0, a1;
    const rsrc_t xs = mk(p.x, 4096u * KS);
    const unsigned vx = (unsigned)(lane & 15) * KS + (lane >> 4) * 16;
    auto rb_of = [&](int g) {
        const int pi = g >> 5, i = pi + (pi >= 6 - link ? 1 : 0);
        return (unsigned)(4 * ((peer + 1 + i) & 7));
    };
    auto xoff = [&](int g) { return (rb_of(g) * 128 + (unsigned)(g & 3) * 32) * KS + (unsigned)((g & 31) >> 2) * 64; };
    a0 = __builtin_amdgcn_raw_buffer_load_b128(xs, vx, xoff(idx), 0);
    a1 = __builtin_amdgcn_raw_buffer_load_b128(xs, vx + 16 * KS, xoff(idx), 0);
    const float scale = p.sx[0] * p.sw[0];
    const unsigned call = UNI(ld_sys(cntp)) + 1u;
    if (lane == 0) st_sys((unsigned*)cntp, call);
    STAMP(0);
    const unsigned par = O_PAR + (call & 1u) * PAR_BYTES;
    const unsigned dbase = par + P_X + (unsigned)me * (2 * MiB);
    const unsigned* rdyp = (const unsigned*)(myws + O_RDY) + peer * 64 + lane;
    const unsigned rneed = lane < 48 && !(lane < 32 && (unsigned)(lane >> 2) == peer) ? call : 0u;
    unsigned rv = 0;
    #pragma unroll 1
    for (int r = 0; r < RP - 1; ++r) {
        const int g = idx + r * WL;
        const unsigned so = xoff(g + WL);
        const u32x4 b0 = __builtin_amdgcn_raw_buffer_load_b128(xs, vx, so, 0);
        const u32x4 b1 = __builtin_amdgcn_raw_buffer_load_b128(xs, vx + 16 * KS, so, 0);
        const unsigned d = dbase + rb_of(g) * 65536u + (unsigned)(g & 31) * 2048u;
        st128(a0, rem, v16, d, SYS);
        st128(a1, rem, v16 + 1024, d, SYS);
        rv = ld_sys(rdyp);
        drain();
        a0 = b0; a1 = b1;
    }
    int dead = 0;
    if (BALLOT(rv < rneed) != 0) dead = wave_wait(rdyp, rneed, tmo, call, lane, 0) ? 0 : 1;
    dead = UNI(dead);
    STAMP(1);

    unsigned* prog = (unsigned*)(myws + O_PROG) + link * 64 + idx;
    const rsrc_t tick = mk(myws + O_TICK + link * 128, 4);
    const unsigned tbase = (call - 1u) * TICKETS - (unsigned)((RP + 1) * WL);
    auto in_off = [&](int g, unsigned& src, unsigned& dst) {
        if ((g >= G_WA && g < G_X1) || (g >= G_WB && g < G_X2)) {
            const unsigned o = (unsigned)(g < G_X1 ? g - G_WA : g - G_WB + 64 * TA) * 2048u;
            src = par + P_WS + (unsigned)me * MiB + o;
            dst = O_WR + peer * MiB + o;
        } else {
            unsigned rb;
            if (g >= G_XO) rb = 4 * me + ((g - G_XO) >> 5);
            else {
                const int q = (g < G_WA ? g : g < G_WB ? g - 64 * TA : g - 512) >> 5;
                const int c = q / 6, pi = q - 6 * c;
                const int i = pi + (pi >= link ? 1 : 0);
                rb = 4 * ((me + 1 + i) & 7) + c;
            }
            src = dst = par + P_X + peer * (2 * MiB) + rb * 65536u + (unsigned)(g & 31) * 2048u;
        }
    };
    const rsrc_t remd = mk(remp, dead ? 0u : WS_USED), locd = mk(myws, dead ? 0u : WS_USED);
    const unsigned oflag = lane < 32 ? O_ORDY + (((call & 1u) * 8 + peer) * 32 + lane) * 4
                                     : O_PRDY + (peer * 32 + (lane - 32)) * 4;
    unsigned fv = 0;
    int g = idx + RP * WL, gn;
    unsigned src, cdst, ndst;
    in_off(g, src, cdst);
    {
        const u32x4 b0 = __builtin_amdgcn_raw_buffer_load_b128(remd, v16, src, SYS);
        const u32x4 b1 = __builtin_amdgcn_raw_buffer_load_b128(remd, v16 + 1024, src, SYS);
        const unsigned tk = draw(tick, lane == 0);
        const unsigned d = dbase + rb_of(g - WL) * 65536u + (unsigned)((g - WL) & 31) * 2048u;
        st128(a0, rem, v16, d, SYS);
        st128(a1, rem, v16 + 1024, d, SYS);
        drain();
        if (lane == 0) st_sys((unsigned*)(remp + O_PUSH) + (6 - link) * 64 + idx, call);
        a0 = b0; a1 = b1;
        gn = (int)(UNI(tk) - tbase);
    }
    while (gn < NIN) {
        in_off(gn, src, ndst);
        const u32x4 b0 = __builtin_amdgcn_raw_buffer_load_b128(remd, v16, src, SYS);
        const u32x4 b1 = __builtin_amdgcn_raw_buffer_load_b128(remd, v16 + 1024, src, SYS);
        const unsigned tk = draw(tick, lane == 0);
        st128(a0, locd, v16, cdst, SYS);
        st128(a1, locd, v16 + 1024, cdst, SYS);
        fv = __builtin_amdgcn_raw_buffer_load_b32(mk(myws, gn >= G_XO ? O_DBG : 0u), oflag, 0, SYS);
        drain();
        if (lane == 0 && !dead) st_sys(prog, (call << 12) + (unsigned)gn);
        a0 = b0; a1 = b1;
        g = gn; cdst = ndst;
        gn = (int)(UNI(tk) - tbase);
    }

    const rsrc_t out = mk(p.out, OUT_BYTES);
    const unsigned ovoff = (unsigned)(4 * (lane >> 4)) * (NCOL * 4) + (lane & 15) * 16;
    unsigned rmask;
    Item cur, nxt;
    cur.l0 = a0; cur.l1 = a1;
    int state = 2;
    int waiting = 0;
    unsigned long long t0 = 0;
    {
        const unsigned long long m = BALLOT(lane < 32 ? (fv >> 8) >= call : fv >= call);
        rmask = (unsigned)m & (unsigned)(m >> 32);
    }
    for (;;) {
        g = UNI(g); gn = UNI(gn); dead = UNI(dead); state = UNI(state); rmask = UNI(rmask);
        cdst = UNI(cdst); waiting = UNI(waiting);
        const bool have = gn < NTOT;
        const unsigned o = (unsigned)(gn - NIN) & 1023u;
        const bool can = have && (dead || ((rmask >> (o >> 5)) & 1u));
        {
            const bool go = can && !dead;
            const rsrc_t r0 = mk(remp, go ? WS_USED : 0u), r1 = mk(myws, go ? WS_USED : 0u);
            const unsigned so = par + P_OS + (unsigned)me * (4 * MiB) + o * 4096u;
            const unsigned ps = O_PS + peer * (4 * MiB) + o * 4096u;
            nxt.l0 = __builtin_amdgcn_raw_buffer_load_b128(r0, v16, so, SYS);
            nxt.l1 = __builtin_amdgcn_raw_buffer_load_b128(r0, v16 + 1024, so, SYS);
            nxt.h3 = __builtin_amdgcn_raw_buffer_load_b96(r0, lane * 12 + 2048, so, SYS);
            nxt.p0 = __builtin_amdgcn_raw_buffer_load_b128(r1, v16, ps, SYS);
            nxt.p1 = __builtin_amdgcn_raw_buffer_load_b128(r1, v16 + 1024, ps, SYS);
            nxt.p2 = __builtin_amdgcn_raw_buffer_load_b128(r1, v16 + 2048, ps, SYS);
            nxt.p3 = __builtin_amdgcn_raw_buffer_load_b128(r1, v16 + 3072, ps, SYS);
        }
        const bool poll = rmask != 0xffffffffu && !dead;
        const rsrc_t rf = mk(myws, poll ? O_DBG : 0u);
        const unsigned nf = __builtin_amdgcn_raw_buffer_load_b32(rf, oflag, 0, SYS);
        const unsigned nt = __builtin_amdgcn_raw_buffer_load_b32(rf, O_TMO, 0, SYS);
        const unsigned tk = draw(tick, can && idx < WLO && lane == 0);

        if (state == 2) {
            st128(cur.l0, locd, v16, cdst, SYS);
            st128(cur.l1, locd, v16 + 1024, cdst, SYS);
        } else if (state == 1) {
            const unsigned oc = (unsigned)(g - NIN);
            const bool raw = (__builtin_amdgcn_readlane(fv, oc >> 5) >> ((oc >> 2) & 7)) & 1u;
            const unsigned L[8] = {cur.l0.x, cur.l0.y, cur.l0.z, cur.l0.w, cur.l1.x, cur.l1.y, cur.l1.z, cur.l1.w};
            const u32x4 P[4] = {cur.p0, cur.p1, cur.p2, cur.p3};
            u32x4 v[4];
            if (raw && !dead) {
                const unsigned so = par + P_OS + (unsigned)me * (4 * MiB) + oc * 4096u;
                const unsigned h1 = __builtin_amdgcn_raw_buffer_load_b32(rem, lane * 4 + 2816, so, SYS);
                const u32x4 h4 = __builtin_amdgcn_raw_buffer_load_b128(rem, v16 + 3072, so, SYS);
                const unsigned H[8] = {cur.h3.x, cur.h3.y, cur.h3.z, h1, h4.x, h4.y, h4.z, h4.w};
                #pragma unroll
                for (int r = 0; r < 4; ++r) {
                    v[r].x = perm(H[2 * r], L[2 * r], 0x05040100u);
                    v[r].y = perm(H[2 * r], L[2 * r], 0x07060302u);
                    v[r].z = perm(H[2 * r + 1], L[2 * r + 1], 0x05040100u);
                    v[r].w = perm(H[2 * r + 1], L[2 * r + 1], 0x07060302u);
                }
            } else {
                unsigned B[4];
                B[0] = cur.h3.x & 0x3f3f3f3fu;
                B[1] = cur.h3.y & 0x3f3f3f3fu;
                B[2] = cur.h3.z & 0x3f3f3f3fu;
                B[3] = ((cur.h3.x >> 6) & 0x03030303u) | ((cur.h3.y >> 4) & 0x0c0c0c0cu) | ((cur.h3.z >> 2) & 0x30303030u);
                #pragma unroll
                for (int r = 0; r < 4; ++r) {
                    v[r].x = perm(B[r], L[2 * r], 0x0c040100u);
                    v[r].y = perm(B[r], L[2 * r], 0x0c050302u);
                    v[r].z = perm(B[r], L[2 * r + 1], 0x0c060100u);
                    v[r].w = perm(B[r], L[2 * r + 1], 0x0c070302u);
                }
            }
            #pragma unroll
            for (int r = 0; r < 4; ++r) {
                const u32x4 t = v[r] + P[r];
                f32x4 f;
                f.x = (float)(int)t.x * scale; f.y = (float)(int)t.y * scale;
                f.z = (float)(int)t.z * scale; f.w = (float)(int)t.w * scale;
                u32x4 o4 = __builtin_bit_cast(u32x4, f);
                if (dead) o4 = mk4(0x7fc00000u, 0x7fc00000u, 0x7fc00000u, 0x7fc00000u);
                st128(o4, out, ovoff + r * (NCOL * 4), cdst, OUT_AUX);
            }
        }
        if (!have && state != 2) break;
        drain();
        if (state == 2) {
            if (lane == 0 && !dead) st_sys(prog, (call << 12) + (unsigned)NIN);
            STAMP(2);
        }
        if (!have) break;
        if (poll) {
            fv = nf;
            const unsigned long long m = BALLOT(lane < 32 ? (nf >> 8) >= call : nf >= call);
            rmask = (unsigned)m & (unsigned)(m >> 32);
            if (UNI(nt) == call) dead = 1;
        }
        if (can) {
            cur = nxt; g = gn; state = 1;
            const unsigned n = o >> 5, w8 = (o >> 2) & 7, mf = o & 3;
            const unsigned c = n >= 16 ? n >> 3 : n < 2 * TA ? (n >= TA ? 1 : 0) : n & 1;
            const unsigned t = n >= 16 ? n & 7 : n < 2 * TA ? n - c * TA : TA + ((n - 2 * TA) >> 1);
            cdst = ((c * 128 + (w8 >> 2) * 64 + mf * 16) * NCOL + peer * 2048 + t * 256 + (w8 & 3) * 64) * 4u;
            gn = idx < WLO ? (int)(UNI(tk) - tbase) : NTOT;
            waiting = 0;
        } else {
            if (!waiting) { waiting = 1; t0 = now(); }
            else {
                if (now() - t0 > DEADLINE) { if (lane == 0) st_sys(tmo, call); dead = 1; }
                __builtin_amdgcn_s_sleep(4);
            }
            state = 0;
        }
    }
    STAMP(3);
}

__device__ __forceinline__ unsigned prep(const Params& p, char* myws, const unsigned* cntp, int id, int tid, int lane, int wv) {
    const int me = p.rank;
    unsigned call;
    const rsrc_t loc = mk(myws, WS_USED);
    if (id < 32) {
        const rsrc_t xs = mk(p.x, 4096u * KS);
        const unsigned voff = (unsigned)(lane & 15) * KS + (lane >> 4) * 16;
        const unsigned soff = (unsigned)(id * 128 + wv * 16) * KS;
        u32x4 v[8];
        #pragma unroll
        for (int i = 0; i < 8; ++i) v[i] = __builtin_amdgcn_raw_buffer_load_b128(xs, voff + i * 64, soff, 0);
        call = UNI(ld_sys(cntp)) + 1u;
        const unsigned dst = O_PAR + (call & 1u) * PAR_BYTES + P_X + (unsigned)me * (2 * MiB) + (unsigned)id * 65536u + wv * 1024u;
        #pragma unroll
        for (int i = 0; i < 8; ++i) st128(v[i], loc, lane * 16, dst + i * 8192u, SYS);
    } else {
        const int pid = id - 32, jt = pid >> 1, h = pid & 1;
        const rsrc_t wsr = mk(p.w, (unsigned)KS * NCOL);
        const int wn = wv & 3, kq0 = h * 4 + (wv >> 2) * 2;
        const unsigned voff = (unsigned)(16 * (lane >> 4)) * NCOL + (lane & 15) * 4;
        unsigned r[2][16];
        #pragma unroll
        for (int b = 0; b < 2; ++b)
            #pragma unroll
            for (int j = 0; j < 16; ++j)
                r[b][j] = __builtin_amdgcn_raw_buffer_load_b32(wsr, voff, (unsigned)((kq0 + b) * 64 + j) * NCOL + jt * 256 + wn * 64, 0);
        call = UNI(ld_sys(cntp)) + 1u;
        const unsigned par = O_PAR + (call & 1u) * PAR_BYTES;
        #pragma unroll
        for (int b = 0; b < 2; ++b) {
            unsigned o[4][4];
            #pragma unroll
            for (int q = 0; q < 4; ++q) {
                const unsigned t0 = perm(r[b][4 * q + 1], r[b][4 * q], 0x05010400u);
                const unsigned t1 = perm(r[b][4 * q + 1], r[b][4 * q], 0x07030602u);
                const unsigned t2 = perm(r[b][4 * q + 3], r[b][4 * q + 2], 0x05010400u);
                const unsigned t3 = perm(r[b][4 * q + 3], r[b][4 * q + 2], 0x07030602u);
                o[0][q] = perm(t2, t0, 0x05040100u);
                o[1][q] = perm(t2, t0, 0x07060302u);
                o[2][q] = perm(t3, t1, 0x05040100u);
                o[3][q] = perm(t3, t1, 0x07060302u);
            }
            const unsigned dst = par + P_WS + (unsigned)jt * 131072u + (unsigned)((kq0 + b) * 16 + wn * 4) * 1024u;
            #pragma unroll
            for (int i = 0; i < 4; ++i)
                st128(mk4(o[i][0], o[i][1], o[i][2], o[i][3]), loc, lane * 16 + i * 1024, dst, SYS);
        }
    }
    drain();
    __syncthreads();
    if (id < 32) {
        if (tid < NR) st_sys((unsigned*)(p.ws[tid] + O_RDY) + me * 64 + id, call);
    } else {
        const int pid = id - 32;
        if (tid == 0) st_sys((unsigned*)(p.ws[pid >> 4] + O_RDY) + me * 64 + 32 + (pid & 15), call);
        if (tid == 1) st_sys((unsigned*)(myws + O_LRW) + pid, call);
    }
    return call;
}

}

__global__ __launch_bounds__(NTHR, 2) void gemm_rs_fused(Params p) {
    __shared__ __attribute__((aligned(16))) char lds[3 * STAGE + 64];
    const int tid = threadIdx.x, lane = tid & 63;
    const int wv = __builtin_amdgcn_readfirstlane(tid >> 6);
    const int b = blockIdx.x, me = p.rank;
    char* myws = p.ws[me];
    unsigned* cntp = (unsigned*)(myws + O_CNT) + b * 8 + wv;

    if (b < NPULL) {
        const int s = wv * NPULL + b;
        if (s < NLW) link_wave(p, myws, cntp, s, lane, wv);
        else if (b == NPULL - 1 && wv == 7) {
            const unsigned call = UNI(ld_sys(cntp)) + 1u;
            if (lane == 0) st_sys(cntp, call);
            monitor(myws, call, lane);
        }
        return;
    }

    const int cb = b - NPULL;
    STAMP(0);
    unsigned call;
    if (cb < 160) call = prep(p, myws, cntp, cb, tid, lane, wv);
    else call = UNI(ld_sys(cntp)) + 1u;
    if (lane == 0) st_sys(cntp, call);
    STAMP(1);

    const int t = b & 7, u = cb >> 3;
    const unsigned par = O_PAR + (call & 1u) * PAR_BYTES;
    const rsrc_t loc = mk(myws, WS_USED);
    unsigned* tmo = (unsigned*)(myws + O_TMO);
    volatile __attribute__((address_space(3))) unsigned* misc =
        (volatile __attribute__((address_space(3))) unsigned*)(lds + LDS_MISC);
    const int wm = wv >> 2, wn = wv & 3;

    for (int step = 0; step < 4; ++step) {
        int task, v;
        constexpr int NP1 = NCU - 7;
        if (step == 0) { task = 0; v = u - 7; if (v < 0) continue; }
        else if (step == 1) { task = 0; v = NP1 + (NCU - 1 - u); if (v >= 28) continue; }
        else if (step == 2) { task = 1; v = u; }
        else {
            task = 1; v = NCU + u;
            if (v >= 28) { task = 2; v = (v - 28) * 7; if (v >= 28) break; }
        }
        const int c = v / 7, d = (me + 1 + (v - 7 * c)) & 7;
        const int rb = task == 1 ? 4 * d + c : 4 * me + c;
        const int pos = tile_pos(c, t);
        const int wj = task == 0 ? d : me;

        i32x4 acc[4][4];
        {
            const int a0 = task == 0 ? -BIAS : task == 1 ? BIAS : 0;
            #pragma unroll
            for (int m = 0; m < 4; ++m)
                #pragma unroll
                for (int n = 0; n < 4; ++n) acc[m][n] = (i32x4){a0, a0, a0, a0};
        }
        unsigned todo = task == 0 ? 1u << me : task == 1 ? 0xffu & ~(1u << d) : 0xffu;
        bool ok = true;
        while (todo) {
            if (wv == 0) {
                const unsigned gend = task == 2 ? (unsigned)(G_XO + 32 * (c + 1))
                                    : c >= 2 ? (unsigned)(G_X2 + 192 * (c - 1))
                                    : t >= TA ? (unsigned)(G_WB + 64 * (t - TA + 1))
                                    : c == 1 ? (unsigned)G_WB : (unsigned)(G_WA + 64 * (t + 1));
                const unsigned* ptr = (const unsigned*)(myws + O_RDY) + me * 64 + rb;
                unsigned need = call;
                if (lane == 8 || lane == 9) ptr = (const unsigned*)(myws + O_LRW) + (wj * 8 + t) * 2 + (lane - 8);
                if (lane == 10) ptr = tmo;
                if (lane < 7) { ptr = (const unsigned*)(myws + O_G + (b & 7) * 128) + lane; need = (call << 11) + gend; }
                const unsigned long long t0 = now();
                unsigned got;
                for (;;) {
                    const unsigned val = ld_sys(ptr);
                    unsigned m = (unsigned)BALLOT(val >= need);
                    if (__builtin_amdgcn_readlane(val, 10) == call) { got = 0; break; }
                    m = (m & 0x7fu) | (((m >> 7) & 7u) == 7u ? 0x80u : 0u);
                    m = ((m | (m << 8)) >> (7 - me)) & 0xffu;
                    got = m & todo;
                    if (got) break;
                    if (now() - t0 > DEADLINE) { if (lane == 0) st_sys(tmo, call); break; }
                    __builtin_amdgcn_s_sleep(8);
                }
                if (lane == 0) misc[0] = got;
            }
            __syncthreads();
            const unsigned got = UNI(misc[0]);
            if (!got) { ok = false; break; }
            todo &= ~got;
            unsigned klist = 0, rest = got;
            int nst = 0;
            while (rest) { klist |= (unsigned)__builtin_ctz(rest) << nst; rest &= rest - 1; nst += 4; }
            const unsigned va = wv * 2048 + lane * 16, vb = wv * 4096 + lane * 16;
            auto issue = [&](int s, int slot) {
                const int k = (klist >> (s & ~3)) & 15;
                const unsigned abase = par + P_X + (unsigned)k * (2 * MiB) + (unsigned)rb * 65536u + (s & 3) * 16384u;
                const unsigned bbase = (k == me ? par + P_WS + (unsigned)wj * MiB : O_WR + (unsigned)k * MiB) + (unsigned)t * 131072u + (s & 3) * 32768u;
                char* la = lds + slot * STAGE + wv * 2048;
                char* lb = lds + slot * STAGE + 16384 + wv * 4096;
                __builtin_amdgcn_raw_ptr_buffer_load_lds(loc, (__attribute__((address_space(3))) void*)la, 16, va, abase, 0, SYS);
                __builtin_amdgcn_raw_ptr_buffer_load_lds(loc, (__attribute__((address_space(3))) void*)la, 16, va, abase, 1024, SYS);
                __builtin_amdgcn_raw_ptr_buffer_load_lds(loc, (__attribute__((address_space(3))) void*)lb, 16, vb, bbase, 0, SYS);
                __builtin_amdgcn_raw_ptr_buffer_load_lds(loc, (__attribute__((address_space(3))) void*)lb, 16, vb, bbase, 1024, SYS);
                __builtin_amdgcn_raw_ptr_buffer_load_lds(loc, (__attribute__((address_space(3))) void*)lb, 16, vb, bbase, 2048, SYS);
                __builtin_amdgcn_raw_ptr_buffer_load_lds(loc, (__attribute__((address_space(3))) void*)lb, 16, vb, bbase, 3072, SYS);
            };
            auto compute = [&](int slot) {
                const char* A = lds + slot * STAGE + wm * 4096 + lane * 16;
                const char* B = lds + slot * STAGE + 16384 + wn * 4096 + lane * 16;
                i32x4 af[2][4], bf[2][4];
                #pragma unroll
                for (int kb = 0; kb < 2; ++kb) {
                    af[kb][0] = *(const i32x4*)(A + kb * 8192);
                    #pragma unroll
                    for (int i = 0; i < 4; ++i) bf[kb][i] = *(const i32x4*)(B + kb * 16384 + i * 1024);
                    #pragma unroll
                    for (int i = 1; i < 4; ++i) af[kb][i] = *(const i32x4*)(A + kb * 8192 + i * 1024);
                }
                __builtin_amdgcn_sched_barrier(0);
                #pragma unroll
                for (int kb = 0; kb < 2; ++kb)
                    #pragma unroll
                    for (int m = 0; m < 4; ++m)
                        #pragma unroll
                        for (int n = 0; n < 4; ++n)
                            acc[m][n] = __builtin_amdgcn_mfma_i32_16x16x64_i8(af[kb][m], bf[kb][n], acc[m][n], 0, 0, 0);
                __builtin_amdgcn_sched_barrier(0);
            };
            issue(0, 0);
            issue(1, 1);
            int slot = 0;
            for (int s = 0; s < nst; ++s) {
                if (s + 1 < nst) asm volatile("s_waitcnt vmcnt(6) lgkmcnt(0)" ::: "memory");
                else asm volatile("s_waitcnt vmcnt(0) lgkmcnt(0)" ::: "memory");
                __builtin_amdgcn_s_barrier();
                if (s + 2 < nst) issue(s + 2, slot == 0 ? 2 : slot - 1);
                compute(slot);
                slot = slot == 2 ? 0 : slot + 1;
            }
        }

        if (task == 0) {
            if (ok) {
                const unsigned dst = O_PS + (unsigned)d * (4 * MiB) + (unsigned)(pos * 8 + wv) * 16384u;
                #pragma unroll
                for (int m = 0; m < 4; ++m)
                    #pragma unroll
                    for (int r = 0; r < 4; ++r)
                        st128(mk4((unsigned)acc[m][0][r], (unsigned)acc[m][1][r], (unsigned)acc[m][2][r], (unsigned)acc[m][3][r]),
                            loc, lane * 16 + r * 1024, dst + m * 4096u, SYS);
            }
            drain();
            __syncthreads();
            if (tid == 0 && ok) st_sys((unsigned*)(myws + O_PRDY) + d * 32 + pos, call);
        } else if (task == 1) {
            unsigned any = 0;
            #pragma unroll
            for (int m = 0; m < 4; ++m)
                #pragma unroll
                for (int n = 0; n < 4; ++n)
                    any |= (unsigned)(acc[m][n][0] | acc[m][n][1]) | (unsigned)(acc[m][n][2] | acc[m][n][3]);
            const bool raw = BALLOT((any >> 22) != 0) != 0;
            if (ok) {
                const unsigned dst = par + P_OS + (unsigned)d * (4 * MiB) + (unsigned)(pos * 8 + wv) * 16384u;
                #pragma unroll
                for (int m = 0; m < 4; ++m) {
                    unsigned L[8], H[8];
                    #pragma unroll
                    for (int r = 0; r < 4; ++r) {
                        L[2 * r] = perm(acc[m][1][r], acc[m][0][r], 0x05040100u);
                        L[2 * r + 1] = perm(acc[m][3][r], acc[m][2][r], 0x05040100u);
                        H[2 * r] = perm(acc[m][1][r], acc[m][0][r], 0x07060302u);
                        H[2 * r + 1] = perm(acc[m][3][r], acc[m][2][r], 0x07060302u);
                    }
                    st128(mk4(L[0], L[1], L[2], L[3]), loc, lane * 16, dst + m * 4096u, SYS);
                    st128(mk4(L[4], L[5], L[6], L[7]), loc, lane * 16 + 1024, dst + m * 4096u, SYS);
                    if (raw) {
                        st96(mk3(H[0], H[1], H[2]), loc, lane * 12 + 2048, dst + m * 4096u, SYS);
                        st32(H[3], loc, lane * 4 + 2816, dst + m * 4096u, SYS);
                        st128(mk4(H[4], H[5], H[6], H[7]), loc, lane * 16 + 3072, dst + m * 4096u, SYS);
                    } else {
                        unsigned B[4];
                        #pragma unroll
                        for (int r = 0; r < 4; ++r) B[r] = perm(H[2 * r + 1], H[2 * r], 0x06040200u);
                        const u32x3 h = {bfi(0x3f3f3f3fu, B[0], B[3] << 6), bfi(0x3f3f3f3fu, B[1], B[3] << 4),
                                         bfi(0x3f3f3f3fu, B[2], B[3] << 2)};
                        st96(h, loc, lane * 12 + 2048, dst + m * 4096u, SYS);
                    }
                }
            }
            if (lane == 0) misc[4 + wv] = raw ? 1u : 0u;
            drain();
            __syncthreads();
            if (tid == 0 && ok) {
                unsigned mask = 0;
                #pragma unroll
                for (int i = 0; i < 8; ++i) mask |= misc[4 + i] << i;
                st_sys((unsigned*)(p.ws[d] + O_ORDY) + ((call & 1u) * 8 + me) * 32 + pos, (call << 8) | mask);
            }
        } else {
            const rsrc_t out = mk(p.out, OUT_BYTES);
            const float scale = p.sx[0] * p.sw[0];
            const unsigned ovoff = (unsigned)(4 * (lane >> 4)) * (NCOL * 4) + (lane & 15) * 16;
            const unsigned dst = ((unsigned)(c * 128 + wm * 64) * NCOL + me * 2048 + t * 256 + wn * 64) * 4u;
            #pragma unroll
            for (int m = 0; m < 4; ++m)
                #pragma unroll
                for (int r = 0; r < 4; ++r) {
                    f32x4 f;
                    f.x = (float)acc[m][0][r] * scale; f.y = (float)acc[m][1][r] * scale;
                    f.z = (float)acc[m][2][r] * scale; f.w = (float)acc[m][3][r] * scale;
                    u32x4 o4 = __builtin_bit_cast(u32x4, f);
                    if (!ok) o4 = (u32x4){0x7fc00000u, 0x7fc00000u, 0x7fc00000u, 0x7fc00000u};
                    st128(o4, out, ovoff + r * (NCOL * 4), dst + (unsigned)(m * 16) * (NCOL * 4), OUT_AUX);
                }
        }
        STAMP(step == 3 ? 7 : 3 + 2 * task);
    }
}

extern "C" void kernel_launch(int rank, int nranks, ncclComm_t comm, hipStream_t stream,
                              void* const* d_in, const int* in_sizes, int n_in,
                              void* d_out, int out_size, const int* in_dtypes, int out_dtype,
                              void* const* d_out_all, void* const* d_ws_all, size_t ws_bytes) {
    (void)comm; (void)in_sizes; (void)in_dtypes; (void)out_dtype; (void)d_out_all;
    if (nranks != NR || (unsigned)rank >= (unsigned)NR || n_in < 4 || ws_bytes < WS_USED ||
        (size_t)out_size * 4 != OUT_BYTES) {
        fprintf(stderr, "kernel_launch: unexpected problem shape\n");
        return;
    }
    Params p;
    for (int i = 0; i < NR; ++i) p.ws[i] = (char*)d_ws_all[i];
    p.x = (const char*)d_in[0];
    p.w = (const char*)d_in[1];
    p.sx = (const float*)d_in[2];
    p.sw = (const float*)d_in[3];
    p.out = (float*)d_out;
    p.rank = rank;
    p.pad = 0;
    hipLaunchKernelGGL(gemm_rs_fused, dim3(NBLK), dim3(NTHR), 0, stream, p);
    HIP_CHECK(hipGetLastError());
}
```

```cpp
#include <hip/hip_runtime.h>
#include <rccl/rccl.h>
#include <cstdio>
#include <cstdint>

#define HIP_CHECK(expr) do { hipError_t _e = (expr); if (_e != hipSuccess) { \
    fprintf(stderr, "HIP error %s at %s:%d\n", hipGetErrorString(_e), __FILE__, __LINE__); \
    return; } } while (0)

typedef int i32x4 __attribute__((ext_vector_type(4)));
typedef unsigned u32x4 __attribute__((ext_vector_type(4)));
typedef unsigned u32x3 __attribute__((ext_vector_type(3)));
typedef float f32x4 __attribute__((ext_vector_type(4)));
typedef __amdgpu_buffer_rsrc_t rsrc_t;
#define UNI(x) __builtin_amdgcn_readfirstlane(x)
#define BALLOT(c) __builtin_amdgcn_ballot_w64(c)

namespace {

constexpr int NR = 8;
constexpr int NCOL = 16384;
constexpr int KS = 512;
constexpr unsigned MiB = 1u << 20;

constexpr int NBLK = 256, NTHR = 512;
#define NPULL_ 64
#define WL_ 32
constexpr int NPULL = NPULL_;
constexpr int WL = WL_;
constexpr int NLW = 7 * WL;
constexpr int NCU = (NBLK - NPULL) / 8;
#define SH_ 2
constexpr int SH = SH_;
static_assert(NCU % SH == 0, "sharing");
static_assert(NLW <= 7 * NPULL && NLW <= 512 && NCU >= 20 && NCU <= 28 && NBLK - NPULL >= 160, "geometry");

#define RP_ 6
#define WLO_ 30
#define TA_ 8
#define RELAY_TH 44
#define LEAD_ 110
#define X1 1
#define X2 1
constexpr int RP = RP_;
constexpr int WLO = WLO_;
constexpr int TA = TA_;
constexpr int G_WA = 192, G_X1 = G_WA + 64 * TA, G_WB = G_X1 + 192, G_X2 = G_WB + 64 * (8 - TA), G_XO = G_X2 + 192;
constexpr int G_X3 = G_XO + 128;
constexpr int G_ALL = 2047;
constexpr int NOUT = 1024;
constexpr int DYN0 = (RP + 1) * WL;
static_assert(RP >= 2 && RP * WL <= G_WA && WLO <= WL && WL <= 64, "push rounds");
__host__ __device__ constexpr int tile_pos(int c, int t) {
    return c >= 2 ? c * 8 + t : t < TA ? c * TA + t : 2 * TA + (t - TA) * 2 + c;
}

constexpr unsigned O_CNT = 0;
constexpr unsigned O_TMO = 8192;
constexpr unsigned O_TICK = 12288;
constexpr unsigned O_TAB = 14336;
constexpr unsigned O_OWN = 14848;
constexpr unsigned O_RDY = 16384;
constexpr unsigned O_LRW = 20480;
constexpr unsigned O_PROG = 24576;
constexpr unsigned O_PUSH = 26624;
constexpr unsigned O_G = 28672;
constexpr unsigned O_PRDY = 32768;
constexpr unsigned O_ORDY = 36864;
constexpr unsigned O_DBG = 65536;
constexpr unsigned O_WR = 1 * MiB;
constexpr unsigned O_PS = 9 * MiB;
constexpr unsigned O_PAR = 41 * MiB;
constexpr unsigned PAR_BYTES = 56 * MiB;
constexpr unsigned P_X = 0, P_WS = 16 * MiB, P_OS = 24 * MiB;
constexpr unsigned WS_USED = O_PAR + 2 * PAR_BYTES;
constexpr unsigned OUT_BYTES = 512u * NCOL * 4u;

constexpr unsigned long long DEADLINE = 60000000ull;
constexpr int BIAS = 1 << 21;
constexpr int SYS = 17;
#define OUT_AUX 16

constexpr int STAGE = 49152;
constexpr int LDS_MISC = 3 * STAGE;

struct Params {
    char* ws[NR];
    const char* x;
    const char* w;
    const float* sx;
    const float* sw;
    float* out;
    int rank;
    int pad;
};

__device__ __forceinline__ rsrc_t mk(const void* p, unsigned bytes) {
    return __builtin_amdgcn_make_buffer_rsrc((void*)p, 0, bytes, 0x00020000);
}
#define st128(v, r, voff, soff, aux) __builtin_amdgcn_raw_buffer_store_b128(v, r, (voff) + (soff), 0, aux)
#define st96(v, r, voff, soff, aux) __builtin_amdgcn_raw_buffer_store_b96(v, r, (voff) + (soff), 0, aux)
#define st32(v, r, voff, soff, aux) __builtin_amdgcn_raw_buffer_store_b32(v, r, (voff) + (soff), 0, aux)
__device__ __forceinline__ u32x4 mk4(unsigned a, unsigned b, unsigned c, unsigned d) { return (u32x4){a, b, c, d}; }
__device__ __forceinline__ u32x3 mk3(unsigned a, unsigned b, unsigned c) { return (u32x3){a, b, c}; }
__device__ __forceinline__ void st1(const rsrc_t r, unsigned off, unsigned v, int lane) {
    __builtin_amdgcn_raw_buffer_store_b32(v, r, lane == 0 ? off : 0xf0000000u, 0, SYS);
}
__device__ __forceinline__ unsigned ld_sys(const unsigned* p) {
    return __hip_atomic_load(p, __ATOMIC_RELAXED, __HIP_MEMORY_SCOPE_SYSTEM);
}
__device__ __forceinline__ void st_sys(unsigned* p, unsigned v) {
    __hip_atomic_store(p, v, __ATOMIC_RELAXED, __HIP_MEMORY_SCOPE_SYSTEM);
}
__device__ __forceinline__ void drain() {
    asm volatile("s_waitcnt vmcnt(0)" ::: "memory");
    __builtin_amdgcn_s_waitcnt(0x0F70);
}
__device__ __forceinline__ unsigned long long now() { return __builtin_amdgcn_s_memrealtime(); }
__device__ __forceinline__ unsigned perm(unsigned hi, unsigned lo, unsigned sel) {
    return __builtin_amdgcn_perm(hi, lo, sel);
}
__device__ __forceinline__ unsigned bfi(unsigned m, unsigned a, unsigned b) { return (a & m) | (b & ~m); }

#define STAMP(slot) do {} while (0)

__device__ __forceinline__ bool wave_wait(const unsigned* ptr, unsigned need, char* myws, unsigned call, int lane) {
    const unsigned* tmo = (const unsigned*)(myws + O_TMO);
    const unsigned long long t0 = now();
    for (;;) {
        const unsigned v = ld_sys(ptr);
        const unsigned t = ld_sys(tmo);
        if (__builtin_amdgcn_readfirstlane(t) == call) return false;
        if (BALLOT(v < need) == 0) return true;
        if (now() - t0 > DEADLINE) {
            st1(mk(myws, O_DBG), O_TMO, call, lane);
            return false;
        }
        __builtin_amdgcn_s_sleep(1);
    }
}

__device__ __forceinline__ void monitor(const Params& p, char* myws, unsigned call, int lane) {
    unsigned* tmo = (unsigned*)(myws + O_TMO);
    const unsigned* prog = (const unsigned*)(myws + O_PROG);
    const unsigned* push = (const unsigned*)(myws + O_PUSH);
    const int l7 = lane & 7, me = p.rank;
    unsigned* ownp = (unsigned*)(p.ws[lane >> 3] + O_OWN) + me * 8 + ((me + 1 + l7) & 7);
    const unsigned base = call << 12;
    unsigned last = 0, nin = 4095;
    bool planned = false, told = false;
    const unsigned long long t0 = now();
    for (;;) {
        unsigned g[7];
        const unsigned tk = __builtin_amdgcn_raw_buffer_load_b32(mk(myws, !planned && BALLOT(last >= G_X3 - LEAD_ - 160 && l7 < 7) != 0 ? O_DBG : 0u), O_TICK + l7 * 128, 0, SYS);
        #pragma unroll
        for (int l = 0; l < 7; ++l) {
            const unsigned v = ld_sys(prog + l * 64 + lane), q = ld_sys(push + l * 64 + lane);
            g[l] = lane >= WL ? 4095u : (v >= base && q >= call) ? v - base : 0u;
        }
        if (!planned) {
            int r = l7 < 7 && tk >= base ? (int)(tk - base) : 0;
            if (BALLOT(r >= G_X3 - LEAD_) != 0) {
                planned = true;
                r = -r;
                unsigned keep = 0x7fu & ~(1u << l7), recv = 0;
                #pragma unroll
                for (int j = 1; j < 7; ++j) {
                    const int to = l7 + j >= 7 ? l7 + j - 7 : l7 + j, from = l7 - j < 0 ? l7 - j + 7 : l7 - j;
                    const int give = r - __shfl(r, to) > RELAY_TH ? 1 : 0;
                    const int got = __shfl(give, from);
                    r += 32 * (got - give);
                    keep &= ~((unsigned)give << to);
                    recv |= (unsigned)got << from;
                }
                nin = G_X3 + 32 * (__builtin_popcount(keep) + __builtin_popcount(recv));
                if (lane < 7) st_sys((unsigned*)(myws + O_TAB) + lane, (call << 14) | (recv << 7) | keep);
            }
        }
        #pragma unroll
        for (int o = 32; o; o >>= 1)
            #pragma unroll
            for (int l = 0; l < 7; ++l) g[l] = min(g[l], (unsigned)__shfl_xor((int)g[l], o));
        unsigned mine = 4095;
        #pragma unroll
        for (int l = 0; l < 7; ++l) if (l7 == l) mine = g[l];
        mine = mine >= nin ? (unsigned)G_ALL : min(mine, (unsigned)G_X3);
        if (mine > last) {
            last = mine;
            if (l7 < 7) st_sys((unsigned*)(myws + O_G + (lane >> 3) * 128) + l7, (call << 11) + mine);
        }
        if (!told && mine >= G_XO + 32) {
            told = true;
            if (l7 < 7 && (lane >> 3) != me) st_sys(ownp, call);
        }
        if (BALLOT(mine < G_ALL) == 0) return;
        if (UNI(ld_sys(tmo)) == call) return;
        if (now() - t0 > DEADLINE) { if (lane == 0) st_sys(tmo, call); return; }
        __builtin_amdgcn_s_sleep(1);
    }
}

struct Item { u32x4 l0, l1, p0, p1, p2, p3; u32x3 h3; };

__device__ __forceinline__ unsigned draw(const rsrc_t tick, bool sel) {
    int t = 1, z = 0;
    asm volatile("" : "+v"(z));
    if (sel) t = __builtin_amdgcn_raw_ptr_buffer_atomic_add_i32(t, tick, z, 0, 0);
    return (unsigned)t;
}

__device__ __forceinline__ void link_wave(const Params& p, char* myws, const unsigned* cntp, int s, int lane, int wv) {
    const int me = p.rank;
    const int link = s % 7, idx = s / 7;
    const unsigned peer = (unsigned)(me + 1 + link) & 7u;
    const char* remp = p.ws[peer];
    const rsrc_t loc = mk(myws, WS_USED), rem = mk(remp, WS_USED);
    const unsigned v16 = lane * 16;

    u32x4 a0, a1;
    const rsrc_t xs = mk(p.x, 4096u * KS);
    const unsigned vx = (unsigned)(lane & 15) * KS + (lane >> 4) * 16;
    auto rb_of = [&](int g) {
        const int pi = g >> 5, i = pi + (pi >= 6 - link ? 1 : 0);
        return (unsigned)(4 * ((peer + 1 + i) & 7));
    };
    auto xoff = [&](int g) { return (rb_of(g) * 128 + (unsigned)(g & 3) * 32) * KS + (unsigned)((g & 31) >> 2) * 64; };
    const unsigned cv = ld_sys(cntp);
    a0 = __builtin_amdgcn_raw_buffer_load_b128(xs, vx, xoff(idx), 0);
    a1 = __builtin_amdgcn_raw_buffer_load_b128(xs, vx + 16 * KS, xoff(idx), 0);
    const unsigned call = UNI(cv) + 1u;
    STAMP(0);
    const unsigned par = O_PAR + (call & 1u) * PAR_BYTES;
    const unsigned dbase = par + P_X + (unsigned)me * (2 * MiB);
    const unsigned* rdyp = (const unsigned*)(myws + O_RDY) + peer * 64 + lane;
    const unsigned rneed = lane < 48 && !(lane < 32 && (unsigned)(lane >> 2) == peer) ? call : 0u;
    unsigned rv = 0;
    #pragma unroll 1
    for (int r = 0; r < RP - 1; ++r) {
        if (lane == 0 && r == 1 + (idx & 3)) __hip_atomic_fetch_max((unsigned*)(myws + O_TICK + link * 128), (call << 12) + DYN0,
                                          __ATOMIC_RELAXED, __HIP_MEMORY_SCOPE_AGENT);
        const int g = idx + r * WL;
        const unsigned so = xoff(g + WL);
        const u32x4 b0 = __builtin_amdgcn_raw_buffer_load_b128(xs, vx, so, 0);
        const u32x4 b1 = __builtin_amdgcn_raw_buffer_load_b128(xs, vx + 16 * KS, so, 0);
        rv = ld_sys(rdyp);
        asm volatile("" ::: "memory");
        const unsigned d = dbase + rb_of(g) * 65536u + (unsigned)(g & 31) * 2048u;
        st128(a0, rem, v16, d, SYS);
        st128(a1, rem, v16 + 1024, d, SYS);
        drain();
        a0 = b0; a1 = b1;
    }
    {
        const int g = idx + (RP - 1) * WL;
        const unsigned d = dbase + rb_of(g) * 65536u + (unsigned)(g & 31) * 2048u;
        st128(a0, rem, v16, d, SYS);
        st128(a1, rem, v16 + 1024, d, SYS);
    }
    const float scale = p.sx[0] * p.sw[0];
    st1(loc, (unsigned)((const char*)cntp - myws), call, lane);
    int dead = 0;
    if (BALLOT(rv < rneed) != 0) dead = wave_wait(rdyp, rneed, myws, call, lane) ? 0 : 1;
    dead = UNI(dead);
    STAMP(1);

    const unsigned prog = O_PROG + (link * 64 + idx) * 4;
    const rsrc_t tick = mk(myws + O_TICK + link * 128, 4);
    const unsigned tbase = call << 12;
    auto x_off = [&](unsigned k, unsigned rb, int g) {
        return par + P_X + k * (2 * MiB) + rb * 65536u + (unsigned)(g & 31) * 2048u;
    };
    auto in_off = [&](int g, unsigned& src, unsigned& dst) {
        if ((g >= G_WA && g < G_X1) || (g >= G_WB && g < G_X2)) {
            const unsigned o = (unsigned)(g < G_X1 ? g - G_WA : g - G_WB + 64 * TA) * 2048u;
            src = par + P_WS + (unsigned)me * MiB + o;
            dst = O_WR + peer * MiB + o;
        } else {
            unsigned rb;
            if (g >= G_XO) rb = 4 * me + ((((g - G_XO) >> 5) + 3) & 3);
            else {
                const int q = (g < G_WA ? g : g < G_WB ? g - 64 * TA : g - 512) >> 5;
                const int c = q / 6, pi = q - 6 * c;
                const int i = pi + (pi >= link ? 1 : 0);
                rb = 4 * ((me + 1 + i) & 7) + c;
            }
            src = dst = x_off(peer, rb, g);
        }
    };
    const unsigned oflag = lane < 32 ? O_ORDY + (((call & 1u) * 8 + peer) * 32 + lane) * 4
                                     : O_PRDY + (peer * 32 + (lane - 32)) * 4;
    const unsigned tflag = lane < 8 ? O_OWN + (peer * 8 + lane) * 4 : lane == 8 ? O_TAB + link * 4 : O_TMO;
    unsigned fv = 0;
    unsigned tv = 0;
    int g = idx + RP * WL, gn;
    int nin = 0;
    int held = 1;
    unsigned plan = 0, src, cdst, ndst;
    in_off(g, src, cdst);
    {
        const rsrc_t remd = mk(remp, dead ? 0u : WS_USED);
        const u32x4 b0 = __builtin_amdgcn_raw_buffer_load_b128(remd, v16, src, SYS);
        const u32x4 b1 = __builtin_amdgcn_raw_buffer_load_b128(remd, v16 + 1024, src, SYS);
        const unsigned tk = draw(tick, lane == 0);
        drain();
        st1(rem, O_PUSH + ((6 - link) * 64 + idx) * 4, call, lane);
        a0 = b0; a1 = b1;
        gn = (int)(UNI(tk) - tbase);
    }
    for (;;) {
        gn = UNI(gn); dead = UNI(dead); held = UNI(held); nin = UNI(nin); plan = UNI(plan); cdst = UNI(cdst);
        if (gn < G_X3) in_off(gn, src, ndst);
        else {
            const unsigned long long t0 = now();
            for (;;) {
                if (!nin && !dead) {
                    plan = __builtin_amdgcn_readlane(tv, 8);
                    if ((plan >> 14) == (call & 0x3ffffu)) nin = G_X3 + 32 * __builtin_popcount(plan & 0x3fffu);
                }
                if (dead && !nin) { plan = 0; nin = G_X3; }
                if (nin) {
                    if (gn >= nin) break;
                    unsigned m = plan & 0x3fffu;
                    for (int j = (gn - G_X3) >> 5; j; --j) m &= m - 1;
                    const unsigned bit = __builtin_ctz(m);
                    const unsigned other = (unsigned)(me + 1 + (bit >= 7 ? bit - 7 : bit)) & 7u;
                    src = ndst = bit >= 7 ? x_off(other, 4 * peer + 3, gn) : x_off(peer, 4 * other + 3, gn);
                    if (bit < 7 || dead || __builtin_amdgcn_readlane(tv, other) >= call) break;
                }
                if (held) {
                    st128(a0, loc, v16, cdst, SYS);
                    st128(a1, loc, v16 + 1024, cdst, SYS);
                    drain();
                    st1(loc, prog, tbase + (unsigned)gn, lane);
                    held = 0;
                } else {
                    if (now() - t0 > DEADLINE) { st1(loc, O_TMO, call, lane); dead = 1; }
                    __builtin_amdgcn_s_sleep(2);
                }
                tv = __builtin_amdgcn_raw_buffer_load_b32(loc, tflag, 0, SYS);
                drain();
                if (__builtin_amdgcn_readlane(tv, 9) == call) dead = 1;
            }
            if (gn >= nin) break;
        }
        const rsrc_t remd = mk(remp, dead ? 0u : WS_USED), locd = mk(myws, dead || !held ? 0u : WS_USED);
        const rsrc_t alive = mk(myws, dead ? 0u : O_DBG);
        const u32x4 b0 = __builtin_amdgcn_raw_buffer_load_b128(remd, v16, src, SYS);
        const u32x4 b1 = __builtin_amdgcn_raw_buffer_load_b128(remd, v16 + 1024, src, SYS);
        const unsigned tk = draw(tick, lane == 0);
        const rsrc_t rf = mk(myws, gn >= G_X3 - LEAD_ ? O_DBG : 0u);
        fv = __builtin_amdgcn_raw_buffer_load_b32(rf, oflag, 0, SYS);
        tv = __builtin_amdgcn_raw_buffer_load_b32(rf, tflag, 0, SYS);
        asm volatile("" ::: "memory");
        st128(a0, locd, v16, cdst, SYS);
        st128(a1, locd, v16 + 1024, cdst, SYS);
        drain();
        st1(alive, prog, tbase + (unsigned)gn, lane);
        a0 = b0; a1 = b1; held = 1;
        g = gn; cdst = ndst;
        gn = (int)(UNI(tk) - tbase);
    }
    const rsrc_t locd = mk(myws, dead || !held ? 0u : WS_USED);
    const int NIN = nin, NTOT = nin + NOUT;

    const rsrc_t out = mk(p.out, OUT_BYTES);
    const unsigned ovoff = (unsigned)(4 * (lane >> 4)) * (NCOL * 4) + (lane & 15) * 16;
    unsigned rmask;
    Item cur, nxt;
    cur.l0 = a0; cur.l1 = a1;
    int state = 2;
    int waiting = 0;
    unsigned long long t0 = 0;
    {
        const unsigned long long m = BALLOT(lane < 32 ? (fv >> 8) >= call : fv >= call);
        rmask = (unsigned)m & (unsigned)(m >> 32);
    }
    for (;;) {
        g = UNI(g); gn = UNI(gn); dead = UNI(dead); state = UNI(state); rmask = UNI(rmask);
        cdst = UNI(cdst); waiting = UNI(waiting);
        const bool have = gn < NTOT;
        const unsigned o = (unsigned)(gn - NIN) & 1023u;
        const bool can = have && (dead || ((rmask >> (o >> 5)) & 1u));
        {
            const bool go = can && !dead;
            const rsrc_t r0 = mk(remp, go ? WS_USED : 0u), r1 = mk(myws, go ? WS_USED : 0u);
            const unsigned so = par + P_OS + (unsigned)me * (4 * MiB) + o * 4096u;
            const unsigned ps = O_PS + peer * (4 * MiB) + o * 4096u;
            nxt.l0 = __builtin_amdgcn_raw_buffer_load_b128(r0, v16, so, SYS);
            nxt.l1 = __builtin_amdgcn_raw_buffer_load_b128(r0, v16 + 1024, so, SYS);
            nxt.h3 = __builtin_amdgcn_raw_buffer_load_b96(r0, lane * 12 + 2048, so, SYS);
            nxt.p0 = __builtin_amdgcn_raw_buffer_load_b128(r1, v16, ps, SYS);
            nxt.p1 = __builtin_amdgcn_raw_buffer_load_b128(r1, v16 + 1024, ps, SYS);
            nxt.p2 = __builtin_amdgcn_raw_buffer_load_b128(r1, v16 + 2048, ps, SYS);
            nxt.p3 = __builtin_amdgcn_raw_buffer_load_b128(r1, v16 + 3072, ps, SYS);
        }
        const bool poll = rmask != 0xffffffffu && !dead;
        const rsrc_t rf = mk(myws, poll ? O_DBG : 0u);
        const unsigned nf = __builtin_amdgcn_raw_buffer_load_b32(rf, oflag, 0, SYS);
        const unsigned nt = __builtin_amdgcn_raw_buffer_load_b32(rf, O_TMO, 0, SYS);
        const unsigned tk = draw(tick, can && idx < WLO && lane == 0);

        if (state == 2) {
            st128(cur.l0, locd, v16, cdst, SYS);
            st128(cur.l1, locd, v16 + 1024, cdst, SYS);
        } else if (state == 1) {
            const unsigned oc = (unsigned)(g - NIN);
            const bool raw = (__builtin_amdgcn_readlane(fv, oc >> 5) >> ((oc >> 2) & 7)) & 1u;
            const unsigned L[8] = {cur.l0.x, cur.l0.y, cur.l0.z, cur.l0.w, cur.l1.x, cur.l1.y, cur.l1.z, cur.l1.w};
            const u32x4 P[4] = {cur.p0, cur.p1, cur.p2, cur.p3};
            u32x4 v[4];
            if (raw && !dead) {
                const unsigned so = par + P_OS + (unsigned)me * (4 * MiB) + oc * 4096u;
                const unsigned h1 = __builtin_amdgcn_raw_buffer_load_b32(rem, lane * 4 + 2816, so, SYS);
                const u32x4 h4 = __builtin_amdgcn_raw_buffer_load_b128(rem, v16 + 3072, so, SYS);
                const unsigned H[8] = {cur.h3.x, cur.h3.y, cur.h3.z, h1, h4.x, h4.y, h4.z, h4.w};
                #pragma unroll
                for (int r = 0; r < 4; ++r) {
                    v[r].x = perm(H[2 * r], L[2 * r], 0x05040100u);
                    v[r].y = perm(H[2 * r], L[2 * r], 0x07060302u);
                    v[r].z = perm(H[2 * r + 1], L[2 * r + 1], 0x05040100u);
                    v[r].w = perm(H[2 * r + 1], L[2 * r + 1], 0x07060302u);
                }
            } else {
                unsigned B[4];
                B[0] = cur.h3.x & 0x3f3f3f3fu;
                B[1] = cur.h3.y & 0x3f3f3f3fu;
                B[2] = cur.h3.z & 0x3f3f3f3fu;
                B[3] = ((cur.h3.x >> 6) & 0x03030303u) | ((cur.h3.y >> 4) & 0x0c0c0c0cu) | ((cur.h3.z >> 2) & 0x30303030u);
                #pragma unroll
                for (int r = 0; r < 4; ++r) {
                    v[r].x = perm(B[r], L[2 * r], 0x0c040100u);
                    v[r].y = perm(B[r], L[2 * r], 0x0c050302u);
                    v[r].z = perm(B[r], L[2 * r + 1], 0x0c060100u);
                    v[r].w = perm(B[r], L[2 * r + 1], 0x0c070302u);
                }
            }
            #pragma unroll
            for (int r = 0; r < 4; ++r) {
                const u32x4 t = v[r] + P[r];
                f32x4 f;
                f.x = (float)(int)t.x * scale; f.y = (float)(int)t.y * scale;
                f.z = (float)(int)t.z * scale; f.w = (float)(int)t.w * scale;
                u32x4 o4 = __builtin_bit_cast(u32x4, f);
                if (dead) o4 = mk4(0x7fc00000u, 0x7fc00000u, 0x7fc00000u, 0x7fc00000u);
                st128(o4, out, ovoff + r * (NCOL * 4), cdst, OUT_AUX);
            }
        }
        if (!have && state != 2) break;
        drain();
        if (state == 2) {
            if (!dead) st1(loc, prog, tbase + 4095u, lane);
            STAMP(2);
        }
        if (!have) break;
        if (poll) {
            fv = nf;
            const unsigned long long m = BALLOT(lane < 32 ? (nf >> 8) >= call : nf >= call);
            rmask = (unsigned)m & (unsigned)(m >> 32);
            if (UNI(nt) == call) dead = 1;
        }
        if (can) {
            cur = nxt; g = gn; state = 1;
            const unsigned n = o >> 5, w8 = (o >> 2) & 7, mf = o & 3;
            const unsigned c = n >= 16 ? n >> 3 : n < 2 * TA ? (n >= TA ? 1 : 0) : n & 1;
            const unsigned t = n >= 16 ? n & 7 : n < 2 * TA ? n - c * TA : TA + ((n - 2 * TA) >> 1);
            cdst = ((c * 128 + (w8 >> 2) * 64 + mf * 16) * NCOL + peer * 2048 + t * 256 + (w8 & 3) * 64) * 4u;
            gn = idx < WLO ? (int)(UNI(tk) - tbase) : NTOT;
            waiting = 0;
        } else {
            if (!waiting) { waiting = 1; t0 = now(); }
            else {
                if (now() - t0 > DEADLINE) { st1(loc, O_TMO, call, lane); dead = 1; }
                __builtin_amdgcn_s_sleep(4);
            }
            state = 0;
        }
    }
    STAMP(3);
}

__device__ __forceinline__ unsigned prep(const Params& p, char* myws, const unsigned* cntp, int id, int tid, int lane, int wv) {
    const int me = p.rank;
    unsigned call;
    const rsrc_t loc = mk(myws, WS_USED);
    if (id < 32) {
        const rsrc_t xs = mk(p.x, 4096u * KS);
        const unsigned voff = (unsigned)(lane & 15) * KS + (lane >> 4) * 16;
        const unsigned soff = (unsigned)(id * 128 + wv * 16) * KS;
        u32x4 v[8];
        #pragma unroll
        for (int i = 0; i < 8; ++i) v[i] = __builtin_amdgcn_raw_buffer_load_b128(xs, voff + i * 64, soff, 0);
        call = UNI(ld_sys(cntp)) + 1u;
        const unsigned dst = O_PAR + (call & 1u) * PAR_BYTES + P_X + (unsigned)me * (2 * MiB) + (unsigned)id * 65536u + wv * 1024u;
        #pragma unroll
        for (int i = 0; i < 8; ++i) st128(v[i], loc, lane * 16, dst + i * 8192u, SYS);
    } else {
        const int pid = id - 32, jt = pid >> 1, h = pid & 1;
        const rsrc_t wsr = mk(p.w, (unsigned)KS * NCOL);
        const int wn = wv & 3, kq0 = h * 4 + (wv >> 2) * 2;
        const unsigned voff = (unsigned)(16 * (lane >> 4)) * NCOL + (lane & 15) * 4;
        unsigned r[2][16];
        #pragma unroll
        for (int b = 0; b < 2; ++b)
            #pragma unroll
            for (int j = 0; j < 16; ++j)
                r[b][j] = __builtin_amdgcn_raw_buffer_load_b32(wsr, voff, (unsigned)((kq0 + b) * 64 + j) * NCOL + jt * 256 + wn * 64, 0);
        call = UNI(ld_sys(cntp)) + 1u;
        const unsigned par = O_PAR + (call & 1u) * PAR_BYTES;
        #pragma unroll
        for (int b = 0; b < 2; ++b) {
            unsigned o[4][4];
            #pragma unroll
            for (int q = 0; q < 4; ++q) {
                const unsigned t0 = perm(r[b][4 * q + 1], r[b][4 * q], 0x05010400u);
                const unsigned t1 = perm(r[b][4 * q + 1], r[b][4 * q], 0x07030602u);
                const unsigned t2 = perm(r[b][4 * q + 3], r[b][4 * q + 2], 0x05010400u);
                const unsigned t3 = perm(r[b][4 * q + 3], r[b][4 * q + 2], 0x07030602u);
                o[0][q] = perm(t2, t0, 0x05040100u);
                o[1][q] = perm(t2, t0, 0x07060302u);
                o[2][q] = perm(t3, t1, 0x05040100u);
                o[3][q] = perm(t3, t1, 0x07060302u);
            }
            const unsigned dst = par + P_WS + (unsigned)jt * 131072u + (unsigned)((kq0 + b) * 16 + wn * 4) * 1024u;
            #pragma unroll
            for (int i = 0; i < 4; ++i)
                st128(mk4(o[i][0], o[i][1], o[i][2], o[i][3]), loc, lane * 16 + i * 1024, dst, SYS);
        }
    }
    drain();
    __syncthreads();
    if (id < 32) {
        if (tid < NR) st_sys((unsigned*)(p.ws[tid] + O_RDY) + me * 64 + id, call);
    } else {
        const int pid = id - 32;
        if (tid == 0) st_sys((unsigned*)(p.ws[pid >> 4] + O_RDY) + me * 64 + 32 + (pid & 15), call);
        if (tid == 1) st_sys((unsigned*)(myws + O_LRW) + pid, call);
    }
    return call;
}

}

__global__ __launch_bounds__(NTHR, 2) void gemm_rs_fused(Params p) {
    __shared__ __attribute__((aligned(16))) char lds[3 * STAGE + 64];
    const int tid = threadIdx.x, lane = tid & 63;
    const int wv = __builtin_amdgcn_readfirstlane(tid >> 6);
    const int b = blockIdx.x, me = p.rank;
    char* myws = p.ws[me];
    unsigned* cntp = (unsigned*)(myws + O_CNT) + b * 8 + wv;

    if (b < NPULL) {
        const int s = wv * NPULL + b;
        if (s < NLW) link_wave(p, myws, cntp, s, lane, wv);
        else if (b == NPULL - 1 && wv == 7) {
            const unsigned call = UNI(ld_sys(cntp)) + 1u;
            if (lane == 0) st_sys(cntp, call);
            monitor(p, myws, call, lane);
        }
        return;
    }

    const int cb = b - NPULL;
    STAMP(0);
    unsigned call;
    if (cb < 160) call = prep(p, myws, cntp, cb, tid, lane, wv);
    else call = UNI(ld_sys(cntp)) + 1u;
    if (lane == 0) st_sys(cntp, call);
    STAMP(1);

    const int t = (b & 7 & ~(SH - 1)) | ((cb >> 3) & (SH - 1));
    const int u = ((cb >> 3) & ~(SH - 1)) | (b & (SH - 1));
    const unsigned par = O_PAR + (call & 1u) * PAR_BYTES;
    const rsrc_t loc = mk(myws, WS_USED);
    unsigned* tmo = (unsigned*)(myws + O_TMO);
    volatile __attribute__((address_space(3))) unsigned* misc =
        (volatile __attribute__((address_space(3))) unsigned*)(lds + LDS_MISC);
    const int wm = wv >> 2, wn = wv & 3;

    for (int step = 0; step < 4; ++step) {
        int task, v;
        constexpr int NP1 = NCU - 7;
        if (step == 0) { task = 0; v = u - 7; if (v < 0) continue; }
        else if (step == 1) { task = 0; v = NP1 + (NCU - 1 - u); if (v >= 28) continue; }
        else if (step == 2) { task = 1; v = u; }
        else {
            task = 1; v = NCU + u;
            if (v >= 28) { task = 2; v = (v - 28) * 7; if (v >= 28) break; }
        }
        const int c = v / 7, d = (me + 1 + (v - 7 * c)) & 7;
        const int rb = task == 1 ? 4 * d + c : 4 * me + c;
        const int pos = tile_pos(c, t);
        const int wj = task == 0 ? d : me;

        i32x4 acc[4][4];
        {
            const int a0 = task == 0 ? -BIAS : task == 1 ? BIAS : 0;
            #pragma unroll
            for (int m = 0; m < 4; ++m)
                #pragma unroll
                for (int n = 0; n < 4; ++n) acc[m][n] = (i32x4){a0, a0, a0, a0};
        }
        unsigned todo = task == 0 ? 1u << me : task == 1 ? 0xffu & ~(1u << d) : 0xffu;
        bool ok = true;
        while (todo) {
            if (wv == 0) {
                const unsigned gend = task == 2 ? (unsigned)(G_XO + 32 * (((c + 1) & 3) + 1))
                                    : c == 3 ? (unsigned)G_ALL : c == 2 ? (unsigned)G_XO
                                    : t >= TA ? (unsigned)(G_WB + 64 * (t - TA + 1))
                                    : c == 1 ? (unsigned)G_WB : (unsigned)(G_WA + 64 * (t + 1));
                const unsigned* ptr = (const unsigned*)(myws + O_RDY) + me * 64 + rb;
                unsigned need = call;
                if (lane == 8 || lane == 9) ptr = (const unsigned*)(myws + O_LRW) + (wj * 8 + t) * 2 + (lane - 8);
                if (lane == 10) ptr = tmo;
                if (lane < 7) { ptr = (const unsigned*)(myws + O_G + (b & 7) * 128) + lane; need = (call << 11) + gend; }
                const unsigned long long t0 = now();
                unsigned got;
                for (;;) {
                    const unsigned val = ld_sys(ptr);
                    unsigned m = (unsigned)BALLOT(val >= need);
                    if (__builtin_amdgcn_readlane(val, 10) == call) { got = 0; break; }
                    if (task == 1 && c == 3 && !((m >> (v - 21)) & 1u)) m &= ~0x7fu;
                    m = (m & 0x7fu) | (((m >> 7) & 7u) == 7u ? 0x80u : 0u);
                    m = ((m | (m << 8)) >> (7 - me)) & 0xffu;
                    got = m & todo;
                    if (got) break;
                    if (now() - t0 > DEADLINE) { if (lane == 0) st_sys(tmo, call); break; }
                    __builtin_amdgcn_s_sleep(8);
                }
                if (lane == 0) misc[0] = got;
            }
            __syncthreads();
            const unsigned got = UNI(misc[0]);
            if (!got) { ok = false; break; }
            todo &= ~got;
            unsigned klist = 0, rest = got;
            int nst = 0;
            while (rest) { klist |= (unsigned)__builtin_ctz(rest) << nst; rest &= rest - 1; nst += 4; }
            const unsigned va = wv * 2048 + lane * 16, vb = wv * 4096 + lane * 16;
            auto issue = [&](int s, int slot) {
                const int k = (klist >> (s & ~3)) & 15;
                const unsigned abase = par + P_X + (unsigned)k * (2 * MiB) + (unsigned)rb * 65536u + (s & 3) * 16384u;
                const unsigned bbase = (k == me ? par + P_WS + (unsigned)wj * MiB : O_WR + (unsigned)k * MiB) + (unsigned)t * 131072u + (s & 3) * 32768u;
                char* la = lds + slot * STAGE + wv * 2048;
                char* lb = lds + slot * STAGE + 16384 + wv * 4096;
                __builtin_amdgcn_raw_ptr_buffer_load_lds(loc, (__attribute__((address_space(3))) void*)la, 16, va, abase, 0, SYS);
                __builtin_amdgcn_raw_ptr_buffer_load_lds(loc, (__attribute__((address_space(3))) void*)la, 16, va, abase, 1024, SYS);
                __builtin_amdgcn_raw_ptr_buffer_load_lds(loc, (__attribute__((address_space(3))) void*)lb, 16, vb, bbase, 0, SYS);
                __builtin_amdgcn_raw_ptr_buffer_load_lds(loc, (__attribute__((address_space(3))) void*)lb, 16, vb, bbase, 1024, SYS);
                __builtin_amdgcn_raw_ptr_buffer_load_lds(loc, (__attribute__((address_space(3))) void*)lb, 16, vb, bbase, 2048, SYS);
                __builtin_amdgcn_raw_ptr_buffer_load_lds(loc, (__attribute__((address_space(3))) void*)lb, 16, vb, bbase, 3072, SYS);
            };
            auto compute = [&](int slot) {
                const char* A = lds + slot * STAGE + wm * 4096 + lane * 16;
                const char* B = lds + slot * STAGE + 16384 + wn * 4096 + lane * 16;
                i32x4 af[2][4], bf[2][4];
                #pragma unroll
                for (int kb = 0; kb < 2; ++kb) {
                    af[kb][0] = *(const i32x4*)(A + kb * 8192);
                    #pragma unroll
                    for (int i = 0; i < 4; ++i) bf[kb][i] = *(const i32x4*)(B + kb * 16384 + i * 1024);
                    #pragma unroll
                    for (int i = 1; i < 4; ++i) af[kb][i] = *(const i32x4*)(A + kb * 8192 + i * 1024);
                }
                __builtin_amdgcn_sched_barrier(0);
                #pragma unroll
                for (int kb = 0; kb < 2; ++kb)
                    #pragma unroll
                    for (int m = 0; m < 4; ++m)
                        #pragma unroll
                        for (int n = 0; n < 4; ++n)
                            acc[m][n] = __builtin_amdgcn_mfma_i32_16x16x64_i8(af[kb][m], bf[kb][n], acc[m][n], 0, 0, 0);
                __builtin_amdgcn_sched_barrier(0);
            };
            issue(0, 0);
            issue(1, 1);
            int slot = 0;
            for (int s = 0; s < nst; ++s) {
                if (s + 1 < nst) asm volatile("s_waitcnt vmcnt(6) lgkmcnt(0)" ::: "memory");
                else asm volatile("s_waitcnt vmcnt(0) lgkmcnt(0)" ::: "memory");
                __builtin_amdgcn_s_barrier();
                if (s + 2 < nst) issue(s + 2, slot == 0 ? 2 : slot - 1);
                compute(slot);
                slot = slot == 2 ? 0 : slot + 1;
            }
        }

        if (task == 0) {
            if (ok) {
                const unsigned dst = O_PS + (unsigned)d * (4 * MiB) + (unsigned)(pos * 8 + wv) * 16384u;
                #pragma unroll
                for (int m = 0; m < 4; ++m)
                    #pragma unroll
                    for (int r = 0; r < 4; ++r)
                        st128(mk4((unsigned)acc[m][0][r], (unsigned)acc[m][1][r], (unsigned)acc[m][2][r], (unsigned)acc[m][3][r]),
                            loc, lane * 16 + r * 1024, dst + m * 4096u, SYS);
            }
            drain();
            __syncthreads();
            if (tid == 0 && ok) st_sys((unsigned*)(myws + O_PRDY) + d * 32 + pos, call);
        } else if (task == 1) {
            unsigned any = 0;
            #pragma unroll
            for (int m = 0; m < 4; ++m)
                #pragma unroll
                for (int n = 0; n < 4; ++n)
                    any |= (unsigned)(acc[m][n][0] | acc[m][n][1]) | (unsigned)(acc[m][n][2] | acc[m][n][3]);
            const bool raw = BALLOT((any >> 22) != 0) != 0;
            if (ok) {
                const unsigned dst = par + P_OS + (unsigned)d * (4 * MiB) + (unsigned)(pos * 8 + wv) * 16384u;
                #pragma unroll
                for (int m = 0; m < 4; ++m) {
                    unsigned L[8], H[8];
                    #pragma unroll
                    for (int r = 0; r < 4; ++r) {
                        L[2 * r] = perm(acc[m][1][r], acc[m][0][r], 0x05040100u);
                        L[2 * r + 1] = perm(acc[m][3][r], acc[m][2][r], 0x05040100u);
                        H[2 * r] = perm(acc[m][1][r], acc[m][0][r], 0x07060302u);
                        H[2 * r + 1] = perm(acc[m][3][r], acc[m][2][r], 0x07060302u);
                    }
                    st128(mk4(L[0], L[1], L[2], L[3]), loc, lane * 16, dst + m * 4096u, SYS);
                    st128(mk4(L[4], L[5], L[6], L[7]), loc, lane * 16 + 1024, dst + m * 4096u, SYS);
                    if (raw) {
                        st96(mk3(H[0], H[1], H[2]), loc, lane * 12 + 2048, dst + m * 4096u, SYS);
                        st32(H[3], loc, lane * 4 + 2816, dst + m * 4096u, SYS);
                        st128(mk4(H[4], H[5], H[6], H[7]), loc, lane * 16 + 3072, dst + m * 4096u, SYS);
                    } else {
                        unsigned B[4];
                        #pragma unroll
                        for (int r = 0; r < 4; ++r) B[r] = perm(H[2 * r + 1], H[2 * r], 0x06040200u);
                        const u32x3 h = {bfi(0x3f3f3f3fu, B[0], B[3] << 6), bfi(0x3f3f3f3fu, B[1], B[3] << 4),
                                         bfi(0x3f3f3f3fu, B[2], B[3] << 2)};
                        st96(h, loc, lane * 12 + 2048, dst + m * 4096u, SYS);
                    }
                }
            }
            if (lane == 0) misc[4 + wv] = raw ? 1u : 0u;
            drain();
            __syncthreads();
            if (tid == 0 && ok) {
                unsigned mask = 0;
                #pragma unroll
                for (int i = 0; i < 8; ++i) mask |= misc[4 + i] << i;
                st_sys((unsigned*)(p.ws[d] + O_ORDY) + ((call & 1u) * 8 + me) * 32 + pos, (call << 8) | mask);
            }
        } else {
            const rsrc_t out = mk(p.out, OUT_BYTES);
            const float scale = p.sx[0] * p.sw[0];
            const unsigned ovoff = (unsigned)(4 * (lane >> 4)) * (NCOL * 4) + (lane & 15) * 16;
            const unsigned dst = ((unsigned)(c * 128 + wm * 64) * NCOL + me * 2048 + t * 256 + wn * 64) * 4u;
            #pragma unroll
            for (int m = 0; m < 4; ++m)
                #pragma unroll
                for (int r = 0; r < 4; ++r) {
                    f32x4 f;
                    f.x = (float)acc[m][0][r] * scale; f.y = (float)acc[m][1][r] * scale;
                    f.z = (float)acc[m][2][r] * scale; f.w = (float)acc[m][3][r] * scale;
                    u32x4 o4 = __builtin_bit_cast(u32x4, f);
                    if (!ok) o4 = (u32x4){0x7fc00000u, 0x7fc00000u, 0x7fc00000u, 0x7fc00000u};
                    st128(o4, out, ovoff + r * (NCOL * 4), dst + (unsigned)(m * 16) * (NCOL * 4), OUT_AUX);
                }
        }
        STAMP(step == 3 ? 7 : 3 + 2 * task);
    }
}

extern "C" void kernel_launch(int rank, int nranks, ncclComm_t comm, hipStream_t stream,
                              void* const* d_in, const int* in_sizes, int n_in,
                              void* d_out, int out_size, const int* in_dtypes, int out_dtype,
                              void* const* d_out_all, void* const* d_ws_all, size_t ws_bytes) {
    (void)comm; (void)in_sizes; (void)in_dtypes; (void)out_dtype; (void)d_out_all;
    if (nranks != NR || (unsigned)rank >= (unsigned)NR || n_in < 4 || ws_bytes < WS_USED ||
        (size_t)out_size * 4 != OUT_BYTES) {
        fprintf(stderr, "kernel_launch: unexpected problem shape\n");
        return;
    }
    Params p;
    for (int i = 0; i < NR; ++i) p.ws[i] = (char*)d_ws_all[i];
    p.x = (const char*)d_in[0];
    p.w = (const char*)d_in[1];
    p.sx = (const float*)d_in[2];
    p.sw = (const float*)d_in[3];
    p.out = (float*)d_out;
    p.rank = rank;
    p.pad = 0;
    hipLaunchKernelGGL(gemm_rs_fused, dim3(NBLK), dim3(NTHR), 0, stream, p);
    HIP_CHECK(hipGetLastError());
}
```

```cpp
#include <hip/hip_runtime.h>
#include <rccl/rccl.h>
#include <cstdio>
#include <cstdint>

#define HIP_CHECK(expr) do { hipError_t _e = (expr); if (_e != hipSuccess) { \
    fprintf(stderr, "HIP error %s at %s:%d\n", hipGetErrorString(_e), __FILE__, __LINE__); \
    return; } } while (0)

typedef int i32x4 __attribute__((ext_vector_type(4)));
typedef unsigned u32x4 __attribute__((ext_vector_type(4)));
typedef unsigned u32x3 __attribute__((ext_vector_type(3)));
typedef unsigned u32x2 __attribute__((ext_vector_type(2)));
typedef unsigned u32x16 __attribute__((ext_vector_type(16)));
typedef float f32x4 __attribute__((ext_vector_type(4)));
typedef __amdgpu_buffer_rsrc_t rsrc_t;
#define UNI(x) __builtin_amdgcn_readfirstlane(x)
#define BALLOT(c) __builtin_amdgcn_ballot_w64(c)

namespace {

constexpr int NR = 8;
constexpr int NCOL = 16384;
constexpr int KS = 512;
constexpr unsigned MiB = 1u << 20;

constexpr int NBLK = 256, NTHR = 512;
#define NPULL_ 64
#define WL_ 36
constexpr int NPULL = NPULL_;
constexpr int WL = WL_;
constexpr int NLW = 7 * WL;
constexpr int NCU = (NBLK - NPULL) / 8;
#define SH_ 2
constexpr int SH = SH_;
static_assert(NCU % SH == 0, "sharing");
static_assert(NLW <= 7 * NPULL && NLW <= 512 && NCU >= 20 && NCU <= 28 && NBLK - NPULL >= 160, "geometry");

#define RP_ 4
#define WLO_ 30
#define TA_ 8
#define RELAY_TH 44
#define LEAD_ 160
#define X1 1
#define X2 1
#define CSLEEP 16
#define TE_ 2
#define PD_W 0
#define PD_X 0
constexpr int TE = TE_;
#define GO_ 1
constexpr int RP = RP_;
constexpr int WLO = WLO_;
constexpr int TA = TA_;
constexpr int G_WA = 192, G_X1 = G_WA + 64 * TA, G_WB = G_X1 + 192, G_X2 = G_WB + 64 * (8 - TA), G_XO = G_X2 + 192;
constexpr int G_X3 = G_XO + 128;
constexpr int G_ALL = 2047;
constexpr int NOUT = 1024;
constexpr int DYN0 = (RP + 1) * WL;
static_assert(RP >= 2 && RP * WL <= G_WA && WLO <= WL && WL <= 64, "push rounds");
__host__ __device__ constexpr int tile_pos(int c, int t) {
    return c >= 2 ? c * 8 + t : t < TA ? c * TA + t : 2 * TA + (t - TA) * 2 + c;
}

constexpr unsigned O_CNT = 0;
constexpr unsigned O_TMO = 8192;
constexpr unsigned O_TICK = 12288;
constexpr unsigned O_TAB = 14336;
constexpr unsigned O_OWN = 14848;
constexpr unsigned O_RDY = 16384;
constexpr unsigned O_LRW = 20480;
constexpr unsigned O_PROG = 24576;
constexpr unsigned O_PUSH = 26624;
constexpr unsigned O_G = 28672;
constexpr unsigned O_PRDY = 32768;
constexpr unsigned O_ORDY = 36864;
constexpr unsigned O_DBG = 65536;
constexpr unsigned O_EX = 512 * 1024;
constexpr unsigned O_WR = 1 * MiB;
constexpr unsigned O_PS = 9 * MiB;
constexpr unsigned O_PAR = 41 * MiB;
constexpr unsigned PAR_BYTES = 56 * MiB;
constexpr unsigned P_X = 0, P_WS = 16 * MiB, P_OS = 24 * MiB;
constexpr unsigned WS_USED = O_PAR + 2 * PAR_BYTES;
constexpr unsigned OUT_BYTES = 512u * NCOL * 4u;

constexpr unsigned long long DEADLINE = 60000000ull;
constexpr int BIAS = 1 << 20;
constexpr int NEX = 16;
constexpr int SYS = 17;
#define OUT_AUX 16

constexpr int STAGE = 49152;
constexpr int LDS_MISC = 3 * STAGE;
constexpr int LDS_EX = LDS_MISC + 64;

struct Params {
    char* ws[NR];
    const char* x;
    const char* w;
    const float* sx;
    const float* sw;
    float* out;
    int rank;
    int pad;
};

__device__ __forceinline__ rsrc_t mk(const void* p, unsigned bytes) {
    return __builtin_amdgcn_make_buffer_rsrc((void*)p, 0, bytes, 0x00020000);
}
#define st128(v, r, voff, soff, aux) __builtin_amdgcn_raw_buffer_store_b128(v, r, (voff) + (soff), 0, aux)
#define st96(v, r, voff, soff, aux) __builtin_amdgcn_raw_buffer_store_b96(v, r, (voff) + (soff), 0, aux)
#define st32(v, r, voff, soff, aux) __builtin_amdgcn_raw_buffer_store_b32(v, r, (voff) + (soff), 0, aux)
__device__ __forceinline__ u32x4 mk4(unsigned a, unsigned b, unsigned c, unsigned d) { return (u32x4){a, b, c, d}; }
__device__ __forceinline__ u32x3 mk3(unsigned a, unsigned b, unsigned c) { return (u32x3){a, b, c}; }
__device__ __forceinline__ void st1(const rsrc_t r, unsigned off, unsigned v, int lane) {
    __builtin_amdgcn_raw_buffer_store_b32(v, r, lane == 0 ? off : 0xf0000000u, 0, SYS);
}
__device__ __forceinline__ unsigned tmo_off() { return O_TMO + (blockIdx.x & 31) * 128; }
__device__ __forceinline__ void set_tmo(char* myws, unsigned call, int lane) {
    __builtin_amdgcn_raw_buffer_store_b32(call, mk(myws, O_DBG), lane < 32 ? O_TMO + lane * 128 : 0xf0000000u, 0, SYS);
}
__device__ __forceinline__ unsigned ld_sys(const unsigned* p) {
    return __hip_atomic_load(p, __ATOMIC_RELAXED, __HIP_MEMORY_SCOPE_SYSTEM);
}
__device__ __forceinline__ void st_sys(unsigned* p, unsigned v) {
    __hip_atomic_store(p, v, __ATOMIC_RELAXED, __HIP_MEMORY_SCOPE_SYSTEM);
}
__device__ __forceinline__ void drain() {
    asm volatile("s_waitcnt vmcnt(0)" ::: "memory");
    __builtin_amdgcn_s_waitcnt(0x0F70);
}
__device__ __forceinline__ unsigned long long now() { return __builtin_amdgcn_s_memrealtime(); }
__device__ __forceinline__ unsigned perm(unsigned hi, unsigned lo, unsigned sel) {
    return __builtin_amdgcn_perm(hi, lo, sel);
}
__device__ __forceinline__ unsigned bfi(unsigned m, unsigned a, unsigned b) { return (a & m) | (b & ~m); }

#define STAMP(slot) do {} while (0)
#define TICKSTAMP(g) do {} while (0)

__device__ __forceinline__ bool wave_wait(const unsigned* ptr, unsigned need, char* myws, unsigned call, int lane) {
    const unsigned* tmo = (const unsigned*)(myws + tmo_off());
    const unsigned long long t0 = now();
    for (;;) {
        const unsigned v = ld_sys(ptr);
        const unsigned t = ld_sys(tmo);
        if (__builtin_amdgcn_readfirstlane(t) == call) return false;
        if (BALLOT(v < need) == 0) return true;
        if (now() - t0 > DEADLINE) {
            set_tmo(myws, call, lane);
            return false;
        }
        __builtin_amdgcn_s_sleep(1);
    }
}

__device__ __forceinline__ void monitor(const Params& p, char* myws, unsigned call, int lane) {
    const unsigned* tmo = (const unsigned*)(myws + tmo_off());
    const unsigned* prog = (const unsigned*)(myws + O_PROG);
    const unsigned* push = (const unsigned*)(myws + O_PUSH);
    const int l7 = lane & 7, me = p.rank;
    unsigned* ownp = (unsigned*)(p.ws[lane >> 3] + O_OWN) + me * 8 + ((me + 1 + l7) & 7);
    const unsigned base = call << 12;
    unsigned last = 0, nin = 4095;
    bool planned = false, told = false;
    const unsigned long long t0 = now();
    for (;;) {
        unsigned g[7];
        const unsigned tk = __builtin_amdgcn_raw_buffer_load_b32(mk(myws, !planned && BALLOT(last >= G_X3 - LEAD_ - 160 && l7 < 7) != 0 ? O_DBG : 0u), O_TICK + l7 * 128, 0, SYS);
        #pragma unroll
        for (int l = 0; l < 7; ++l) {
            const unsigned v = ld_sys(prog + l * 64 + lane), q = ld_sys(push + l * 64 + lane);
            g[l] = lane >= WL ? 4095u : (v >= base && q >= 2 * call) ? v - base : 0u;
        }
        if (!planned) {
            int r = l7 < 7 && tk >= base ? (int)(tk - base) : 0;
            if (BALLOT(r >= G_X3 - LEAD_) != 0) {
                planned = true;
                r = -r;
                unsigned keep = 0x7fu & ~(1u << l7), recv = 0;
                #pragma unroll
                for (int j = 1; j < 7; ++j) {
                    const int to = l7 + j >= 7 ? l7 + j - 7 : l7 + j, from = l7 - j < 0 ? l7 - j + 7 : l7 - j;
                    const int give = r - __shfl(r, to) > RELAY_TH ? 1 : 0;
                    const int got = __shfl(give, from);
                    r += 32 * (got - give);
                    keep &= ~((unsigned)give << to);
                    recv |= (unsigned)got << from;
                }
                nin = G_X3 + 32 * (__builtin_popcount(keep) + __builtin_popcount(recv));
                if (lane < 7) st_sys((unsigned*)(myws + O_TAB) + lane, (call << 14) | (recv << 7) | keep);
            }
        }
        #pragma unroll
        for (int o = 32; o; o >>= 1)
            #pragma unroll
            for (int l = 0; l < 7; ++l) g[l] = min(g[l], (unsigned)__shfl_xor((int)g[l], o));
        unsigned mine = 4095;
        #pragma unroll
        for (int l = 0; l < 7; ++l) if (l7 == l) mine = g[l];
        mine = mine >= nin ? (unsigned)G_ALL : min(mine, (unsigned)G_X3);
        if (mine > last) {
            last = mine;
            if (l7 < 7) st_sys((unsigned*)(myws + O_G + (lane >> 3) * 128) + l7, (call << 11) + mine);
        }
        if (!told && mine >= G_XO + 32) {
            told = true;
            if (l7 < 7 && (lane >> 3) != me) st_sys(ownp, call);
        }
        if (BALLOT(mine < G_ALL) == 0) return;
        if (UNI(ld_sys(tmo)) == call) return;
        if (now() - t0 > DEADLINE) { set_tmo(myws, call, lane); return; }
        __builtin_amdgcn_s_sleep(1);
    }
}

struct Item { u32x4 l0, l1, p0, p1, p2, p3; u32x2 h4; unsigned h1, ex; };

__device__ __forceinline__ unsigned draw(const rsrc_t tick, bool sel) {
    int t = 1, z = 0;
    asm volatile("" : "+v"(z));
    if (sel) t = __builtin_amdgcn_raw_ptr_buffer_atomic_add_i32(t, tick, z, 0, 0);
    return (unsigned)t;
}

__device__ __forceinline__ void link_wave(const Params& p, char* myws, const unsigned* cntp, int s, int lane, int wv) {
    const int me = p.rank;
    const int link = s % 7, idx = s / 7;
    const unsigned peer = (unsigned)(me + 1 + link) & 7u;
    const char* remp = p.ws[peer];
    const rsrc_t loc = mk(myws, WS_USED), rem = mk(remp, WS_USED);
    const unsigned v16 = lane * 16;

    u32x4 a0, a1;
    const rsrc_t xs = mk(p.x, 4096u * KS);
    const unsigned vx = (unsigned)(lane & 15) * KS + (lane >> 4) * 16;
    auto rb_of = [&](int g) {
        const int pi = g >> 5, i = pi + (pi >= 6 - link ? 1 : 0);
        return (unsigned)(4 * ((peer + 1 + i) & 7));
    };
    auto xoff = [&](int g) { return (rb_of(g) * 128 + (unsigned)(g & 3) * 32) * KS + (unsigned)((g & 31) >> 2) * 64; };
    const unsigned cv = ld_sys(cntp);
    a0 = __builtin_amdgcn_raw_buffer_load_b128(xs, vx, xoff(idx), 0);
    a1 = __builtin_amdgcn_raw_buffer_load_b128(xs, vx + 16 * KS, xoff(idx), 0);
    const unsigned call = UNI(cv) + 1u;
    STAMP(0);
    const unsigned par = O_PAR + (call & 1u) * PAR_BYTES;
    const unsigned dbase = par + P_X + (unsigned)me * (2 * MiB);
    const unsigned roff = lane == 63 ? O_PUSH + (link * 64 + idx) * 4 : O_RDY + (peer * 64 + lane) * 4;
    const unsigned* rdyp = (const unsigned*)(myws + roff);
    const unsigned rall = lane < 48 && !(lane < 32 && (unsigned)(lane >> 2) == peer) ? call : 0u;
    const unsigned rneed = lane == 63 ? (GO_ ? 2 * call - 1 : 0u)
                         : (lane < 32 ? (lane & 3) == 0 : lane < 32 + 2 * TE) ? rall : 0u;
    unsigned rv = 0;
    #pragma unroll 1
    for (int r = 0; r < RP - 1; ++r) {
        if (lane == 0 && r == idx % (RP - 1)) __hip_atomic_fetch_max((unsigned*)(myws + O_TICK + link * 128), (call << 12) + DYN0,
                                          __ATOMIC_RELAXED, __HIP_MEMORY_SCOPE_AGENT);
        const int g = idx + r * WL;
        const unsigned so = xoff(g + WL);
        const u32x4 b0 = __builtin_amdgcn_raw_buffer_load_b128(xs, vx, so, 0);
        const u32x4 b1 = __builtin_amdgcn_raw_buffer_load_b128(xs, vx + 16 * KS, so, 0);
        if (r == RP - 2) st1(rem, O_PUSH + ((6 - link) * 64 + idx) * 4, 2 * call - 1, lane);
        asm volatile("" ::: "memory");
        const unsigned d = dbase + rb_of(g) * 65536u + (unsigned)(g & 31) * 2048u;
        st128(a0, rem, v16, d, SYS);
        st128(a1, rem, v16 + 1024, d, SYS);
        drain();
        a0 = b0; a1 = b1;
    }
    {
        rv = __builtin_amdgcn_raw_buffer_load_b32(loc, roff, 0, SYS);
        asm volatile("" ::: "memory");
        const int g = idx + (RP - 1) * WL;
        const unsigned d = dbase + rb_of(g) * 65536u + (unsigned)(g & 31) * 2048u;
        st128(a0, rem, v16, d, SYS);
        st128(a1, rem, v16 + 1024, d, SYS);
    }
    const float scale = p.sx[0] * p.sw[0];
    st1(loc, (unsigned)((const char*)cntp - myws), call, lane);
    int dead = 0;
    if (BALLOT(rv < rneed) != 0) dead = wave_wait(rdyp, rneed, myws, call, lane) ? 0 : 1;
    dead = UNI(dead);
    int early = BALLOT(rv < rall) != 0;
    STAMP(1);

    const unsigned prog = O_PROG + (link * 64 + idx) * 4;
    const rsrc_t tick = mk(myws + O_TICK + link * 128, 4);
    const unsigned tbase = call << 12;
    auto x_off = [&](unsigned k, unsigned rb, int g) {
        return par + P_X + k * (2 * MiB) + rb * 65536u + (unsigned)(g & 31) * 2048u;
    };
    auto in_off = [&](int g, unsigned& src, unsigned& dst) {
        if ((g >= G_WA && g < G_X1) || (g >= G_WB && g < G_X2)) {
            const unsigned o = (unsigned)(g < G_X1 ? g - G_WA : g - G_WB + 64 * TA) * 2048u;
            src = par + P_WS + (unsigned)me * MiB + o;
            dst = O_WR + peer * MiB + o;
        } else {
            unsigned rb;
            if (g >= G_XO) rb = 4 * me + ((((g - G_XO) >> 5) + 3) & 3);
            else {
                const int q = (g < G_WA ? g : g < G_WB ? g - 64 * TA : g - 512) >> 5;
                const int c = q / 6, pi = q - 6 * c;
                const int i = pi + (pi >= link ? 1 : 0);
                rb = 4 * ((me + 1 + i) & 7) + c;
            }
            src = dst = x_off(peer, rb, g);
        }
    };
    const unsigned oflag = lane < 32 ? O_ORDY + (((call & 1u) * 8 + peer) * 32 + lane) * 4
                                     : O_PRDY + (peer * 32 + (lane - 32)) * 4;
    const unsigned tflag = lane < 8 ? O_OWN + (peer * 8 + lane) * 4 : lane == 8 ? O_TAB + link * 4 : tmo_off();
    unsigned fv = 0;
    unsigned tv = 0;
    int g = idx + RP * WL, gn;
    int nin = 0;
    int held = 1;
    unsigned plan = 0, lack = 0, src, cdst, ndst;
    auto read_plan = [&]() {
        if (!nin) {
            plan = __builtin_amdgcn_readlane(tv, 8);
            if ((plan >> 14) == (call & 0x3ffffu)) nin = G_X3 + 32 * __builtin_popcount(plan & 0x3fffu);
        }
        const unsigned recv = (plan >> 7) & 0x7fu;
        lack = ((recv | (recv << 8)) >> (7 - me)) & 0xffu & ~(unsigned)BALLOT(lane < 8 && tv >= call);
    };
    in_off(g, src, cdst);
    {
        const rsrc_t remd = mk(remp, dead ? 0u : WS_USED);
        const u32x4 b0 = __builtin_amdgcn_raw_buffer_load_b128(remd, v16, src, SYS);
        const u32x4 b1 = __builtin_amdgcn_raw_buffer_load_b128(remd, v16 + 1024, src, SYS);
        const unsigned tk = draw(tick, lane == 0);
        drain();
        st1(rem, O_PUSH + ((6 - link) * 64 + idx) * 4, 2 * call, lane);
        a0 = b0; a1 = b1;
        gn = (int)(UNI(tk) - tbase);
    }
    for (;;) {
        gn = UNI(gn); dead = UNI(dead); held = UNI(held); nin = UNI(nin); plan = UNI(plan); cdst = UNI(cdst); lack = UNI(lack);
        early = UNI(early);
        if (early && gn >= G_WA + 64 * TE) {
            if (!dead) dead = wave_wait(rdyp, rall, myws, call, lane) ? 0 : 1;
            early = 0;
        }
        if (gn < G_X3) in_off(gn, src, ndst);
        else {
            unsigned long long t0 = 0;
            for (;;) {
                if (dead && !nin) { plan = 0; nin = G_X3; }
                if (nin) {
                    if (gn >= nin) break;
                    unsigned m = plan & 0x3fffu;
                    for (int j = (gn - G_X3) >> 5; j; --j) m &= m - 1;
                    const unsigned bit = __builtin_ctz(m);
                    const unsigned other = (unsigned)(me + 1 + (bit >= 7 ? bit - 7 : bit)) & 7u;
                    src = ndst = bit >= 7 ? x_off(other, 4 * peer + 3, gn) : x_off(peer, 4 * other + 3, gn);
                    if (bit < 7 || dead || !((lack >> other) & 1u)) break;
                }
                if (held) {
                    st128(a0, loc, v16, cdst, SYS);
                    st128(a1, loc, v16 + 1024, cdst, SYS);
                    drain();
                    st1(loc, prog, tbase + (unsigned)gn, lane);
                    held = 0;
                } else {
                    if (!t0) t0 = now();
                    if (now() - t0 > DEADLINE) { set_tmo(myws, call, lane); dead = 1; }
                    __builtin_amdgcn_s_sleep(2);
                }
                tv = __builtin_amdgcn_raw_buffer_load_b32(loc, tflag, 0, SYS);
                drain();
                if (__builtin_amdgcn_readlane(tv, 9) == call) dead = 1;
                read_plan();
            }
            if (gn >= nin) break;
        }
        const rsrc_t remd = mk(remp, dead ? 0u : WS_USED), locd = mk(myws, dead || !held ? 0u : WS_USED);
        const rsrc_t alive = mk(myws, dead ? 0u : O_DBG);
        const u32x4 b0 = __builtin_amdgcn_raw_buffer_load_b128(remd, v16, src, SYS);
        const u32x4 b1 = __builtin_amdgcn_raw_buffer_load_b128(remd, v16 + 1024, src, SYS);
        const unsigned tk = draw(tick, lane == 0);
        TICKSTAMP(gn);
        const bool look = gn >= G_X3 - 80 && (!nin || lack);
        fv = __builtin_amdgcn_raw_buffer_load_b32(mk(myws, nin && gn >= nin - 48 ? O_DBG : 0u), oflag, 0, SYS);
        const bool peek = early && gn >= G_WA + 64 * TE - 80;
        const unsigned tn = __builtin_amdgcn_raw_buffer_load_b32(mk(myws, look || peek ? O_DBG : 0u),
                                                                 peek ? roff : tflag, 0, SYS);
        asm volatile("" ::: "memory");
        st128(a0, locd, v16, cdst, SYS);
        st128(a1, locd, v16 + 1024, cdst, SYS);
        drain();
        st1(alive, prog, tbase + (unsigned)gn, lane);
        if (look) { tv = tn; read_plan(); }
        if (peek) early = BALLOT(tn < rall) != 0;
        a0 = b0; a1 = b1; held = 1;
        g = gn; cdst = ndst;
        gn = (int)(UNI(tk) - tbase);
    }
    const rsrc_t locd = mk(myws, dead || !held ? 0u : WS_USED);
    const int NIN = nin, NTOT = nin + NOUT;

    const rsrc_t out = mk(p.out, OUT_BYTES);
    const unsigned ovoff = (unsigned)(4 * (lane >> 4)) * (NCOL * 4) + (lane & 15) * 16;
    const unsigned exb = O_EX + ((call & 1u) * 8 + peer) * (256 * NEX * 4);
    unsigned rmask;
    Item cur, nxt;
    cur.l0 = a0; cur.l1 = a1;
    int state = 2;
    int waiting = 0;
    unsigned long long t0 = 0;
    {
        const unsigned long long m = BALLOT(lane < 32 ? (fv >> 8) >= call : fv >= call);
        rmask = (unsigned)m & (unsigned)(m >> 32);
    }
    for (;;) {
        g = UNI(g); gn = UNI(gn); dead = UNI(dead); state = UNI(state); rmask = UNI(rmask);
        cdst = UNI(cdst); waiting = UNI(waiting);
        const bool have = gn < NTOT;
        const unsigned o = (unsigned)(gn - NIN) & 1023u;
        const bool can = have && (dead || ((rmask >> (o >> 5)) & 1u));
        {
            const bool go = can && !dead;
            const rsrc_t r0 = mk(remp, go ? WS_USED : 0u), r1 = mk(myws, go ? WS_USED : 0u);
            const unsigned so = par + P_OS + (unsigned)me * (4 * MiB) + o * 4096u;
            const unsigned ps = O_PS + peer * (4 * MiB) + o * 4096u;
            nxt.l0 = __builtin_amdgcn_raw_buffer_load_b128(r0, v16, so, SYS);
            nxt.l1 = __builtin_amdgcn_raw_buffer_load_b128(r0, v16 + 1024, so, SYS);
            nxt.h4 = __builtin_amdgcn_raw_buffer_load_b64(r0, lane * 8 + 2048, so, SYS);
            nxt.h1 = __builtin_amdgcn_raw_buffer_load_b16(r0, lane * 2 + 2560, so, SYS);
            nxt.ex = __builtin_amdgcn_raw_buffer_load_b32(r1, (lane & (NEX - 1)) * 4, exb + (o >> 2) * (NEX * 4), SYS);
            nxt.p0 = __builtin_amdgcn_raw_buffer_load_b128(r1, v16, ps, SYS);
            nxt.p1 = __builtin_amdgcn_raw_buffer_load_b128(r1, v16 + 1024, ps, SYS);
            nxt.p2 = __builtin_amdgcn_raw_buffer_load_b128(r1, v16 + 2048, ps, SYS);
            nxt.p3 = __builtin_amdgcn_raw_buffer_load_b128(r1, v16 + 3072, ps, SYS);
        }
        const bool poll = have && !dead && (~rmask & (7u << (o >> 5))) != 0;
        const rsrc_t rf = mk(myws, poll ? O_DBG : 0u);
        const unsigned nf = __builtin_amdgcn_raw_buffer_load_b32(rf, oflag, 0, SYS);
        const unsigned nt = __builtin_amdgcn_raw_buffer_load_b32(rf, tmo_off(), 0, SYS);
        const unsigned tk = draw(tick, can && idx < WLO && lane == 0);

        if (state == 2) {
            st128(cur.l0, locd, v16, cdst, SYS);
            st128(cur.l1, locd, v16 + 1024, cdst, SYS);
        } else if (state == 1) {
            const unsigned oc = (unsigned)(g - NIN);
            const bool raw = (__builtin_amdgcn_readlane(fv, oc >> 5) >> ((oc >> 2) & 7)) & 1u;
            const unsigned L[8] = {cur.l0.x, cur.l0.y, cur.l0.z, cur.l0.w, cur.l1.x, cur.l1.y, cur.l1.z, cur.l1.w};
            u32x16 pv = {cur.p0.x, cur.p0.y, cur.p0.z, cur.p0.w, cur.p1.x, cur.p1.y, cur.p1.z, cur.p1.w,
                         cur.p2.x, cur.p2.y, cur.p2.z, cur.p2.w, cur.p3.x, cur.p3.y, cur.p3.z, cur.p3.w};
            u32x4 v[4];
            if (raw && !dead) {
                const unsigned so = par + P_OS + (unsigned)me * (4 * MiB) + oc * 4096u;
                const u32x4 ha = __builtin_amdgcn_raw_buffer_load_b128(rem, v16 + 2048, so, SYS);
                const u32x4 hb = __builtin_amdgcn_raw_buffer_load_b128(rem, v16 + 3072, so, SYS);
                const unsigned H[8] = {ha.x, ha.y, ha.z, ha.w, hb.x, hb.y, hb.z, hb.w};
                #pragma unroll
                for (int r = 0; r < 4; ++r) {
                    v[r].x = perm(H[2 * r], L[2 * r], 0x05040100u);
                    v[r].y = perm(H[2 * r], L[2 * r], 0x07060302u);
                    v[r].z = perm(H[2 * r + 1], L[2 * r + 1], 0x05040100u);
                    v[r].w = perm(H[2 * r + 1], L[2 * r + 1], 0x07060302u);
                }
            } else {
                unsigned em = (unsigned)BALLOT(lane < NEX && (cur.ex >> 21) == (oc & 3u));
                while (em) {
                    const unsigned e = __builtin_amdgcn_readlane(cur.ex, __builtin_ctz(em));
                    em &= em - 1;
                    pv[(e >> 17) & 15u] += (unsigned)lane == ((e >> 11) & 63u) ? e << 21 : 0u;
                }
                const unsigned t = (cur.h1 & 0x0f0fu) | ((cur.h1 & 0xf0f0u) << 12);
                unsigned B[4];
                B[0] = (cur.h4.x & 0x0f0f0f0fu) | ((t << 4) & 0x10101010u);
                B[1] = ((cur.h4.x >> 4) & 0x0f0f0f0fu) | ((t << 3) & 0x10101010u);
                B[2] = (cur.h4.y & 0x0f0f0f0fu) | ((t << 2) & 0x10101010u);
                B[3] = ((cur.h4.y >> 4) & 0x0f0f0f0fu) | ((t << 1) & 0x10101010u);
                #pragma unroll
                for (int r = 0; r < 4; ++r) {
                    v[r].x = perm(B[r], L[2 * r], 0x0c040100u);
                    v[r].y = perm(B[r], L[2 * r], 0x0c050302u);
                    v[r].z = perm(B[r], L[2 * r + 1], 0x0c060100u);
                    v[r].w = perm(B[r], L[2 * r + 1], 0x0c070302u);
                }
            }
            const u32x4 P[4] = {mk4(pv[0], pv[1], pv[2], pv[3]), mk4(pv[4], pv[5], pv[6], pv[7]),
                                mk4(pv[8], pv[9], pv[10], pv[11]), mk4(pv[12], pv[13], pv[14], pv[15])};
            #pragma unroll
            for (int r = 0; r < 4; ++r) {
                const u32x4 t = v[r] + P[r];
                f32x4 f;
                f.x = (float)(int)t.x * scale; f.y = (float)(int)t.y * scale;
                f.z = (float)(int)t.z * scale; f.w = (float)(int)t.w * scale;
                u32x4 o4 = __builtin_bit_cast(u32x4, f);
                if (dead) o4 = mk4(0x7fc00000u, 0x7fc00000u, 0x7fc00000u, 0x7fc00000u);
                st128(o4, out, ovoff + r * (NCOL * 4), cdst, OUT_AUX);
            }
        }
        if (!have && state != 2) break;
        drain();
        if (state == 2) {
            if (!dead) st1(loc, prog, tbase + 4095u, lane);
            STAMP(2);
        }
        if (!have) break;
        if (poll) {
            fv = nf;
            const unsigned long long m = BALLOT(lane < 32 ? (nf >> 8) >= call : nf >= call);
            rmask = (unsigned)m & (unsigned)(m >> 32);
            if (UNI(nt) == call) dead = 1;
        }
        if (can) {
            cur = nxt; g = gn; state = 1;
            const unsigned n = o >> 5, w8 = (o >> 2) & 7, mf = o & 3;
            const unsigned c = n >= 16 ? n >> 3 : n < 2 * TA ? (n >= TA ? 1 : 0) : n & 1;
            const unsigned t = n >= 16 ? n & 7 : n < 2 * TA ? n - c * TA : TA + ((n - 2 * TA) >> 1);
            cdst = ((c * 128 + (w8 >> 2) * 64 + mf * 16) * NCOL + peer * 2048 + t * 256 + (w8 & 3) * 64) * 4u;
            gn = idx < WLO ? (int)(UNI(tk) - tbase) : NTOT;
            waiting = 0;
        } else {
            if (!waiting) { waiting = 1; t0 = now(); }
            else {
                if (now() - t0 > DEADLINE) { set_tmo(myws, call, lane); dead = 1; }
                __builtin_amdgcn_s_sleep(4);
            }
            state = 0;
        }
    }
    STAMP(3);
}

__device__ __forceinline__ unsigned prep(const Params& p, char* myws, const unsigned* cntp, int id, int tid, int lane, int wv) {
    const int me = p.rank;
    unsigned call;
    {
        const int tw = ((id - 32) >> 1) & 7;
        const unsigned long long hold = id < 32 ? ((id & 3) ? PD_X : 0) : (tw >= TE ? (tw - TE + 2) * PD_W : 0);
        const unsigned long long t0 = now();
        while (now() - t0 < hold) __builtin_amdgcn_s_sleep(4);
    }
    const rsrc_t loc = mk(myws, WS_USED);
    if (id < 32) {
        const rsrc_t xs = mk(p.x, 4096u * KS);
        const unsigned voff = (unsigned)(lane & 15) * KS + (lane >> 4) * 16;
        const unsigned soff = (unsigned)(id * 128 + wv * 16) * KS;
        u32x4 v[8];
        #pragma unroll
        for (int i = 0; i < 8; ++i) v[i] = __builtin_amdgcn_raw_buffer_load_b128(xs, voff + i * 64, soff, 0);
        call = UNI(ld_sys(cntp)) + 1u;
        const unsigned dst = O_PAR + (call & 1u) * PAR_BYTES + P_X + (unsigned)me * (2 * MiB) + (unsigned)id * 65536u + wv * 1024u;
        #pragma unroll
        for (int i = 0; i < 8; ++i) st128(v[i], loc, lane * 16, dst + i * 8192u, SYS);
    } else {
        const int pid = id - 32, jt = pid >> 1, h = pid & 1;
        const rsrc_t wsr = mk(p.w, (unsigned)KS * NCOL);
        const int wn = wv & 3, kq0 = h * 4 + (wv >> 2) * 2;
        const unsigned voff = (unsigned)(16 * (lane >> 4)) * NCOL + (lane & 15) * 4;
        unsigned r[2][16];
        #pragma unroll
        for (int b = 0; b < 2; ++b)
            #pragma unroll
            for (int j = 0; j < 16; ++j)
                r[b][j] = __builtin_amdgcn_raw_buffer_load_b32(wsr, voff, (unsigned)((kq0 + b) * 64 + j) * NCOL + jt * 256 + wn * 64, 0);
        call = UNI(ld_sys(cntp)) + 1u;
        const unsigned par = O_PAR + (call & 1u) * PAR_BYTES;
        #pragma unroll
        for (int b = 0; b < 2; ++b) {
            unsigned o[4][4];
            #pragma unroll
            for (int q = 0; q < 4; ++q) {
                const unsigned t0 = perm(r[b][4 * q + 1], r[b][4 * q], 0x05010400u);
                const unsigned t1 = perm(r[b][4 * q + 1], r[b][4 * q], 0x07030602u);
                const unsigned t2 = perm(r[b][4 * q + 3], r[b][4 * q + 2], 0x05010400u);
                const unsigned t3 = perm(r[b][4 * q + 3], r[b][4 * q + 2], 0x07030602u);
                o[0][q] = perm(t2, t0, 0x05040100u);
                o[1][q] = perm(t2, t0, 0x07060302u);
                o[2][q] = perm(t3, t1, 0x05040100u);
                o[3][q] = perm(t3, t1, 0x07060302u);
            }
            const unsigned dst = par + P_WS + (unsigned)jt * 131072u + (unsigned)((kq0 + b) * 16 + wn * 4) * 1024u;
            #pragma unroll
            for (int i = 0; i < 4; ++i)
                st128(mk4(o[i][0], o[i][1], o[i][2], o[i][3]), loc, lane * 16 + i * 1024, dst, SYS);
        }
    }
    drain();
    __syncthreads();
    if (id < 32) {
        if (tid < NR) st_sys((unsigned*)(p.ws[tid] + O_RDY) + me * 64 + id, call);
    } else {
        const int pid = id - 32;
        if (tid == 0) st_sys((unsigned*)(p.ws[pid >> 4] + O_RDY) + me * 64 + 32 + (pid & 15), call);
        if (tid == 1) st_sys((unsigned*)(myws + O_LRW) + pid, call);
    }
    return call;
}

}

__global__ __launch_bounds__(NTHR, 2) void gemm_rs_fused(Params p) {
    __shared__ __attribute__((aligned(16))) char lds[LDS_EX + 8 * NEX * 4];
    const int tid = threadIdx.x, lane = tid & 63;
    const int wv = __builtin_amdgcn_readfirstlane(tid >> 6);
    const int b = blockIdx.x, me = p.rank;
    char* myws = p.ws[me];
    unsigned* cntp = (unsigned*)(myws + O_CNT) + b * 8 + wv;

    if (b < NPULL) {
        const int s = wv * NPULL + b;
        if (s < NLW) link_wave(p, myws, cntp, s, lane, wv);
        else if (b == NPULL - 1 && wv == 7) {
            const unsigned call = UNI(ld_sys(cntp)) + 1u;
            if (lane == 0) st_sys(cntp, call);
            monitor(p, myws, call, lane);
        }
        return;
    }

    const int cb = b - NPULL;
    STAMP(0);
    unsigned call;
    if (cb < 160) call = prep(p, myws, cntp, cb, tid, lane, wv);
    else call = UNI(ld_sys(cntp)) + 1u;
    if (lane == 0) st_sys(cntp, call);
    STAMP(1);

    const int t = (b & 7 & ~(SH - 1)) | ((cb >> 3) & (SH - 1));
    const int u = ((cb >> 3) & ~(SH - 1)) | (b & (SH - 1));
    const unsigned par = O_PAR + (call & 1u) * PAR_BYTES;
    const rsrc_t loc = mk(myws, WS_USED);
    volatile __attribute__((address_space(3))) unsigned* misc =
        (volatile __attribute__((address_space(3))) unsigned*)(lds + LDS_MISC);
    const int wm = wv >> 2, wn = wv & 3;

    for (int step = 0; step < 4; ++step) {
        int task, v;
        constexpr int NP1 = NCU - 7;
        if (step == 0) { task = 0; v = u - 7; if (v < 0) continue; }
        else if (step == 1) { task = 0; v = NP1 + (NCU - 1 - u); if (v >= 28) continue; }
        else if (step == 2) { task = 1; v = u; }
        else {
            task = 1; v = NCU + u;
            if (v >= 28) { task = 2; v = (v - 28) * 7; if (v >= 28) break; }
        }
        const int c = v / 7, d = (me + 1 + (v - 7 * c)) & 7;
        const int rb = task == 1 ? 4 * d + c : 4 * me + c;
        const int pos = tile_pos(c, t);
        const int wj = task == 0 ? d : me;

        i32x4 acc[4][4];
        {
            const int a0 = task == 0 ? -BIAS : task == 1 ? BIAS : 0;
            #pragma unroll
            for (int m = 0; m < 4; ++m)
                #pragma unroll
                for (int n = 0; n < 4; ++n) acc[m][n] = (i32x4){a0, a0, a0, a0};
        }
        unsigned todo = task == 0 ? 1u << me : task == 1 ? 0xffu & ~(1u << d) : 0xffu;
        bool ok = true;
        while (todo) {
            if (wv == 0) {
                const unsigned gend = task == 2 ? (unsigned)(G_XO + 32 * (((c + 1) & 3) + 1))
                                    : c == 3 ? (unsigned)G_ALL : c == 2 ? (unsigned)G_XO
                                    : t >= TA ? (unsigned)(G_WB + 64 * (t - TA + 1))
                                    : c == 1 ? (unsigned)G_WB : (unsigned)(G_WA + 64 * (t + 1));
                const unsigned* ptr = (const unsigned*)(myws + tmo_off());
                unsigned need = call;
                if ((todo >> me) & 1u) {
                    if (lane == 8 || lane == 9) ptr = (const unsigned*)(myws + O_LRW) + (wj * 8 + t) * 2 + (lane - 8);
                    if (lane == 10) ptr = (const unsigned*)(myws + O_RDY) + me * 64 + rb;
                }
                if (lane < 7 && task) { ptr = (const unsigned*)(myws + O_G + (b & 7) * 128) + lane; need = (call << 11) + gend; }
                const unsigned long long t0 = now();
                unsigned got;
                for (;;) {
                    const unsigned val = ld_sys(ptr);
                    unsigned m = (unsigned)BALLOT(val >= need);
                    if (__builtin_amdgcn_readlane(val, 7) == call) { got = 0; break; }
                    if (task == 1 && c == 3 && !((m >> (v - 21)) & 1u)) m &= ~0x7fu;
                    m = (m & 0x7fu) | (((m >> 8) & 7u) == 7u ? 0x80u : 0u);
                    m = ((m | (m << 8)) >> (7 - me)) & 0xffu;
                    got = m & todo;
                    if (got) break;
                    if (now() - t0 > DEADLINE) { set_tmo(myws, call, lane); break; }
                    __builtin_amdgcn_s_sleep(CSLEEP);
                }
                if (lane == 0) misc[0] = got;
            }
            __syncthreads();
            const unsigned got = UNI(misc[0]);
            if (!got) { ok = false; break; }
            todo &= ~got;
            unsigned klist = 0, rest = got;
            int nst = 0;
            while (rest) { klist |= (unsigned)__builtin_ctz(rest) << nst; rest &= rest - 1; nst += 4; }
            const unsigned va = wv * 2048 + lane * 16, vb = wv * 4096 + lane * 16;
            auto issue = [&](int s, int slot) {
                const int k = (klist >> (s & ~3)) & 15;
                const unsigned abase = par + P_X + (unsigned)k * (2 * MiB) + (unsigned)rb * 65536u + (s & 3) * 16384u;
                const unsigned bbase = (k == me ? par + P_WS + (unsigned)wj * MiB : O_WR + (unsigned)k * MiB) + (unsigned)t * 131072u + (s & 3) * 32768u;
                char* la = lds + slot * STAGE + wv * 2048;
                char* lb = lds + slot * STAGE + 16384 + wv * 4096;
                __builtin_amdgcn_raw_ptr_buffer_load_lds(loc, (__attribute__((address_space(3))) void*)la, 16, va, abase, 0, SYS);
                __builtin_amdgcn_raw_ptr_buffer_load_lds(loc, (__attribute__((address_space(3))) void*)la, 16, va, abase, 1024, SYS);
                __builtin_amdgcn_raw_ptr_buffer_load_lds(loc, (__attribute__((address_space(3))) void*)lb, 16, vb, bbase, 0, SYS);
                __builtin_amdgcn_raw_ptr_buffer_load_lds(loc, (__attribute__((address_space(3))) void*)lb, 16, vb, bbase, 1024, SYS);
                __builtin_amdgcn_raw_ptr_buffer_load_lds(loc, (__attribute__((address_space(3))) void*)lb, 16, vb, bbase, 2048, SYS);
                __builtin_amdgcn_raw_ptr_buffer_load_lds(loc, (__attribute__((address_space(3))) void*)lb, 16, vb, bbase, 3072, SYS);
            };
            auto compute = [&](int slot) {
                const char* A = lds + slot * STAGE + wm * 4096 + lane * 16;
                const char* B = lds + slot * STAGE + 16384 + wn * 4096 + lane * 16;
                i32x4 af[2][4], bf[2][4];
                #pragma unroll
                for (int kb = 0; kb < 2; ++kb) {
                    af[kb][0] = *(const i32x4*)(A + kb * 8192);
                    #pragma unroll
                    for (int i = 0; i < 4; ++i) bf[kb][i] = *(const i32x4*)(B + kb * 16384 + i * 1024);
                    #pragma unroll
                    for (int i = 1; i < 4; ++i) af[kb][i] = *(const i32x4*)(A + kb * 8192 + i * 1024);
                }
                __builtin_amdgcn_sched_barrier(0);
                #pragma unroll
                for (int kb = 0; kb < 2; ++kb)
                    #pragma unroll
                    for (int m = 0; m < 4; ++m)
                        #pragma unroll
                        for (int n = 0; n < 4; ++n)
                            acc[m][n] = __builtin_amdgcn_mfma_i32_16x16x64_i8(af[kb][m], bf[kb][n], acc[m][n], 0, 0, 0);
                __builtin_amdgcn_sched_barrier(0);
            };
            issue(0, 0);
            issue(1, 1);
            int slot = 0;
            for (int s = 0; s < nst; ++s) {
                if (s + 1 < nst) asm volatile("s_waitcnt vmcnt(6) lgkmcnt(0)" ::: "memory");
                else asm volatile("s_waitcnt vmcnt(0) lgkmcnt(0)" ::: "memory");
                __builtin_amdgcn_s_barrier();
                if (s + 2 < nst) issue(s + 2, slot == 0 ? 2 : slot - 1);
                compute(slot);
                slot = slot == 2 ? 0 : slot + 1;
            }
        }

        if (task == 0) {
            if (ok) {
                const unsigned dst = O_PS + (unsigned)d * (4 * MiB) + (unsigned)(pos * 8 + wv) * 16384u;
                #pragma unroll
                for (int m = 0; m < 4; ++m)
                    #pragma unroll
                    for (int r = 0; r < 4; ++r)
                        st128(mk4((unsigned)acc[m][0][r], (unsigned)acc[m][1][r], (unsigned)acc[m][2][r], (unsigned)acc[m][3][r]),
                            loc, lane * 16 + r * 1024, dst + m * 4096u, SYS);
            }
            drain();
            __syncthreads();
            if (tid == 0 && ok) st_sys((unsigned*)(myws + O_PRDY) + d * 32 + pos, call);
        } else if (task == 1) {
            volatile __attribute__((address_space(3))) unsigned* exl =
                (volatile __attribute__((address_space(3))) unsigned*)(lds + LDS_EX) + wv * NEX;
            if (lane < NEX) exl[lane] = 0xffffffffu;
            unsigned nex = 0;
            #pragma unroll
            for (int m = 0; m < 4; ++m)
                #pragma unroll
                for (int r = 0; r < 4; ++r) {
                    const unsigned any = (unsigned)(acc[m][0][r] | acc[m][1][r]) | (unsigned)(acc[m][2][r] | acc[m][3][r]);
                    if (BALLOT((any >> 21) != 0) != 0) {
                        #pragma unroll
                        for (int n = 0; n < 4; ++n) {
                            const unsigned hi = (unsigned)acc[m][n][r] >> 21;
                            const unsigned long long bm = BALLOT(hi != 0);
                            const unsigned at = nex + __builtin_amdgcn_mbcnt_hi((unsigned)(bm >> 32), __builtin_amdgcn_mbcnt_lo((unsigned)bm, 0u));
                            if (hi != 0 && at < NEX) exl[at] = (unsigned)(m << 21 | (r * 4 + n) << 17) | ((unsigned)lane << 11) | hi;
                            nex += __builtin_popcountll(bm);
                        }
                    }
                }
            const bool raw = nex > NEX;
            if (ok) {
                const unsigned e = exl[lane & (NEX - 1)];
                __builtin_amdgcn_raw_buffer_store_b32(e, mk(p.ws[d], WS_USED),
                    lane < NEX ? O_EX + ((((call & 1u) * 8 + me) * 32 + pos) * 8 + wv) * (NEX * 4) + lane * 4 : 0xf0000000u, 0, SYS);
            }
            if (ok) {
                const unsigned dst = par + P_OS + (unsigned)d * (4 * MiB) + (unsigned)(pos * 8 + wv) * 16384u;
                #pragma unroll
                for (int m = 0; m < 4; ++m) {
                    unsigned L[8], H[8];
                    #pragma unroll
                    for (int r = 0; r < 4; ++r) {
                        L[2 * r] = perm(acc[m][1][r], acc[m][0][r], 0x05040100u);
                        L[2 * r + 1] = perm(acc[m][3][r], acc[m][2][r], 0x05040100u);
                        H[2 * r] = perm(acc[m][1][r], acc[m][0][r], 0x07060302u);
                        H[2 * r + 1] = perm(acc[m][3][r], acc[m][2][r], 0x07060302u);
                    }
                    st128(mk4(L[0], L[1], L[2], L[3]), loc, lane * 16, dst + m * 4096u, SYS);
                    st128(mk4(L[4], L[5], L[6], L[7]), loc, lane * 16 + 1024, dst + m * 4096u, SYS);
                    if (raw) {
                        st128(mk4(H[0], H[1], H[2], H[3]), loc, lane * 16 + 2048, dst + m * 4096u, SYS);
                        st128(mk4(H[4], H[5], H[6], H[7]), loc, lane * 16 + 3072, dst + m * 4096u, SYS);
                    } else {
                        unsigned B[4];
                        #pragma unroll
                        for (int r = 0; r < 4; ++r) B[r] = perm(H[2 * r + 1], H[2 * r], 0x06040200u);
                        const u32x2 h4 = {bfi(0x0f0f0f0fu, B[0], B[1] << 4), bfi(0x0f0f0f0fu, B[2], B[3] << 4)};
                        const unsigned t = ((B[0] >> 4) & 0x01010101u) | ((B[1] >> 3) & 0x02020202u)
                                         | ((B[2] >> 2) & 0x04040404u) | ((B[3] >> 1) & 0x08080808u);
                        __builtin_amdgcn_raw_buffer_store_b64(h4, loc, lane * 8 + 2048 + dst + m * 4096u, 0, SYS);
                        __builtin_amdgcn_raw_buffer_store_b16((unsigned short)(t | (t >> 12)), loc, lane * 2 + 2560 + dst + m * 4096u, 0, SYS);
                    }
                }
            }
            if (lane == 0) misc[4 + wv] = raw ? 1u : 0u;
            drain();
            __syncthreads();
            if (tid == 0 && ok) {
                unsigned mask = 0;
                #pragma unroll
                for (int i = 0; i < 8; ++i) mask |= misc[4 + i] << i;
                st_sys((unsigned*)(p.ws[d] + O_ORDY) + ((call & 1u) * 8 + me) * 32 + pos, (call << 8) | mask);
            }
        } else {
            const rsrc_t out = mk(p.out, OUT_BYTES);
            const float scale = p.sx[0] * p.sw[0];
            const unsigned ovoff = (unsigned)(4 * (lane >> 4)) * (NCOL * 4) + (lane & 15) * 16;
            const unsigned dst = ((unsigned)(c * 128 + wm * 64) * NCOL + me * 2048 + t * 256 + wn * 64) * 4u;
            #pragma unroll
            for (int m = 0; m < 4; ++m)
                #pragma unroll
                for (int r = 0; r < 4; ++r) {
                    f32x4 f;
                    f.x = (float)acc[m][0][r] * scale; f.y = (float)acc[m][1][r] * scale;
                    f.z = (float)acc[m][2][r] * scale; f.w = (float)acc[m][3][r] * scale;
                    u32x4 o4 = __builtin_bit_cast(u32x4, f);
                    if (!ok) o4 = (u32x4){0x7fc00000u, 0x7fc00000u, 0x7fc00000u, 0x7fc00000u};
                    st128(o4, out, ovoff + r * (NCOL * 4), dst + (unsigned)(m * 16) * (NCOL * 4), OUT_AUX);
                }
        }
        STAMP(step == 3 ? 7 : 3 + 2 * task);
    }
}

extern "C" void kernel_launch(int rank, int nranks, ncclComm_t comm, hipStream_t stream,
                              void* const* d_in, const int* in_sizes, int n_in,
                              void* d_out, int out_size, const int* in_dtypes, int out_dtype,
                              void* const* d_out_all, void* const* d_ws_all, size_t ws_bytes) {
    (void)comm; (void)in_sizes; (void)in_dtypes; (void)out_dtype; (void)d_out_all;
    if (nranks != NR || (unsigned)rank >= (unsigned)NR || n_in < 4 || ws_bytes < WS_USED ||
        (size_t)out_size * 4 != OUT_BYTES) {
        fprintf(stderr, "kernel_launch: unexpected problem shape\n");
        return;
    }
    Params p;
    for (int i = 0; i < NR; ++i) p.ws[i] = (char*)d_ws_all[i];
    p.x = (const char*)d_in[0];
    p.w = (const char*)d_in[1];
    p.sx = (const float*)d_in[2];
    p.sw = (const float*)d_in[3];
    p.out = (float*)d_out;
    p.rank = rank;
    p.pad = 0;
    hipLaunchKernelGGL(gemm_rs_fused, dim3(NBLK), dim3(NTHR), 0, stream, p);
    HIP_CHECK(hipGetLastError());
}
```

```cpp
#include <hip/hip_runtime.h>
#include <rccl/rccl.h>
#include <cstdio>
#include <cstdint>

#define HIP_CHECK(expr) do { hipError_t _e = (expr); if (_e != hipSuccess) { \
    fprintf(stderr, "HIP error %s at %s:%d\n", hipGetErrorString(_e), __FILE__, __LINE__); \
    return; } } while (0)

typedef int i32x4 __attribute__((ext_vector_type(4)));
typedef unsigned u32x4 __attribute__((ext_vector_type(4)));
typedef unsigned u32x3 __attribute__((ext_vector_type(3)));
typedef unsigned u32x2 __attribute__((ext_vector_type(2)));
typedef unsigned u32x16 __attribute__((ext_vector_type(16)));
typedef float f32x4 __attribute__((ext_vector_type(4)));
typedef __amdgpu_buffer_rsrc_t rsrc_t;
#define UNI(x) __builtin_amdgcn_readfirstlane(x)
#define BALLOT(c) __builtin_amdgcn_ballot_w64(c)

namespace {

constexpr int NR = 8;
constexpr int NCOL = 16384;
constexpr int KS = 512;
constexpr unsigned MiB = 1u << 20;

constexpr int NBLK = 256, NTHR = 512;
#define NPULL_ 64
#define WL_ 36
constexpr int NPULL = NPULL_;
constexpr int WL = WL_;
constexpr int NLW = 7 * WL;
constexpr int NCU = (NBLK - NPULL) / 8;
#define SH_ 2
constexpr int SH = SH_;
static_assert(NCU % SH == 0, "sharing");
static_assert(NLW <= 7 * NPULL && NLW <= 512 && NCU >= 20 && NCU <= 28 && NBLK - NPULL >= 160, "geometry");

#define RP_ 4
#define WLO_ 30
#define TA_ 8
#define RELAY_TH 44
#define LEAD_ 160
#define X1 1
#define X2 1
#define CSLEEP 16
#define TE_ 2
#define PD_W 60
#define PD_X 300
#define TE2_ 4
#define PD_0 120
#define PEEK 60
constexpr int TE2 = TE2_;
static_assert(TE2_ <= TA_ && TE_ <= TE2_, "stages");
constexpr int TE = TE_;
#define GO_ 1
constexpr int RP = RP_;
constexpr int WLO = WLO_;
constexpr int TA = TA_;
constexpr int G_WA = 192, G_X1 = G_WA + 64 * TA, G_WB = G_X1 + 192, G_X2 = G_WB + 64 * (8 - TA), G_XO = G_X2 + 192;
constexpr int G_X3 = G_XO + 128;
constexpr int G_ALL = 2047;
constexpr int NOUT = 1024;
constexpr int DYN0 = (RP + 1) * WL;
static_assert(RP >= 2 && RP * WL <= G_WA && WLO <= WL && WL <= 64, "push rounds");
__host__ __device__ constexpr int tile_pos(int c, int t) {
    return c >= 2 ? c * 8 + t : t < TA ? c * TA + t : 2 * TA + (t - TA) * 2 + c;
}

constexpr unsigned O_CNT = 0;
constexpr unsigned O_TMO = 8192;
constexpr unsigned O_TICK = 12288;
constexpr unsigned O_TAB = 14336;
constexpr unsigned O_OWN = 14848;
constexpr unsigned O_RDY = 16384;
constexpr unsigned O_LRW = 20480;
constexpr unsigned O_PROG = 24576;
constexpr unsigned O_PUSH = 26624;
constexpr unsigned O_G = 28672;
constexpr unsigned O_PRDY = 32768;
constexpr unsigned O_ORDY = 36864;
constexpr unsigned O_DBG = 65536;
constexpr unsigned O_EX = 512 * 1024;
constexpr unsigned O_WR = 1 * MiB;
constexpr unsigned O_PS = 9 * MiB;
constexpr unsigned O_PAR = 41 * MiB;
constexpr unsigned PAR_BYTES = 56 * MiB;
constexpr unsigned P_X = 0, P_WS = 16 * MiB, P_OS = 24 * MiB;
constexpr unsigned WS_USED = O_PAR + 2 * PAR_BYTES;
constexpr unsigned OUT_BYTES = 512u * NCOL * 4u;

constexpr unsigned long long DEADLINE = 60000000ull;
constexpr int BIAS = 1 << 20;
constexpr int NEX = 16;
constexpr int SYS = 17;
#define OUT_AUX 16

constexpr int STAGE = 49152;
constexpr int LDS_MISC = 3 * STAGE;
constexpr int LDS_EX = LDS_MISC + 64;

struct Params {
    char* ws[NR];
    const char* x;
    const char* w;
    const float* sx;
    const float* sw;
    float* out;
    int rank;
    int pad;
};

__device__ __forceinline__ rsrc_t mk(const void* p, unsigned bytes) {
    return __builtin_amdgcn_make_buffer_rsrc((void*)p, 0, bytes, 0x00020000);
}
#define st128(v, r, voff, soff, aux) __builtin_amdgcn_raw_buffer_store_b128(v, r, (voff) + (soff), 0, aux)
#define st96(v, r, voff, soff, aux) __builtin_amdgcn_raw_buffer_store_b96(v, r, (voff) + (soff), 0, aux)
#define st32(v, r, voff, soff, aux) __builtin_amdgcn_raw_buffer_store_b32(v, r, (voff) + (soff), 0, aux)
__device__ __forceinline__ u32x4 mk4(unsigned a, unsigned b, unsigned c, unsigned d) { return (u32x4){a, b, c, d}; }
__device__ __forceinline__ u32x3 mk3(unsigned a, unsigned b, unsigned c) { return (u32x3){a, b, c}; }
__device__ __forceinline__ void st1(const rsrc_t r, unsigned off, unsigned v, int lane) {
    __builtin_amdgcn_raw_buffer_store_b32(v, r, lane == 0 ? off : 0xf0000000u, 0, SYS);
}
__device__ __forceinline__ unsigned tmo_off() { return O_TMO + (blockIdx.x & 31) * 128; }
__device__ __forceinline__ void set_tmo(char* myws, unsigned call, int lane) {
    __builtin_amdgcn_raw_buffer_store_b32(call, mk(myws, O_DBG), lane < 32 ? O_TMO + lane * 128 : 0xf0000000u, 0, SYS);
}
__device__ __forceinline__ unsigned ld_sys(const unsigned* p) {
    return __hip_atomic_load(p, __ATOMIC_RELAXED, __HIP_MEMORY_SCOPE_SYSTEM);
}
__device__ __forceinline__ void st_sys(unsigned* p, unsigned v) {
    __hip_atomic_store(p, v, __ATOMIC_RELAXED, __HIP_MEMORY_SCOPE_SYSTEM);
}
__device__ __forceinline__ void drain() {
    asm volatile("s_waitcnt vmcnt(0)" ::: "memory");
    __builtin_amdgcn_s_waitcnt(0x0F70);
}
__device__ __forceinline__ unsigned long long now() { return __builtin_amdgcn_s_memrealtime(); }
__device__ __forceinline__ unsigned perm(unsigned hi, unsigned lo, unsigned sel) {
    return __builtin_amdgcn_perm(hi, lo, sel);
}
__device__ __forceinline__ unsigned bfi(unsigned m, unsigned a, unsigned b) { return (a & m) | (b & ~m); }

#define STAMP(slot) do {} while (0)
#define TICKSTAMP(g) do {} while (0)

__device__ __forceinline__ bool wave_wait(const unsigned* ptr, unsigned need, char* myws, unsigned call, int lane) {
    const unsigned* tmo = (const unsigned*)(myws + tmo_off());
    const unsigned long long t0 = now();
    for (;;) {
        const unsigned v = ld_sys(ptr);
        const unsigned t = ld_sys(tmo);
        if (__builtin_amdgcn_readfirstlane(t) == call) return false;
        if (BALLOT(v < need) == 0) return true;
        if (now() - t0 > DEADLINE) {
            set_tmo(myws, call, lane);
            return false;
        }
        __builtin_amdgcn_s_sleep(1);
    }
}

__device__ __forceinline__ void monitor(const Params& p, char* myws, unsigned call, int lane) {
    const unsigned* tmo = (const unsigned*)(myws + tmo_off());
    const unsigned* prog = (const unsigned*)(myws + O_PROG);
    const unsigned* push = (const unsigned*)(myws + O_PUSH);
    const int l7 = lane & 7, me = p.rank;
    unsigned* ownp = (unsigned*)(p.ws[lane >> 3] + O_OWN) + me * 8 + ((me + 1 + l7) & 7);
    const unsigned base = call << 12;
    unsigned last = 0, nin = 4095;
    bool planned = false, told = false;
    const unsigned long long t0 = now();
    for (;;) {
        unsigned g[7];
        const unsigned tk = __builtin_amdgcn_raw_buffer_load_b32(mk(myws, !planned && BALLOT(last >= G_X3 - LEAD_ - 160 && l7 < 7) != 0 ? O_DBG : 0u), O_TICK + l7 * 128, 0, SYS);
        #pragma unroll
        for (int l = 0; l < 7; ++l) {
            const unsigned v = ld_sys(prog + l * 64 + lane), q = ld_sys(push + l * 64 + lane);
            g[l] = lane >= WL ? 4095u : (v >= base && q >= 2 * call) ? v - base : 0u;
        }
        if (!planned) {
            int r = l7 < 7 && tk >= base ? (int)(tk - base) : 0;
            if (BALLOT(r >= G_X3 - LEAD_) != 0) {
                planned = true;
                r = -r;
                unsigned keep = 0x7fu & ~(1u << l7), recv = 0;
                #pragma unroll
                for (int j = 1; j < 7; ++j) {
                    const int to = l7 + j >= 7 ? l7 + j - 7 : l7 + j, from = l7 - j < 0 ? l7 - j + 7 : l7 - j;
                    const int give = r - __shfl(r, to) > RELAY_TH ? 1 : 0;
                    const int got = __shfl(give, from);
                    r += 32 * (got - give);
                    keep &= ~((unsigned)give << to);
                    recv |= (unsigned)got << from;
                }
                nin = G_X3 + 32 * (__builtin_popcount(keep) + __builtin_popcount(recv));
                if (lane < 7) st_sys((unsigned*)(myws + O_TAB) + lane, (call << 14) | (recv << 7) | keep);
            }
        }
        #pragma unroll
        for (int o = 32; o; o >>= 1)
            #pragma unroll
            for (int l = 0; l < 7; ++l) g[l] = min(g[l], (unsigned)__shfl_xor((int)g[l], o));
        unsigned mine = 4095;
        #pragma unroll
        for (int l = 0; l < 7; ++l) if (l7 == l) mine = g[l];
        mine = mine >= nin ? (unsigned)G_ALL : min(mine, (unsigned)G_X3);
        if (mine > last) {
            last = mine;
            if (l7 < 7) st_sys((unsigned*)(myws + O_G + (lane >> 3) * 128) + l7, (call << 11) + mine);
        }
        if (!told && mine >= G_XO + 32) {
            told = true;
            if (l7 < 7 && (lane >> 3) != me) st_sys(ownp, call);
        }
        if (BALLOT(mine < G_ALL) == 0) return;
        if (UNI(ld_sys(tmo)) == call) return;
        if (now() - t0 > DEADLINE) { set_tmo(myws, call, lane); return; }
        __builtin_amdgcn_s_sleep(1);
    }
}

struct Item { u32x4 l0, l1, p0, p1, p2, p3; u32x2 h4; unsigned h1, ex; };

__device__ __forceinline__ unsigned draw(const rsrc_t tick, bool sel) {
    int t = 1, z = 0;
    asm volatile("" : "+v"(z));
    if (sel) t = __builtin_amdgcn_raw_ptr_buffer_atomic_add_i32(t, tick, z, 0, 0);
    return (unsigned)t;
}

__device__ __forceinline__ void link_wave(const Params& p, char* myws, const unsigned* cntp, int s, int lane, int wv) {
    const int me = p.rank;
    const int link = s % 7, idx = s / 7;
    const unsigned peer = (unsigned)(me + 1 + link) & 7u;
    const char* remp = p.ws[peer];
    const rsrc_t loc = mk(myws, WS_USED), rem = mk(remp, WS_USED);
    const unsigned v16 = lane * 16;

    u32x4 a0, a1;
    const rsrc_t xs = mk(p.x, 4096u * KS);
    const unsigned vx = (unsigned)(lane & 15) * KS + (lane >> 4) * 16;
    auto rb_of = [&](int g) {
        const int pi = g >> 5, i = pi + (pi >= 6 - link ? 1 : 0);
        return (unsigned)(4 * ((peer + 1 + i) & 7));
    };
    auto xoff = [&](int g) { return (rb_of(g) * 128 + (unsigned)(g & 3) * 32) * KS + (unsigned)((g & 31) >> 2) * 64; };
    const unsigned cv = ld_sys(cntp);
    a0 = __builtin_amdgcn_raw_buffer_load_b128(xs, vx, xoff(idx), 0);
    a1 = __builtin_amdgcn_raw_buffer_load_b128(xs, vx + 16 * KS, xoff(idx), 0);
    const unsigned call = UNI(cv) + 1u;
    STAMP(0);
    const unsigned par = O_PAR + (call & 1u) * PAR_BYTES;
    const unsigned dbase = par + P_X + (unsigned)me * (2 * MiB);
    const unsigned roff = lane == 63 ? O_PUSH + (link * 64 + idx) * 4 : O_RDY + (peer * 64 + lane) * 4;
    const unsigned* rdyp = (const unsigned*)(myws + roff);
    const unsigned rall = lane < 48 && !(lane < 32 && (unsigned)(lane >> 2) == peer) ? call : 0u;
    const unsigned rneed = lane == 63 ? (GO_ ? 2 * call - 1 : 0u)
                         : (lane < 32 ? (lane & 3) == 0 : lane < 32 + 2 * TE) ? rall : 0u;
    unsigned rv = 0;
    #pragma unroll 1
    for (int r = 0; r < RP - 1; ++r) {
        if (lane == 0 && r == idx % (RP - 1)) __hip_atomic_fetch_max((unsigned*)(myws + O_TICK + link * 128), (call << 12) + DYN0,
                                          __ATOMIC_RELAXED, __HIP_MEMORY_SCOPE_AGENT);
        const int g = idx + r * WL;
        const unsigned so = xoff(g + WL);
        const u32x4 b0 = __builtin_amdgcn_raw_buffer_load_b128(xs, vx, so, 0);
        const u32x4 b1 = __builtin_amdgcn_raw_buffer_load_b128(xs, vx + 16 * KS, so, 0);
        if (r == RP - 2) st1(rem, O_PUSH + ((6 - link) * 64 + idx) * 4, 2 * call - 1, lane);
        asm volatile("" ::: "memory");
        const unsigned d = dbase + rb_of(g) * 65536u + (unsigned)(g & 31) * 2048u;
        st128(a0, rem, v16, d, SYS);
        st128(a1, rem, v16 + 1024, d, SYS);
        drain();
        a0 = b0; a1 = b1;
    }
    {
        rv = __builtin_amdgcn_raw_buffer_load_b32(loc, roff, 0, SYS);
        asm volatile("" ::: "memory");
        const int g = idx + (RP - 1) * WL;
        const unsigned d = dbase + rb_of(g) * 65536u + (unsigned)(g & 31) * 2048u;
        st128(a0, rem, v16, d, SYS);
        st128(a1, rem, v16 + 1024, d, SYS);
    }
    const float scale = p.sx[0] * p.sw[0];
    st1(loc, (unsigned)((const char*)cntp - myws), call, lane);
    int dead = 0;
    if (BALLOT(rv < rneed) != 0) dead = wave_wait(rdyp, rneed, myws, call, lane) ? 0 : 1;
    dead = UNI(dead);
    unsigned long long unseen = BALLOT(rv < rall);
    auto wanted = [&](int g) {
        return g < G_WA + 64 * TE ? 0ull : g < G_WA + 64 * TE2 ? ((1ull << (2 * TE2)) - 1) << 32 : ~0ull;
    };
    STAMP(1);

    const unsigned prog = O_PROG + (link * 64 + idx) * 4;
    const rsrc_t tick = mk(myws + O_TICK + link * 128, 4);
    const unsigned tbase = call << 12;
    auto x_off = [&](unsigned k, unsigned rb, int g) {
        return par + P_X + k * (2 * MiB) + rb * 65536u + (unsigned)(g & 31) * 2048u;
    };
    auto in_off = [&](int g, unsigned& src, unsigned& dst) {
        if ((g >= G_WA && g < G_X1) || (g >= G_WB && g < G_X2)) {
            const unsigned o = (unsigned)(g < G_X1 ? g - G_WA : g - G_WB + 64 * TA) * 2048u;
            src = par + P_WS + (unsigned)me * MiB + o;
            dst = O_WR + peer * MiB + o;
        } else {
            unsigned rb;
            if (g >= G_XO) rb = 4 * me + ((((g - G_XO) >> 5) + 3) & 3);
            else {
                const int q = (g < G_WA ? g : g < G_WB ? g - 64 * TA : g - 512) >> 5;
                const int c = q / 6, pi = q - 6 * c;
                const int i = pi + (pi >= link ? 1 : 0);
                rb = 4 * ((me + 1 + i) & 7) + c;
            }
            src = dst = x_off(peer, rb, g);
        }
    };
    const unsigned oflag = lane < 32 ? O_ORDY + (((call & 1u) * 8 + peer) * 32 + lane) * 4
                                     : O_PRDY + (peer * 32 + (lane - 32)) * 4;
    const unsigned tflag = lane < 8 ? O_OWN + (peer * 8 + lane) * 4 : lane == 8 ? O_TAB + link * 4 : tmo_off();
    unsigned fv = 0;
    unsigned tv = 0;
    int g = idx + RP * WL, gn;
    int nin = 0;
    int held = 1;
    unsigned plan = 0, lack = 0, src, cdst, ndst;
    auto read_plan = [&]() {
        if (!nin) {
            plan = __builtin_amdgcn_readlane(tv, 8);
            if ((plan >> 14) == (call & 0x3ffffu)) nin = G_X3 + 32 * __builtin_popcount(plan & 0x3fffu);
        }
        const unsigned recv = (plan >> 7) & 0x7fu;
        lack = ((recv | (recv << 8)) >> (7 - me)) & 0xffu & ~(unsigned)BALLOT(lane < 8 && tv >= call);
    };
    in_off(g, src, cdst);
    {
        const rsrc_t remd = mk(remp, dead ? 0u : WS_USED);
        const u32x4 b0 = __builtin_amdgcn_raw_buffer_load_b128(remd, v16, src, SYS);
        const u32x4 b1 = __builtin_amdgcn_raw_buffer_load_b128(remd, v16 + 1024, src, SYS);
        const unsigned tk = draw(tick, lane == 0);
        drain();
        st1(rem, O_PUSH + ((6 - link) * 64 + idx) * 4, 2 * call, lane);
        a0 = b0; a1 = b1;
        gn = (int)(UNI(tk) - tbase);
    }
    for (;;) {
        gn = UNI(gn); dead = UNI(dead); held = UNI(held); nin = UNI(nin); plan = UNI(plan); cdst = UNI(cdst); lack = UNI(lack);
        unseen = ((unsigned long long)UNI((unsigned)(unseen >> 32)) << 32) | UNI((unsigned)unseen);
        if (unseen & wanted(gn)) {
            const unsigned long long w = wanted(gn);
            if (!dead) dead = wave_wait(rdyp, (w >> lane) & 1 ? rall : 0u, myws, call, lane) ? 0 : 1;
            unseen &= ~w;
        }
        if (gn < G_X3) in_off(gn, src, ndst);
        else {
            unsigned long long t0 = 0;
            for (;;) {
                if (dead && !nin) { plan = 0; nin = G_X3; }
                if (nin) {
                    if (gn >= nin) break;
                    unsigned m = plan & 0x3fffu;
                    for (int j = (gn - G_X3) >> 5; j; --j) m &= m - 1;
                    const unsigned bit = __builtin_ctz(m);
                    const unsigned other = (unsigned)(me + 1 + (bit >= 7 ? bit - 7 : bit)) & 7u;
                    src = ndst = bit >= 7 ? x_off(other, 4 * peer + 3, gn) : x_off(peer, 4 * other + 3, gn);
                    if (bit < 7 || dead || !((lack >> other) & 1u)) break;
                }
                if (held) {
                    st128(a0, loc, v16, cdst, SYS);
                    st128(a1, loc, v16 + 1024, cdst, SYS);
                    drain();
                    st1(loc, prog, tbase + (unsigned)gn, lane);
                    held = 0;
                } else {
                    if (!t0) t0 = now();
                    if (now() - t0 > DEADLINE) { set_tmo(myws, call, lane); dead = 1; }
                    __builtin_amdgcn_s_sleep(2);
                }
                tv = __builtin_amdgcn_raw_buffer_load_b32(loc, tflag, 0, SYS);
                drain();
                if (__builtin_amdgcn_readlane(tv, 9) == call) dead = 1;
                read_plan();
            }
            if (gn >= nin) break;
        }
        const rsrc_t remd = mk(remp, dead ? 0u : WS_USED), locd = mk(myws, dead || !held ? 0u : WS_USED);
        const rsrc_t alive = mk(myws, dead ? 0u : O_DBG);
        const u32x4 b0 = __builtin_amdgcn_raw_buffer_load_b128(remd, v16, src, SYS);
        const u32x4 b1 = __builtin_amdgcn_raw_buffer_load_b128(remd, v16 + 1024, src, SYS);
        const unsigned tk = draw(tick, lane == 0);
        TICKSTAMP(gn);
        const bool look = gn >= G_X3 - 80 && (!nin || lack);
        fv = __builtin_amdgcn_raw_buffer_load_b32(mk(myws, nin && gn >= nin - 48 ? O_DBG : 0u), oflag, 0, SYS);
        const bool peek = (unseen & wanted(gn + PEEK)) != 0;
        const unsigned tn = __builtin_amdgcn_raw_buffer_load_b32(mk(myws, look || peek ? O_DBG : 0u),
                                                                 peek ? roff : tflag, 0, SYS);
        asm volatile("" ::: "memory");
        st128(a0, locd, v16, cdst, SYS);
        st128(a1, locd, v16 + 1024, cdst, SYS);
        drain();
        st1(alive, prog, tbase + (unsigned)gn, lane);
        if (look) { tv = tn; read_plan(); }
        if (peek) unseen = BALLOT(tn < rall);
        a0 = b0; a1 = b1; held = 1;
        g = gn; cdst = ndst;
        gn = (int)(UNI(tk) - tbase);
    }
    const rsrc_t locd = mk(myws, dead || !held ? 0u : WS_USED);
    const int NIN = nin, NTOT = nin + NOUT;

    const rsrc_t out = mk(p.out, OUT_BYTES);
    const unsigned ovoff = (unsigned)(4 * (lane >> 4)) * (NCOL * 4) + (lane & 15) * 16;
    const unsigned exb = O_EX + ((call & 1u) * 8 + peer) * (256 * NEX * 4);
    unsigned rmask;
    Item cur, nxt;
    cur.l0 = a0; cur.l1 = a1;
    int state = 2;
    int waiting = 0;
    unsigned long long t0 = 0;
    {
        const unsigned long long m = BALLOT(lane < 32 ? (fv >> 8) >= call : fv >= call);
        rmask = (unsigned)m & (unsigned)(m >> 32);
    }
    for (;;) {
        g = UNI(g); gn = UNI(gn); dead = UNI(dead); state = UNI(state); rmask = UNI(rmask);
        cdst = UNI(cdst); waiting = UNI(waiting);
        const bool have = gn < NTOT;
        const unsigned o = (unsigned)(gn - NIN) & 1023u;
        const bool can = have && (dead || ((rmask >> (o >> 5)) & 1u));
        {
            const bool go = can && !dead;
            const rsrc_t r0 = mk(remp, go ? WS_USED : 0u), r1 = mk(myws, go ? WS_USED : 0u);
            const unsigned so = par + P_OS + (unsigned)me * (4 * MiB) + o * 4096u;
            const unsigned ps = O_PS + peer * (4 * MiB) + o * 4096u;
            nxt.l0 = __builtin_amdgcn_raw_buffer_load_b128(r0, v16, so, SYS);
            nxt.l1 = __builtin_amdgcn_raw_buffer_load_b128(r0, v16 + 1024, so, SYS);
            nxt.h4 = __builtin_amdgcn_raw_buffer_load_b64(r0, lane * 8 + 2048, so, SYS);
            nxt.h1 = __builtin_amdgcn_raw_buffer_load_b16(r0, lane * 2 + 2560, so, SYS);
            nxt.ex = __builtin_amdgcn_raw_buffer_load_b32(r1, (lane & (NEX - 1)) * 4, exb + (o >> 2) * (NEX * 4), SYS);
            nxt.p0 = __builtin_amdgcn_raw_buffer_load_b128(r1, v16, ps, SYS);
            nxt.p1 = __builtin_amdgcn_raw_buffer_load_b128(r1, v16 + 1024, ps, SYS);
            nxt.p2 = __builtin_amdgcn_raw_buffer_load_b128(r1, v16 + 2048, ps, SYS);
            nxt.p3 = __builtin_amdgcn_raw_buffer_load_b128(r1, v16 + 3072, ps, SYS);
        }
        const bool poll = have && !dead && (~rmask & (7u << (o >> 5))) != 0;
        const rsrc_t rf = mk(myws, poll ? O_DBG : 0u);
        const unsigned nf = __builtin_amdgcn_raw_buffer_load_b32(rf, oflag, 0, SYS);
        const unsigned nt = __builtin_amdgcn_raw_buffer_load_b32(rf, tmo_off(), 0, SYS);
        const unsigned tk = draw(tick, can && idx < WLO && lane == 0);

        if (state == 2) {
            st128(cur.l0, locd, v16, cdst, SYS);
            st128(cur.l1, locd, v16 + 1024, cdst, SYS);
        } else if (state == 1) {
            const unsigned oc = (unsigned)(g - NIN);
            const bool raw = (__builtin_amdgcn_readlane(fv, oc >> 5) >> ((oc >> 2) & 7)) & 1u;
            const unsigned L[8] = {cur.l0.x, cur.l0.y, cur.l0.z, cur.l0.w, cur.l1.x, cur.l1.y, cur.l1.z, cur.l1.w};
            u32x16 pv = {cur.p0.x, cur.p0.y, cur.p0.z, cur.p0.w, cur.p1.x, cur.p1.y, cur.p1.z, cur.p1.w,
                         cur.p2.x, cur.p2.y, cur.p2.z, cur.p2.w, cur.p3.x, cur.p3.y, cur.p3.z, cur.p3.w};
            u32x4 v[4];
            if (raw && !dead) {
                const unsigned so = par + P_OS + (unsigned)me * (4 * MiB) + oc * 4096u;
                const u32x4 ha = __builtin_amdgcn_raw_buffer_load_b128(rem, v16 + 2048, so, SYS);
                const u32x4 hb = __builtin_amdgcn_raw_buffer_load_b128(rem, v16 + 3072, so, SYS);
                const unsigned H[8] = {ha.x, ha.y, ha.z, ha.w, hb.x, hb.y, hb.z, hb.w};
                #pragma unroll
                for (int r = 0; r < 4; ++r) {
                    v[r].x = perm(H[2 * r], L[2 * r], 0x05040100u);
                    v[r].y = perm(H[2 * r], L[2 * r], 0x07060302u);
                    v[r].z = perm(H[2 * r + 1], L[2 * r + 1], 0x05040100u);
                    v[r].w = perm(H[2 * r + 1], L[2 * r + 1], 0x07060302u);
                }
            } else {
                unsigned em = (unsigned)BALLOT(lane < NEX && (cur.ex >> 21) == (oc & 3u));
                while (em) {
                    const unsigned e = __builtin_amdgcn_readlane(cur.ex, __builtin_ctz(em));
                    em &= em - 1;
                    pv[(e >> 17) & 15u] += (unsigned)lane == ((e >> 11) & 63u) ? e << 21 : 0u;
                }
                const unsigned t = (cur.h1 & 0x0f0fu) | ((cur.h1 & 0xf0f0u) << 12);
                unsigned B[4];
                B[0] = (cur.h4.x & 0x0f0f0f0fu) | ((t << 4) & 0x10101010u);
                B[1] = ((cur.h4.x >> 4) & 0x0f0f0f0fu) | ((t << 3) & 0x10101010u);
                B[2] = (cur.h4.y & 0x0f0f0f0fu) | ((t << 2) & 0x10101010u);
                B[3] = ((cur.h4.y >> 4) & 0x0f0f0f0fu) | ((t << 1) & 0x10101010u);
                #pragma unroll
                for (int r = 0; r < 4; ++r) {
                    v[r].x = perm(B[r], L[2 * r], 0x0c040100u);
                    v[r].y = perm(B[r], L[2 * r], 0x0c050302u);
                    v[r].z = perm(B[r], L[2 * r + 1], 0x0c060100u);
                    v[r].w = perm(B[r], L[2 * r + 1], 0x0c070302u);
                }
            }
            const u32x4 P[4] = {mk4(pv[0], pv[1], pv[2], pv[3]), mk4(pv[4], pv[5], pv[6], pv[7]),
                                mk4(pv[8], pv[9], pv[10], pv[11]), mk4(pv[12], pv[13], pv[14], pv[15])};
            #pragma unroll
            for (int r = 0; r < 4; ++r) {
                const u32x4 t = v[r] + P[r];
                f32x4 f;
                f.x = (float)(int)t.x * scale; f.y = (float)(int)t.y * scale;
                f.z = (float)(int)t.z * scale; f.w = (float)(int)t.w * scale;
                u32x4 o4 = __builtin_bit_cast(u32x4, f);
                if (dead) o4 = mk4(0x7fc00000u, 0x7fc00000u, 0x7fc00000u, 0x7fc00000u);
                st128(o4, out, ovoff + r * (NCOL * 4), cdst, OUT_AUX);
            }
        }
        if (!have && state != 2) break;
        drain();
        if (state == 2) {
            if (!dead) st1(loc, prog, tbase + 4095u, lane);
            STAMP(2);
        }
        if (!have) break;
        if (poll) {
            fv = nf;
            const unsigned long long m = BALLOT(lane < 32 ? (nf >> 8) >= call : nf >= call);
            rmask = (unsigned)m & (unsigned)(m >> 32);
            if (UNI(nt) == call) dead = 1;
        }
        if (can) {
            cur = nxt; g = gn; state = 1;
            const unsigned n = o >> 5, w8 = (o >> 2) & 7, mf = o & 3;
            const unsigned c = n >= 16 ? n >> 3 : n < 2 * TA ? (n >= TA ? 1 : 0) : n & 1;
            const unsigned t = n >= 16 ? n & 7 : n < 2 * TA ? n - c * TA : TA + ((n - 2 * TA) >> 1);
            cdst = ((c * 128 + (w8 >> 2) * 64 + mf * 16) * NCOL + peer * 2048 + t * 256 + (w8 & 3) * 64) * 4u;
            gn = idx < WLO ? (int)(UNI(tk) - tbase) : NTOT;
            waiting = 0;
        } else {
            if (!waiting) { waiting = 1; t0 = now(); }
            else {
                if (now() - t0 > DEADLINE) { set_tmo(myws, call, lane); dead = 1; }
                __builtin_amdgcn_s_sleep(4);
            }
            state = 0;
        }
    }
    STAMP(3);
}

__device__ __forceinline__ unsigned prep(const Params& p, char* myws, const unsigned* cntp, int id, int tid, int lane, int wv) {
    const int me = p.rank;
    unsigned call;
    {
        const int tw = ((id - 32) >> 1) & 7;
        const unsigned long long hold = id < 32 ? ((id & 3) ? PD_X : 0) : (tw >= TE ? PD_0 + (tw - TE) * PD_W : 0);
        const unsigned long long t0 = now();
        while (now() - t0 < hold) __builtin_amdgcn_s_sleep(4);
    }
    const rsrc_t loc = mk(myws, WS_USED);
    if (id < 32) {
        const rsrc_t xs = mk(p.x, 4096u * KS);
        const unsigned voff = (unsigned)(lane & 15) * KS + (lane >> 4) * 16;
        const unsigned soff = (unsigned)(id * 128 + wv * 16) * KS;
        u32x4 v[8];
        #pragma unroll
        for (int i = 0; i < 8; ++i) v[i] = __builtin_amdgcn_raw_buffer_load_b128(xs, voff + i * 64, soff, 0);
        call = UNI(ld_sys(cntp)) + 1u;
        const unsigned dst = O_PAR + (call & 1u) * PAR_BYTES + P_X + (unsigned)me * (2 * MiB) + (unsigned)id * 65536u + wv * 1024u;
        #pragma unroll
        for (int i = 0; i < 8; ++i) st128(v[i], loc, lane * 16, dst + i * 8192u, SYS);
    } else {
        const int pid = id - 32, jt = pid >> 1, h = pid & 1;
        const rsrc_t wsr = mk(p.w, (unsigned)KS * NCOL);
        const int wn = wv & 3, kq0 = h * 4 + (wv >> 2) * 2;
        const unsigned voff = (unsigned)(16 * (lane >> 4)) * NCOL + (lane & 15) * 4;
        unsigned r[2][16];
        #pragma unroll
        for (int b = 0; b < 2; ++b)
            #pragma unroll
            for (int j = 0; j < 16; ++j)
                r[b][j] = __builtin_amdgcn_raw_buffer_load_b32(wsr, voff, (unsigned)((kq0 + b) * 64 + j) * NCOL + jt * 256 + wn * 64, 0);
        call = UNI(ld_sys(cntp)) + 1u;
        const unsigned par = O_PAR + (call & 1u) * PAR_BYTES;
        #pragma unroll
        for (int b = 0; b < 2; ++b) {
            unsigned o[4][4];
            #pragma unroll
            for (int q = 0; q < 4; ++q) {
                const unsigned t0 = perm(r[b][4 * q + 1], r[b][4 * q], 0x05010400u);
                const unsigned t1 = perm(r[b][4 * q + 1], r[b][4 * q], 0x07030602u);
                const unsigned t2 = perm(r[b][4 * q + 3], r[b][4 * q + 2], 0x05010400u);
                const unsigned t3 = perm(r[b][4 * q + 3], r[b][4 * q + 2], 0x07030602u);
                o[0][q] = perm(t2, t0, 0x05040100u);
                o[1][q] = perm(t2, t0, 0x07060302u);
                o[2][q] = perm(t3, t1, 0x05040100u);
                o[3][q] = perm(t3, t1, 0x07060302u);
            }
            const unsigned dst = par + P_WS + (unsigned)jt * 131072u + (unsigned)((kq0 + b) * 16 + wn * 4) * 1024u;
            #pragma unroll
            for (int i = 0; i < 4; ++i)
                st128(mk4(o[i][0], o[i][1], o[i][2], o[i][3]), loc, lane * 16 + i * 1024, dst, SYS);
        }
    }
    drain();
    __syncthreads();
    if (id < 32) {
        if (tid < NR) st_sys((unsigned*)(p.ws[tid] + O_RDY) + me * 64 + id, call);
    } else {
        const int pid = id - 32;
        if (tid == 0) st_sys((unsigned*)(p.ws[pid >> 4] + O_RDY) + me * 64 + 32 + (pid & 15), call);
        if (tid == 1) st_sys((unsigned*)(myws + O_LRW) + pid, call);
    }
    return call;
}

}

__global__ __launch_bounds__(NTHR, 2) void gemm_rs_fused(Params p) {
    __shared__ __attribute__((aligned(16))) char lds[LDS_EX + 8 * NEX * 4];
    const int tid = threadIdx.x, lane = tid & 63;
    const int wv = __builtin_amdgcn_readfirstlane(tid >> 6);
    const int b = blockIdx.x, me = p.rank;
    char* myws = p.ws[me];
    unsigned* cntp = (unsigned*)(myws + O_CNT) + b * 8 + wv;

    if (b < NPULL) {
        const int s = wv * NPULL + b;
        if (s < NLW) link_wave(p, myws, cntp, s, lane, wv);
        else if (b == NPULL - 1 && wv == 7) {
            const unsigned call = UNI(ld_sys(cntp)) + 1u;
            if (lane == 0) st_sys(cntp, call);
            monitor(p, myws, call, lane);
        }
        return;
    }

    const int cb = b - NPULL;
    STAMP(0);
    unsigned call;
    if (cb < 160) call = prep(p, myws, cntp, cb, tid, lane, wv);
    else call = UNI(ld_sys(cntp)) + 1u;
    if (lane == 0) st_sys(cntp, call);
    STAMP(1);

    const int t = (b & 7 & ~(SH - 1)) | ((cb >> 3) & (SH - 1));
    const int u = ((cb >> 3) & ~(SH - 1)) | (b & (SH - 1));
    const unsigned par = O_PAR + (call & 1u) * PAR_BYTES;
    const rsrc_t loc = mk(myws, WS_USED);
    volatile __attribute__((address_space(3))) unsigned* misc =
        (volatile __attribute__((address_space(3))) unsigned*)(lds + LDS_MISC);
    const int wm = wv >> 2, wn = wv & 3;

    for (int step = 0; step < 4; ++step) {
        int task, v;
        constexpr int NP1 = NCU - 7;
        if (step == 0) { task = 0; v = u - 7; if (v < 0) continue; }
        else if (step == 1) { task = 0; v = NP1 + (NCU - 1 - u); if (v >= 28) continue; }
        else if (step == 2) { task = 1; v = u; }
        else {
            task = 1; v = NCU + u;
            if (v >= 28) { task = 2; v = (v - 28) * 7; if (v >= 28) break; }
        }
        const int c = v / 7, d = (me + 1 + (v - 7 * c)) & 7;
        const int rb = task == 1 ? 4 * d + c : 4 * me + c;
        const int pos = tile_pos(c, t);
        const int wj = task == 0 ? d : me;

        i32x4 acc[4][4];
        {
            const int a0 = task == 0 ? -BIAS : task == 1 ? BIAS : 0;
            #pragma unroll
            for (int m = 0; m < 4; ++m)
                #pragma unroll
                for (int n = 0; n < 4; ++n) acc[m][n] = (i32x4){a0, a0, a0, a0};
        }
        unsigned todo = task == 0 ? 1u << me : task == 1 ? 0xffu & ~(1u << d) : 0xffu;
        bool ok = true;
        while (todo) {
            if (wv == 0) {
                const unsigned gend = task == 2 ? (unsigned)(G_XO + 32 * (((c + 1) & 3) + 1))
                                    : c == 3 ? (unsigned)G_ALL : c == 2 ? (unsigned)G_XO
                                    : t >= TA ? (unsigned)(G_WB + 64 * (t - TA + 1))
                                    : c == 1 ? (unsigned)G_WB : (unsigned)(G_WA + 64 * (t + 1));
                const unsigned* ptr = (const unsigned*)(myws + tmo_off());
                unsigned need = call;
                if ((todo >> me) & 1u) {
                    if (lane == 8 || lane == 9) ptr = (const unsigned*)(myws + O_LRW) + (wj * 8 + t) * 2 + (lane - 8);
                    if (lane == 10) ptr = (const unsigned*)(myws + O_RDY) + me * 64 + rb;
                }
                if (lane < 7 && task) { ptr = (const unsigned*)(myws + O_G + (b & 7) * 128) + lane; need = (call << 11) + gend; }
                const unsigned long long t0 = now();
                unsigned got;
                for (;;) {
                    const unsigned val = ld_sys(ptr);
                    unsigned m = (unsigned)BALLOT(val >= need);
                    if (__builtin_amdgcn_readlane(val, 7) == call) { got = 0; break; }
                    if (task == 1 && c == 3 && !((m >> (v - 21)) & 1u)) m &= ~0x7fu;
                    m = (m & 0x7fu) | (((m >> 8) & 7u) == 7u ? 0x80u : 0u);
                    m = ((m | (m << 8)) >> (7 - me)) & 0xffu;
                    got = m & todo;
                    if (got) break;
                    if (now() - t0 > DEADLINE) { set_tmo(myws, call, lane); break; }
                    __builtin_amdgcn_s_sleep(CSLEEP);
                }
                if (lane == 0) misc[0] = got;
            }
            __syncthreads();
            const unsigned got = UNI(misc[0]);
            if (!got) { ok = false; break; }
            todo &= ~got;
            unsigned klist = 0, rest = got;
            int nst = 0;
            while (rest) { klist |= (unsigned)__builtin_ctz(rest) << nst; rest &= rest - 1; nst += 4; }
            const unsigned va = wv * 2048 + lane * 16, vb = wv * 4096 + lane * 16;
            auto issue = [&](int s, int slot) {
                const int k = (klist >> (s & ~3)) & 15;
                const unsigned abase = par + P_X + (unsigned)k * (2 * MiB) + (unsigned)rb * 65536u + (s & 3) * 16384u;
                const unsigned bbase = (k == me ? par + P_WS + (unsigned)wj * MiB : O_WR + (unsigned)k * MiB) + (unsigned)t * 131072u + (s & 3) * 32768u;
                char* la = lds + slot * STAGE + wv * 2048;
                char* lb = lds + slot * STAGE + 16384 + wv * 4096;
                __builtin_amdgcn_raw_ptr_buffer_load_lds(loc, (__attribute__((address_space(3))) void*)la, 16, va, abase, 0, SYS);
                __builtin_amdgcn_raw_ptr_buffer_load_lds(loc, (__attribute__((address_space(3))) void*)la, 16, va, abase, 1024, SYS);
                __builtin_amdgcn_raw_ptr_buffer_load_lds(loc, (__attribute__((address_space(3))) void*)lb, 16, vb, bbase, 0, SYS);
                __builtin_amdgcn_raw_ptr_buffer_load_lds(loc, (__attribute__((address_space(3))) void*)lb, 16, vb, bbase, 1024, SYS);
                __builtin_amdgcn_raw_ptr_buffer_load_lds(loc, (__attribute__((address_space(3))) void*)lb, 16, vb, bbase, 2048, SYS);
                __builtin_amdgcn_raw_ptr_buffer_load_lds(loc, (__attribute__((address_space(3))) void*)lb, 16, vb, bbase, 3072, SYS);
            };
            auto compute = [&](int slot) {
                const char* A = lds + slot * STAGE + wm * 4096 + lane * 16;
                const char* B = lds + slot * STAGE + 16384 + wn * 4096 + lane * 16;
                i32x4 af[2][4], bf[2][4];
                #pragma unroll
                for (int kb = 0; kb < 2; ++kb) {
                    af[kb][0] = *(const i32x4*)(A + kb * 8192);
                    #pragma unroll
                    for (int i = 0; i < 4; ++i) bf[kb][i] = *(const i32x4*)(B + kb * 16384 + i * 1024);
                    #pragma unroll
                    for (int i = 1; i < 4; ++i) af[kb][i] = *(const i32x4*)(A + kb * 8192 + i * 1024);
                }
                __builtin_amdgcn_sched_barrier(0);
                #pragma unroll
                for (int kb = 0; kb < 2; ++kb)
                    #pragma unroll
                    for (int m = 0; m < 4; ++m)
                        #pragma unroll
                        for (int n = 0; n < 4; ++n)
                            acc[m][n] = __builtin_amdgcn_mfma_i32_16x16x64_i8(af[kb][m], bf[kb][n], acc[m][n], 0, 0, 0);
                __builtin_amdgcn_sched_barrier(0);
            };
            issue(0, 0);
            issue(1, 1);
            int slot = 0;
            for (int s = 0; s < nst; ++s) {
                if (s + 1 < nst) asm volatile("s_waitcnt vmcnt(6) lgkmcnt(0)" ::: "memory");
                else asm volatile("s_waitcnt vmcnt(0) lgkmcnt(0)" ::: "memory");
                __builtin_amdgcn_s_barrier();
                if (s + 2 < nst) issue(s + 2, slot == 0 ? 2 : slot - 1);
                compute(slot);
                slot = slot == 2 ? 0 : slot + 1;
            }
        }

        if (task == 0) {
            if (ok) {
                const unsigned dst = O_PS + (unsigned)d * (4 * MiB) + (unsigned)(pos * 8 + wv) * 16384u;
                #pragma unroll
                for (int m = 0; m < 4; ++m)
                    #pragma unroll
                    for (int r = 0; r < 4; ++r)
                        st128(mk4((unsigned)acc[m][0][r], (unsigned)acc[m][1][r], (unsigned)acc[m][2][r], (unsigned)acc[m][3][r]),
                            loc, lane * 16 + r * 1024, dst + m * 4096u, SYS);
            }
            drain();
            __syncthreads();
            if (tid == 0 && ok) st_sys((unsigned*)(myws + O_PRDY) + d * 32 + pos, call);
        } else if (task == 1) {
            volatile __attribute__((address_space(3))) unsigned* exl =
                (volatile __attribute__((address_space(3))) unsigned*)(lds + LDS_EX) + wv * NEX;
            if (lane < NEX) exl[lane] = 0xffffffffu;
            unsigned nex = 0;
            #pragma unroll
            for (int m = 0; m < 4; ++m)
                #pragma unroll
                for (int r = 0; r < 4; ++r) {
                    const unsigned any = (unsigned)(acc[m][0][r] | acc[m][1][r]) | (unsigned)(acc[m][2][r] | acc[m][3][r]);
                    if (BALLOT((any >> 21) != 0) != 0) {
                        #pragma unroll
                        for (int n = 0; n < 4; ++n) {
                            const unsigned hi = (unsigned)acc[m][n][r] >> 21;
                            const unsigned long long bm = BALLOT(hi != 0);
                            const unsigned at = nex + __builtin_amdgcn_mbcnt_hi((unsigned)(bm >> 32), __builtin_amdgcn_mbcnt_lo((unsigned)bm, 0u));
                            if (hi != 0 && at < NEX) exl[at] = (unsigned)(m << 21 | (r * 4 + n) << 17) | ((unsigned)lane << 11) | hi;
                            nex += __builtin_popcountll(bm);
                        }
                    }
                }
            const bool raw = nex > NEX;
            if (ok) {
                const unsigned e = exl[lane & (NEX - 1)];
                __builtin_amdgcn_raw_buffer_store_b32(e, mk(p.ws[d], WS_USED),
                    lane < NEX ? O_EX + ((((call & 1u) * 8 + me) * 32 + pos) * 8 + wv) * (NEX * 4) + lane * 4 : 0xf0000000u, 0, SYS);
            }
            if (ok) {
                const unsigned dst = par + P_OS + (unsigned)d * (4 * MiB) + (unsigned)(pos * 8 + wv) * 16384u;
                #pragma unroll
                for (int m = 0; m < 4; ++m) {
                    unsigned L[8], H[8];
                    #pragma unroll
                    for (int r = 0; r < 4; ++r) {
                        L[2 * r] = perm(acc[m][1][r], acc[m][0][r], 0x05040100u);
                        L[2 * r + 1] = perm(acc[m][3][r], acc[m][2][r], 0x05040100u);
                        H[2 * r] = perm(acc[m][1][r], acc[m][0][r], 0x07060302u);
                        H[2 * r + 1] = perm(acc[m][3][r], acc[m][2][r], 0x07060302u);
                    }
                    st128(mk4(L[0], L[1], L[2], L[3]), loc, lane * 16, dst + m * 4096u, SYS);
                    st128(mk4(L[4], L[5], L[6], L[7]), loc, lane * 16 + 1024, dst + m * 4096u, SYS);
                    if (raw) {
                        st128(mk4(H[0], H[1], H[2], H[3]), loc, lane * 16 + 2048, dst + m * 4096u, SYS);
                        st128(mk4(H[4], H[5], H[6], H[7]), loc, lane * 16 + 3072, dst + m * 4096u, SYS);
                    } else {
                        unsigned B[4];
                        #pragma unroll
                        for (int r = 0; r < 4; ++r) B[r] = perm(H[2 * r + 1], H[2 * r], 0x06040200u);
                        const u32x2 h4 = {bfi(0x0f0f0f0fu, B[0], B[1] << 4), bfi(0x0f0f0f0fu, B[2], B[3] << 4)};
                        const unsigned t = ((B[0] >> 4) & 0x01010101u) | ((B[1] >> 3) & 0x02020202u)
                                         | ((B[2] >> 2) & 0x04040404u) | ((B[3] >> 1) & 0x08080808u);
                        __builtin_amdgcn_raw_buffer_store_b64(h4, loc, lane * 8 + 2048 + dst + m * 4096u, 0, SYS);
                        __builtin_amdgcn_raw_buffer_store_b16((unsigned short)(t | (t >> 12)), loc, lane * 2 + 2560 + dst + m * 4096u, 0, SYS);
                    }
                }
            }
            if (lane == 0) misc[4 + wv] = raw ? 1u : 0u;
            drain();
            __syncthreads();
            if (tid == 0 && ok) {
                unsigned mask = 0;
                #pragma unroll
                for (int i = 0; i < 8; ++i) mask |= misc[4 + i] << i;
                st_sys((unsigned*)(p.ws[d] + O_ORDY) + ((call & 1u) * 8 + me) * 32 + pos, (call << 8) | mask);
            }
        } else {
            const rsrc_t out = mk(p.out, OUT_BYTES);
            const float scale = p.sx[0] * p.sw[0];
            const unsigned ovoff = (unsigned)(4 * (lane >> 4)) * (NCOL * 4) + (lane & 15) * 16;
            const unsigned dst = ((unsigned)(c * 128 + wm * 64) * NCOL + me * 2048 + t * 256 + wn * 64) * 4u;
            #pragma unroll
            for (int m = 0; m < 4; ++m)
                #pragma unroll
                for (int r = 0; r < 4; ++r) {
                    f32x4 f;
                    f.x = (float)acc[m][0][r] * scale; f.y = (float)acc[m][1][r] * scale;
                    f.z = (float)acc[m][2][r] * scale; f.w = (float)acc[m][3][r] * scale;
                    u32x4 o4 = __builtin_bit_cast(u32x4, f);
                    if (!ok) o4 = (u32x4){0x7fc00000u, 0x7fc00000u, 0x7fc00000u, 0x7fc00000u};
                    st128(o4, out, ovoff + r * (NCOL * 4), dst + (unsigned)(m * 16) * (NCOL * 4), OUT_AUX);
                }
        }
        STAMP(step == 3 ? 7 : 3 + 2 * task);
    }
}

extern "C" void kernel_launch(int rank, int nranks, ncclComm_t comm, hipStream_t stream,
                              void* const* d_in, const int* in_sizes, int n_in,
                              void* d_out, int out_size, const int* in_dtypes, int out_dtype,
                              void* const* d_out_all, void* const* d_ws_all, size_t ws_bytes) {
    (void)comm; (void)in_sizes; (void)in_dtypes; (void)out_dtype; (void)d_out_all;
    if (nranks != NR || (unsigned)rank >= (unsigned)NR || n_in < 4 || ws_bytes < WS_USED ||
        (size_t)out_size * 4 != OUT_BYTES) {
        fprintf(stderr, "kernel_launch: unexpected problem shape\n");
        return;
    }
    Params p;
    for (int i = 0; i < NR; ++i) p.ws[i] = (char*)d_ws_all[i];
    p.x = (const char*)d_in[0];
    p.w = (const char*)d_in[1];
    p.sx = (const float*)d_in[2];
    p.sw = (const float*)d_in[3];
    p.out = (float*)d_out;
    p.rank = rank;
    p.pad = 0;
    hipLaunchKernelGGL(gemm_rs_fused, dim3(NBLK), dim3(NTHR), 0, stream, p);
    HIP_CHECK(hipGetLastError());
}
```

```cpp
#include <hip/hip_runtime.h>
#include <rccl/rccl.h>
#include <cstdio>
#include <cstdint>

#define HIP_CHECK(expr) do { hipError_t _e = (expr); if (_e != hipSuccess) { \
    fprintf(stderr, "HIP error %s at %s:%d\n", hipGetErrorString(_e), __FILE__, __LINE__); \
    return; } } while (0)

typedef int i32x4 __attribute__((ext_vector_type(4)));
typedef unsigned u32x4 __attribute__((ext_vector_type(4)));
typedef unsigned u32x3 __attribute__((ext_vector_type(3)));
typedef unsigned u32x2 __attribute__((ext_vector_type(2)));
typedef unsigned u32x16 __attribute__((ext_vector_type(16)));
typedef float f32x4 __attribute__((ext_vector_type(4)));
typedef __amdgpu_buffer_rsrc_t rsrc_t;
#define UNI(x) __builtin_amdgcn_readfirstlane(x)
#define BALLOT(c) __builtin_amdgcn_ballot_w64(c)

namespace {

constexpr int NR = 8;
constexpr int NCOL = 16384;
constexpr int KS = 512;
constexpr unsigned MiB = 1u << 20;

constexpr int NBLK = 256, NTHR = 512;
#define NPULL_ 80
#define WL_ 36
constexpr int NPULL = NPULL_;
constexpr int WL = WL_;
constexpr int NLW = 7 * WL;
constexpr int NCU = (NBLK - NPULL) / 8;
#define SH_ 2
constexpr int SH = SH_;
static_assert(NCU % SH == 0, "sharing");
static_assert(NLW <= 7 * NPULL && NLW <= 512 && NCU >= 20 && NCU <= 28 && NBLK - NPULL >= 160, "geometry");

#define RP_ 4
#define WLO_ 30
#define TA_ 4
#define RELAY_TH 36
#define LEAD_ 160
#define X1 1
#define X2 1
#define CSLEEP 16
#define TE_ 2
#define PD_W 60
#define PD_X 300
#define TE2_ 4
#define PD_0 120
#define PEEK 60
constexpr int TE2 = TE2_;
static_assert(TE2_ <= TA_ && TE_ <= TE2_, "stages");
constexpr int TE = TE_;
#define GO_ 1
#define Y1 0
#define Y2 1
#define LOOK 7u
constexpr int RP = RP_;
constexpr int WLO = WLO_;
constexpr int TA = TA_;
constexpr int G_WA = 192, G_X1 = G_WA + 64 * TA, G_WB = G_X1 + 192, G_X2 = G_WB + 64 * (8 - TA), G_XO = G_X2 + 192;
constexpr int G_X3 = G_XO + 128;
constexpr int G_ALL = 2047;
constexpr int NOUT = 1024;
constexpr int DYN0 = (RP + 1) * WL;
static_assert(RP >= 2 && RP * WL <= G_WA && WLO <= WL && WL <= 64, "push rounds");
__host__ __device__ constexpr int tile_pos(int c, int t) {
    return c >= 2 ? c * 8 + t : t < TA ? c * TA + t : 2 * TA + (t - TA) * 2 + c;
}

constexpr unsigned O_CNT = 0;
constexpr unsigned O_TMO = 8192;
constexpr unsigned O_TICK = 12288;
constexpr unsigned O_OWN = 14336;
constexpr unsigned O_RDY = 16384;
constexpr unsigned O_LRW = 20480;
constexpr unsigned O_PROG = 24576;
constexpr unsigned O_PUSH = 26624;
constexpr unsigned O_G = 28672;
constexpr unsigned O_PRDY = 32768;
constexpr unsigned O_ORDY = 36864;
constexpr unsigned O_DBG = 65536;
constexpr unsigned O_EX = 512 * 1024;
constexpr unsigned O_WR = 1 * MiB;
constexpr unsigned O_PS = 9 * MiB;
constexpr unsigned O_PAR = 41 * MiB;
constexpr unsigned PAR_BYTES = 56 * MiB;
constexpr unsigned P_X = 0, P_WS = 16 * MiB, P_OS = 24 * MiB;
constexpr unsigned WS_USED = O_PAR + 2 * PAR_BYTES;
constexpr unsigned OUT_BYTES = 512u * NCOL * 4u;

constexpr unsigned long long DEADLINE = 60000000ull;
constexpr int BIAS = 1 << 20;
constexpr int NEX = 16;
constexpr int SYS = 17;
#define OUT_AUX 16

constexpr int STAGE = 49152;
constexpr int LDS_MISC = 3 * STAGE;
constexpr int LDS_EX = LDS_MISC + 64;

struct Params {
    char* ws[NR];
    const char* x;
    const char* w;
    const float* sx;
    const float* sw;
    float* out;
    int rank;
    int pad;
};

__device__ __forceinline__ rsrc_t mk(const void* p, unsigned bytes) {
    return __builtin_amdgcn_make_buffer_rsrc((void*)p, 0, bytes, 0x00020000);
}
#define st128(v, r, voff, soff, aux) __builtin_amdgcn_raw_buffer_store_b128(v, r, (voff) + (soff), 0, aux)
#define st96(v, r, voff, soff, aux) __builtin_amdgcn_raw_buffer_store_b96(v, r, (voff) + (soff), 0, aux)
#define st32(v, r, voff, soff, aux) __builtin_amdgcn_raw_buffer_store_b32(v, r, (voff) + (soff), 0, aux)
__device__ __forceinline__ u32x4 mk4(unsigned a, unsigned b, unsigned c, unsigned d) { return (u32x4){a, b, c, d}; }
__device__ __forceinline__ u32x3 mk3(unsigned a, unsigned b, unsigned c) { return (u32x3){a, b, c}; }
__device__ __forceinline__ void st1(const rsrc_t r, unsigned off, unsigned v, int lane) {
    __builtin_amdgcn_raw_buffer_store_b32(v, r, lane == 0 ? off : 0xf0000000u, 0, SYS);
}
__device__ __forceinline__ unsigned tmo_off() { return O_TMO + (blockIdx.x & 31) * 128; }
__device__ __forceinline__ void set_tmo(char* myws, unsigned call, int lane) {
    __builtin_amdgcn_raw_buffer_store_b32(call, mk(myws, O_DBG), lane < 32 ? O_TMO + lane * 128 : 0xf0000000u, 0, SYS);
}
__device__ __forceinline__ unsigned ld_sys(const unsigned* p) {
    return __hip_atomic_load(p, __ATOMIC_RELAXED, __HIP_MEMORY_SCOPE_SYSTEM);
}
__device__ __forceinline__ void st_sys(unsigned* p, unsigned v) {
    __hip_atomic_store(p, v, __ATOMIC_RELAXED, __HIP_MEMORY_SCOPE_SYSTEM);
}
__device__ __forceinline__ void drain() {
    asm volatile("s_waitcnt vmcnt(0)" ::: "memory");
    __builtin_amdgcn_s_waitcnt(0x0F70);
}
__device__ __forceinline__ unsigned long long now() { return __builtin_amdgcn_s_memrealtime(); }
__device__ __forceinline__ unsigned perm(unsigned hi, unsigned lo, unsigned sel) {
    return __builtin_amdgcn_perm(hi, lo, sel);
}
__device__ __forceinline__ unsigned bfi(unsigned m, unsigned a, unsigned b) { return (a & m) | (b & ~m); }

#define STAMP(slot) do {} while (0)
#define TICKSTAMP(g) do {} while (0)

__device__ __forceinline__ bool wave_wait(const unsigned* ptr, unsigned need, char* myws, unsigned call, int lane) {
    const unsigned* tmo = (const unsigned*)(myws + tmo_off());
    const unsigned long long t0 = now();
    for (;;) {
        const unsigned v = ld_sys(ptr);
        const unsigned t = ld_sys(tmo);
        if (__builtin_amdgcn_readfirstlane(t) == call) return false;
        if (BALLOT(v < need) == 0) return true;
        if (now() - t0 > DEADLINE) {
            set_tmo(myws, call, lane);
            return false;
        }
        __builtin_amdgcn_s_sleep(1);
    }
}

__device__ __forceinline__ void monitor(const Params& p, char* myws, unsigned call, int lane) {
    const unsigned* tmo = (const unsigned*)(myws + tmo_off());
    const unsigned* prog = (const unsigned*)(myws + O_PROG);
    const unsigned* push = (const unsigned*)(myws + O_PUSH);
    const int l7 = lane & 7, me = p.rank;
    unsigned* ownp = (unsigned*)(p.ws[lane >> 3] + O_OWN) + me * 32 + ((me + 1 + l7) & 7);
    const unsigned base = call << 12;
    unsigned last = 0, nin = 4095;
    bool planned = false, told = false;
    const unsigned long long t0 = now();
    for (;;) {
        unsigned g[7];
        const unsigned tk = __builtin_amdgcn_raw_buffer_load_b32(mk(myws, !planned && BALLOT(last >= G_X3 - LEAD_ - 160 && l7 < 7) != 0 ? O_DBG : 0u), O_TICK + l7 * 128, 0, SYS);
        #pragma unroll
        for (int l = 0; l < 7; ++l) {
            const unsigned v = ld_sys(prog + l * 64 + lane), q = ld_sys(push + l * 64 + lane);
            g[l] = lane >= WL ? 4095u : (v >= base && q >= 2 * call) ? v - base : 0u;
        }
        if (!planned) {
            int r = l7 < 7 && tk >= base ? (int)(tk - base) : 0;
            if (BALLOT(r >= G_X3 - LEAD_) != 0) {
                planned = true;
                r = -r;
                unsigned keep = 0x7fu & ~(1u << l7), recv = 0;
                #pragma unroll
                for (int j = 1; j < 7; ++j) {
                    const int to = l7 + j >= 7 ? l7 + j - 7 : l7 + j, from = l7 - j < 0 ? l7 - j + 7 : l7 - j;
                    const int give = r - __shfl(r, to) > RELAY_TH ? 1 : 0;
                    const int got = __shfl(give, from);
                    r += 32 * (got - give);
                    keep &= ~((unsigned)give << to);
                    recv |= (unsigned)got << from;
                }
                nin = G_X3 + 32 * (__builtin_popcount(keep) + __builtin_popcount(recv));
                if (lane < 7) st_sys((unsigned*)(myws + O_OWN) + ((me + 1 + lane) & 7) * 32 + 8, (call << 14) | (recv << 7) | keep);
            }
        }
        #pragma unroll
        for (int o = 32; o; o >>= 1)
            #pragma unroll
            for (int l = 0; l < 7; ++l) g[l] = min(g[l], (unsigned)__shfl_xor((int)g[l], o));
        unsigned mine = 4095;
        #pragma unroll
        for (int l = 0; l < 7; ++l) if (l7 == l) mine = g[l];
        mine = mine >= nin ? (unsigned)G_ALL : min(mine, (unsigned)G_X3);
        if (mine > last) {
            last = mine;
            if (l7 < 7) st_sys((unsigned*)(myws + O_G + (lane >> 3) * 128) + l7, (call << 11) + mine);
        }
        if (!told && mine >= G_XO + 32) {
            told = true;
            if (l7 < 7 && (lane >> 3) != me) st_sys(ownp, call);
        }
        if (BALLOT(mine < G_ALL) == 0) return;
        if (UNI(ld_sys(tmo)) == call) return;
        if (now() - t0 > DEADLINE) { set_tmo(myws, call, lane); return; }
        __builtin_amdgcn_s_sleep(1);
    }
}

struct Item { u32x4 l0, l1, p0, p1, p2, p3; u32x2 h4; unsigned h1, ex; };

__device__ __forceinline__ unsigned draw(const rsrc_t tick, bool sel) {
    int t = 1, z = 0;
    asm volatile("" : "+v"(z));
    if (sel) t = __builtin_amdgcn_raw_ptr_buffer_atomic_add_i32(t, tick, z, 0, 0);
    return (unsigned)t;
}

__device__ __forceinline__ void link_wave(const Params& p, char* myws, const unsigned* cntp, int s, int lane, int wv) {
    const int me = p.rank;
    const int link = s % 7, idx = s / 7;
    const unsigned peer = (unsigned)(me + 1 + link) & 7u;
    const char* remp = p.ws[peer];
    const rsrc_t loc = mk(myws, WS_USED), rem = mk(remp, WS_USED);
    const unsigned v16 = lane * 16;

    u32x4 a0, a1;
    const rsrc_t xs = mk(p.x, 4096u * KS);
    const unsigned vx = (unsigned)(lane & 15) * KS + (lane >> 4) * 16;
    auto rb_of = [&](int g) {
        const int pi = g >> 5, i = pi + (pi >= 6 - link ? 1 : 0);
        return (unsigned)(4 * ((peer + 1 + i) & 7));
    };
    auto xoff = [&](int g) { return (rb_of(g) * 128 + (unsigned)(g & 3) * 32) * KS + (unsigned)((g & 31) >> 2) * 64; };
    const unsigned cv = ld_sys(cntp);
    a0 = __builtin_amdgcn_raw_buffer_load_b128(xs, vx, xoff(idx), 0);
    a1 = __builtin_amdgcn_raw_buffer_load_b128(xs, vx + 16 * KS, xoff(idx), 0);
    const unsigned call = UNI(cv) + 1u;
    STAMP(0);
    const unsigned par = O_PAR + (call & 1u) * PAR_BYTES;
    const unsigned dbase = par + P_X + (unsigned)me * (2 * MiB);
    const unsigned roff = lane == 63 ? O_PUSH + (link * 64 + idx) * 4 : O_RDY + (peer * 64 + lane) * 4;
    const unsigned* rdyp = (const unsigned*)(myws + roff);
    const unsigned rall = lane < 48 && !(lane < 32 && (unsigned)(lane >> 2) == peer) ? call : 0u;
    const unsigned rneed = lane == 63 ? (GO_ ? 2 * call - 1 : 0u)
                         : (lane < 32 ? (lane & 3) == 0 : lane < 32 + 2 * TE) ? rall : 0u;
    unsigned rv = 0;
    #pragma unroll 1
    for (int r = 0; r < RP - 1; ++r) {
        if (lane == 0 && r == idx % (RP - 1)) __hip_atomic_fetch_max((unsigned*)(myws + O_TICK + link * 128), (call << 12) + DYN0,
                                          __ATOMIC_RELAXED, __HIP_MEMORY_SCOPE_AGENT);
        const int g = idx + r * WL;
        const unsigned so = xoff(g + WL);
        const u32x4 b0 = __builtin_amdgcn_raw_buffer_load_b128(xs, vx, so, 0);
        const u32x4 b1 = __builtin_amdgcn_raw_buffer_load_b128(xs, vx + 16 * KS, so, 0);
        if (r == RP - 2) st1(rem, O_PUSH + ((6 - link) * 64 + idx) * 4, 2 * call - 1, lane);
        asm volatile("" ::: "memory");
        const unsigned d = dbase + rb_of(g) * 65536u + (unsigned)(g & 31) * 2048u;
        st128(a0, rem, v16, d, SYS);
        st128(a1, rem, v16 + 1024, d, SYS);
        drain();
        a0 = b0; a1 = b1;
    }
    {
        rv = __builtin_amdgcn_raw_buffer_load_b32(loc, roff, 0, SYS);
        asm volatile("" ::: "memory");
        const int g = idx + (RP - 1) * WL;
        const unsigned d = dbase + rb_of(g) * 65536u + (unsigned)(g & 31) * 2048u;
        st128(a0, rem, v16, d, SYS);
        st128(a1, rem, v16 + 1024, d, SYS);
    }
    const float scale = p.sx[0] * p.sw[0];
    st1(loc, (unsigned)((const char*)cntp - myws), call, lane);
    int dead = 0;
    if (BALLOT(rv < rneed) != 0) dead = wave_wait(rdyp, rneed, myws, call, lane) ? 0 : 1;
    dead = UNI(dead);
    unsigned long long unseen = BALLOT(rv < rall);
    auto wanted = [&](int g) {
        return g < G_WA + 64 * TE ? 0ull : g < G_WA + 64 * TE2 ? ((1ull << (2 * TE2)) - 1) << 32 : ~0ull;
    };
    STAMP(1);

    const unsigned prog = O_PROG + (link * 64 + idx) * 4;
    const rsrc_t tick = mk(myws + O_TICK + link * 128, 4);
    const unsigned tbase = call << 12;
    auto x_off = [&](unsigned k, unsigned rb, int g) {
        return par + P_X + k * (2 * MiB) + rb * 65536u + (unsigned)(g & 31) * 2048u;
    };
    auto in_off = [&](int g, unsigned& src, unsigned& dst) {
        if ((g >= G_WA && g < G_X1) || (g >= G_WB && g < G_X2)) {
            const unsigned o = (unsigned)(g < G_X1 ? g - G_WA : g - G_WB + 64 * TA) * 2048u;
            src = par + P_WS + (unsigned)me * MiB + o;
            dst = O_WR + peer * MiB + o;
        } else {
            unsigned rb;
            if (g >= G_XO) rb = 4 * me + ((((g - G_XO) >> 5) + 3) & 3);
            else {
                const int q = (g < G_WA ? g : g < G_WB ? g - 64 * TA : g - 512) >> 5;
                const int c = q / 6, pi = q - 6 * c;
                const int i = pi + (pi >= link ? 1 : 0);
                rb = 4 * ((me + 1 + i) & 7) + c;
            }
            src = dst = x_off(peer, rb, g);
        }
    };
    const unsigned oflag = lane < 32 ? O_ORDY + (((call & 1u) * 8 + peer) * 32 + lane) * 4
                                     : O_PRDY + (peer * 32 + (lane - 32)) * 4;
    const unsigned tflag = lane < 9 ? O_OWN + (peer * 32 + lane) * 4 : lane == 9 ? tmo_off() : 0xf0000000u;
    unsigned fv = 0;
    unsigned tv = 0;
    int g = idx + RP * WL, gn;
    int nin = 0;
    int held = 1;
    unsigned plan = 0, lack = 0, src, cdst, ndst;
    auto read_plan = [&]() {
        if (!nin) {
            plan = __builtin_amdgcn_readlane(tv, 8);
            if ((plan >> 14) == (call & 0x3ffffu)) nin = G_X3 + 32 * __builtin_popcount(plan & 0x3fffu);
        }
        const unsigned recv = (plan >> 7) & 0x7fu;
        lack = ((recv | (recv << 8)) >> (7 - me)) & 0xffu & ~(unsigned)BALLOT(lane < 8 && tv >= call);
    };
    in_off(g, src, cdst);
    {
        const rsrc_t remd = mk(remp, dead ? 0u : WS_USED);
        const u32x4 b0 = __builtin_amdgcn_raw_buffer_load_b128(remd, v16, src, SYS);
        const u32x4 b1 = __builtin_amdgcn_raw_buffer_load_b128(remd, v16 + 1024, src, SYS);
        const unsigned tk = draw(tick, lane == 0);
        drain();
        st1(rem, O_PUSH + ((6 - link) * 64 + idx) * 4, 2 * call, lane);
        a0 = b0; a1 = b1;
        gn = (int)(UNI(tk) - tbase);
    }
    for (;;) {
        gn = UNI(gn); dead = UNI(dead); held = UNI(held); nin = UNI(nin); plan = UNI(plan); cdst = UNI(cdst); lack = UNI(lack);
        unseen = ((unsigned long long)UNI((unsigned)(unseen >> 32)) << 32) | UNI((unsigned)unseen);
        if (unseen & wanted(gn)) {
            const unsigned long long w = wanted(gn);
            if (!dead) dead = wave_wait(rdyp, (w >> lane) & 1 ? rall : 0u, myws, call, lane) ? 0 : 1;
            unseen &= ~w;
        }
        if (gn < G_X3) in_off(gn, src, ndst);
        else {
            unsigned long long t0 = 0;
            for (;;) {
                if (dead && !nin) { plan = 0; nin = G_X3; }
                if (nin) {
                    if (gn >= nin) break;
                    unsigned m = plan & 0x3fffu;
                    for (int j = (gn - G_X3) >> 5; j; --j) m &= m - 1;
                    const unsigned bit = __builtin_ctz(m);
                    const unsigned other = (unsigned)(me + 1 + (bit >= 7 ? bit - 7 : bit)) & 7u;
                    src = ndst = bit >= 7 ? x_off(other, 4 * peer + 3, gn) : x_off(peer, 4 * other + 3, gn);
                    if (bit < 7 || dead || !((lack >> other) & 1u)) break;
                }
                if (held) {
                    st128(a0, loc, v16, cdst, SYS);
                    st128(a1, loc, v16 + 1024, cdst, SYS);
                    drain();
                    st1(loc, prog, tbase + (unsigned)gn, lane);
                    held = 0;
                } else {
                    if (!t0) t0 = now();
                    if (now() - t0 > DEADLINE) { set_tmo(myws, call, lane); dead = 1; }
                    __builtin_amdgcn_s_sleep(2);
                }
                tv = __builtin_amdgcn_raw_buffer_load_b32(loc, tflag, 0, SYS);
                drain();
                if (__builtin_amdgcn_readlane(tv, 9) == call) dead = 1;
                read_plan();
            }
            if (gn >= nin) break;
        }
        const rsrc_t remd = mk(remp, dead ? 0u : WS_USED), locd = mk(myws, dead || !held ? 0u : WS_USED);
        const rsrc_t alive = mk(myws, dead ? 0u : O_DBG);
        const u32x4 b0 = __builtin_amdgcn_raw_buffer_load_b128(remd, v16, src, SYS);
        const u32x4 b1 = __builtin_amdgcn_raw_buffer_load_b128(remd, v16 + 1024, src, SYS);
        const unsigned tk = draw(tick, lane == 0);
        TICKSTAMP(gn);
        const bool look = gn >= G_X3 - 80 && (!nin || lack);
        fv = __builtin_amdgcn_raw_buffer_load_b32(mk(myws, nin && gn >= nin - 48 ? O_DBG : 0u), oflag, 0, SYS);
        const bool peek = (unseen & wanted(gn + PEEK)) != 0;
        const unsigned tn = __builtin_amdgcn_raw_buffer_load_b32(mk(myws, look || peek ? O_DBG : 0u),
                                                                 peek ? roff : tflag, 0, SYS);
        asm volatile("" ::: "memory");
        st128(a0, locd, v16, cdst, SYS);
        st128(a1, locd, v16 + 1024, cdst, SYS);
        drain();
        st1(alive, prog, tbase + (unsigned)gn, lane);
        if (look) { tv = tn; read_plan(); }
        if (peek) unseen = BALLOT(tn < rall);
        a0 = b0; a1 = b1; held = 1;
        g = gn; cdst = ndst;
        gn = (int)(UNI(tk) - tbase);
    }
    const rsrc_t locd = mk(myws, dead || !held ? 0u : WS_USED);
    const int NIN = nin, NTOT = nin + NOUT;

    const unsigned ovoff = (unsigned)(4 * (lane >> 4)) * (NCOL * 4) + (lane & 15) * 16;
    const unsigned exb = O_EX + ((call & 1u) * 8 + peer) * (256 * NEX * 4);
    unsigned rmask;
    Item cur, nxt;
    st128(a0, locd, v16, cdst, SYS);
    st128(a1, locd, v16 + 1024, cdst, SYS);
    int state = 2;
    int waiting = 0;
    unsigned long long t0 = 0;
    {
        const unsigned long long m = BALLOT(lane < 32 ? (fv >> 8) >= call : fv >= call);
        rmask = (unsigned)m;
        if ((unsigned)(m >> 32) != 0xffffffffu && !dead)
            dead = wave_wait((const unsigned*)(myws + O_PRDY) + peer * 32 + (lane & 31), call, myws, call, lane) ? 0 : 1;
    }
    const unsigned pflag = lane < 32 ? oflag : lane == 32 ? tmo_off() : 0xf0000000u;
    for (;;) {
        g = UNI(g); gn = UNI(gn); dead = UNI(dead); state = UNI(state); rmask = UNI(rmask);
        cdst = UNI(cdst); waiting = UNI(waiting);
        const bool have = gn < NTOT;
        const unsigned o = (unsigned)(gn - NIN) & 1023u;
        const bool can = have && (dead || ((rmask >> (o >> 5)) & 1u));
        {
            const bool go = can && !dead;
            const rsrc_t r0 = mk(remp, go ? WS_USED : 0u), r1 = mk(myws, go ? WS_USED : 0u);
            const unsigned so = par + P_OS + (unsigned)me * (4 * MiB) + o * 4096u;
            const unsigned ps = O_PS + peer * (4 * MiB) + o * 4096u;
            nxt.l0 = __builtin_amdgcn_raw_buffer_load_b128(r0, v16, so, SYS);
            nxt.l1 = __builtin_amdgcn_raw_buffer_load_b128(r0, v16 + 1024, so, SYS);
            nxt.h4 = __builtin_amdgcn_raw_buffer_load_b64(r0, lane * 8 + 2048, so, SYS);
            nxt.h1 = __builtin_amdgcn_raw_buffer_load_b16(r0, lane * 2 + 2560, so, SYS);
            nxt.ex = __builtin_amdgcn_raw_buffer_load_b32(r1, (lane & (NEX - 1)) * 4, exb + (o >> 2) * (NEX * 4), SYS);
            nxt.p0 = __builtin_amdgcn_raw_buffer_load_b128(r1, v16, ps, SYS);
            nxt.p1 = __builtin_amdgcn_raw_buffer_load_b128(r1, v16 + 1024, ps, SYS);
            nxt.p2 = __builtin_amdgcn_raw_buffer_load_b128(r1, v16 + 2048, ps, SYS);
            nxt.p3 = __builtin_amdgcn_raw_buffer_load_b128(r1, v16 + 3072, ps, SYS);
        }
        const bool poll = have && !dead && (~rmask & (LOOK << (o >> 5))) != 0;
        const unsigned nf = __builtin_amdgcn_raw_buffer_load_b32(mk(myws, poll ? O_DBG : 0u), pflag, 0, SYS);
        const unsigned tk = draw(tick, can && idx < WLO && lane == 0);

        u32x4 o4[4];
        if (state == 1) {
            const unsigned oc = (unsigned)(g - NIN);
            const bool raw = (__builtin_amdgcn_readlane(fv, oc >> 5) >> ((oc >> 2) & 7)) & 1u;
            const unsigned L[8] = {cur.l0.x, cur.l0.y, cur.l0.z, cur.l0.w, cur.l1.x, cur.l1.y, cur.l1.z, cur.l1.w};
            u32x16 pv = {cur.p0.x, cur.p0.y, cur.p0.z, cur.p0.w, cur.p1.x, cur.p1.y, cur.p1.z, cur.p1.w,
                         cur.p2.x, cur.p2.y, cur.p2.z, cur.p2.w, cur.p3.x, cur.p3.y, cur.p3.z, cur.p3.w};
            u32x4 v[4];
            if (raw && !dead) {
                const unsigned so = par + P_OS + (unsigned)me * (4 * MiB) + oc * 4096u;
                const u32x4 ha = __builtin_amdgcn_raw_buffer_load_b128(rem, v16 + 2048, so, SYS);
                const u32x4 hb = __builtin_amdgcn_raw_buffer_load_b128(rem, v16 + 3072, so, SYS);
                const unsigned H[8] = {ha.x, ha.y, ha.z, ha.w, hb.x, hb.y, hb.z, hb.w};
                #pragma unroll
                for (int r = 0; r < 4; ++r) {
                    v[r].x = perm(H[2 * r], L[2 * r], 0x05040100u);
                    v[r].y = perm(H[2 * r], L[2 * r], 0x07060302u);
                    v[r].z = perm(H[2 * r + 1], L[2 * r + 1], 0x05040100u);
                    v[r].w = perm(H[2 * r + 1], L[2 * r + 1], 0x07060302u);
                }
            } else {
                unsigned em = (unsigned)BALLOT(lane < NEX && (cur.ex >> 21) == (oc & 3u));
                while (em) {
                    const unsigned e = __builtin_amdgcn_readlane(cur.ex, __builtin_ctz(em));
                    em &= em - 1;
                    pv[(e >> 17) & 15u] += (unsigned)lane == ((e >> 11) & 63u) ? e << 21 : 0u;
                }
                const unsigned t = (cur.h1 & 0x0f0fu) | ((cur.h1 & 0xf0f0u) << 12);
                unsigned B[4];
                B[0] = (cur.h4.x & 0x0f0f0f0fu) | ((t << 4) & 0x10101010u);
                B[1] = ((cur.h4.x >> 4) & 0x0f0f0f0fu) | ((t << 3) & 0x10101010u);
                B[2] = (cur.h4.y & 0x0f0f0f0fu) | ((t << 2) & 0x10101010u);
                B[3] = ((cur.h4.y >> 4) & 0x0f0f0f0fu) | ((t << 1) & 0x10101010u);
                #pragma unroll
                for (int r = 0; r < 4; ++r) {
                    v[r].x = perm(B[r], L[2 * r], 0x0c040100u);
                    v[r].y = perm(B[r], L[2 * r], 0x0c050302u);
                    v[r].z = perm(B[r], L[2 * r + 1], 0x0c060100u);
                    v[r].w = perm(B[r], L[2 * r + 1], 0x0c070302u);
                }
            }
            const u32x4 P[4] = {mk4(pv[0], pv[1], pv[2], pv[3]), mk4(pv[4], pv[5], pv[6], pv[7]),
                                mk4(pv[8], pv[9], pv[10], pv[11]), mk4(pv[12], pv[13], pv[14], pv[15])};
            #pragma unroll
            for (int r = 0; r < 4; ++r) {
                const u32x4 t = v[r] + P[r];
                f32x4 f;
                f.x = (float)(int)t.x * scale; f.y = (float)(int)t.y * scale;
                f.z = (float)(int)t.z * scale; f.w = (float)(int)t.w * scale;
                o4[r] = __builtin_bit_cast(u32x4, f);
                if (dead) o4[r] = mk4(0x7fc00000u, 0x7fc00000u, 0x7fc00000u, 0x7fc00000u);
            }
        }
        {
            const rsrc_t out = mk(p.out, state == 1 ? OUT_BYTES : 0u);
            #pragma unroll
            for (int r = 0; r < 4; ++r) st128(o4[r], out, ovoff + r * (NCOL * 4), cdst, OUT_AUX);
        }
        if (!have && state != 2) break;
        asm volatile("s_waitcnt vmcnt(4)" ::: "memory");
        __builtin_amdgcn_s_waitcnt(0x0F74);
        if (state == 2) {
            asm volatile("s_waitcnt vmcnt(0)" ::: "memory");
            if (!dead) st1(loc, prog, tbase + 4095u, lane);
            STAMP(2);
        }
        if (!have) break;
        if (poll) {
            fv = nf;
            rmask = (unsigned)BALLOT((nf >> 8) >= call);
            if (__builtin_amdgcn_readlane(nf, 32) == call) dead = 1;
        }
        if (can) {
            cur = nxt; g = gn; state = 1;
            const unsigned n = o >> 5, w8 = (o >> 2) & 7, mf = o & 3;
            const unsigned c = n >= 16 ? n >> 3 : n < 2 * TA ? (n >= TA ? 1 : 0) : n & 1;
            const unsigned t = n >= 16 ? n & 7 : n < 2 * TA ? n - c * TA : TA + ((n - 2 * TA) >> 1);
            cdst = ((c * 128 + (w8 >> 2) * 64 + mf * 16) * NCOL + peer * 2048 + t * 256 + (w8 & 3) * 64) * 4u;
            gn = idx < WLO ? (int)(UNI(tk) - tbase) : NTOT;
            waiting = 0;
        } else {
            if (!waiting) { waiting = 1; t0 = now(); }
            else {
                if (now() - t0 > DEADLINE) { set_tmo(myws, call, lane); dead = 1; }
                __builtin_amdgcn_s_sleep(4);
            }
            state = 0;
        }
    }
    STAMP(3);
}

__device__ __forceinline__ unsigned prep(const Params& p, char* myws, const unsigned* cntp, int id, int tid, int lane, int wv) {
    const int me = p.rank;
    unsigned call;
    {
        const int tw = ((id - 32) >> 1) & 7;
        const unsigned long long hold = id < 32 ? ((id & 3) ? PD_X : 0) : (tw >= TE ? PD_0 + (tw - TE) * PD_W : 0);
        const unsigned long long t0 = now();
        while (now() - t0 < hold) __builtin_amdgcn_s_sleep(4);
    }
    const rsrc_t loc = mk(myws, WS_USED);
    if (id < 32) {
        const rsrc_t xs = mk(p.x, 4096u * KS);
        const unsigned voff = (unsigned)(lane & 15) * KS + (lane >> 4) * 16;
        const unsigned soff = (unsigned)(id * 128 + wv * 16) * KS;
        u32x4 v[8];
        #pragma unroll
        for (int i = 0; i < 8; ++i) v[i] = __builtin_amdgcn_raw_buffer_load_b128(xs, voff + i * 64, soff, 0);
        call = UNI(ld_sys(cntp)) + 1u;
        const unsigned dst = O_PAR + (call & 1u) * PAR_BYTES + P_X + (unsigned)me * (2 * MiB) + (unsigned)id * 65536u + wv * 1024u;
        #pragma unroll
        for (int i = 0; i < 8; ++i) st128(v[i], loc, lane * 16, dst + i * 8192u, SYS);
    } else {
        const int pid = id - 32, jt = pid >> 1, h = pid & 1;
        const rsrc_t wsr = mk(p.w, (unsigned)KS * NCOL);
        const int wn = wv & 3, kq0 = h * 4 + (wv >> 2) * 2;
        const unsigned voff = (unsigned)(16 * (lane >> 4)) * NCOL + (lane & 15) * 4;
        unsigned r[2][16];
        #pragma unroll
        for (int b = 0; b < 2; ++b)
            #pragma unroll
            for (int j = 0; j < 16; ++j)
                r[b][j] = __builtin_amdgcn_raw_buffer_load_b32(wsr, voff, (unsigned)((kq0 + b) * 64 + j) * NCOL + jt * 256 + wn * 64, 0);
        call = UNI(ld_sys(cntp)) + 1u;
        const unsigned par = O_PAR + (call & 1u) * PAR_BYTES;
        #pragma unroll
        for (int b = 0; b < 2; ++b) {
            unsigned o[4][4];
            #pragma unroll
            for (int q = 0; q < 4; ++q) {
                const unsigned t0 = perm(r[b][4 * q + 1], r[b][4 * q], 0x05010400u);
                const unsigned t1 = perm(r[b][4 * q + 1], r[b][4 * q], 0x07030602u);
                const unsigned t2 = perm(r[b][4 * q + 3], r[b][4 * q + 2], 0x05010400u);
                const unsigned t3 = perm(r[b][4 * q + 3], r[b][4 * q + 2], 0x07030602u);
                o[0][q] = perm(t2, t0, 0x05040100u);
                o[1][q] = perm(t2, t0, 0x07060302u);
                o[2][q] = perm(t3, t1, 0x05040100u);
                o[3][q] = perm(t3, t1, 0x07060302u);
            }
            const unsigned dst = par + P_WS + (unsigned)jt * 131072u + (unsigned)((kq0 + b) * 16 + wn * 4) * 1024u;
            #pragma unroll
            for (int i = 0; i < 4; ++i)
                st128(mk4(o[i][0], o[i][1], o[i][2], o[i][3]), loc, lane * 16 + i * 1024, dst, SYS);
        }
    }
    drain();
    __syncthreads();
    if (id < 32) {
        if (tid < NR) st_sys((unsigned*)(p.ws[tid] + O_RDY) + me * 64 + id, call);
    } else {
        const int pid = id - 32;
        if (tid == 0) st_sys((unsigned*)(p.ws[pid >> 4] + O_RDY) + me * 64 + 32 + (pid & 15), call);
        if (tid == 1) st_sys((unsigned*)(myws + O_LRW) + pid, call);
    }
    return call;
}

}

__global__ __launch_bounds__(NTHR, 2) void gemm_rs_fused(Params p) {
    __shared__ __attribute__((aligned(16))) char lds[LDS_EX + 8 * NEX * 4];
    const int tid = threadIdx.x, lane = tid & 63;
    const int wv = __builtin_amdgcn_readfirstlane(tid >> 6);
    const int b = blockIdx.x, me = p.rank;
    char* myws = p.ws[me];
    unsigned* cntp = (unsigned*)(myws + O_CNT) + b * 8 + wv;

    if (b < NPULL) {
        const int s = wv * NPULL + b;
        if (s < NLW) link_wave(p, myws, cntp, s, lane, wv);
        else if (b == NPULL - 1 && wv == 7) {
            const unsigned call = UNI(ld_sys(cntp)) + 1u;
            if (lane == 0) st_sys(cntp, call);
            monitor(p, myws, call, lane);
        }
        return;
    }

    const int cb = b - NPULL;
    STAMP(0);
    unsigned call;
    if (cb < 160) call = prep(p, myws, cntp, cb, tid, lane, wv);
    else call = UNI(ld_sys(cntp)) + 1u;
    if (lane == 0) st_sys(cntp, call);
    STAMP(1);

    const int t = (b & 7 & ~(SH - 1)) | ((cb >> 3) & (SH - 1));
    const int u = ((cb >> 3) & ~(SH - 1)) | (b & (SH - 1));
    const unsigned par = O_PAR + (call & 1u) * PAR_BYTES;
    const rsrc_t loc = mk(myws, WS_USED);
    volatile __attribute__((address_space(3))) unsigned* misc =
        (volatile __attribute__((address_space(3))) unsigned*)(lds + LDS_MISC);
    const int wm = wv >> 2, wn = wv & 3;

    for (int step = 0; step < 4; ++step) {
        int task, v;
        constexpr int NP1 = NCU - 7;
        if (step == 0) { task = 0; v = u - 7; if (v < 0) continue; }
        else if (step == 1) { task = 0; v = NP1 + (NCU - 1 - u); if (v >= 28) continue; }
        else if (step == 2) { task = 1; v = u; }
        else {
            task = 1; v = NCU + u;
            if (v >= 28) { task = 2; v = (v - 28) * 7; if (v >= 28) break; }
        }
        const int c = v / 7, d = (me + 1 + (v - 7 * c)) & 7;
        const int rb = task == 1 ? 4 * d + c : 4 * me + c;
        const int pos = tile_pos(c, t);
        const int wj = task == 0 ? d : me;

        i32x4 acc[4][4];
        {
            const int a0 = task == 0 ? -BIAS : task == 1 ? BIAS : 0;
            #pragma unroll
            for (int m = 0; m < 4; ++m)
                #pragma unroll
                for (int n = 0; n < 4; ++n) acc[m][n] = (i32x4){a0, a0, a0, a0};
        }
        unsigned todo = task == 0 ? 1u << me : task == 1 ? 0xffu & ~(1u << d) : 0xffu;
        bool ok = true;
        while (todo) {
            if (wv == 0) {
                const unsigned gend = task == 2 ? (unsigned)(G_XO + 32 * (((c + 1) & 3) + 1))
                                    : c == 3 ? (unsigned)G_ALL : c == 2 ? (unsigned)G_XO
                                    : t >= TA ? (unsigned)(G_WB + 64 * (t - TA + 1))
                                    : c == 1 ? (unsigned)G_WB : (unsigned)(G_WA + 64 * (t + 1));
                const unsigned* ptr = (const unsigned*)(myws + tmo_off());
                unsigned need = call;
                if ((todo >> me) & 1u) {
                    if (lane == 8 || lane == 9) ptr = (const unsigned*)(myws + O_LRW) + (wj * 8 + t) * 2 + (lane - 8);
                    if (lane == 10) ptr = (const unsigned*)(myws + O_RDY) + me * 64 + rb;
                }
                if (lane < 7 && task) { ptr = (const unsigned*)(myws + O_G + (b & 7) * 128) + lane; need = (call << 11) + gend; }
                const unsigned long long t0 = now();
                unsigned got;
                for (;;) {
                    const unsigned val = ld_sys(ptr);
                    unsigned m = (unsigned)BALLOT(val >= need);
                    if (__builtin_amdgcn_readlane(val, 7) == call) { got = 0; break; }
                    if (task == 1 && c == 3 && !((m >> (v - 21)) & 1u)) m &= ~0x7fu;
                    m = (m & 0x7fu) | (((m >> 8) & 7u) == 7u ? 0x80u : 0u);
                    m = ((m | (m << 8)) >> (7 - me)) & 0xffu;
                    got = m & todo;
                    if (got) break;
                    if (now() - t0 > DEADLINE) { set_tmo(myws, call, lane); break; }
                    __builtin_amdgcn_s_sleep(CSLEEP);
                }
                if (lane == 0) misc[0] = got;
            }
            __syncthreads();
            const unsigned got = UNI(misc[0]);
            if (!got) { ok = false; break; }
            todo &= ~got;
            unsigned klist = 0, rest = got;
            int nst = 0;
            while (rest) { klist |= (unsigned)__builtin_ctz(rest) << nst; rest &= rest - 1; nst += 4; }
            const unsigned va = wv * 2048 + lane * 16, vb = wv * 4096 + lane * 16;
            auto issue = [&](int s, int slot) {
                const int k = (klist >> (s & ~3)) & 15;
                const unsigned abase = par + P_X + (unsigned)k * (2 * MiB) + (unsigned)rb * 65536u + (s & 3) * 16384u;
                const unsigned bbase = (k == me ? par + P_WS + (unsigned)wj * MiB : O_WR + (unsigned)k * MiB) + (unsigned)t * 131072u + (s & 3) * 32768u;
                char* la = lds + slot * STAGE + wv * 2048;
                char* lb = lds + slot * STAGE + 16384 + wv * 4096;
                __builtin_amdgcn_raw_ptr_buffer_load_lds(loc, (__attribute__((address_space(3))) void*)la, 16, va, abase, 0, SYS);
                __builtin_amdgcn_raw_ptr_buffer_load_lds(loc, (__attribute__((address_space(3))) void*)la, 16, va, abase, 1024, SYS);
                __builtin_amdgcn_raw_ptr_buffer_load_lds(loc, (__attribute__((address_space(3))) void*)lb, 16, vb, bbase, 0, SYS);
                __builtin_amdgcn_raw_ptr_buffer_load_lds(loc, (__attribute__((address_space(3))) void*)lb, 16, vb, bbase, 1024, SYS);
                __builtin_amdgcn_raw_ptr_buffer_load_lds(loc, (__attribute__((address_space(3))) void*)lb, 16, vb, bbase, 2048, SYS);
                __builtin_amdgcn_raw_ptr_buffer_load_lds(loc, (__attribute__((address_space(3))) void*)lb, 16, vb, bbase, 3072, SYS);
            };
            auto compute = [&](int slot) {
                const char* A = lds + slot * STAGE + wm * 4096 + lane * 16;
                const char* B = lds + slot * STAGE + 16384 + wn * 4096 + lane * 16;
                i32x4 af[2][4], bf[2][4];
                #pragma unroll
                for (int kb = 0; kb < 2; ++kb) {
                    af[kb][0] = *(const i32x4*)(A + kb * 8192);
                    #pragma unroll
                    for (int i = 0; i < 4; ++i) bf[kb][i] = *(const i32x4*)(B + kb * 16384 + i * 1024);
                    #pragma unroll
                    for (int i = 1; i < 4; ++i) af[kb][i] = *(const i32x4*)(A + kb * 8192 + i * 1024);
                }
                __builtin_amdgcn_sched_barrier(0);
                #pragma unroll
                for (int kb = 0; kb < 2; ++kb)
                    #pragma unroll
                    for (int m = 0; m < 4; ++m)
                        #pragma unroll
                        for (int n = 0; n < 4; ++n)
                            acc[m][n] = __builtin_amdgcn_mfma_i32_16x16x64_i8(af[kb][m], bf[kb][n], acc[m][n], 0, 0, 0);
                __builtin_amdgcn_sched_barrier(0);
            };
            issue(0, 0);
            issue(1, 1);
            int slot = 0;
            for (int s = 0; s < nst; ++s) {
                if (s + 1 < nst) asm volatile("s_waitcnt vmcnt(6) lgkmcnt(0)" ::: "memory");
                else asm volatile("s_waitcnt vmcnt(0) lgkmcnt(0)" ::: "memory");
                __builtin_amdgcn_s_barrier();
                if (s + 2 < nst) issue(s + 2, slot == 0 ? 2 : slot - 1);
                compute(slot);
                slot = slot == 2 ? 0 : slot + 1;
            }
        }

        if (task == 0) {
            if (ok) {
                const unsigned dst = O_PS + (unsigned)d * (4 * MiB) + (unsigned)(pos * 8 + wv) * 16384u;
                #pragma unroll
                for (int m = 0; m < 4; ++m)
                    #pragma unroll
                    for (int r = 0; r < 4; ++r)
                        st128(mk4((unsigned)acc[m][0][r], (unsigned)acc[m][1][r], (unsigned)acc[m][2][r], (unsigned)acc[m][3][r]),
                            loc, lane * 16 + r * 1024, dst + m * 4096u, SYS);
            }
            drain();
            __syncthreads();
            if (tid == 0 && ok) st_sys((unsigned*)(myws + O_PRDY) + d * 32 + pos, call);
        } else if (task == 1) {
            volatile __attribute__((address_space(3))) unsigned* exl =
                (volatile __attribute__((address_space(3))) unsigned*)(lds + LDS_EX) + wv * NEX;
            if (lane < NEX) exl[lane] = 0xffffffffu;
            unsigned nex = 0;
            #pragma unroll
            for (int m = 0; m < 4; ++m)
                #pragma unroll
                for (int r = 0; r < 4; ++r) {
                    const unsigned any = (unsigned)(acc[m][0][r] | acc[m][1][r]) | (unsigned)(acc[m][2][r] | acc[m][3][r]);
                    if (BALLOT((any >> 21) != 0) != 0) {
                        #pragma unroll
                        for (int n = 0; n < 4; ++n) {
                            const unsigned hi = (unsigned)acc[m][n][r] >> 21;
                            const unsigned long long bm = BALLOT(hi != 0);
                            const unsigned at = nex + __builtin_amdgcn_mbcnt_hi((unsigned)(bm >> 32), __builtin_amdgcn_mbcnt_lo((unsigned)bm, 0u));
                            if (hi != 0 && at < NEX) exl[at] = (unsigned)(m << 21 | (r * 4 + n) << 17) | ((unsigned)lane << 11) | hi;
                            nex += __builtin_popcountll(bm);
                        }
                    }
                }
            const bool raw = nex > NEX;
            if (ok) {
                const unsigned e = exl[lane & (NEX - 1)];
                __builtin_amdgcn_raw_buffer_store_b32(e, mk(p.ws[d], WS_USED),
                    lane < NEX ? O_EX + ((((call & 1u) * 8 + me) * 32 + pos) * 8 + wv) * (NEX * 4) + lane * 4 : 0xf0000000u, 0, SYS);
            }
            if (ok) {
                const unsigned dst = par + P_OS + (unsigned)d * (4 * MiB) + (unsigned)(pos * 8 + wv) * 16384u;
                #pragma unroll
                for (int m = 0; m < 4; ++m) {
                    unsigned L[8], H[8];
                    #pragma unroll
                    for (int r = 0; r < 4; ++r) {
                        L[2 * r] = perm(acc[m][1][r], acc[m][0][r], 0x05040100u);
                        L[2 * r + 1] = perm(acc[m][3][r], acc[m][2][r], 0x05040100u);
                        H[2 * r] = perm(acc[m][1][r], acc[m][0][r], 0x07060302u);
                        H[2 * r + 1] = perm(acc[m][3][r], acc[m][2][r], 0x07060302u);
                    }
                    st128(mk4(L[0], L[1], L[2], L[3]), loc, lane * 16, dst + m * 4096u, SYS);
                    st128(mk4(L[4], L[5], L[6], L[7]), loc, lane * 16 + 1024, dst + m * 4096u, SYS);
                    if (raw) {
                        st128(mk4(H[0], H[1], H[2], H[3]), loc, lane * 16 + 2048, dst + m * 4096u, SYS);
                        st128(mk4(H[4], H[5], H[6], H[7]), loc, lane * 16 + 3072, dst + m * 4096u, SYS);
                    } else {
                        unsigned B[4];
                        #pragma unroll
                        for (int r = 0; r < 4; ++r) B[r] = perm(H[2 * r + 1], H[2 * r], 0x06040200u);
                        const u32x2 h4 = {bfi(0x0f0f0f0fu, B[0], B[1] << 4), bfi(0x0f0f0f0fu, B[2], B[3] << 4)};
                        const unsigned t = ((B[0] >> 4) & 0x01010101u) | ((B[1] >> 3) & 0x02020202u)
                                         | ((B[2] >> 2) & 0x04040404u) | ((B[3] >> 1) & 0x08080808u);
                        __builtin_amdgcn_raw_buffer_store_b64(h4, loc, lane * 8 + 2048 + dst + m * 4096u, 0, SYS);
                        __builtin_amdgcn_raw_buffer_store_b16((unsigned short)(t | (t >> 12)), loc, lane * 2 + 2560 + dst + m * 4096u, 0, SYS);
                    }
                }
            }
            if (lane == 0) misc[4 + wv] = raw ? 1u : 0u;
            drain();
            __syncthreads();
            if (tid == 0 && ok) {
                unsigned mask = 0;
                #pragma unroll
                for (int i = 0; i < 8; ++i) mask |= misc[4 + i] << i;
                st_sys((unsigned*)(p.ws[d] + O_ORDY) + ((call & 1u) * 8 + me) * 32 + pos, (call << 8) | mask);
            }
        } else {
            const rsrc_t out = mk(p.out, OUT_BYTES);
            const float scale = p.sx[0] * p.sw[0];
            const unsigned ovoff = (unsigned)(4 * (lane >> 4)) * (NCOL * 4) + (lane & 15) * 16;
            const unsigned dst = ((unsigned)(c * 128 + wm * 64) * NCOL + me * 2048 + t * 256 + wn * 64) * 4u;
            #pragma unroll
            for (int m = 0; m < 4; ++m)
                #pragma unroll
                for (int r = 0; r < 4; ++r) {
                    f32x4 f;
                    f.x = (float)acc[m][0][r] * scale; f.y = (float)acc[m][1][r] * scale;
                    f.z = (float)acc[m][2][r] * scale; f.w = (float)acc[m][3][r] * scale;
                    u32x4 o4 = __builtin_bit_cast(u32x4, f);
                    if (!ok) o4 = (u32x4){0x7fc00000u, 0x7fc00000u, 0x7fc00000u, 0x7fc00000u};
                    st128(o4, out, ovoff + r * (NCOL * 4), dst + (unsigned)(m * 16) * (NCOL * 4), OUT_AUX);
                }
        }
        STAMP(step == 3 ? 7 : 3 + 2 * task);
    }
}

extern "C" void kernel_launch(int rank, int nranks, ncclComm_t comm, hipStream_t stream,
                              void* const* d_in, const int* in_sizes, int n_in,
                              void* d_out, int out_size, const int* in_dtypes, int out_dtype,
                              void* const* d_out_all, void* const* d_ws_all, size_t ws_bytes) {
    (void)comm; (void)in_sizes; (void)in_dtypes; (void)out_dtype; (void)d_out_all;
    if (nranks != NR || (unsigned)rank >= (unsigned)NR || n_in < 4 || ws_bytes < WS_USED ||
        (size_t)out_size * 4 != OUT_BYTES) {
        fprintf(stderr, "kernel_launch: unexpected problem shape\n");
        return;
    }
    Params p;
    for (int i = 0; i < NR; ++i) p.ws[i] = (char*)d_ws_all[i];
    p.x = (const char*)d_in[0];
    p.w = (const char*)d_in[1];
    p.sx = (const float*)d_in[2];
    p.sw = (const float*)d_in[3];
    p.out = (float*)d_out;
    p.rank = rank;
    p.pad = 0;
    hipLaunchKernelGGL(gemm_rs_fused, dim3(NBLK), dim3(NTHR), 0, stream, p);
    HIP_CHECK(hipGetLastError());
}
```

```cpp
#include <hip/hip_runtime.h>
#include <rccl/rccl.h>
#include <cstdio>
#include <cstdint>

#define HIP_CHECK(expr) do { hipError_t _e = (expr); if (_e != hipSuccess) { \
    fprintf(stderr, "HIP error %s at %s:%d\n", hipGetErrorString(_e), __FILE__, __LINE__); \
    return; } } while (0)

typedef int i32x4 __attribute__((ext_vector_type(4)));
typedef unsigned u32x4 __attribute__((ext_vector_type(4)));
typedef unsigned u32x2 __attribute__((ext_vector_type(2)));
typedef unsigned u32x16 __attribute__((ext_vector_type(16)));
typedef float f32x4 __attribute__((ext_vector_type(4)));
typedef __amdgpu_buffer_rsrc_t rsrc_t;
typedef volatile __attribute__((address_space(3))) unsigned* ldsw_t;
#define UNI(x) __builtin_amdgcn_readfirstlane(x)
#define BALLOT(c) __builtin_amdgcn_ballot_w64(c)

#define NPULL_ 80
#define WL_ 40
#define WLO_ 30
#define RP_ 4
#define TA_ 2
#define TE_ 2
#define VS_ 8
#define TE2_ 4
#define RELAY_TH 36
#define QBIAS 64
#define LEAD 160
#define PD_0 120
#define PD_W 30
#define PD_X 0
#define CSLEEP 6
#define NP0_ 48

#define PAIRV 2
namespace {

constexpr int NR = 8;
constexpr int NCOL = 16384;
constexpr int KS = 512;
constexpr unsigned MiB = 1u << 20;

constexpr int NBLK = 256, NTHR = 512;
constexpr int NPULL = NPULL_;
constexpr int WL = WL_;
constexpr int WLO = WLO_;
constexpr int NLW = 7 * WL;
constexpr int NCOMP = NBLK - NPULL, NCU = NCOMP / 8;
constexpr int NP0 = NP0_;
static_assert(NLW <= 7 * NPULL && NCOMP >= 160 && NCOMP <= 192 && 2 * (NCOMP - NP0) >= 192 && WLO <= WL && WL <= 64, "geometry");

constexpr int RP = RP_, TA = TA_, TE = TE_, VS = VS_, TE2 = TE2_;
constexpr int G_OWN = 320 + 64 * 14, G_REL = 384 + 64 * 16, G_OWNQ = G_REL + 768;
constexpr int G_ALL = 4095;
constexpr int NOUT = 1024;
constexpr int DYN0 = (RP + 1) * WL;
static_assert(RP >= 2 && RP * WL <= 160 && TE <= TA && TA <= TE2_ && TE2_ <= VS && VS <= 12, "lists");
__host__ __device__ constexpr int tile_pos(int cc, int V) { return V < TA ? cc * TA + V : 2 * V + cc; }
__host__ __device__ constexpr int tile_of(int V, int q) {
    return V < VS ? V : V < VS + 2 ? 12 + 2 * (V - VS) + q : V < 14 ? V - 2 : 13 + 2 * (V - 14) - q;
}
__host__ __device__ constexpr int w_done(int xa, int V) { return (V < TA ? xa : 2 * xa) + 64 * (V + 1); }

constexpr unsigned O_CNT = 0;
constexpr unsigned O_TMO = 8192;
constexpr unsigned O_TICK = 12288;
constexpr unsigned O_OWN = 14336;
constexpr unsigned O_RDY = 16384;
constexpr unsigned O_LRW = 20480;
constexpr unsigned O_PROG = 24576;
constexpr unsigned O_PUSH = 26624;
constexpr unsigned O_G = 28672;
constexpr unsigned O_PRDY = 32768;
constexpr unsigned O_ORDY = 36864;
constexpr unsigned O_DBG = 65536;
constexpr unsigned O_EX = 512 * 1024;
constexpr unsigned O_PS = 1 * MiB;
constexpr unsigned O_PAR = 33 * MiB;
constexpr unsigned P_X = 0;
constexpr unsigned P_WS = 16 * MiB;
constexpr unsigned P_WR = 24 * MiB;
constexpr unsigned P_OS = 40 * MiB;
constexpr unsigned PAR_BYTES = 72 * MiB;
constexpr unsigned WS_USED = O_PAR + 2 * PAR_BYTES;
constexpr unsigned OUT_BYTES = 512u * NCOL * 4u;
constexpr unsigned NOWHERE = 0xf0000000u;

constexpr unsigned long long DEADLINE = 40000000ull;
constexpr int BIAS = 1 << 20;
constexpr int NEX = 16;
constexpr int SYS = 17;
constexpr int OUT_AUX = 16;

constexpr int STAGE = 49152;
constexpr int LDS_MISC = 3 * STAGE;
constexpr int LDS_EX = LDS_MISC + 64;

struct Params {
    char* ws[NR];
    const char* x;
    const char* w;
    const float* sx;
    const float* sw;
    float* out;
    int rank;
    int pad;
};

__device__ __forceinline__ rsrc_t mk(const void* p, unsigned bytes) {
    return __builtin_amdgcn_make_buffer_rsrc((void*)p, 0, bytes, 0x00020000);
}
#define st128(v, r, voff, soff, aux) __builtin_amdgcn_raw_buffer_store_b128(v, r, (voff) + (soff), 0, aux)
#define ld128(r, voff, soff, aux) __builtin_amdgcn_raw_buffer_load_b128(r, voff, soff, aux)
#define ld32(r, voff) __builtin_amdgcn_raw_buffer_load_b32(r, voff, 0, SYS)
__device__ __forceinline__ u32x4 mk4(unsigned a, unsigned b, unsigned c, unsigned d) { return (u32x4){a, b, c, d}; }
__device__ __forceinline__ void st1(const rsrc_t r, unsigned off, unsigned v, int lane) {
    __builtin_amdgcn_raw_buffer_store_b32(v, r, lane == 0 ? off : NOWHERE, 0, SYS);
}
__device__ __forceinline__ unsigned tmo_off() { return O_TMO + (blockIdx.x & 31) * 128; }
__device__ __forceinline__ void set_tmo(char* myws, unsigned call, int lane) {
    __builtin_amdgcn_raw_buffer_store_b32(call, mk(myws, O_DBG), lane < 32 ? O_TMO + lane * 128 : NOWHERE, 0, SYS);
}
__device__ __forceinline__ unsigned ld_sys(const unsigned* p) {
    return __hip_atomic_load(p, __ATOMIC_RELAXED, __HIP_MEMORY_SCOPE_SYSTEM);
}
__device__ __forceinline__ void st_sys(unsigned* p, unsigned v) {
    __hip_atomic_store(p, v, __ATOMIC_RELAXED, __HIP_MEMORY_SCOPE_SYSTEM);
}
__device__ __forceinline__ void drain() {
    asm volatile("s_waitcnt vmcnt(0)" ::: "memory");
    __builtin_amdgcn_s_waitcnt(0x0F70);
}
__device__ __forceinline__ unsigned long long now() { return __builtin_amdgcn_s_memrealtime(); }
__device__ __forceinline__ unsigned perm(unsigned hi, unsigned lo, unsigned sel) {
    return __builtin_amdgcn_perm(hi, lo, sel);
}
__device__ __forceinline__ unsigned bfi(unsigned m, unsigned a, unsigned b) { return (a & m) | (b & ~m); }
__device__ __forceinline__ unsigned links_to_ranks(unsigned m, int me) { return ((m | (m << 8)) >> (7 - me)) & 0xffu; }

#define STAMP(slot) do {} while (0)
#define TICKSTAMP(g) do {} while (0)

__device__ __forceinline__ bool wave_wait(const unsigned* ptr, unsigned need, char* myws, unsigned call, int lane) {
    const unsigned* tmo = (const unsigned*)(myws + tmo_off());
    const unsigned long long t0 = now();
    for (;;) {
        const unsigned v = ld_sys(ptr);
        const unsigned t = ld_sys(tmo);
        if (UNI(t) == call) return false;
        if (BALLOT(v < need) == 0) return true;
        if (now() - t0 > DEADLINE) {
            set_tmo(myws, call, lane);
            return false;
        }
        __builtin_amdgcn_s_sleep(1);
    }
}

__device__ __forceinline__ void monitor(const Params& p, char* myws, unsigned call, int lane) {
    const unsigned* tmo = (const unsigned*)(myws + tmo_off());
    const unsigned* prog = (const unsigned*)(myws + O_PROG);
    const int l7 = lane & 7, me = p.rank, lp = (me & 1) ? 6 : 0;
    unsigned* ownp = (unsigned*)(p.ws[lane >> 3] + O_OWN) + me * 32 + ((me + 1 + l7) & 7);
    const int xa = l7 == lp ? 192 : 160;
    const unsigned tell = (lane >> 3) == (me ^ 1) ? w_done(160, VS + 1) : 2 * xa + 64 * TA;
    const unsigned gown = l7 == lp ? G_OWNQ : G_OWN;
    const unsigned base = call << 12;
    unsigned last = 0, nin = 4095, mine, open = 0x7f;
    bool pushed = false;
    bool planned = false, told = l7 == 7 || (lane >> 3) == me || (l7 == lp && (lane >> 3) == (me ^ 1));
    const unsigned long long t0 = now();
    for (;;) {
        asm volatile("" ::: "memory");
        const unsigned tk = ld32(mk(myws, planned ? 0u : O_DBG), O_TICK + l7 * 128);
        if (open == 1u << lp && pushed) {
            const unsigned v = ld_sys(prog + lp * 64 + lane);
            mine = lane >= WL ? 4095u : v >= base ? v - base : 0u;
            #pragma unroll
            for (int o = 32; o; o >>= 1) mine = min(mine, (unsigned)__shfl_xor((int)mine, o));
            if (l7 != lp) mine = 4095;
        } else {
            unsigned g[7];
            bool all = true;
            #pragma unroll
            for (int l = 0; l < 7; ++l) {
                const unsigned v = ld_sys(prog + l * 64 + lane);
                const unsigned q = ld32(mk(myws, pushed ? 0u : O_DBG), O_PUSH + (l * 64 + lane) * 4);
                const bool in = pushed || q >= call || lane >= WL;
                all = all && in;
                g[l] = lane >= WL ? 4095u : (v >= base && in) ? v - base : 0u;
            }
            pushed = BALLOT(!all) == 0;
            #pragma unroll
            for (int o = 32; o; o >>= 1)
                #pragma unroll
                for (int l = 0; l < 7; ++l) g[l] = min(g[l], (unsigned)__shfl_xor((int)g[l], o));
            mine = 4095;
            #pragma unroll
            for (int l = 0; l < 7; ++l) if (l7 == l) mine = g[l];
        }
        if (!planned) {
            int r = l7 < 7 && tk >= base ? (int)(tk - base) : 0;
            if (BALLOT(l7 != lp && r >= G_OWN - LEAD) != 0) {
                planned = true;
                r = -r - (l7 == lp ? QBIAS : 0);
                unsigned keep = 15, recv = 0;
                #pragma unroll
                for (int j = 1; j < 7; ++j) {
                    const int to = l7 + j >= 7 ? l7 + j - 7 : l7 + j, from = l7 - j < 0 ? l7 - j + 7 : l7 - j;
                    const unsigned two = (keep >> (2 * ((me + 1 + to) & 1))) & 3u;
                    const int rto = __shfl(r, to);
                    const int give = to != lp && two && r - rto > RELAY_TH ? (two == 3 ? 3 : 1) : 0;
                    const int got = __shfl(give, from);
                    r += 32 * ((got & 1) - (give & 1));
                    if (give) keep &= ~(1u << (2 * ((me + 1 + to) & 1) + (give >> 1)));
                    recv |= (unsigned)(got & 1) << from | (unsigned)(got >> 1) << (7 + from);
                }
                nin = gown + 32 * (__builtin_popcount(keep) + __builtin_popcount(recv & 0x7fu));
                if (lane < 7) st_sys((unsigned*)(myws + O_OWN) + ((me + 1 + lane) & 7) * 32 + 8, (call << 18) | (recv << 4) | keep);
            }
        }
        mine = mine >= nin ? (unsigned)G_ALL : min(mine, gown);
        if (mine > last) {
            last = mine;
            if (l7 < 7) st_sys((unsigned*)(myws + O_G + (lane >> 3) * 128) + l7, base + mine);
        }
        if (!told && mine >= tell) {
            told = true;
            st_sys(ownp, call);
        }
        open = (unsigned)BALLOT(mine < G_ALL) & 0x7fu;
        if (!open) return;
        if (now() - t0 > 100000) {
            if (UNI(ld_sys(tmo)) == call) return;
            if (now() - t0 > DEADLINE) { set_tmo(myws, call, lane); return; }
            __builtin_amdgcn_s_sleep(1);
        }
    }
}

struct Item { u32x4 l0, l1, p0, p1, p2, p3; u32x2 h4; unsigned h1, ex; };

__device__ __forceinline__ unsigned draw(const rsrc_t tick, bool sel) {
    int t = 1, z = 0;
    asm volatile("" : "+v"(z));
    if (sel) t = __builtin_amdgcn_raw_ptr_buffer_atomic_add_i32(t, tick, z, 0, 0);
    return (unsigned)t;
}

__device__ __forceinline__ void link_wave(const Params& p, char* myws, const unsigned* cntp, int s, int lane, int wv) {
    const int me = p.rank, hf = me & 1;
    const int link = s % 7, idx = s / 7;
    const unsigned peer = (unsigned)(me + 1 + link) & 7u;
    const bool isq = peer == (unsigned)(me ^ 1);
    const char* remp = p.ws[peer];
    const rsrc_t loc = mk(myws, WS_USED), rem = mk(remp, WS_USED);
    const unsigned v16 = lane * 16;
    auto third = [&](int r, int l, int i) {
        const int j = i + 1 - (r & 1);
        return (unsigned)(r + 1 + j + (!isq && j >= l ? 1 : 0)) & 7u;
    };

    u32x4 a0, a1;
    const rsrc_t xs = mk(p.x, 4096u * KS);
    const unsigned vx = (unsigned)(lane & 15) * KS + (lane >> 4) * 16;
    auto rb_of = [&](int g) { return 4 * third((int)peer, 6 - link, g >> 5) + 2 * (peer & 1); };
    auto xoff = [&](int g) { return (rb_of(g) * 128 + (unsigned)(g & 3) * 32) * KS + (unsigned)((g & 31) >> 2) * 64; };
    const unsigned cv = ld_sys(cntp);
    a0 = ld128(xs, vx, xoff(idx), 0);
    a1 = ld128(xs, vx + 16 * KS, xoff(idx), 0);
    const unsigned call = UNI(cv) + 1u;
    STAMP(0);
    const unsigned par = O_PAR + (call & 1u) * PAR_BYTES;
    const unsigned dbase = par + P_X + (unsigned)me * (2 * MiB);
    const unsigned tbase = call << 12;
    #pragma unroll 1
    for (int r = 0; r < RP - 1; ++r) {
        if (lane == 0 && r == idx % (RP - 1))
            __hip_atomic_fetch_max((unsigned*)(myws + O_TICK + link * 128), tbase + DYN0, __ATOMIC_RELAXED, __HIP_MEMORY_SCOPE_AGENT);
        const int g = idx + r * WL;
        const unsigned so = xoff(g + WL);
        const u32x4 b0 = ld128(xs, vx, so, 0);
        const u32x4 b1 = ld128(xs, vx + 16 * KS, so, 0);
        asm volatile("" ::: "memory");
        const unsigned d = dbase + rb_of(g) * 65536u + (unsigned)(g & 31) * 2048u;
        st128(a0, rem, v16, d, SYS);
        st128(a1, rem, v16 + 1024, d, SYS);
        drain();
        a0 = b0; a1 = b1;
    }
    const unsigned roff = O_RDY + (peer * 64 + lane) * 4;
    const unsigned* rdyp = (const unsigned*)(myws + roff);
    const unsigned rall = lane < 32 && (unsigned)(lane >> 2) == peer ? 0u : call;
    unsigned long long unseen;
    int dead = 0;
    {
        const unsigned rv = ld32(loc, roff);
        asm volatile("" ::: "memory");
        const int g = idx + (RP - 1) * WL;
        const unsigned d = dbase + rb_of(g) * 65536u + (unsigned)(g & 31) * 2048u;
        st128(a0, rem, v16, d, SYS);
        st128(a1, rem, v16 + 1024, d, SYS);
        st1(loc, (unsigned)((const char*)cntp - myws), call, lane);
        unseen = BALLOT(rv < rall);
    }
    const float scale = p.sx[0] * p.sw[0];
    const int xa = isq ? 192 : 160, gxb = xa + 64 * TA, gown = isq ? G_OWNQ : G_OWN;
    auto wanted = [&](int g) {
        return g < xa + 64 * TE ? ((1ull << (2 * TE)) - 1) << 32 | 0x55555555ull
             : g < 2 * xa + 64 * TE2 ? (1ull << (32 + 2 * TE2)) - 1 : ~0ull;
    };
    auto await_pieces = [&](int g) {
        const unsigned long long w = wanted(g);
        if (unseen & w) {
            if (!dead) dead = wave_wait(rdyp, (w >> lane) & 1 ? rall : 0u, myws, call, lane) ? 0 : 1;
            unseen &= ~w;
        }
    };
    await_pieces(0);
    dead = UNI(dead);
    STAMP(1);

    const unsigned prog = O_PROG + (link * 64 + idx) * 4;
    const rsrc_t tick = mk(myws + O_TICK + link * 128, 4);
    auto x_off = [&](unsigned k, unsigned rb, int g) {
        return par + P_X + k * (2 * MiB) + rb * 65536u + (unsigned)(g & 31) * 2048u;
    };
    auto w_off = [&](int V, int h, unsigned& src, unsigned& dst) {
        const unsigned o = (unsigned)tile_of(V, hf) * 131072u + (unsigned)(h & 63) * 2048u;
        src = par + P_WS + (unsigned)(me >> 1) * (2 * MiB) + o;
        dst = par + P_WR + peer * (2 * MiB) + o;
    };
    auto head_off = [&](int g, unsigned& src, unsigned& dst) {
        if (g < xa) src = dst = x_off(peer, 4 * third(me, link, g >> 5) + 2 * hf, g);
        else if (g < gxb) w_off((g - xa) >> 6, g - xa, src, dst);
        else if (g < gxb + xa) src = dst = x_off(peer, 4 * third(me, link, (g - gxb) >> 5) + 2 * hf + 1, g);
        else w_off((g - 2 * xa) >> 6, g - 2 * xa, src, dst);
    };
    const int ghead = isq ? G_REL : G_OWN;
    const unsigned oflag = lane < 32 ? O_ORDY + (((call & 1u) * 8 + peer) * 32 + lane) * 4
                                     : O_PRDY + (peer * 32 + (lane - 32)) * 4;
    const unsigned tflag = lane < 9 ? O_OWN + (peer * 32 + lane) * 4 : lane == 9 ? tmo_off() : NOWHERE;
    unsigned fv = 0;
    int g = idx + RP * WL, gn;
    int nin = 0;
    int held = 1;
    unsigned plan = 0, lack = 0xff, src, cdst, ndst;
    auto read_plan = [&](unsigned tv) {
        if (!nin) {
            plan = __builtin_amdgcn_readlane(tv, 8);
            if ((plan >> 18) == (call & 0x3fffu)) nin = gown + 32 * __builtin_popcount(plan & 0x7ffu);
        }
        lack = (isq ? 0xffu & ~(3u << (me & 6)) : links_to_ranks((plan >> 4) & 0x7fu, me)) & ~(unsigned)BALLOT(lane < 8 && tv >= call);
    };
    head_off(g, src, cdst);
    {
        const rsrc_t remd = mk(remp, dead ? 0u : WS_USED);
        const u32x4 b0 = ld128(remd, v16, src, SYS);
        const u32x4 b1 = ld128(remd, v16 + 1024, src, SYS);
        const unsigned tk = draw(tick, lane == 0);
        drain();
        st1(rem, O_PUSH + ((6 - link) * 64 + idx) * 4, call, lane);
        a0 = b0; a1 = b1;
        gn = (int)(UNI(tk) - tbase);
    }
    for (;;) {
        gn = UNI(gn); dead = UNI(dead); held = UNI(held); nin = UNI(nin);
        plan = UNI(plan); cdst = UNI(cdst); lack = UNI(lack);
        unseen = ((unsigned long long)UNI((unsigned)(unseen >> 32)) << 32) | UNI((unsigned)unseen);
        await_pieces(gn);
        if (gn < ghead) head_off(gn, src, ndst);
        else {
            unsigned long long t0 = 0;
            for (;;) {
                if (dead && !nin) { plan = 0; nin = gown; }
                if (gn < gown) {
                    const int r = gn - G_REL;
                    const unsigned via = third(me, link, r >> 7);
                    src = ndst = par + P_WR + via * (2 * MiB) + (unsigned)tile_of(14 + ((r >> 6) & 1), hf) * 131072u + (unsigned)(r & 63) * 2048u;
                    if (dead || !((lack >> via) & 1u)) break;
                } else if (nin) {
                    if (gn >= nin) break;
                    unsigned m = plan & 0x7ffu;
                    for (int j = (gn - gown) >> 5; j; --j) m &= m - 1;
                    const unsigned bit = __builtin_ctz(m);
                    const unsigned via = (unsigned)(me + 1 + bit - 4) & 7u;
                    src = ndst = bit < 4 ? x_off(peer, 4 * me + bit, gn)
                                         : x_off(via, 4 * me + 2 * (peer & 1) + ((plan >> (bit + 7)) & 1u), gn);
                    if (dead || bit < 4 || !((lack >> via) & 1u)) break;
                }
                if (held) {
                    st128(a0, loc, v16, cdst, SYS);
                    st128(a1, loc, v16 + 1024, cdst, SYS);
                    drain();
                    st1(loc, prog, tbase + (unsigned)gn, lane);
                    held = 0;
                } else {
                    if (!t0) t0 = now();
                    if (now() - t0 > DEADLINE) { set_tmo(myws, call, lane); dead = 1; }
                    __builtin_amdgcn_s_sleep(2);
                }
                const unsigned tv = ld32(loc, tflag);
                drain();
                if (__builtin_amdgcn_readlane(tv, 9) == call) dead = 1;
                read_plan(tv);
            }
            if (nin && gn >= nin) break;
        }
        src = UNI(src); ndst = UNI(ndst);
        const rsrc_t remd = mk(remp, dead ? 0u : WS_USED), locd = mk(myws, dead || !held ? 0u : WS_USED);
        const u32x4 b0 = ld128(remd, v16, src, SYS);
        const u32x4 b1 = ld128(remd, v16 + 1024, src, SYS);
        const unsigned tk = draw(tick, lane == 0);
        TICKSTAMP(gn);
        const bool look = gn + 80 >= ghead && (!nin || lack);
        const bool peek = !look && (unseen & wanted(gn + 60)) != 0;
        fv = ld32(mk(myws, nin && !isq && gn >= nin - 48 ? O_DBG : 0u), oflag);
        const unsigned tn = ld32(mk(myws, look || peek ? O_DBG : 0u), peek ? roff : tflag);
        asm volatile("" ::: "memory");
        st128(a0, locd, v16, cdst, SYS);
        st128(a1, locd, v16 + 1024, cdst, SYS);
        drain();
        st1(mk(myws, dead ? 0u : O_DBG), prog, tbase + (unsigned)gn, lane);
        if (look) read_plan(tn);
        if (peek) unseen = BALLOT(tn < rall);
        a0 = b0; a1 = b1; held = 1;
        g = gn; cdst = ndst;
        gn = (int)(UNI(tk) - tbase);
    }
    {
        const rsrc_t locd = mk(myws, dead || !held ? 0u : WS_USED);
        st128(a0, locd, v16, cdst, SYS);
        st128(a1, locd, v16 + 1024, cdst, SYS);
    }
    if (isq) {
        drain();
        if (!dead) st1(loc, prog, tbase + 4095u, lane);
        STAMP(2); STAMP(3);
        return;
    }

    const int NTOT = nin + NOUT;
    const unsigned ovoff = (unsigned)(4 * (lane >> 4)) * (NCOL * 4) + (lane & 15) * 16;
    const unsigned exb = O_EX + ((call & 1u) * 8 + peer) * (256 * NEX * 4);
    unsigned rmask;
    Item cur, nxt;
    int state = 2;
    int waiting = 0;
    unsigned long long t0 = 0;
    {
        const unsigned long long m = BALLOT(lane < 32 ? (fv >> 8) >= call : fv >= call);
        rmask = (unsigned)m;
        if ((unsigned)(m >> 32) != 0xffffffffu && !dead)
            dead = wave_wait((const unsigned*)(myws + O_PRDY) + peer * 32 + (lane & 31), call, myws, call, lane) ? 0 : 1;
    }
    const unsigned pflag = lane < 32 ? oflag : lane == 32 ? tmo_off() : NOWHERE;
    for (;;) {
        g = UNI(g); gn = UNI(gn); dead = UNI(dead); state = UNI(state); rmask = UNI(rmask);
        cdst = UNI(cdst); waiting = UNI(waiting);
        const bool have = gn < NTOT;
        const unsigned o = (unsigned)(gn - nin) & 1023u;
        const bool can = have && (dead || ((rmask >> (o >> 5)) & 1u));
        {
            const bool go = can && !dead;
            const rsrc_t r0 = mk(remp, go ? WS_USED : 0u), r1 = mk(myws, go ? WS_USED : 0u);
            const unsigned so = par + P_OS + (unsigned)me * (4 * MiB) + o * 4096u;
            const unsigned ps = O_PS + peer * (4 * MiB) + o * 4096u;
            nxt.l0 = ld128(r0, v16, so, SYS);
            nxt.l1 = ld128(r0, v16 + 1024, so, SYS);
            nxt.h4 = __builtin_amdgcn_raw_buffer_load_b64(r0, lane * 8 + 2048, so, SYS);
            nxt.h1 = __builtin_amdgcn_raw_buffer_load_b16(r0, lane * 2 + 2560, so, SYS);
            nxt.ex = __builtin_amdgcn_raw_buffer_load_b32(r1, (lane & (NEX - 1)) * 4, exb + (o >> 2) * (NEX * 4), SYS);
            nxt.p0 = ld128(r1, v16, ps, SYS);
            nxt.p1 = ld128(r1, v16 + 1024, ps, SYS);
            nxt.p2 = ld128(r1, v16 + 2048, ps, SYS);
            nxt.p3 = ld128(r1, v16 + 3072, ps, SYS);
        }
        const bool poll = have && !dead && (~rmask & (7u << (o >> 5))) != 0;
        const unsigned nf = ld32(mk(myws, poll ? O_DBG : 0u), pflag);
        const unsigned tk = draw(tick, can && idx < WLO && lane == 0);

        u32x4 o4[4];
        if (state == 1) {
            const unsigned oc = (unsigned)(g - nin);
            const bool raw = (__builtin_amdgcn_readlane(fv, oc >> 5) >> ((oc >> 2) & 7)) & 1u;
            const unsigned L[8] = {cur.l0.x, cur.l0.y, cur.l0.z, cur.l0.w, cur.l1.x, cur.l1.y, cur.l1.z, cur.l1.w};
            u32x16 pv = {cur.p0.x, cur.p0.y, cur.p0.z, cur.p0.w, cur.p1.x, cur.p1.y, cur.p1.z, cur.p1.w,
                         cur.p2.x, cur.p2.y, cur.p2.z, cur.p2.w, cur.p3.x, cur.p3.y, cur.p3.z, cur.p3.w};
            u32x4 v[4];
            if (raw && !dead) {
                const unsigned so = par + P_OS + (unsigned)me * (4 * MiB) + oc * 4096u;
                const u32x4 ha = ld128(rem, v16 + 2048, so, SYS);
                const u32x4 hb = ld128(rem, v16 + 3072, so, SYS);
                const unsigned H[8] = {ha.x, ha.y, ha.z, ha.w, hb.x, hb.y, hb.z, hb.w};
                #pragma unroll
                for (int r = 0; r < 4; ++r) {
                    v[r].x = perm(H[2 * r], L[2 * r], 0x05040100u);
                    v[r].y = perm(H[2 * r], L[2 * r], 0x07060302u);
                    v[r].z = perm(H[2 * r + 1], L[2 * r + 1], 0x05040100u);
                    v[r].w = perm(H[2 * r + 1], L[2 * r + 1], 0x07060302u);
                }
            } else {
                unsigned em = (unsigned)BALLOT(lane < NEX && (cur.ex >> 21) == (oc & 3u));
                while (em) {
                    const unsigned e = __builtin_amdgcn_readlane(cur.ex, __builtin_ctz(em));
                    em &= em - 1;
                    pv[(e >> 17) & 15u] += (unsigned)lane == ((e >> 11) & 63u) ? e << 21 : 0u;
                }
                const unsigned t = (cur.h1 & 0x0f0fu) | ((cur.h1 & 0xf0f0u) << 12);
                unsigned B[4];
                B[0] = (cur.h4.x & 0x0f0f0f0fu) | ((t << 4) & 0x10101010u);
                B[1] = ((cur.h4.x >> 4) & 0x0f0f0f0fu) | ((t << 3) & 0x10101010u);
                B[2] = (cur.h4.y & 0x0f0f0f0fu) | ((t << 2) & 0x10101010u);
                B[3] = ((cur.h4.y >> 4) & 0x0f0f0f0fu) | ((t << 1) & 0x10101010u);
                #pragma unroll
                for (int r = 0; r < 4; ++r) {
                    v[r].x = perm(B[r], L[2 * r], 0x0c040100u);
                    v[r].y = perm(B[r], L[2 * r], 0x0c050302u);
                    v[r].z = perm(B[r], L[2 * r + 1], 0x0c060100u);
                    v[r].w = perm(B[r], L[2 * r + 1], 0x0c070302u);
                }
            }
            #pragma unroll
            for (int r = 0; r < 4; ++r) {
                f32x4 f;
                f.x = (float)(int)(v[r].x + pv[4 * r]) * scale;     f.y = (float)(int)(v[r].y + pv[4 * r + 1]) * scale;
                f.z = (float)(int)(v[r].z + pv[4 * r + 2]) * scale; f.w = (float)(int)(v[r].w + pv[4 * r + 3]) * scale;
                o4[r] = __builtin_bit_cast(u32x4, f);
                if (dead) o4[r] = mk4(0x7fc00000u, 0x7fc00000u, 0x7fc00000u, 0x7fc00000u);
            }
        }
        {
            const rsrc_t out = mk(p.out, state == 1 ? OUT_BYTES : 0u);
            #pragma unroll
            for (int r = 0; r < 4; ++r) st128(o4[r], out, ovoff + r * (NCOL * 4), cdst, OUT_AUX);
        }
        if (!have && state != 2) break;
        asm volatile("s_waitcnt vmcnt(4)" ::: "memory");
        __builtin_amdgcn_s_waitcnt(0x0F74);
        if (state == 2) {
            asm volatile("s_waitcnt vmcnt(0)" ::: "memory");
            if (!dead) st1(loc, prog, tbase + 4095u, lane);
            STAMP(2);
        }
        if (!have) break;
        if (poll) {
            fv = nf;
            rmask = (unsigned)BALLOT((nf >> 8) >= call);
            if (__builtin_amdgcn_readlane(nf, 32) == call) dead = 1;
        }
        if (can) {
            cur = nxt; g = gn; state = 1;
            const unsigned n = o >> 5, w8 = (o >> 2) & 7, mf = o & 3;
            const unsigned cc = n < 2 * TA ? (n >= TA ? 1 : 0) : n & 1;
            const unsigned V = n < 2 * TA ? n - cc * TA : n >> 1;
            cdst = (((2 * (peer & 1) + cc) * 128 + (w8 >> 2) * 64 + mf * 16) * NCOL
                    + (peer >> 1) * 4096 + (unsigned)tile_of((int)V, (int)(peer & 1)) * 256 + (w8 & 3) * 64) * 4u;
            gn = idx < WLO ? (int)(UNI(tk) - tbase) : NTOT;
            waiting = 0;
        } else {
            if (!waiting) { waiting = 1; t0 = now(); }
            else {
                if (now() - t0 > DEADLINE) { set_tmo(myws, call, lane); dead = 1; }
                __builtin_amdgcn_s_sleep(4);
            }
            state = 0;
        }
    }
    STAMP(3);
}

__device__ __forceinline__ unsigned prep(const Params& p, char* myws, const unsigned* cntp, int id, int tid, int lane, int wv) {
    const int me = p.rank;
    unsigned call;
    {
        const int tw = ((id - 32) >> 1) & 15;
        const unsigned long long hold = id < 32 ? ((id & 1) ? PD_X : 0) : (tw >= TE ? PD_0 + (tw - TE) * PD_W : 0);
        const unsigned long long t0 = now();
        while (now() - t0 < hold) __builtin_amdgcn_s_sleep(4);
    }
    const rsrc_t loc = mk(myws, WS_USED);
    if (id < 32) {
        const rsrc_t xs = mk(p.x, 4096u * KS);
        const unsigned voff = (unsigned)(lane & 15) * KS + (lane >> 4) * 16;
        const unsigned soff = (unsigned)(id * 128 + wv * 16) * KS;
        u32x4 v[8];
        #pragma unroll
        for (int i = 0; i < 8; ++i) v[i] = ld128(xs, voff + i * 64, soff, 0);
        call = UNI(ld_sys(cntp)) + 1u;
        const unsigned dst = O_PAR + (call & 1u) * PAR_BYTES + P_X + (unsigned)me * (2 * MiB) + (unsigned)id * 65536u + wv * 1024u;
        #pragma unroll
        for (int i = 0; i < 8; ++i) st128(v[i], loc, lane * 16, dst + i * 8192u, SYS);
    } else {
        const int pid = id - 32, jt = pid >> 1, h = pid & 1;
        const rsrc_t wsr = mk(p.w, (unsigned)KS * NCOL);
        const int wn = wv & 3, kq0 = h * 4 + (wv >> 2) * 2;
        const unsigned voff = (unsigned)(16 * (lane >> 4)) * NCOL + (lane & 15) * 4;
        unsigned r[2][16];
        #pragma unroll
        for (int b = 0; b < 2; ++b)
            #pragma unroll
            for (int j = 0; j < 16; ++j)
                r[b][j] = __builtin_amdgcn_raw_buffer_load_b32(wsr, voff, (unsigned)((kq0 + b) * 64 + j) * NCOL + jt * 256 + wn * 64, 0);
        call = UNI(ld_sys(cntp)) + 1u;
        const unsigned par = O_PAR + (call & 1u) * PAR_BYTES;
        #pragma unroll
        for (int b = 0; b < 2; ++b) {
            unsigned o[4][4];
            #pragma unroll
            for (int q = 0; q < 4; ++q) {
                const unsigned t0 = perm(r[b][4 * q + 1], r[b][4 * q], 0x05010400u);
                const unsigned t1 = perm(r[b][4 * q + 1], r[b][4 * q], 0x07030602u);
                const unsigned t2 = perm(r[b][4 * q + 3], r[b][4 * q + 2], 0x05010400u);
                const unsigned t3 = perm(r[b][4 * q + 3], r[b][4 * q + 2], 0x07030602u);
                o[0][q] = perm(t2, t0, 0x05040100u);
                o[1][q] = perm(t2, t0, 0x07060302u);
                o[2][q] = perm(t3, t1, 0x05040100u);
                o[3][q] = perm(t3, t1, 0x07060302u);
            }
            const unsigned dst = par + P_WS + (unsigned)jt * 131072u + (unsigned)((kq0 + b) * 16 + wn * 4) * 1024u;
            #pragma unroll
            for (int i = 0; i < 4; ++i)
                st128(mk4(o[i][0], o[i][1], o[i][2], o[i][3]), loc, lane * 16 + i * 1024, dst, SYS);
        }
    }
    drain();
    __syncthreads();
    if (id < 32) {
        if (tid < NR) st_sys((unsigned*)(p.ws[tid] + O_RDY) + me * 64 + id, call);
    } else {
        const int pid = id - 32;
        if (tid < 2) st_sys((unsigned*)(p.ws[(pid >> 5) * 2 + tid] + O_RDY) + me * 64 + 32 + (pid & 31), call);
        if (tid == 2) st_sys((unsigned*)(myws + O_LRW) + pid, call);
    }
    return call;
}

}

__global__ __launch_bounds__(NTHR, 2) void gemm_rs_fused(Params p) {
    __shared__ __attribute__((aligned(16))) char lds[LDS_EX + 8 * NEX * 4];
    const int tid = threadIdx.x, lane = tid & 63;
    const int wv = UNI(tid >> 6);
    const int b = blockIdx.x, me = p.rank;
    char* myws = p.ws[me];
    unsigned* cntp = (unsigned*)(myws + O_CNT) + b * 8 + wv;

    if (b < NPULL) {
        const int s = wv * NPULL + b;
        if (s < NLW) link_wave(p, myws, cntp, s, lane, wv);
        else if (b == NPULL - 1 && wv == 7) {
            const unsigned call = UNI(ld_sys(cntp)) + 1u;
            if (lane == 0) st_sys(cntp, call);
            monitor(p, myws, call, lane);
        }
        return;
    }

    const int cb = b - NPULL;
    STAMP(0);
    unsigned call;
    if (cb < 160) call = prep(p, myws, cntp, cb, tid, lane, wv);
    else call = UNI(ld_sys(cntp)) + 1u;
    if (lane == 0) st_sys(cntp, call);
    STAMP(1);

    const int ci = (b & 7) * NCU + (cb >> 3);
    const int hf = me & 1, lp = hf ? 6 : 0, lb = 1 - hf;
    const unsigned par = O_PAR + (call & 1u) * PAR_BYTES;
    const rsrc_t loc = mk(myws, WS_USED);
    ldsw_t misc = (ldsw_t)(lds + LDS_MISC);
    const int wm = wv >> 2, wn = wv & 3;

    for (int step = 0; step < 4; ++step) {
        int task, di, cc, V;
        if (step < 2) {
            const int j = step == 0 ? ci - NP0 : (NCOMP - NP0) + (NCOMP - 1 - ci);
            if (j < 0 || j >= 192) continue;
            const int n = j / 6;
            task = 0; di = j - 6 * n;
            cc = n < 2 * TA ? (n >= TA ? 1 : 0) : n & 1;
            V = n < 2 * TA ? n - cc * TA : n >> 1;
        } else {
            int i = ci;
            if (step == 3) {
                if (tid == 0) misc[1] = __hip_atomic_fetch_add((unsigned*)(myws + O_TICK + 7 * 128), 1u, __ATOMIC_RELAXED, __HIP_MEMORY_SCOPE_AGENT);
                __syncthreads();
                i = NCOMP + (int)(UNI(misc[1]) - (call - 1u) * NCOMP);
                if (i >= 256) break;
            }
            if (i < 192) {
                const int r = i % 24;
                task = 1; V = 2 * (i / 24) + (r & 1); cc = r >= 12 ? 1 : 0; di = (r >> 1) - 6 * cc;
            } else {
                task = 2; di = 0; V = (i - 192) >> 2; cc = i & 3;
            }
        }
        const int d = (me + 1 + lb + di) & 7;
        const int rb = task == 0 ? 4 * me + 2 * (d & 1) + cc : task == 1 ? 4 * d + 2 * hf + cc : 4 * me + cc;
        const int pos = tile_pos(cc, V);
        const int T = tile_of(V, task == 0 ? d & 1 : hf);
        const int jt = (task == 0 ? d >> 1 : me >> 1) * 16 + T;

        i32x4 acc[4][4];
        {
            const int a0 = task == 0 ? -BIAS : task == 1 ? BIAS : 0;
            #pragma unroll
            for (int m = 0; m < 4; ++m)
                #pragma unroll
                for (int n = 0; n < 4; ++n) acc[m][n] = (i32x4){a0, a0, a0, a0};
        }
        unsigned todo = task == 0 ? 1u << me : task == 1 ? 0xffu & ~(1u << d) : 0xffu;
        bool ok = true;
        while (todo) {
            if (wv == 0) {
                const int l = lane & 15;
                const bool direct = V < 14 || l == lp;
                const unsigned* ptr = (const unsigned*)(myws + tmo_off());
                unsigned need = call;
                if ((todo >> me) & 1u) {
                    if (lane == 8 || lane == 9) ptr = (const unsigned*)(myws + O_LRW) + jt * 2 + (lane - 8);
                    if (lane == 10) ptr = (const unsigned*)(myws + O_RDY) + me * 64 + rb;
                }
                if (task && lane < 7) {
                    const int xa = l == lp ? 192 : 160;
                    const int xn = task == 2 ? G_ALL : cc ? 2 * xa + 64 * TA : xa;
                    ptr = (const unsigned*)(myws + O_G + (b & 7) * 128) + l;
                    need = (call << 12) + (unsigned)max(xn, direct ? w_done(xa, V) : 0);
                }
                if (task && lane >= 16 && lane < 23) {
                    ptr = (const unsigned*)(myws + O_G + (b & 7) * 128) + lp;
                    need = direct ? 0u : (call << 12) + (unsigned)(G_REL + 128 * (l - lb) + 64 * (V - 13));
                }
                const unsigned long long t0 = now();
                unsigned got;
                for (;;) {
                    const unsigned val = ld_sys(ptr);
                    unsigned m = (unsigned)BALLOT(val >= need);
                    if (__builtin_amdgcn_readlane(val, 7) == call) { got = 0; break; }
                    unsigned lm = m & (m >> 16) & 0x7fu;
                    if (task == 2 && ((m | (1u << lp)) & 0x7fu) != 0x7fu) lm = 0;
                    got = links_to_ranks(lm | (((m >> 8) & 7u) == 7u ? 0x80u : 0u), me) & todo;
                    if (got) break;
                    if (now() - t0 > DEADLINE) { set_tmo(myws, call, lane); break; }
                    __builtin_amdgcn_s_sleep(CSLEEP);
                }
                if (lane == 0) misc[0] = got;
            }
            __syncthreads();
            const unsigned got = UNI(misc[0]);
            if (!got) { ok = false; break; }
            todo &= ~got;
            unsigned klist = 0, rest = got;
            int nst = 0;
            while (rest) { klist |= (unsigned)__builtin_ctz(rest) << nst; rest &= rest - 1; nst += 4; }
            const unsigned va = wv * 2048 + lane * 16, vb = wv * 4096 + lane * 16;
            auto issue = [&](int s, int slot) {
                const int k = (klist >> (s & ~3)) & 15;
                const unsigned abase = par + P_X + (unsigned)k * (2 * MiB) + (unsigned)rb * 65536u + (s & 3) * 16384u;
                const unsigned bbase = par + (k == me ? P_WS + (unsigned)jt * 131072u : P_WR + (unsigned)(k * 16 + T) * 131072u) + (s & 3) * 32768u;
                char* la = lds + slot * STAGE + wv * 2048;
                char* lb_ = lds + slot * STAGE + 16384 + wv * 4096;
                __builtin_amdgcn_raw_ptr_buffer_load_lds(loc, (__attribute__((address_space(3))) void*)la, 16, va, abase, 0, SYS);
                __builtin_amdgcn_raw_ptr_buffer_load_lds(loc, (__attribute__((address_space(3))) void*)la, 16, va, abase, 1024, SYS);
                __builtin_amdgcn_raw_ptr_buffer_load_lds(loc, (__attribute__((address_space(3))) void*)lb_, 16, vb, bbase, 0, SYS);
                __builtin_amdgcn_raw_ptr_buffer_load_lds(loc, (__attribute__((address_space(3))) void*)lb_, 16, vb, bbase, 1024, SYS);
                __builtin_amdgcn_raw_ptr_buffer_load_lds(loc, (__attribute__((address_space(3))) void*)lb_, 16, vb, bbase, 2048, SYS);
                __builtin_amdgcn_raw_ptr_buffer_load_lds(loc, (__attribute__((address_space(3))) void*)lb_, 16, vb, bbase, 3072, SYS);
            };
            auto compute = [&](int slot) {
                const char* A = lds + slot * STAGE + wm * 4096 + lane * 16;
                const char* B = lds + slot * STAGE + 16384 + wn * 4096 + lane * 16;
                i32x4 af[2][4], bf[2][4];
                #pragma unroll
                for (int kb = 0; kb < 2; ++kb) {
                    af[kb][0] = *(const i32x4*)(A + kb * 8192);
                    #pragma unroll
                    for (int i = 0; i < 4; ++i) bf[kb][i] = *(const i32x4*)(B + kb * 16384 + i * 1024);
                    #pragma unroll
                    for (int i = 1; i < 4; ++i) af[kb][i] = *(const i32x4*)(A + kb * 8192 + i * 1024);
                }
                __builtin_amdgcn_sched_barrier(0);
                #pragma unroll
                for (int kb = 0; kb < 2; ++kb)
                    #pragma unroll
                    for (int m = 0; m < 4; ++m)
                        #pragma unroll
                        for (int n = 0; n < 4; ++n)
                            acc[m][n] = __builtin_amdgcn_mfma_i32_16x16x64_i8(af[kb][m], bf[kb][n], acc[m][n], 0, 0, 0);
                __builtin_amdgcn_sched_barrier(0);
            };
            issue(0, 0);
            issue(1, 1);
            int slot = 0;
            for (int s = 0; s < nst; ++s) {
                if (s + 1 < nst) asm volatile("s_waitcnt vmcnt(6) lgkmcnt(0)" ::: "memory");
                else asm volatile("s_waitcnt vmcnt(0) lgkmcnt(0)" ::: "memory");
                __builtin_amdgcn_s_barrier();
                if (s + 2 < nst) issue(s + 2, slot == 0 ? 2 : slot - 1);
                compute(slot);
                slot = slot == 2 ? 0 : slot + 1;
            }
        }

        if (task == 0) {
            if (ok) {
                const unsigned dst = O_PS + (unsigned)d * (4 * MiB) + (unsigned)(pos * 8 + wv) * 16384u;
                #pragma unroll
                for (int m = 0; m < 4; ++m)
                    #pragma unroll
                    for (int r = 0; r < 4; ++r)
                        st128(mk4((unsigned)acc[m][0][r], (unsigned)acc[m][1][r], (unsigned)acc[m][2][r], (unsigned)acc[m][3][r]),
                              loc, lane * 16 + r * 1024, dst + m * 4096u, SYS);
            }
            drain();
            __syncthreads();
            if (tid == 0 && ok) st_sys((unsigned*)(myws + O_PRDY) + d * 32 + pos, call);
        } else if (task == 1) {
            ldsw_t exl = (ldsw_t)(lds + LDS_EX) + wv * NEX;
            if (lane < NEX) exl[lane] = 0xffffffffu;
            unsigned nex = 0;
            #pragma unroll
            for (int m = 0; m < 4; ++m)
                #pragma unroll
                for (int r = 0; r < 4; ++r) {
                    const unsigned any = (unsigned)(acc[m][0][r] | acc[m][1][r]) | (unsigned)(acc[m][2][r] | acc[m][3][r]);
                    if (BALLOT((any >> 21) != 0) != 0) {
                        #pragma unroll
                        for (int n = 0; n < 4; ++n) {
                            const unsigned hi = (unsigned)acc[m][n][r] >> 21;
                            const unsigned long long bm = BALLOT(hi != 0);
                            const unsigned at = nex + __builtin_amdgcn_mbcnt_hi((unsigned)(bm >> 32), __builtin_amdgcn_mbcnt_lo((unsigned)bm, 0u));
                            if (hi != 0 && at < NEX) exl[at] = (unsigned)(m << 21 | (r * 4 + n) << 17) | ((unsigned)lane << 11) | hi;
                            nex += __builtin_popcountll(bm);
                        }
                    }
                }
            const bool raw = nex > NEX;
            if (ok) {
                const unsigned e = exl[lane & (NEX - 1)];
                __builtin_amdgcn_raw_buffer_store_b32(e, mk(p.ws[d], WS_USED),
                    lane < NEX ? O_EX + ((((call & 1u) * 8 + me) * 32 + pos) * 8 + wv) * (NEX * 4) + lane * 4 : NOWHERE, 0, SYS);
                const unsigned dst = par + P_OS + (unsigned)d * (4 * MiB) + (unsigned)(pos * 8 + wv) * 16384u;
                #pragma unroll
                for (int m = 0; m < 4; ++m) {
                    unsigned L[8], H[8];
                    #pragma unroll
                    for (int r = 0; r < 4; ++r) {
                        L[2 * r] = perm(acc[m][1][r], acc[m][0][r], 0x05040100u);
                        L[2 * r + 1] = perm(acc[m][3][r], acc[m][2][r], 0x05040100u);
                        H[2 * r] = perm(acc[m][1][r], acc[m][0][r], 0x07060302u);
                        H[2 * r + 1] = perm(acc[m][3][r], acc[m][2][r], 0x07060302u);
                    }
                    st128(mk4(L[0], L[1], L[2], L[3]), loc, lane * 16, dst + m * 4096u, SYS);
                    st128(mk4(L[4], L[5], L[6], L[7]), loc, lane * 16 + 1024, dst + m * 4096u, SYS);
                    if (raw) {
                        st128(mk4(H[0], H[1], H[2], H[3]), loc, lane * 16 + 2048, dst + m * 4096u, SYS);
                        st128(mk4(H[4], H[5], H[6], H[7]), loc, lane * 16 + 3072, dst + m * 4096u, SYS);
                    } else {
                        unsigned B[4];
                        #pragma unroll
                        for (int r = 0; r < 4; ++r) B[r] = perm(H[2 * r + 1], H[2 * r], 0x06040200u);
                        const u32x2 h4 = {bfi(0x0f0f0f0fu, B[0], B[1] << 4), bfi(0x0f0f0f0fu, B[2], B[3] << 4)};
                        const unsigned t = ((B[0] >> 4) & 0x01010101u) | ((B[1] >> 3) & 0x02020202u)
                                         | ((B[2] >> 2) & 0x04040404u) | ((B[3] >> 1) & 0x08080808u);
                        __builtin_amdgcn_raw_buffer_store_b64(h4, loc, lane * 8 + 2048 + dst + m * 4096u, 0, SYS);
                        __builtin_amdgcn_raw_buffer_store_b16((unsigned short)(t | (t >> 12)), loc, lane * 2 + 2560 + dst + m * 4096u, 0, SYS);
                    }
                }
            }
            if (lane == 0) misc[4 + wv] = raw ? 1u : 0u;
            drain();
            __syncthreads();
            if (tid == 0 && ok) {
                unsigned mask = 0;
                #pragma unroll
                for (int i = 0; i < 8; ++i) mask |= misc[4 + i] << i;
                st_sys((unsigned*)(p.ws[d] + O_ORDY) + ((call & 1u) * 8 + me) * 32 + pos, (call << 8) | mask);
            }
        } else {
            const rsrc_t out = mk(p.out, OUT_BYTES);
            const float scale = p.sx[0] * p.sw[0];
            const unsigned ovoff = (unsigned)(4 * (lane >> 4)) * (NCOL * 4) + (lane & 15) * 16;
            const unsigned dst = ((unsigned)(cc * 128 + wm * 64) * NCOL + jt * 256 + wn * 64) * 4u;
            #pragma unroll
            for (int m = 0; m < 4; ++m)
                #pragma unroll
                for (int r = 0; r < 4; ++r) {
                    f32x4 f;
                    f.x = (float)acc[m][0][r] * scale; f.y = (float)acc[m][1][r] * scale;
                    f.z = (float)acc[m][2][r] * scale; f.w = (float)acc[m][3][r] * scale;
                    u32x4 o4 = __builtin_bit_cast(u32x4, f);
                    if (!ok) o4 = mk4(0x7fc00000u, 0x7fc00000u, 0x7fc00000u, 0x7fc00000u);
                    st128(o4, out, ovoff + r * (NCOL * 4), dst + (unsigned)(m * 16) * (NCOL * 4), OUT_AUX);
                }
        }
        STAMP(step == 3 ? 7 : 3 + 2 * task);
    }
}

extern "C" void kernel_launch(int rank, int nranks, ncclComm_t comm, hipStream_t stream,
                              void* const* d_in, const int* in_sizes, int n_in,
                              void* d_out, int out_size, const int* in_dtypes, int out_dtype,
                              void* const* d_out_all, void* const* d_ws_all, size_t ws_bytes) {
    (void)comm; (void)in_sizes; (void)in_dtypes; (void)out_dtype; (void)d_out_all;
    if (nranks != NR || (unsigned)rank >= (unsigned)NR || n_in < 4 || ws_bytes < WS_USED ||
        (size_t)out_size * 4 != OUT_BYTES) {
        fprintf(stderr, "kernel_launch: unexpected problem shape\n");
        return;
    }
    Params p;
    for (int i = 0; i < NR; ++i) p.ws[i] = (char*)d_ws_all[i];
    p.x = (const char*)d_in[0];
    p.w = (const char*)d_in[1];
    p.sx = (const float*)d_in[2];
    p.sw = (const float*)d_in[3];
    p.out = (float*)d_out;
    p.rank = rank;
    p.pad = 0;
    hipLaunchKernelGGL(gemm_rs_fused, dim3(NBLK), dim3(NTHR), 0, stream, p);
    HIP_CHECK(hipGetLastError());
}
```

```cpp
#include <hip/hip_runtime.h>
#include <rccl/rccl.h>
#include <cstdio>
#include <cstdint>

#define HIP_CHECK(expr) do { hipError_t _e = (expr); if (_e != hipSuccess) { \
    fprintf(stderr, "HIP error %s at %s:%d\n", hipGetErrorString(_e), __FILE__, __LINE__); \
    return; } } while (0)

typedef int i32x4 __attribute__((ext_vector_type(4)));
typedef unsigned u32x4 __attribute__((ext_vector_type(4)));
typedef unsigned u32x2 __attribute__((ext_vector_type(2)));
typedef unsigned u32x16 __attribute__((ext_vector_type(16)));
typedef float f32x4 __attribute__((ext_vector_type(4)));
typedef __amdgpu_buffer_rsrc_t rsrc_t;
typedef volatile __attribute__((address_space(3))) unsigned* ldsw_t;
#define UNI(x) __builtin_amdgcn_readfirstlane(x)
#define BALLOT(c) __builtin_amdgcn_ballot_w64(c)

namespace {

constexpr int NR = 8;
constexpr int NCOL = 16384;
constexpr int KS = 512;
constexpr unsigned MiB = 1u << 20;

constexpr int NBLK = 256, NTHR = 512;
constexpr int NPULL = 80;
constexpr int WL = 40;
constexpr int WLO = 30;
constexpr int NLW = 7 * WL;
constexpr int NCOMP = NBLK - NPULL, NCU = NCOMP / 8;
constexpr int NP0 = 48;
static_assert(NLW <= 7 * NPULL && NCOMP >= 160 && NCOMP <= 192 && 2 * (NCOMP - NP0) >= 192 && WLO <= WL && WL <= 64, "geometry");

constexpr int RP = 4, TA = 2, VS = 8;
constexpr int TE = 2, TE2 = 4;
constexpr int HOLD0 = 120, HOLD1 = 30;
constexpr int LEAD = 160;
constexpr int RELAY_TH = 36, QBIAS = 64;
constexpr int G_OWN = 320 + 64 * 14, G_REL = 384 + 64 * 16, G_OWNQ = G_REL + 768;
constexpr int G_ALL = 4095;
constexpr int NOUT = 1024;
constexpr int DYN0 = (RP + 1) * WL;
static_assert(RP >= 2 && RP * WL <= 160 && TE <= TA && TA <= TE2 && TE2 <= VS && VS <= 12, "lists");
__host__ __device__ constexpr int tile_pos(int cc, int V) { return V < TA ? cc * TA + V : 2 * V + cc; }
__host__ __device__ constexpr int tile_of(int V, int q) {
    return V < VS ? V : V < VS + 2 ? 12 + 2 * (V - VS) + q : V < 14 ? V - 2 : 13 + 2 * (V - 14) - q;
}
__host__ __device__ constexpr int w_done(int xa, int V) { return (V < TA ? xa : 2 * xa) + 64 * (V + 1); }

constexpr unsigned O_CNT = 0;
constexpr unsigned O_TMO = 8192;
constexpr unsigned O_TICK = 12288;
constexpr unsigned O_OWN = 14336;
constexpr unsigned O_RDY = 16384;
constexpr unsigned O_LRW = 20480;
constexpr unsigned O_PROG = 24576;
constexpr unsigned O_PUSH = 26624;
constexpr unsigned O_G = 28672;
constexpr unsigned O_PRDY = 32768;
constexpr unsigned O_ORDY = 36864;
constexpr unsigned O_END = 65536;
constexpr unsigned O_EX = 512 * 1024;
constexpr unsigned O_PS = 1 * MiB;
constexpr unsigned O_PAR = 33 * MiB;
constexpr unsigned P_X = 0;
constexpr unsigned P_WS = 16 * MiB;
constexpr unsigned P_WR = 24 * MiB;
constexpr unsigned P_OS = 40 * MiB;
constexpr unsigned PAR_BYTES = 72 * MiB;
constexpr unsigned WS_USED = O_PAR + 2 * PAR_BYTES;
constexpr unsigned OUT_BYTES = 512u * NCOL * 4u;
constexpr unsigned NOWHERE = 0xf0000000u;

constexpr unsigned long long DEADLINE = 40000000ull;
constexpr int BIAS = 1 << 20;
constexpr int NEX = 16;
constexpr int SYS = 17;
constexpr int OUT_AUX = 16;

constexpr int STAGE = 49152;
constexpr int LDS_MISC = 3 * STAGE;
constexpr int LDS_EX = LDS_MISC + 64;

struct Params {
    char* ws[NR];
    const char* x;
    const char* w;
    const float* sx;
    const float* sw;
    float* out;
    int rank;
    int pad;
};

__device__ __forceinline__ rsrc_t mk(const void* p, unsigned bytes) {
    return __builtin_amdgcn_make_buffer_rsrc((void*)p, 0, bytes, 0x00020000);
}
#define st128(v, r, voff, soff, aux) __builtin_amdgcn_raw_buffer_store_b128(v, r, (voff) + (soff), 0, aux)
#define ld128(r, voff, soff, aux) __builtin_amdgcn_raw_buffer_load_b128(r, voff, soff, aux)
#define ld32(r, voff) __builtin_amdgcn_raw_buffer_load_b32(r, voff, 0, SYS)
__device__ __forceinline__ u32x4 mk4(unsigned a, unsigned b, unsigned c, unsigned d) { return (u32x4){a, b, c, d}; }
__device__ __forceinline__ void st1(const rsrc_t r, unsigned off, unsigned v, int lane) {
    __builtin_amdgcn_raw_buffer_store_b32(v, r, lane == 0 ? off : NOWHERE, 0, SYS);
}
__device__ __forceinline__ unsigned tmo_off() { return O_TMO + (blockIdx.x & 31) * 128; }
__device__ __forceinline__ void set_tmo(char* myws, unsigned call, int lane) {
    __builtin_amdgcn_raw_buffer_store_b32(call, mk(myws, O_END), lane < 32 ? O_TMO + lane * 128 : NOWHERE, 0, SYS);
}
__device__ __forceinline__ unsigned ld_sys(const unsigned* p) {
    return __hip_atomic_load(p, __ATOMIC_RELAXED, __HIP_MEMORY_SCOPE_SYSTEM);
}
__device__ __forceinline__ void st_sys(unsigned* p, unsigned v) {
    __hip_atomic_store(p, v, __ATOMIC_RELAXED, __HIP_MEMORY_SCOPE_SYSTEM);
}
__device__ __forceinline__ void drain() {
    asm volatile("s_waitcnt vmcnt(0)" ::: "memory");
    __builtin_amdgcn_s_waitcnt(0x0F70);
}
__device__ __forceinline__ unsigned long long now() { return __builtin_amdgcn_s_memrealtime(); }
__device__ __forceinline__ unsigned perm(unsigned hi, unsigned lo, unsigned sel) {
    return __builtin_amdgcn_perm(hi, lo, sel);
}
__device__ __forceinline__ unsigned bfi(unsigned m, unsigned a, unsigned b) { return (a & m) | (b & ~m); }
__device__ __forceinline__ unsigned links_to_ranks(unsigned m, int me) { return ((m | (m << 8)) >> (7 - me)) & 0xffu; }

__device__ __forceinline__ bool wave_wait(const unsigned* ptr, unsigned need, char* myws, unsigned call, int lane) {
    const unsigned* tmo = (const unsigned*)(myws + tmo_off());
    const unsigned long long t0 = now();
    for (;;) {
        const unsigned v = ld_sys(ptr);
        const unsigned t = ld_sys(tmo);
        if (UNI(t) == call) return false;
        if (BALLOT(v < need) == 0) return true;
        if (now() - t0 > DEADLINE) {
            set_tmo(myws, call, lane);
            return false;
        }
        __builtin_amdgcn_s_sleep(1);
    }
}

__device__ __forceinline__ void monitor(const Params& p, char* myws, unsigned call, int lane) {
    const unsigned* tmo = (const unsigned*)(myws + tmo_off());
    const unsigned* prog = (const unsigned*)(myws + O_PROG);
    const int l7 = lane & 7, me = p.rank, lp = (me & 1) ? 6 : 0;
    unsigned* ownp = (unsigned*)(p.ws[lane >> 3] + O_OWN) + me * 32 + ((me + 1 + l7) & 7);
    const int xa = l7 == lp ? 192 : 160;
    const unsigned tell = (lane >> 3) == (me ^ 1) ? w_done(160, VS + 1) : 2 * xa + 64 * TA;
    const unsigned gown = l7 == lp ? G_OWNQ : G_OWN;
    const unsigned base = call << 12;
    unsigned last = 0, nin = 4095, mine, open = 0x7f;
    bool pushed = false;
    bool planned = false, told = l7 == 7 || (lane >> 3) == me || (l7 == lp && (lane >> 3) == (me ^ 1));
    const unsigned long long t0 = now();
    for (;;) {
        asm volatile("" ::: "memory");
        const unsigned tk = ld32(mk(myws, planned ? 0u : O_END), O_TICK + l7 * 128);
        if (open == 1u << lp && pushed) {
            const unsigned v = ld_sys(prog + lp * 64 + lane);
            mine = lane >= WL ? 4095u : v >= base ? v - base : 0u;
            #pragma unroll
            for (int o = 32; o; o >>= 1) mine = min(mine, (unsigned)__shfl_xor((int)mine, o));
            if (l7 != lp) mine = 4095;
        } else {
            unsigned g[7];
            bool all = true;
            #pragma unroll
            for (int l = 0; l < 7; ++l) {
                const unsigned v = ld_sys(prog + l * 64 + lane);
                const unsigned q = ld32(mk(myws, pushed ? 0u : O_END), O_PUSH + (l * 64 + lane) * 4);
                const bool in = pushed || q >= call || lane >= WL;
                all = all && in;
                g[l] = lane >= WL ? 4095u : (v >= base && in) ? v - base : 0u;
            }
            pushed = BALLOT(!all) == 0;
            #pragma unroll
            for (int o = 32; o; o >>= 1)
                #pragma unroll
                for (int l = 0; l < 7; ++l) g[l] = min(g[l], (unsigned)__shfl_xor((int)g[l], o));
            mine = 4095;
            #pragma unroll
            for (int l = 0; l < 7; ++l) if (l7 == l) mine = g[l];
        }
        if (!planned) {
            int r = l7 < 7 && tk >= base ? (int)(tk - base) : 0;
            if (BALLOT(l7 != lp && r >= G_OWN - LEAD) != 0) {
                planned = true;
                r = -r - (l7 == lp ? QBIAS : 0);
                unsigned keep = 15, recv = 0;
                #pragma unroll
                for (int j = 1; j < 7; ++j) {
                    const int to = l7 + j >= 7 ? l7 + j - 7 : l7 + j, from = l7 - j < 0 ? l7 - j + 7 : l7 - j;
                    const unsigned two = (keep >> (2 * ((me + 1 + to) & 1))) & 3u;
                    const int rto = __shfl(r, to);
                    const int give = to != lp && two && r - rto > RELAY_TH ? (two == 3 ? 3 : 1) : 0;
                    const int got = __shfl(give, from);
                    r += 32 * ((got & 1) - (give & 1));
                    if (give) keep &= ~(1u << (2 * ((me + 1 + to) & 1) + (give >> 1)));
                    recv |= (unsigned)(got & 1) << from | (unsigned)(got >> 1) << (7 + from);
                }
                nin = gown + 32 * (__builtin_popcount(keep) + __builtin_popcount(recv & 0x7fu));
                if (lane < 7) st_sys((unsigned*)(myws + O_OWN) + ((me + 1 + lane) & 7) * 32 + 8, (call << 18) | (recv << 4) | keep);
            }
        }
        mine = mine >= nin ? (unsigned)G_ALL : min(mine, gown);
        if (mine > last) {
            last = mine;
            if (l7 < 7) st_sys((unsigned*)(myws + O_G + (lane >> 3) * 128) + l7, base + mine);
        }
        if (!told && mine >= tell) {
            told = true;
            st_sys(ownp, call);
        }
        open = (unsigned)BALLOT(mine < G_ALL) & 0x7fu;
        if (!open) return;
        if (now() - t0 > 100000) {
            if (UNI(ld_sys(tmo)) == call) return;
            if (now() - t0 > DEADLINE) { set_tmo(myws, call, lane); return; }
            __builtin_amdgcn_s_sleep(1);
        }
    }
}

struct Item { u32x4 l0, l1, p0, p1, p2, p3; u32x2 h4; unsigned h1, ex; };

__device__ __forceinline__ unsigned draw(const rsrc_t tick, bool sel) {
    int t = 1, z = 0;
    asm volatile("" : "+v"(z));
    if (sel) t = __builtin_amdgcn_raw_ptr_buffer_atomic_add_i32(t, tick, z, 0, 0);
    return (unsigned)t;
}

__device__ __forceinline__ void link_wave(const Params& p, char* myws, const unsigned* cntp, int s, int lane) {
    const int me = p.rank, hf = me & 1;
    const int link = s % 7, idx = s / 7;
    const unsigned peer = (unsigned)(me + 1 + link) & 7u;
    const bool isq = peer == (unsigned)(me ^ 1);
    const char* remp = p.ws[peer];
    const rsrc_t loc = mk(myws, WS_USED), rem = mk(remp, WS_USED);
    const unsigned v16 = lane * 16;
    auto third = [&](int r, int l, int i) {
        const int j = i + 1 - (r & 1);
        return (unsigned)(r + 1 + j + (!isq && j >= l ? 1 : 0)) & 7u;
    };

    u32x4 a0, a1;
    const rsrc_t xs = mk(p.x, 4096u * KS);
    const unsigned vx = (unsigned)(lane & 15) * KS + (lane >> 4) * 16;
    auto rb_of = [&](int g) { return 4 * third((int)peer, 6 - link, g >> 5) + 2 * (peer & 1); };
    auto xoff = [&](int g) { return (rb_of(g) * 128 + (unsigned)(g & 3) * 32) * KS + (unsigned)((g & 31) >> 2) * 64; };
    const unsigned cv = ld_sys(cntp);
    a0 = ld128(xs, vx, xoff(idx), 0);
    a1 = ld128(xs, vx + 16 * KS, xoff(idx), 0);
    const unsigned call = UNI(cv) + 1u;
    const unsigned par = O_PAR + (call & 1u) * PAR_BYTES;
    const unsigned dbase = par + P_X + (unsigned)me * (2 * MiB);
    const unsigned tbase = call << 12;
    #pragma unroll 1
    for (int r = 0; r < RP - 1; ++r) {
        if (lane == 0 && r == idx % (RP - 1))
            __hip_atomic_fetch_max((unsigned*)(myws + O_TICK + link * 128), tbase + DYN0, __ATOMIC_RELAXED, __HIP_MEMORY_SCOPE_AGENT);
        const int g = idx + r * WL;
        const unsigned so = xoff(g + WL);
        const u32x4 b0 = ld128(xs, vx, so, 0);
        const u32x4 b1 = ld128(xs, vx + 16 * KS, so, 0);
        asm volatile("" ::: "memory");
        const unsigned d = dbase + rb_of(g) * 65536u + (unsigned)(g & 31) * 2048u;
        st128(a0, rem, v16, d, SYS);
        st128(a1, rem, v16 + 1024, d, SYS);
        drain();
        a0 = b0; a1 = b1;
    }
    const unsigned roff = O_RDY + (peer * 64 + lane) * 4;
    const unsigned* rdyp = (const unsigned*)(myws + roff);
    const unsigned rall = lane < 32 && (unsigned)(lane >> 2) == peer ? 0u : call;
    unsigned long long unseen;
    int dead = 0;
    {
        const unsigned rv = ld32(loc, roff);
        asm volatile("" ::: "memory");
        const int g = idx + (RP - 1) * WL;
        const unsigned d = dbase + rb_of(g) * 65536u + (unsigned)(g & 31) * 2048u;
        st128(a0, rem, v16, d, SYS);
        st128(a1, rem, v16 + 1024, d, SYS);
        st1(loc, (unsigned)((const char*)cntp - myws), call, lane);
        unseen = BALLOT(rv < rall);
    }
    const float scale = p.sx[0] * p.sw[0];
    const int xa = isq ? 192 : 160, gxb = xa + 64 * TA, gown = isq ? G_OWNQ : G_OWN;
    auto wanted = [&](int g) {
        return g < xa + 64 * TE ? ((1ull << (2 * TE)) - 1) << 32 | 0x55555555ull
             : g < 2 * xa + 64 * TE2 ? (1ull << (32 + 2 * TE2)) - 1 : ~0ull;
    };
    auto await_pieces = [&](int g) {
        const unsigned long long w = wanted(g);
        if (unseen & w) {
            if (!dead) dead = wave_wait(rdyp, (w >> lane) & 1 ? rall : 0u, myws, call, lane) ? 0 : 1;
            unseen &= ~w;
        }
    };
    await_pieces(0);
    asm volatile("" ::: "memory");
    dead = UNI(dead);

    const unsigned prog = O_PROG + (link * 64 + idx) * 4;
    const rsrc_t tick = mk(myws + O_TICK + link * 128, 4);
    auto x_off = [&](unsigned k, unsigned rb, int g) {
        return par + P_X + k * (2 * MiB) + rb * 65536u + (unsigned)(g & 31) * 2048u;
    };
    auto w_off = [&](int V, int h, unsigned& src, unsigned& dst) {
        const unsigned o = (unsigned)tile_of(V, hf) * 131072u + (unsigned)(h & 63) * 2048u;
        src = par + P_WS + (unsigned)(me >> 1) * (2 * MiB) + o;
        dst = par + P_WR + peer * (2 * MiB) + o;
    };
    auto head_off = [&](int g, unsigned& src, unsigned& dst) {
        if (g < xa) src = dst = x_off(peer, 4 * third(me, link, g >> 5) + 2 * hf, g);
        else if (g < gxb) w_off((g - xa) >> 6, g - xa, src, dst);
        else if (g < gxb + xa) src = dst = x_off(peer, 4 * third(me, link, (g - gxb) >> 5) + 2 * hf + 1, g);
        else w_off((g - 2 * xa) >> 6, g - 2 * xa, src, dst);
    };
    const int ghead = isq ? G_REL : G_OWN;
    const unsigned oflag = lane < 32 ? O_ORDY + (((call & 1u) * 8 + peer) * 32 + lane) * 4
                                     : O_PRDY + (peer * 32 + (lane - 32)) * 4;
    const unsigned tflag = lane < 9 ? O_OWN + (peer * 32 + lane) * 4 : lane == 9 ? tmo_off() : NOWHERE;
    unsigned fv = 0;
    int g = idx + RP * WL, gn;
    int nin = 0;
    int held = 1;
    unsigned plan = 0, lack = 0xff, src, cdst, ndst;
    auto read_plan = [&](unsigned tv) {
        if (!nin) {
            plan = __builtin_amdgcn_readlane(tv, 8);
            if ((plan >> 18) == (call & 0x3fffu)) nin = gown + 32 * __builtin_popcount(plan & 0x7ffu);
        }
        lack = (isq ? 0xffu & ~(3u << (me & 6)) : links_to_ranks((plan >> 4) & 0x7fu, me)) & ~(unsigned)BALLOT(lane < 8 && tv >= call);
    };
    head_off(g, src, cdst);
    {
        const rsrc_t remd = mk(remp, dead ? 0u : WS_USED);
        const u32x4 b0 = ld128(remd, v16, src, SYS);
        const u32x4 b1 = ld128(remd, v16 + 1024, src, SYS);
        const unsigned tk = draw(tick, lane == 0);
        drain();
        st1(rem, O_PUSH + ((6 - link) * 64 + idx) * 4, call, lane);
        a0 = b0; a1 = b1;
        gn = (int)(UNI(tk) - tbase);
    }
    for (;;) {
        gn = UNI(gn); dead = UNI(dead); held = UNI(held); nin = UNI(nin);
        plan = UNI(plan); cdst = UNI(cdst); lack = UNI(lack);
        unseen = ((unsigned long long)UNI((unsigned)(unseen >> 32)) << 32) | UNI((unsigned)unseen);
        await_pieces(gn);
        if (gn < ghead) head_off(gn, src, ndst);
        else {
            unsigned long long t0 = 0;
            for (;;) {
                if (dead && !nin) { plan = 0; nin = gown; }
                if (gn < gown) {
                    const int r = gn - G_REL;
                    const unsigned via = third(me, link, r >> 7);
                    src = ndst = par + P_WR + via * (2 * MiB) + (unsigned)tile_of(14 + ((r >> 6) & 1), hf) * 131072u + (unsigned)(r & 63) * 2048u;
                    if (dead || !((lack >> via) & 1u)) break;
                } else if (nin) {
                    if (gn >= nin) break;
                    unsigned m = plan & 0x7ffu;
                    for (int j = (gn - gown) >> 5; j; --j) m &= m - 1;
                    const unsigned bit = __builtin_ctz(m);
                    const unsigned via = (unsigned)(me + 1 + bit - 4) & 7u;
                    src = ndst = bit < 4 ? x_off(peer, 4 * me + bit, gn)
                                         : x_off(via, 4 * me + 2 * (peer & 1) + ((plan >> (bit + 7)) & 1u), gn);
                    if (dead || bit < 4 || !((lack >> via) & 1u)) break;
                }
                if (held) {
                    st128(a0, loc, v16, cdst, SYS);
                    st128(a1, loc, v16 + 1024, cdst, SYS);
                    drain();
                    st1(loc, prog, tbase + (unsigned)gn, lane);
                    held = 0;
                } else {
                    if (!t0) t0 = now();
                    if (now() - t0 > DEADLINE) { set_tmo(myws, call, lane); dead = 1; }
                    __builtin_amdgcn_s_sleep(2);
                }
                const unsigned tv = ld32(loc, tflag);
                drain();
                if (__builtin_amdgcn_readlane(tv, 9) == call) dead = 1;
                read_plan(tv);
            }
            if (nin && gn >= nin) break;
        }
        src = UNI(src); ndst = UNI(ndst);
        const rsrc_t remd = mk(remp, dead ? 0u : WS_USED), locd = mk(myws, dead || !held ? 0u : WS_USED);
        const u32x4 b0 = ld128(remd, v16, src, SYS);
        const u32x4 b1 = ld128(remd, v16 + 1024, src, SYS);
        const unsigned tk = draw(tick, lane == 0);
        const bool look = gn + 80 >= ghead && (!nin || lack);
        const bool peek = !look && (unseen & wanted(gn + 60)) != 0;
        fv = ld32(mk(myws, nin && !isq && gn >= nin - 48 ? O_END : 0u), oflag);
        const unsigned tn = ld32(mk(myws, look || peek ? O_END : 0u), peek ? roff : tflag);
        asm volatile("" ::: "memory");
        st128(a0, locd, v16, cdst, SYS);
        st128(a1, locd, v16 + 1024, cdst, SYS);
        drain();
        st1(mk(myws, dead ? 0u : O_END), prog, tbase + (unsigned)gn, lane);
        if (look) read_plan(tn);
        if (peek) unseen = BALLOT(tn < rall);
        a0 = b0; a1 = b1; held = 1;
        g = gn; cdst = ndst;
        gn = (int)(UNI(tk) - tbase);
    }
    {
        const rsrc_t locd = mk(myws, dead || !held ? 0u : WS_USED);
        st128(a0, locd, v16, cdst, SYS);
        st128(a1, locd, v16 + 1024, cdst, SYS);
    }
    if (isq) {
        drain();
        if (!dead) st1(loc, prog, tbase + 4095u, lane);
        return;
    }

    const int NTOT = nin + NOUT;
    const unsigned ovoff = (unsigned)(4 * (lane >> 4)) * (NCOL * 4) + (lane & 15) * 16;
    const unsigned exb = O_EX + ((call & 1u) * 8 + peer) * (256 * NEX * 4);
    unsigned rmask;
    Item cur, nxt;
    int state = 2;
    int waiting = 0;
    unsigned long long t0 = 0;
    {
        const unsigned long long m = BALLOT(lane < 32 ? (fv >> 8) >= call : fv >= call);
        rmask = (unsigned)m;
        if ((unsigned)(m >> 32) != 0xffffffffu && !dead)
            dead = wave_wait((const unsigned*)(myws + O_PRDY) + peer * 32 + (lane & 31), call, myws, call, lane) ? 0 : 1;
    }
    const unsigned pflag = lane < 32 ? oflag : lane == 32 ? tmo_off() : NOWHERE;
    for (;;) {
        g = UNI(g); gn = UNI(gn); dead = UNI(dead); state = UNI(state); rmask = UNI(rmask);
        cdst = UNI(cdst); waiting = UNI(waiting);
        const bool have = gn < NTOT;
        const unsigned o = (unsigned)(gn - nin) & 1023u;
        const bool can = have && (dead || ((rmask >> (o >> 5)) & 1u));
        {
            const bool go = can && !dead;
            const rsrc_t r0 = mk(remp, go ? WS_USED : 0u), r1 = mk(myws, go ? WS_USED : 0u);
            const unsigned so = par + P_OS + (unsigned)me * (4 * MiB) + o * 4096u;
            const unsigned ps = O_PS + peer * (4 * MiB) + o * 4096u;
            nxt.l0 = ld128(r0, v16, so, SYS);
            nxt.l1 = ld128(r0, v16 + 1024, so, SYS);
            nxt.h4 = __builtin_amdgcn_raw_buffer_load_b64(r0, lane * 8 + 2048, so, SYS);
            nxt.h1 = __builtin_amdgcn_raw_buffer_load_b16(r0, lane * 2 + 2560, so, SYS);
            nxt.ex = __builtin_amdgcn_raw_buffer_load_b32(r1, (lane & (NEX - 1)) * 4, exb + (o >> 2) * (NEX * 4), SYS);
            nxt.p0 = ld128(r1, v16, ps, SYS);
            nxt.p1 = ld128(r1, v16 + 1024, ps, SYS);
            nxt.p2 = ld128(r1, v16 + 2048, ps, SYS);
            nxt.p3 = ld128(r1, v16 + 3072, ps, SYS);
        }
        const bool poll = have && !dead && (~rmask & (7u << (o >> 5))) != 0;
        const unsigned nf = ld32(mk(myws, poll ? O_END : 0u), pflag);
        const unsigned tk = draw(tick, can && idx < WLO && lane == 0);

        u32x4 o4[4];
        if (state == 1) {
            const unsigned oc = (unsigned)(g - nin);
            const bool raw = (__builtin_amdgcn_readlane(fv, oc >> 5) >> ((oc >> 2) & 7)) & 1u;
            const unsigned L[8] = {cur.l0.x, cur.l0.y, cur.l0.z, cur.l0.w, cur.l1.x, cur.l1.y, cur.l1.z, cur.l1.w};
            u32x16 pv = {cur.p0.x, cur.p0.y, cur.p0.z, cur.p0.w, cur.p1.x, cur.p1.y, cur.p1.z, cur.p1.w,
                         cur.p2.x, cur.p2.y, cur.p2.z, cur.p2.w, cur.p3.x, cur.p3.y, cur.p3.z, cur.p3.w};
            u32x4 v[4];
            if (raw && !dead) {
                const unsigned so = par + P_OS + (unsigned)me * (4 * MiB) + oc * 4096u;
                const u32x4 ha = ld128(rem, v16 + 2048, so, SYS);
                const u32x4 hb = ld128(rem, v16 + 3072, so, SYS);
                const unsigned H[8] = {ha.x, ha.y, ha.z, ha.w, hb.x, hb.y, hb.z, hb.w};
                #pragma unroll
                for (int r = 0; r < 4; ++r) {
                    v[r].x = perm(H[2 * r], L[2 * r], 0x05040100u);
                    v[r].y = perm(H[2 * r], L[2 * r], 0x07060302u);
                    v[r].z = perm(H[2 * r + 1], L[2 * r + 1], 0x05040100u);
                    v[r].w = perm(H[2 * r + 1], L[2 * r + 1], 0x07060302u);
                }
            } else {
                unsigned em = (unsigned)BALLOT(lane < NEX && (cur.ex >> 21) == (oc & 3u));
                while (em) {
                    const unsigned e = __builtin_amdgcn_readlane(cur.ex, __builtin_ctz(em));
                    em &= em - 1;
                    pv[(e >> 17) & 15u] += (unsigned)lane == ((e >> 11) & 63u) ? e << 21 : 0u;
                }
                const unsigned t = (cur.h1 & 0x0f0fu) | ((cur.h1 & 0xf0f0u) << 12);
                unsigned B[4];
                B[0] = (cur.h4.x & 0x0f0f0f0fu) | ((t << 4) & 0x10101010u);
                B[1] = ((cur.h4.x >> 4) & 0x0f0f0f0fu) | ((t << 3) & 0x10101010u);
                B[2] = (cur.h4.y & 0x0f0f0f0fu) | ((t << 2) & 0x10101010u);
                B[3] = ((cur.h4.y >> 4) & 0x0f0f0f0fu) | ((t << 1) & 0x10101010u);
                #pragma unroll
                for (int r = 0; r < 4; ++r) {
                    v[r].x = perm(B[r], L[2 * r], 0x0c040100u);
                    v[r].y = perm(B[r], L[2 * r], 0x0c050302u);
                    v[r].z = perm(B[r], L[2 * r + 1], 0x0c060100u);
                    v[r].w = perm(B[r], L[2 * r + 1], 0x0c070302u);
                }
            }
            #pragma unroll
            for (int r = 0; r < 4; ++r) {
                f32x4 f;
                f.x = (float)(int)(v[r].x + pv[4 * r]) * scale;     f.y = (float)(int)(v[r].y + pv[4 * r + 1]) * scale;
                f.z = (float)(int)(v[r].z + pv[4 * r + 2]) * scale; f.w = (float)(int)(v[r].w + pv[4 * r + 3]) * scale;
                o4[r] = __builtin_bit_cast(u32x4, f);
                if (dead) o4[r] = mk4(0x7fc00000u, 0x7fc00000u, 0x7fc00000u, 0x7fc00000u);
            }
        }
        {
            const rsrc_t out = mk(p.out, state == 1 ? OUT_BYTES : 0u);
            #pragma unroll
            for (int r = 0; r < 4; ++r) st128(o4[r], out, ovoff + r * (NCOL * 4), cdst, OUT_AUX);
        }
        if (!have && state != 2) break;
        asm volatile("s_waitcnt vmcnt(4)" ::: "memory");
        __builtin_amdgcn_s_waitcnt(0x0F74);
        if (state == 2) {
            asm volatile("s_waitcnt vmcnt(0)" ::: "memory");
            if (!dead) st1(loc, prog, tbase + 4095u, lane);
        }
        if (!have) break;
        if (poll) {
            fv = nf;
            rmask = (unsigned)BALLOT((nf >> 8) >= call);
            if (__builtin_amdgcn_readlane(nf, 32) == call) dead = 1;
        }
        if (can) {
            cur = nxt; g = gn; state = 1;
            const unsigned n = o >> 5, w8 = (o >> 2) & 7, mf = o & 3;
            const unsigned cc = n < 2 * TA ? (n >= TA ? 1 : 0) : n & 1;
            const unsigned V = n < 2 * TA ? n - cc * TA : n >> 1;
            cdst = (((2 * (peer & 1) + cc) * 128 + (w8 >> 2) * 64 + mf * 16) * NCOL
                    + (peer >> 1) * 4096 + (unsigned)tile_of((int)V, (int)(peer & 1)) * 256 + (w8 & 3) * 64) * 4u;
            gn = idx < WLO ? (int)(UNI(tk) - tbase) : NTOT;
            waiting = 0;
        } else {
            if (!waiting) { waiting = 1; t0 = now(); }
            else {
                if (now() - t0 > DEADLINE) { set_tmo(myws, call, lane); dead = 1; }
                __builtin_amdgcn_s_sleep(4);
            }
            state = 0;
        }
    }
}

__device__ __forceinline__ unsigned prep(const Params& p, char* myws, const unsigned* cntp, int id, int tid, int lane, int wv) {
    const int me = p.rank;
    unsigned call;
    {
        const int tw = ((id - 32) >> 1) & 15;
        const unsigned long long hold = id >= 32 && tw >= TE ? HOLD0 + (tw - TE) * HOLD1 : 0;
        const unsigned long long t0 = now();
        while (now() - t0 < hold) __builtin_amdgcn_s_sleep(4);
    }
    const rsrc_t loc = mk(myws, WS_USED);
    if (id < 32) {
        const rsrc_t xs = mk(p.x, 4096u * KS);
        const unsigned voff = (unsigned)(lane & 15) * KS + (lane >> 4) * 16;
        const unsigned soff = (unsigned)(id * 128 + wv * 16) * KS;
        u32x4 v[8];
        #pragma unroll
        for (int i = 0; i < 8; ++i) v[i] = ld128(xs, voff + i * 64, soff, 0);
        call = UNI(ld_sys(cntp)) + 1u;
        const unsigned dst = O_PAR + (call & 1u) * PAR_BYTES + P_X + (unsigned)me * (2 * MiB) + (unsigned)id * 65536u + wv * 1024u;
        #pragma unroll
        for (int i = 0; i < 8; ++i) st128(v[i], loc, lane * 16, dst + i * 8192u, SYS);
    } else {
        const int pid = id - 32, jt = pid >> 1, h = pid & 1;
        const rsrc_t wsr = mk(p.w, (unsigned)KS * NCOL);
        const int wn = wv & 3, kq0 = h * 4 + (wv >> 2) * 2;
        const unsigned voff = (unsigned)(16 * (lane >> 4)) * NCOL + (lane & 15) * 4;
        unsigned r[2][16];
        #pragma unroll
        for (int b = 0; b < 2; ++b)
            #pragma unroll
            for (int j = 0; j < 16; ++j)
                r[b][j] = __builtin_amdgcn_raw_buffer_load_b32(wsr, voff, (unsigned)((kq0 + b) * 64 + j) * NCOL + jt * 256 + wn * 64, 0);
        call = UNI(ld_sys(cntp)) + 1u;
        const unsigned par = O_PAR + (call & 1u) * PAR_BYTES;
        #pragma unroll
        for (int b = 0; b < 2; ++b) {
            unsigned o[4][4];
            #pragma unroll
            for (int q = 0; q < 4; ++q) {
                const unsigned t0 = perm(r[b][4 * q + 1], r[b][4 * q], 0x05010400u);
                const unsigned t1 = perm(r[b][4 * q + 1], r[b][4 * q], 0x07030602u);
                const unsigned t2 = perm(r[b][4 * q + 3], r[b][4 * q + 2], 0x05010400u);
                const unsigned t3 = perm(r[b][4 * q + 3], r[b][4 * q + 2], 0x07030602u);
                o[0][q] = perm(t2, t0, 0x05040100u);
                o[1][q] = perm(t2, t0, 0x07060302u);
                o[2][q] = perm(t3, t1, 0x05040100u);
                o[3][q] = perm(t3, t1, 0x07060302u);
            }
            const unsigned dst = par + P_WS + (unsigned)jt * 131072u + (unsigned)((kq0 + b) * 16 + wn * 4) * 1024u;
            #pragma unroll
            for (int i = 0; i < 4; ++i)
                st128(mk4(o[i][0], o[i][1], o[i][2], o[i][3]), loc, lane * 16 + i * 1024, dst, SYS);
        }
    }
    drain();
    __syncthreads();
    if (id < 32) {
        if (tid < NR) st_sys((unsigned*)(p.ws[tid] + O_RDY) + me * 64 + id, call);
    } else {
        const int pid = id - 32;
        if (tid < 2) st_sys((unsigned*)(p.ws[(pid >> 5) * 2 + tid] + O_RDY) + me * 64 + 32 + (pid & 31), call);
        if (tid == 2) st_sys((unsigned*)(myws + O_LRW) + pid, call);
    }
    return call;
}

}

__global__ __launch_bounds__(NTHR, 2) void gemm_rs_fused(Params p) {
    __shared__ __attribute__((aligned(16))) char lds[LDS_EX + 8 * NEX * 4];
    const int tid = threadIdx.x, lane = tid & 63;
    const int wv = UNI(tid >> 6);
    const int b = blockIdx.x, me = p.rank;
    char* myws = p.ws[me];
    unsigned* cntp = (unsigned*)(myws + O_CNT) + b * 8 + wv;

    if (b < NPULL) {
        const int s = wv * NPULL + b;
        if (s < NLW) link_wave(p, myws, cntp, s, lane);
        else if (b == NPULL - 1 && wv == 7) {
            const unsigned call = UNI(ld_sys(cntp)) + 1u;
            if (lane == 0) st_sys(cntp, call);
            monitor(p, myws, call, lane);
        }
        return;
    }

    const int cb = b - NPULL;
    unsigned call;
    if (cb < 160) call = prep(p, myws, cntp, cb, tid, lane, wv);
    else call = UNI(ld_sys(cntp)) + 1u;
    if (lane == 0) st_sys(cntp, call);

    const int ci = (b & 7) * NCU + (cb >> 3);
    const int hf = me & 1, lp = hf ? 6 : 0, lb = 1 - hf;
    const unsigned par = O_PAR + (call & 1u) * PAR_BYTES;
    const rsrc_t loc = mk(myws, WS_USED);
    ldsw_t misc = (ldsw_t)(lds + LDS_MISC);
    const int wm = wv >> 2, wn = wv & 3;

    for (int step = 0; step < 4; ++step) {
        int task, di, cc, V;
        if (step < 2) {
            const int j = step == 0 ? ci - NP0 : (NCOMP - NP0) + (NCOMP - 1 - ci);
            if (j < 0 || j >= 192) continue;
            const int n = j / 6;
            task = 0; di = j - 6 * n;
            cc = n < 2 * TA ? (n >= TA ? 1 : 0) : n & 1;
            V = n < 2 * TA ? n - cc * TA : n >> 1;
        } else {
            int i = ci;
            if (step == 3) {
                if (tid == 0) misc[1] = __hip_atomic_fetch_add((unsigned*)(myws + O_TICK + 7 * 128), 1u, __ATOMIC_RELAXED, __HIP_MEMORY_SCOPE_AGENT);
                __syncthreads();
                const unsigned extra = UNI(misc[1]) - (call - 1u) * NCOMP;
                if (extra >= 256u - NCOMP) break;
                i = NCOMP + (int)extra;
            }
            if (i < 192) {
                const int r = i % 24;
                task = 1; V = 2 * (i / 24) + (r & 1); cc = r >= 12 ? 1 : 0; di = (r >> 1) - 6 * cc;
            } else {
                task = 2; di = 0; V = (i - 192) >> 2; cc = i & 3;
            }
        }
        const int d = (me + 1 + lb + di) & 7;
        const int rb = task == 0 ? 4 * me + 2 * (d & 1) + cc : task == 1 ? 4 * d + 2 * hf + cc : 4 * me + cc;
        const int pos = tile_pos(cc, V);
        const int T = tile_of(V, task == 0 ? d & 1 : hf);
        const int jt = (task == 0 ? d >> 1 : me >> 1) * 16 + T;

        i32x4 acc[4][4];
        {
            const int a0 = task == 0 ? -BIAS : task == 1 ? BIAS : 0;
            #pragma unroll
            for (int m = 0; m < 4; ++m)
                #pragma unroll
                for (int n = 0; n < 4; ++n) acc[m][n] = (i32x4){a0, a0, a0, a0};
        }
        unsigned todo = task == 0 ? 1u << me : task == 1 ? 0xffu & ~(1u << d) : 0xffu;
        bool ok = true;
        while (todo) {
            if (wv == 0) {
                const int l = lane & 15;
                const bool direct = V < 14 || l == lp;
                const unsigned* ptr = (const unsigned*)(myws + tmo_off());
                unsigned need = call;
                if ((todo >> me) & 1u) {
                    if (lane == 8 || lane == 9) ptr = (const unsigned*)(myws + O_LRW) + jt * 2 + (lane - 8);
                    if (lane == 10) ptr = (const unsigned*)(myws + O_RDY) + me * 64 + rb;
                }
                if (task && lane < 7) {
                    const int xa = l == lp ? 192 : 160;
                    const int xn = task == 2 ? G_ALL : cc ? 2 * xa + 64 * TA : xa;
                    ptr = (const unsigned*)(myws + O_G + (b & 7) * 128) + l;
                    need = (call << 12) + (unsigned)max(xn, direct ? w_done(xa, V) : 0);
                }
                if (task && lane >= 16 && lane < 23) {
                    ptr = (const unsigned*)(myws + O_G + (b & 7) * 128) + lp;
                    need = direct ? 0u : (call << 12) + (unsigned)(G_REL + 128 * (l - lb) + 64 * (V - 13));
                }
                const unsigned long long t0 = now();
                unsigned got;
                for (;;) {
                    const unsigned val = ld_sys(ptr);
                    unsigned m = (unsigned)BALLOT(val >= need);
                    if (__builtin_amdgcn_readlane(val, 7) == call) { got = 0; break; }
                    unsigned lm = m & (m >> 16) & 0x7fu;
                    if (task == 2 && ((m | (1u << lp)) & 0x7fu) != 0x7fu) lm = 0;
                    got = links_to_ranks(lm | (((m >> 8) & 7u) == 7u ? 0x80u : 0u), me) & todo;
                    if (got) break;
                    if (now() - t0 > DEADLINE) { set_tmo(myws, call, lane); break; }
                    __builtin_amdgcn_s_sleep(6);
                }
                if (lane == 0) misc[0] = got;
            }
            __syncthreads();
            const unsigned got = UNI(misc[0]);
            if (!got) { ok = false; break; }
            todo &= ~got;
            unsigned klist = 0, rest = got;
            int nst = 0;
            while (rest) { klist |= (unsigned)__builtin_ctz(rest) << nst; rest &= rest - 1; nst += 4; }
            const unsigned va = wv * 2048 + lane * 16, vb = wv * 4096 + lane * 16;
            auto issue = [&](int s, int slot) {
                const int k = (klist >> (s & ~3)) & 15;
                const unsigned abase = par + P_X + (unsigned)k * (2 * MiB) + (unsigned)rb * 65536u + (s & 3) * 16384u;
                const unsigned bbase = par + (k == me ? P_WS + (unsigned)jt * 131072u : P_WR + (unsigned)(k * 16 + T) * 131072u) + (s & 3) * 32768u;
                char* la = lds + slot * STAGE + wv * 2048;
                char* lb_ = lds + slot * STAGE + 16384 + wv * 4096;
                __builtin_amdgcn_raw_ptr_buffer_load_lds(loc, (__attribute__((address_space(3))) void*)la, 16, va, abase, 0, SYS);
                __builtin_amdgcn_raw_ptr_buffer_load_lds(loc, (__attribute__((address_space(3))) void*)la, 16, va, abase, 1024, SYS);
                __builtin_amdgcn_raw_ptr_buffer_load_lds(loc, (__attribute__((address_space(3))) void*)lb_, 16, vb, bbase, 0, SYS);
                __builtin_amdgcn_raw_ptr_buffer_load_lds(loc, (__attribute__((address_space(3))) void*)lb_, 16, vb, bbase, 1024, SYS);
                __builtin_amdgcn_raw_ptr_buffer_load_lds(loc, (__attribute__((address_space(3))) void*)lb_, 16, vb, bbase, 2048, SYS);
                __builtin_amdgcn_raw_ptr_buffer_load_lds(loc, (__attribute__((address_space(3))) void*)lb_, 16, vb, bbase, 3072, SYS);
            };
            auto compute = [&](int slot) {
                const char* A = lds + slot * STAGE + wm * 4096 + lane * 16;
                const char* B = lds + slot * STAGE + 16384 + wn * 4096 + lane * 16;
                i32x4 af[2][4], bf[2][4];
                #pragma unroll
                for (int kb = 0; kb < 2; ++kb) {
                    af[kb][0] = *(const i32x4*)(A + kb * 8192);
                    #pragma unroll
                    for (int i = 0; i < 4; ++i) bf[kb][i] = *(const i32x4*)(B + kb * 16384 + i * 1024);
                    #pragma unroll
                    for (int i = 1; i < 4; ++i) af[kb][i] = *(const i32x4*)(A + kb * 8192 + i * 1024);
                }
                __builtin_amdgcn_sched_barrier(0);
                #pragma unroll
                for (int kb = 0; kb < 2; ++kb)
                    #pragma unroll
                    for (int m = 0; m < 4; ++m)
                        #pragma unroll
                        for (int n = 0; n < 4; ++n)
                            acc[m][n] = __builtin_amdgcn_mfma_i32_16x16x64_i8(af[kb][m], bf[kb][n], acc[m][n], 0, 0, 0);
                __builtin_amdgcn_sched_barrier(0);
            };
            issue(0, 0);
            issue(1, 1);
            issue(2, 2);
            int slot = 0;
            for (int s = 0; s < nst; ++s) {
                if (s == 0) asm volatile("s_waitcnt vmcnt(12) lgkmcnt(0)" ::: "memory");
                else if (s + 1 < nst) asm volatile("s_waitcnt vmcnt(6) lgkmcnt(0)" ::: "memory");
                else asm volatile("s_waitcnt vmcnt(0) lgkmcnt(0)" ::: "memory");
                __builtin_amdgcn_s_barrier();
                if (s && s + 2 < nst) issue(s + 2, slot == 0 ? 2 : slot - 1);
                compute(slot);
                slot = slot == 2 ? 0 : slot + 1;
            }
        }

        if (task == 0) {
            if (ok) {
                const unsigned dst = O_PS + (unsigned)d * (4 * MiB) + (unsigned)(pos * 8 + wv) * 16384u;
                #pragma unroll
                for (int m = 0; m < 4; ++m)
                    #pragma unroll
                    for (int r = 0; r < 4; ++r)
                        st128(mk4((unsigned)acc[m][0][r], (unsigned)acc[m][1][r], (unsigned)acc[m][2][r], (unsigned)acc[m][3][r]),
                              loc, lane * 16 + r * 1024, dst + m * 4096u, SYS);
            }
            drain();
            __syncthreads();
            if (tid == 0 && ok) st_sys((unsigned*)(myws + O_PRDY) + d * 32 + pos, call);
        } else if (task == 1) {
            ldsw_t exl = (ldsw_t)(lds + LDS_EX) + wv * NEX;
            if (lane < NEX) exl[lane] = 0xffffffffu;
            unsigned nex = 0;
            #pragma unroll
            for (int m = 0; m < 4; ++m)
                #pragma unroll
                for (int r = 0; r < 4; ++r) {
                    const unsigned any = (unsigned)(acc[m][0][r] | acc[m][1][r]) | (unsigned)(acc[m][2][r] | acc[m][3][r]);
                    if (BALLOT((any >> 21) != 0) != 0) {
                        #pragma unroll
                        for (int n = 0; n < 4; ++n) {
                            const unsigned hi = (unsigned)acc[m][n][r] >> 21;
                            const unsigned long long bm = BALLOT(hi != 0);
                            const unsigned at = nex + __builtin_amdgcn_mbcnt_hi((unsigned)(bm >> 32), __builtin_amdgcn_mbcnt_lo((unsigned)bm, 0u));
                            if (hi != 0 && at < NEX) exl[at] = (unsigned)(m << 21 | (r * 4 + n) << 17) | ((unsigned)lane << 11) | hi;
                            nex += __builtin_popcountll(bm);
                        }
                    }
                }
            const bool raw = nex > NEX;
            if (ok) {
                const unsigned e = exl[lane & (NEX - 1)];
                __builtin_amdgcn_raw_buffer_store_b32(e, mk(p.ws[d], WS_USED),
                    lane < NEX ? O_EX + ((((call & 1u) * 8 + me) * 32 + pos) * 8 + wv) * (NEX * 4) + lane * 4 : NOWHERE, 0, SYS);
                const unsigned dst = par + P_OS + (unsigned)d * (4 * MiB) + (unsigned)(pos * 8 + wv) * 16384u;
                #pragma unroll
                for (int m = 0; m < 4; ++m) {
                    unsigned L[8], H[8];
                    #pragma unroll
                    for (int r = 0; r < 4; ++r) {
                        L[2 * r] = perm(acc[m][1][r], acc[m][0][r], 0x05040100u);
                        L[2 * r + 1] = perm(acc[m][3][r], acc[m][2][r], 0x05040100u);
                        H[2 * r] = perm(acc[m][1][r], acc[m][0][r], 0x07060302u);
                        H[2 * r + 1] = perm(acc[m][3][r], acc[m][2][r], 0x07060302u);
                    }
                    st128(mk4(L[0], L[1], L[2], L[3]), loc, lane * 16, dst + m * 4096u, SYS);
                    st128(mk4(L[4], L[5], L[6], L[7]), loc, lane * 16 + 1024, dst + m * 4096u, SYS);
                    if (raw) {
                        st128(mk4(H[0], H[1], H[2], H[3]), loc, lane * 16 + 2048, dst + m * 4096u, SYS);
                        st128(mk4(H[4], H[5], H[6], H[7]), loc, lane * 16 + 3072, dst + m * 4096u, SYS);
                    } else {
                        unsigned B[4];
                        #pragma unroll
                        for (int r = 0; r < 4; ++r) B[r] = perm(H[2 * r + 1], H[2 * r], 0x06040200u);
                        const u32x2 h4 = {bfi(0x0f0f0f0fu, B[0], B[1] << 4), bfi(0x0f0f0f0fu, B[2], B[3] << 4)};
                        const unsigned t = ((B[0] >> 4) & 0x01010101u) | ((B[1] >> 3) & 0x02020202u)
                                         | ((B[2] >> 2) & 0x04040404u) | ((B[3] >> 1) & 0x08080808u);
                        __builtin_amdgcn_raw_buffer_store_b64(h4, loc, lane * 8 + 2048 + dst + m * 4096u, 0, SYS);
                        __builtin_amdgcn_raw_buffer_store_b16((unsigned short)(t | (t >> 12)), loc, lane * 2 + 2560 + dst + m * 4096u, 0, SYS);
                    }
                }
            }
            if (lane == 0) misc[4 + wv] = raw ? 1u : 0u;
            drain();
            __syncthreads();
            if (tid == 0 && ok) {
                unsigned mask = 0;
                #pragma unroll
                for (int i = 0; i < 8; ++i) mask |= misc[4 + i] << i;
                st_sys((unsigned*)(p.ws[d] + O_ORDY) + ((call & 1u) * 8 + me) * 32 + pos, (call << 8) | mask);
            }
        } else {
            const rsrc_t out = mk(p.out, OUT_BYTES);
            const float scale = p.sx[0] * p.sw[0];
            const unsigned ovoff = (unsigned)(4 * (lane >> 4)) * (NCOL * 4) + (lane & 15) * 16;
            const unsigned dst = ((unsigned)(cc * 128 + wm * 64) * NCOL + jt * 256 + wn * 64) * 4u;
            #pragma unroll
            for (int m = 0; m < 4; ++m)
                #pragma unroll
                for (int r = 0; r < 4; ++r) {
                    f32x4 f;
                    f.x = (float)acc[m][0][r] * scale; f.y = (float)acc[m][1][r] * scale;
                    f.z = (float)acc[m][2][r] * scale; f.w = (float)acc[m][3][r] * scale;
                    u32x4 o4 = __builtin_bit_cast(u32x4, f);
                    if (!ok) o4 = mk4(0x7fc00000u, 0x7fc00000u, 0x7fc00000u, 0x7fc00000u);
                    st128(o4, out, ovoff + r * (NCOL * 4), dst + (unsigned)(m * 16) * (NCOL * 4), OUT_AUX);
                }
        }
    }
}

extern "C" void kernel_launch(int rank, int nranks, ncclComm_t comm, hipStream_t stream,
                              void* const* d_in, const int* in_sizes, int n_in,
                              void* d_out, int out_size, const int* in_dtypes, int out_dtype,
                              void* const* d_out_all, void* const* d_ws_all, size_t ws_bytes) {
    (void)comm; (void)in_sizes; (void)in_dtypes; (void)out_dtype; (void)d_out_all;
    if (nranks != NR || (unsigned)rank >= (unsigned)NR || n_in < 4 || ws_bytes < WS_USED ||
        (size_t)out_size * 4 != OUT_BYTES) {
        fprintf(stderr, "kernel_launch: unexpected problem shape\n");
        return;
    }
    Params p;
    for (int i = 0; i < NR; ++i) p.ws[i] = (char*)d_ws_all[i];
    p.x = (const char*)d_in[0];
    p.w = (const char*)d_in[1];
    p.sx = (const float*)d_in[2];
    p.sw = (const float*)d_in[3];
    p.out = (float*)d_out;
    p.rank = rank;
    p.pad = 0;
    hipLaunchKernelGGL(gemm_rs_fused, dim3(NBLK), dim3(NTHR), 0, stream, p);
    HIP_CHECK(hipGetLastError());
}
```

```cpp
#include <hip/hip_runtime.h>
#include <rccl/rccl.h>
#include <cstdio>
#include <cstdint>

#define HIP_CHECK(expr) do { hipError_t _e = (expr); if (_e != hipSuccess) { \
    fprintf(stderr, "HIP error %s at %s:%d\n", hipGetErrorString(_e), __FILE__, __LINE__); \
    return; } } while (0)

typedef int i32x4 __attribute__((ext_vector_type(4)));
typedef unsigned u32x4 __attribute__((ext_vector_type(4)));
typedef unsigned u32x2 __attribute__((ext_vector_type(2)));
typedef unsigned u32x16 __attribute__((ext_vector_type(16)));
typedef float f32x4 __attribute__((ext_vector_type(4)));
typedef __amdgpu_buffer_rsrc_t rsrc_t;
typedef volatile __attribute__((address_space(3))) unsigned* ldsw_t;
#define UNI(x) __builtin_amdgcn_readfirstlane(x)
#define BALLOT(c) __builtin_amdgcn_ballot_w64(c)

namespace {

constexpr int NR = 8;
constexpr int NCOL = 16384;
constexpr int KS = 512;
constexpr unsigned MiB = 1u << 20;

constexpr int NBLK = 256, NTHR = 512;
constexpr int NPULL = 80;
constexpr int WL = 40;
constexpr int WLO = 30;
constexpr int NLW = 7 * WL;
constexpr int NCOMP = NBLK - NPULL, NCU = NCOMP / 8;
constexpr int NP0 = 48;
static_assert(NLW <= 7 * NPULL && NCOMP >= 160 && NCOMP <= 192 && 2 * (NCOMP - NP0) >= 192 && WLO <= WL && WL <= 64, "geometry");

constexpr int RP = 4, TA = 2, VS = 8;
constexpr int TE = 2, TE2 = 4;
constexpr int HOLD0 = 120, HOLD1 = 30;
constexpr int LEAD = 160;
constexpr int RELAY_TH = 36, QBIAS = 64;
constexpr int G_OWN = 320 + 64 * 14, G_REL = 384 + 64 * 16, G_OWNQ = G_REL + 768;
constexpr int G_ALL = 4095;
constexpr int NOUT = 1024;
constexpr int DYN0 = (RP + 1) * WL;
static_assert(RP >= 2 && RP * WL <= 160 && TE <= TA && TA <= TE2 && TE2 <= VS && VS <= 12, "lists");
__host__ __device__ constexpr int tile_pos(int cc, int V) { return V < TA ? cc * TA + V : 2 * V + cc; }
__host__ __device__ constexpr int tile_of(int V, int q) {
    return V < VS ? V : V < VS + 2 ? 12 + 2 * (V - VS) + q : V < 14 ? V - 2 : 13 + 2 * (V - 14) - q;
}
__host__ __device__ constexpr int w_done(int xa, int V) { return (V < TA ? xa : 2 * xa) + 64 * (V + 1); }

constexpr unsigned O_CNT = 0;
constexpr unsigned O_TMO = 8192;
constexpr unsigned O_TICK = 12288;
constexpr unsigned O_OWN = 14336;
constexpr unsigned O_RDY = 16384;
constexpr unsigned O_LRW = 20480;
constexpr unsigned O_PROG = 24576;
constexpr unsigned O_PUSH = 26624;
constexpr unsigned O_G = 28672;
constexpr unsigned O_PRDY = 32768;
constexpr unsigned O_ORDY = 36864;
constexpr unsigned O_END = 65536;
constexpr unsigned O_EX = 512 * 1024;
constexpr unsigned O_PS = 1 * MiB;
constexpr unsigned O_PAR = 33 * MiB;
constexpr unsigned P_X = 0;
constexpr unsigned P_WS = 16 * MiB;
constexpr unsigned P_WR = 24 * MiB;
constexpr unsigned P_OS = 40 * MiB;
constexpr unsigned PAR_BYTES = 72 * MiB;
constexpr unsigned WS_USED = O_PAR + 2 * PAR_BYTES;
constexpr unsigned OUT_BYTES = 512u * NCOL * 4u;
constexpr unsigned NOWHERE = 0xf0000000u;

constexpr unsigned long long DEADLINE = 40000000ull;
constexpr int BIAS = 1 << 20;
constexpr int NEX = 16;
constexpr int SYS = 17;
constexpr int OUT_AUX = 16;

constexpr int STAGE = 49152;
constexpr int LDS_MISC = 3 * STAGE;
constexpr int LDS_EX = LDS_MISC + 64;

struct Params {
    char* ws[NR];
    const char* x;
    const char* w;
    const float* sx;
    const float* sw;
    float* out;
    int rank;
    int pad;
};

__device__ __forceinline__ rsrc_t mk(const void* p, unsigned bytes) {
    return __builtin_amdgcn_make_buffer_rsrc((void*)p, 0, bytes, 0x00020000);
}
#define st128(v, r, voff, soff, aux) __builtin_amdgcn_raw_buffer_store_b128(v, r, (voff) + (soff), 0, aux)
#define ld128(r, voff, soff, aux) __builtin_amdgcn_raw_buffer_load_b128(r, voff, soff, aux)
#define ld32(r, voff) __builtin_amdgcn_raw_buffer_load_b32(r, voff, 0, SYS)
__device__ __forceinline__ u32x4 mk4(unsigned a, unsigned b, unsigned c, unsigned d) { return (u32x4){a, b, c, d}; }
__device__ __forceinline__ void st1(const rsrc_t r, unsigned off, unsigned v, int lane) {
    __builtin_amdgcn_raw_buffer_store_b32(v, r, lane == 0 ? off : NOWHERE, 0, SYS);
}
__device__ __forceinline__ unsigned tmo_off() { return O_TMO + (blockIdx.x & 31) * 128; }
__device__ __forceinline__ void set_tmo(char* myws, unsigned call, int lane) {
    __builtin_amdgcn_raw_buffer_store_b32(call, mk(myws, O_END), lane < 32 ? O_TMO + lane * 128 : NOWHERE, 0, SYS);
}
__device__ __forceinline__ unsigned ld_sys(const unsigned* p) {
    return __hip_atomic_load(p, __ATOMIC_RELAXED, __HIP_MEMORY_SCOPE_SYSTEM);
}
__device__ __forceinline__ void st_sys(unsigned* p, unsigned v) {
    __hip_atomic_store(p, v, __ATOMIC_RELAXED, __HIP_MEMORY_SCOPE_SYSTEM);
}
__device__ __forceinline__ void drain() {
    asm volatile("s_waitcnt vmcnt(0)" ::: "memory");
    __builtin_amdgcn_s_waitcnt(0x0F70);
}
__device__ __forceinline__ unsigned long long now() { return __builtin_amdgcn_s_memrealtime(); }
__device__ __forceinline__ unsigned perm(unsigned hi, unsigned lo, unsigned sel) {
    return __builtin_amdgcn_perm(hi, lo, sel);
}
__device__ __forceinline__ unsigned bfi(unsigned m, unsigned a, unsigned b) { return (a & m) | (b & ~m); }
__device__ __forceinline__ unsigned links_to_ranks(unsigned m, int me) { return ((m | (m << 8)) >> (7 - me)) & 0xffu; }

__device__ __forceinline__ bool wave_wait(const unsigned* ptr, unsigned need, char* myws, unsigned call, int lane) {
    const unsigned* tmo = (const unsigned*)(myws + tmo_off());
    const unsigned long long t0 = now();
    for (;;) {
        const unsigned v = ld_sys(ptr);
        const unsigned t = ld_sys(tmo);
        if (UNI(t) == call) return false;
        if (BALLOT(v < need) == 0) return true;
        if (now() - t0 > DEADLINE) {
            set_tmo(myws, call, lane);
            return false;
        }
        __builtin_amdgcn_s_sleep(1);
    }
}

__device__ __forceinline__ void monitor(const Params& p, char* myws, unsigned call, int lane) {
    const unsigned* tmo = (const unsigned*)(myws + tmo_off());
    const unsigned* prog = (const unsigned*)(myws + O_PROG);
    const int l7 = lane & 7, me = p.rank, lp = (me & 1) ? 6 : 0;
    unsigned* ownp = (unsigned*)(p.ws[lane >> 3] + O_OWN) + me * 32 + ((me + 1 + l7) & 7);
    const int xa = l7 == lp ? 192 : 160;
    const unsigned tell = (lane >> 3) == (me ^ 1) ? w_done(160, VS + 1) : 2 * xa + 64 * TA;
    const unsigned gown = l7 == lp ? G_OWNQ : G_OWN;
    const unsigned base = call << 12;
    unsigned last = 0, nin = 4095, mine, open = 0x7f;
    bool pushed = false;
    bool planned = false, told = l7 == 7 || (lane >> 3) == me || (l7 == lp && (lane >> 3) == (me ^ 1));
    const unsigned long long t0 = now();
    for (;;) {
        asm volatile("" ::: "memory");
        const unsigned tk = ld32(mk(myws, planned ? 0u : O_END), O_TICK + l7 * 128);
        if (open == 1u << lp && pushed) {
            const unsigned v = ld_sys(prog + lp * 64 + lane);
            mine = lane >= WL ? 4095u : v >= base ? v - base : 0u;
            #pragma unroll
            for (int o = 32; o; o >>= 1) mine = min(mine, (unsigned)__shfl_xor((int)mine, o));
            if (l7 != lp) mine = 4095;
        } else {
            unsigned g[7];
            bool all = true;
            #pragma unroll
            for (int l = 0; l < 7; ++l) {
                const unsigned v = ld_sys(prog + l * 64 + lane);
                const unsigned q = ld32(mk(myws, pushed ? 0u : O_END), O_PUSH + (l * 64 + lane) * 4);
                const bool in = pushed || q >= call || lane >= WL;
                all = all && in;
                g[l] = lane >= WL ? 4095u : (v >= base && in) ? v - base : 0u;
            }
            pushed = BALLOT(!all) == 0;
            #pragma unroll
            for (int o = 32; o; o >>= 1)
                #pragma unroll
                for (int l = 0; l < 7; ++l) g[l] = min(g[l], (unsigned)__shfl_xor((int)g[l], o));
            mine = 4095;
            #pragma unroll
            for (int l = 0; l < 7; ++l) if (l7 == l) mine = g[l];
        }
        if (!planned) {
            int r = l7 < 7 && tk >= base ? (int)(tk - base) : 0;
            if (BALLOT(l7 != lp && r >= G_OWN - LEAD) != 0) {
                planned = true;
                r = -r - (l7 == lp ? QBIAS : 0);
                unsigned keep = 15, recv = 0;
                #pragma unroll
                for (int j = 1; j < 7; ++j) {
                    const int to = l7 + j >= 7 ? l7 + j - 7 : l7 + j, from = l7 - j < 0 ? l7 - j + 7 : l7 - j;
                    const unsigned two = (keep >> (2 * ((me + 1 + to) & 1))) & 3u;
                    const int rto = __shfl(r, to);
                    const int give = to != lp && two && r - rto > RELAY_TH ? (two == 3 ? 3 : 1) : 0;
                    const int got = __shfl(give, from);
                    r += 32 * ((got & 1) - (give & 1));
                    if (give) keep &= ~(1u << (2 * ((me + 1 + to) & 1) + (give >> 1)));
                    recv |= (unsigned)(got & 1) << from | (unsigned)(got >> 1) << (7 + from);
                }
                nin = gown + 32 * (__builtin_popcount(keep) + __builtin_popcount(recv & 0x7fu));
                if (lane < 7) st_sys((unsigned*)(myws + O_OWN) + ((me + 1 + lane) & 7) * 32 + 8, (call << 18) | (recv << 4) | keep);
            }
        }
        mine = mine >= nin ? (unsigned)G_ALL : min(mine, gown);
        if (mine > last) {
            last = mine;
            if (l7 < 7) st_sys((unsigned*)(myws + O_G + (lane >> 3) * 128) + l7, base + mine);
        }
        if (!told && mine >= tell) {
            told = true;
            st_sys(ownp, call);
        }
        open = (unsigned)BALLOT(mine < G_ALL) & 0x7fu;
        if (!open) return;
        if (now() - t0 > 100000) {
            if (UNI(ld_sys(tmo)) == call) return;
            if (now() - t0 > DEADLINE) { set_tmo(myws, call, lane); return; }
            __builtin_amdgcn_s_sleep(1);
        }
    }
}

struct Item { u32x4 l0, l1, p0, p1, p2, p3; u32x2 h4; unsigned h1, ex; };

__device__ __forceinline__ unsigned draw(const rsrc_t tick, bool sel) {
    int t = 1, z = 0;
    asm volatile("" : "+v"(z));
    if (sel) t = __builtin_amdgcn_raw_ptr_buffer_atomic_add_i32(t, tick, z, 0, 0);
    return (unsigned)t;
}

__device__ __forceinline__ void link_wave(const Params& p, char* myws, const unsigned* cntp, int s, int lane) {
    const int me = p.rank, hf = me & 1;
    const int link = s % 7, idx = s / 7;
    const unsigned peer = (unsigned)(me + 1 + link) & 7u;
    const bool isq = peer == (unsigned)(me ^ 1);
    const char* remp = p.ws[peer];
    const rsrc_t loc = mk(myws, WS_USED), rem = mk(remp, WS_USED);
    const unsigned v16 = lane * 16;
    auto third = [&](int r, int l, int i) {
        const int j = i + 1 - (r & 1);
        return (unsigned)(r + 1 + j + (!isq && j >= l ? 1 : 0)) & 7u;
    };

    u32x4 a0, a1;
    const rsrc_t xs = mk(p.x, 4096u * KS);
    const unsigned vx = (unsigned)(lane & 15) * KS + (lane >> 4) * 16;
    auto rb_of = [&](int g) { return 4 * third((int)peer, 6 - link, g >> 5) + 2 * (peer & 1); };
    auto xoff = [&](int g) { return (rb_of(g) * 128 + (unsigned)(g & 3) * 32) * KS + (unsigned)((g & 31) >> 2) * 64; };
    const unsigned cv = ld_sys(cntp);
    a0 = ld128(xs, vx, xoff(idx), 0);
    a1 = ld128(xs, vx + 16 * KS, xoff(idx), 0);
    const unsigned call = UNI(cv) + 1u;
    const unsigned par = O_PAR + (call & 1u) * PAR_BYTES;
    const unsigned dbase = par + P_X + (unsigned)me * (2 * MiB);
    const unsigned tbase = call << 12;
    #pragma unroll 1
    for (int r = 0; r < RP - 1; ++r) {
        if (lane == 0 && r == idx % (RP - 1))
            __hip_atomic_fetch_max((unsigned*)(myws + O_TICK + link * 128), tbase + DYN0, __ATOMIC_RELAXED, __HIP_MEMORY_SCOPE_AGENT);
        const int g = idx + r * WL;
        const unsigned so = xoff(g + WL);
        const u32x4 b0 = ld128(xs, vx, so, 0);
        const u32x4 b1 = ld128(xs, vx + 16 * KS, so, 0);
        asm volatile("" ::: "memory");
        const unsigned d = dbase + rb_of(g) * 65536u + (unsigned)(g & 31) * 2048u;
        st128(a0, rem, v16, d, SYS);
        st128(a1, rem, v16 + 1024, d, SYS);
        drain();
        a0 = b0; a1 = b1;
    }
    const unsigned roff = O_RDY + (peer * 64 + lane) * 4;
    const unsigned* rdyp = (const unsigned*)(myws + roff);
    const unsigned rall = lane < 32 && (unsigned)(lane >> 2) == peer ? 0u : call;
    unsigned long long unseen;
    int dead = 0;
    {
        const unsigned rv = ld32(loc, roff);
        asm volatile("" ::: "memory");
        const int g = idx + (RP - 1) * WL;
        const unsigned d = dbase + rb_of(g) * 65536u + (unsigned)(g & 31) * 2048u;
        st128(a0, rem, v16, d, SYS);
        st128(a1, rem, v16 + 1024, d, SYS);
        st1(loc, (unsigned)((const char*)cntp - myws), call, lane);
        unseen = BALLOT(rv < rall);
    }
    const float scale = p.sx[0] * p.sw[0];
    const int xa = isq ? 192 : 160, gxb = xa + 64 * TA, gown = isq ? G_OWNQ : G_OWN;
    auto wanted = [&](int g) {
        return g < xa + 64 * TE ? ((1ull << (2 * TE)) - 1) << 32 | 0x55555555ull
             : g < 2 * xa + 64 * TE2 ? (1ull << (32 + 2 * TE2)) - 1 : ~0ull;
    };
    auto await_pieces = [&](int g) {
        const unsigned long long w = wanted(g);
        if (unseen & w) {
            if (!dead) dead = wave_wait(rdyp, (w >> lane) & 1 ? rall : 0u, myws, call, lane) ? 0 : 1;
            unseen &= ~w;
        }
    };
    await_pieces(0);
    asm volatile("" ::: "memory");
    dead = UNI(dead);

    const unsigned prog = O_PROG + (link * 64 + idx) * 4;
    const rsrc_t tick = mk(myws + O_TICK + link * 128, 4);
    auto x_off = [&](unsigned k, unsigned rb, int g) {
        return par + P_X + k * (2 * MiB) + rb * 65536u + (unsigned)(g & 31) * 2048u;
    };
    auto w_off = [&](int V, int h, unsigned& src, unsigned& dst) {
        const unsigned o = (unsigned)tile_of(V, hf) * 131072u + (unsigned)(h & 63) * 2048u;
        src = par + P_WS + (unsigned)(me >> 1) * (2 * MiB) + o;
        dst = par + P_WR + peer * (2 * MiB) + o;
    };
    auto head_off = [&](int g, unsigned& src, unsigned& dst) {
        if (g < xa) src = dst = x_off(peer, 4 * third(me, link, g >> 5) + 2 * hf, g);
        else if (g < gxb) w_off((g - xa) >> 6, g - xa, src, dst);
        else if (g < gxb + xa) src = dst = x_off(peer, 4 * third(me, link, (g - gxb) >> 5) + 2 * hf + 1, g);
        else w_off((g - 2 * xa) >> 6, g - 2 * xa, src, dst);
    };
    const int ghead = isq ? G_REL : G_OWN;
    const unsigned oflag = lane < 32 ? O_ORDY + (((call & 1u) * 8 + peer) * 32 + lane) * 4
                                     : O_PRDY + (peer * 32 + (lane - 32)) * 4;
    const unsigned tflag = lane < 9 ? O_OWN + (peer * 32 + lane) * 4 : lane == 9 ? tmo_off() : NOWHERE;
    unsigned fv = 0;
    int g = idx + RP * WL, gn;
    int nin = 0;
    int held = 1;
    unsigned plan = 0, lack = 0xff, src, cdst, ndst;
    auto read_plan = [&](unsigned tv) {
        if (!nin) {
            plan = __builtin_amdgcn_readlane(tv, 8);
            if ((plan >> 18) == (call & 0x3fffu)) nin = gown + 32 * __builtin_popcount(plan & 0x7ffu);
        }
        lack = (isq ? 0xffu & ~(3u << (me & 6)) : links_to_ranks((plan >> 4) & 0x7fu, me)) & ~(unsigned)BALLOT(lane < 8 && tv >= call);
    };
    head_off(g, src, cdst);
    {
        const rsrc_t remd = mk(remp, dead ? 0u : WS_USED);
        const u32x4 b0 = ld128(remd, v16, src, SYS);
        const u32x4 b1 = ld128(remd, v16 + 1024, src, SYS);
        const unsigned tk = draw(tick, lane == 0);
        drain();
        st1(rem, O_PUSH + ((6 - link) * 64 + idx) * 4, call, lane);
        a0 = b0; a1 = b1;
        gn = (int)(UNI(tk) - tbase);
    }
    for (;;) {
        gn = UNI(gn); dead = UNI(dead); held = UNI(held); nin = UNI(nin);
        plan = UNI(plan); cdst = UNI(cdst); lack = UNI(lack);
        unseen = ((unsigned long long)UNI((unsigned)(unseen >> 32)) << 32) | UNI((unsigned)unseen);
        await_pieces(gn);
        if (gn < ghead) head_off(gn, src, ndst);
        else {
            unsigned long long t0 = 0;
            for (;;) {
                if (dead && !nin) { plan = 0; nin = gown; }
                if (gn < gown) {
                    const int r = gn - G_REL;
                    const unsigned via = third(me, link, r >> 7);
                    src = ndst = par + P_WR + via * (2 * MiB) + (unsigned)tile_of(14 + ((r >> 6) & 1), hf) * 131072u + (unsigned)(r & 63) * 2048u;
                    if (dead || !((lack >> via) & 1u)) break;
                } else if (nin) {
                    if (gn >= nin) break;
                    unsigned m = plan & 0x7ffu;
                    for (int j = (gn - gown) >> 5; j; --j) m &= m - 1;
                    const unsigned bit = __builtin_ctz(m);
                    const unsigned via = (unsigned)(me + 1 + bit - 4) & 7u;
                    src = ndst = bit < 4 ? x_off(peer, 4 * me + bit, gn)
                                         : x_off(via, 4 * me + 2 * (peer & 1) + ((plan >> (bit + 7)) & 1u), gn);
                    if (dead || bit < 4 || !((lack >> via) & 1u)) break;
                }
                if (held) {
                    st128(a0, loc, v16, cdst, SYS);
                    st128(a1, loc, v16 + 1024, cdst, SYS);
                    drain();
                    st1(loc, prog, tbase + (unsigned)gn, lane);
                    held = 0;
                } else {
                    if (!t0) t0 = now();
                    if (now() - t0 > DEADLINE) { set_tmo(myws, call, lane); dead = 1; }
                    __builtin_amdgcn_s_sleep(2);
                }
                const unsigned tv = ld32(loc, tflag);
                drain();
                if (__builtin_amdgcn_readlane(tv, 9) == call) dead = 1;
                read_plan(tv);
            }
            if (nin && gn >= nin) break;
        }
        src = UNI(src); ndst = UNI(ndst);
        const rsrc_t remd = mk(remp, dead ? 0u : WS_USED), locd = mk(myws, dead || !held ? 0u : WS_USED);
        const u32x4 b0 = ld128(remd, v16, src, SYS);
        const u32x4 b1 = ld128(remd, v16 + 1024, src, SYS);
        const unsigned tk = draw(tick, lane == 0);
        const bool look = gn + 80 >= ghead && (!nin || lack);
        const bool peek = !look && (unseen & wanted(gn + 60)) != 0;
        fv = ld32(mk(myws, nin && !isq && gn >= nin - 48 ? O_END : 0u), oflag);
        const unsigned tn = ld32(mk(myws, look || peek ? O_END : 0u), peek ? roff : tflag);
        asm volatile("" ::: "memory");
        st128(a0, locd, v16, cdst, SYS);
        st128(a1, locd, v16 + 1024, cdst, SYS);
        drain();
        st1(mk(myws, dead ? 0u : O_END), prog, tbase + (unsigned)gn, lane);
        if (look) read_plan(tn);
        if (peek) unseen = BALLOT(tn < rall);
        a0 = b0; a1 = b1; held = 1;
        g = gn; cdst = ndst;
        gn = (int)(UNI(tk) - tbase);
    }
    {
        const rsrc_t locd = mk(myws, dead || !held ? 0u : WS_USED);
        st128(a0, locd, v16, cdst, SYS);
        st128(a1, locd, v16 + 1024, cdst, SYS);
    }
    if (isq) {
        drain();
        if (!dead) st1(loc, prog, tbase + 4095u, lane);
        return;
    }

    const int NTOT = nin + NOUT;
    const unsigned ovoff = (unsigned)(4 * (lane >> 4)) * (NCOL * 4) + (lane & 15) * 16;
    const unsigned exb = O_EX + ((call & 1u) * 8 + peer) * (256 * NEX * 4);
    unsigned rmask;
    Item cur, nxt;
    int state = 2;
    int waiting = 0;
    unsigned long long t0 = 0;
    {
        const unsigned long long m = BALLOT(lane < 32 ? (fv >> 8) >= call : fv >= call);
        rmask = (unsigned)m;
        if ((unsigned)(m >> 32) != 0xffffffffu && !dead)
            dead = wave_wait((const unsigned*)(myws + O_PRDY) + peer * 32 + (lane & 31), call, myws, call, lane) ? 0 : 1;
    }
    const unsigned pflag = lane < 32 ? oflag : lane == 32 ? tmo_off() : NOWHERE;
    for (;;) {
        g = UNI(g); gn = UNI(gn); dead = UNI(dead); state = UNI(state); rmask = UNI(rmask);
        cdst = UNI(cdst); waiting = UNI(waiting);
        const bool have = gn < NTOT;
        const unsigned o = (unsigned)(gn - nin) & 1023u;
        const bool can = have && (dead || ((rmask >> (o >> 5)) & 1u));
        {
            const bool go = can && !dead;
            const rsrc_t r0 = mk(remp, go ? WS_USED : 0u), r1 = mk(myws, go ? WS_USED : 0u);
            const unsigned so = par + P_OS + (unsigned)me * (4 * MiB) + o * 4096u;
            const unsigned ps = O_PS + peer * (4 * MiB) + o * 4096u;
            nxt.l0 = ld128(r0, v16, so, SYS);
            nxt.l1 = ld128(r0, v16 + 1024, so, SYS);
            nxt.h4 = __builtin_amdgcn_raw_buffer_load_b64(r0, lane * 8 + 2048, so, SYS);
            nxt.h1 = __builtin_amdgcn_raw_buffer_load_b16(r0, lane * 2 + 2560, so, SYS);
            nxt.ex = __builtin_amdgcn_raw_buffer_load_b32(r1, (lane & (NEX - 1)) * 4, exb + (o >> 2) * (NEX * 4), SYS);
            nxt.p0 = ld128(r1, v16, ps, SYS);
            nxt.p1 = ld128(r1, v16 + 1024, ps, SYS);
            nxt.p2 = ld128(r1, v16 + 2048, ps, SYS);
            nxt.p3 = ld128(r1, v16 + 3072, ps, SYS);
        }
        const bool poll = have && !dead && (~rmask & (7u << (o >> 5))) != 0;
        const unsigned nf = ld32(mk(myws, poll ? O_END : 0u), pflag);
        const unsigned tk = draw(tick, can && idx < WLO && lane == 0);

        u32x4 o4[4];
        if (state == 1) {
            const unsigned oc = (unsigned)(g - nin);
            const bool raw = (__builtin_amdgcn_readlane(fv, oc >> 5) >> ((oc >> 2) & 7)) & 1u;
            const unsigned L[8] = {cur.l0.x, cur.l0.y, cur.l0.z, cur.l0.w, cur.l1.x, cur.l1.y, cur.l1.z, cur.l1.w};
            u32x16 pv = {cur.p0.x, cur.p0.y, cur.p0.z, cur.p0.w, cur.p1.x, cur.p1.y, cur.p1.z, cur.p1.w,
                         cur.p2.x, cur.p2.y, cur.p2.z, cur.p2.w, cur.p3.x, cur.p3.y, cur.p3.z, cur.p3.w};
            u32x4 v[4];
            if (raw && !dead) {
                const unsigned so = par + P_OS + (unsigned)me * (4 * MiB) + oc * 4096u;
                const u32x4 ha = ld128(rem, v16 + 2048, so, SYS);
                const u32x4 hb = ld128(rem, v16 + 3072, so, SYS);
                const unsigned H[8] = {ha.x, ha.y, ha.z, ha.w, hb.x, hb.y, hb.z, hb.w};
                #pragma unroll
                for (int r = 0; r < 4; ++r) {
                    v[r].x = perm(H[2 * r], L[2 * r], 0x05040100u);
                    v[r].y = perm(H[2 * r], L[2 * r], 0x07060302u);
                    v[r].z = perm(H[2 * r + 1], L[2 * r + 1], 0x05040100u);
                    v[r].w = perm(H[2 * r + 1], L[2 * r + 1], 0x07060302u);
                }
            } else {
                unsigned em = (unsigned)BALLOT(lane < NEX && (cur.ex >> 21) == (oc & 3u));
                while (em) {
                    const unsigned e = __builtin_amdgcn_readlane(cur.ex, __builtin_ctz(em));
                    em &= em - 1;
                    pv[(e >> 17) & 15u] += (unsigned)lane == ((e >> 11) & 63u) ? e << 21 : 0u;
                }
                const unsigned t = (cur.h1 & 0x0f0fu) | ((cur.h1 & 0xf0f0u) << 12);
                unsigned B[4];
                B[0] = (cur.h4.x & 0x0f0f0f0fu) | ((t << 4) & 0x10101010u);
                B[1] = ((cur.h4.x >> 4) & 0x0f0f0f0fu) | ((t << 3) & 0x10101010u);
                B[2] = (cur.h4.y & 0x0f0f0f0fu) | ((t << 2) & 0x10101010u);
                B[3] = ((cur.h4.y >> 4) & 0x0f0f0f0fu) | ((t << 1) & 0x10101010u);
                #pragma unroll
                for (int r = 0; r < 4; ++r) {
                    v[r].x = perm(B[r], L[2 * r], 0x0c040100u);
                    v[r].y = perm(B[r], L[2 * r], 0x0c050302u);
                    v[r].z = perm(B[r], L[2 * r + 1], 0x0c060100u);
                    v[r].w = perm(B[r], L[2 * r + 1], 0x0c070302u);
                }
            }
            #pragma unroll
            for (int r = 0; r < 4; ++r) {
                f32x4 f;
                f.x = (float)(int)(v[r].x + pv[4 * r]) * scale;     f.y = (float)(int)(v[r].y + pv[4 * r + 1]) * scale;
                f.z = (float)(int)(v[r].z + pv[4 * r + 2]) * scale; f.w = (float)(int)(v[r].w + pv[4 * r + 3]) * scale;
                o4[r] = __builtin_bit_cast(u32x4, f);
                if (dead) o4[r] = mk4(0x7fc00000u, 0x7fc00000u, 0x7fc00000u, 0x7fc00000u);
            }
        }
        {
            const rsrc_t out = mk(p.out, state == 1 ? OUT_BYTES : 0u);
            #pragma unroll
            for (int r = 0; r < 4; ++r) st128(o4[r], out, ovoff + r * (NCOL * 4), cdst, OUT_AUX);
        }
        if (!have && state != 2) break;
        asm volatile("s_waitcnt vmcnt(4)" ::: "memory");
        __builtin_amdgcn_s_waitcnt(0x0F74);
        if (state == 2) {
            asm volatile("s_waitcnt vmcnt(0)" ::: "memory");
            if (!dead) st1(loc, prog, tbase + 4095u, lane);
        }
        if (!have) break;
        if (poll) {
            fv = nf;
            rmask = (unsigned)BALLOT((nf >> 8) >= call);
            if (__builtin_amdgcn_readlane(nf, 32) == call) dead = 1;
        }
        if (can) {
            cur = nxt; g = gn; state = 1;
            const unsigned n = o >> 5, w8 = (o >> 2) & 7, mf = o & 3;
            const unsigned cc = n < 2 * TA ? (n >= TA ? 1 : 0) : n & 1;
            const unsigned V = n < 2 * TA ? n - cc * TA : n >> 1;
            cdst = (((2 * (peer & 1) + cc) * 128 + (w8 >> 2) * 64 + mf * 16) * NCOL
                    + (peer >> 1) * 4096 + (unsigned)tile_of((int)V, (int)(peer & 1)) * 256 + (w8 & 3) * 64) * 4u;
            gn = idx < WLO ? (int)(UNI(tk) - tbase) : NTOT;
            waiting = 0;
        } else {
            if (!waiting) { waiting = 1; t0 = now(); }
            else {
                if (now() - t0 > DEADLINE) { set_tmo(myws, call, lane); dead = 1; }
                __builtin_amdgcn_s_sleep(4);
            }
            state = 0;
        }
    }
}

__device__ __forceinline__ unsigned prep(const Params& p, char* myws, const unsigned* cntp, int id, int tid, int lane, int wv) {
    const int me = p.rank;
    unsigned call;
    {
        const int tw = ((id - 32) >> 1) & 15;
        const unsigned long long hold = id >= 32 && tw >= TE ? HOLD0 + (tw - TE) * HOLD1 : 0;
        const unsigned long long t0 = now();
        while (now() - t0 < hold) __builtin_amdgcn_s_sleep(4);
    }
    const rsrc_t loc = mk(myws, WS_USED);
    if (id < 32) {
        const rsrc_t xs = mk(p.x, 4096u * KS);
        const unsigned voff = (unsigned)(lane & 15) * KS + (lane >> 4) * 16;
        const unsigned soff = (unsigned)(id * 128 + wv * 16) * KS;
        u32x4 v[8];
        #pragma unroll
        for (int i = 0; i < 8; ++i) v[i] = ld128(xs, voff + i * 64, soff, 0);
        call = UNI(ld_sys(cntp)) + 1u;
        const unsigned dst = O_PAR + (call & 1u) * PAR_BYTES + P_X + (unsigned)me * (2 * MiB) + (unsigned)id * 65536u + wv * 1024u;
        #pragma unroll
        for (int i = 0; i < 8; ++i) st128(v[i], loc, lane * 16, dst + i * 8192u, SYS);
    } else {
        const int pid = id - 32, jt = pid >> 1, h = pid & 1;
        const rsrc_t wsr = mk(p.w, (unsigned)KS * NCOL);
        const int wn = wv & 3, kq0 = h * 4 + (wv >> 2) * 2;
        const unsigned voff = (unsigned)(16 * (lane >> 4)) * NCOL + (lane & 15) * 4;
        unsigned r[2][16];
        #pragma unroll
        for (int b = 0; b < 2; ++b)
            #pragma unroll
            for (int j = 0; j < 16; ++j)
                r[b][j] = __builtin_amdgcn_raw_buffer_load_b32(wsr, voff, (unsigned)((kq0 + b) * 64 + j) * NCOL + jt * 256 + wn * 64, 0);
        call = UNI(ld_sys(cntp)) + 1u;
        const unsigned par = O_PAR + (call & 1u) * PAR_BYTES;
        #pragma unroll
        for (int b = 0; b < 2; ++b) {
            unsigned o[4][4];
            #pragma unroll
            for (int q = 0; q < 4; ++q) {
                const unsigned t0 = perm(r[b][4 * q + 1], r[b][4 * q], 0x05010400u);
                const unsigned t1 = perm(r[b][4 * q + 1], r[b][4 * q], 0x07030602u);
                const unsigned t2 = perm(r[b][4 * q + 3], r[b][4 * q + 2], 0x05010400u);
                const unsigned t3 = perm(r[b][4 * q + 3], r[b][4 * q + 2], 0x07030602u);
                o[0][q] = perm(t2, t0, 0x05040100u);
                o[1][q] = perm(t2, t0, 0x07060302u);
                o[2][q] = perm(t3, t1, 0x05040100u);
                o[3][q] = perm(t3, t1, 0x07060302u);
            }
            const unsigned dst = par + P_WS + (unsigned)jt * 131072u + (unsigned)((kq0 + b) * 16 + wn * 4) * 1024u;
            #pragma unroll
            for (int i = 0; i < 4; ++i)
                st128(mk4(o[i][0], o[i][1], o[i][2], o[i][3]), loc, lane * 16 + i * 1024, dst, SYS);
        }
    }
    drain();
    __syncthreads();
    if (id < 32) {
        if (tid < NR) st_sys((unsigned*)(p.ws[tid] + O_RDY) + me * 64 + id, call);
    } else {
        const int pid = id - 32;
        if (tid < 2) st_sys((unsigned*)(p.ws[(pid >> 5) * 2 + tid] + O_RDY) + me * 64 + 32 + (pid & 31), call);
        if (tid == 2) st_sys((unsigned*)(myws + O_LRW) + pid, call);
    }
    return call;
}

}

__global__ __launch_bounds__(NTHR, 2) void gemm_rs_fused(Params p) {
    __shared__ __attribute__((aligned(16))) char lds[LDS_EX + 8 * NEX * 4];
    const int tid = threadIdx.x, lane = tid & 63;
    const int wv = UNI(tid >> 6);
    const int b = blockIdx.x, me = p.rank;
    char* myws = p.ws[me];
    unsigned* cntp = (unsigned*)(myws + O_CNT) + b * 8 + wv;

    if (b < NPULL) {
        const int s = wv * NPULL + b;
        if (s < NLW) link_wave(p, myws, cntp, s, lane);
        else if (b == NPULL - 1 && wv == 7) {
            const unsigned call = UNI(ld_sys(cntp)) + 1u;
            if (lane == 0) st_sys(cntp, call);
            monitor(p, myws, call, lane);
        }
        return;
    }

    const int cb = b - NPULL;
    unsigned call;
    if (cb < 160) call = prep(p, myws, cntp, cb, tid, lane, wv);
    else call = UNI(ld_sys(cntp)) + 1u;
    if (lane == 0) st_sys(cntp, call);

    const int ci = (b & 7) * NCU + (cb >> 3);
    const int hf = me & 1, lp = hf ? 6 : 0, lb = 1 - hf;
    const unsigned par = O_PAR + (call & 1u) * PAR_BYTES;
    const rsrc_t loc = mk(myws, WS_USED);
    ldsw_t misc = (ldsw_t)(lds + LDS_MISC);
    const int wm = wv >> 2, wn = wv & 3;

    for (int step = 0; step < 4; ++step) {
        int task, di, cc, V;
        if (step < 2) {
            const int j = step == 0 ? ci - NP0 : (NCOMP - NP0) + (NCOMP - 1 - ci);
            if (j < 0 || j >= 192) continue;
            const int n = j / 6;
            task = 0; di = j - 6 * n;
            cc = n < 2 * TA ? (n >= TA ? 1 : 0) : n & 1;
            V = n < 2 * TA ? n - cc * TA : n >> 1;
        } else {
            int i = ci;
            if (step == 3) {
                if (tid == 0) misc[1] = __hip_atomic_fetch_add((unsigned*)(myws + O_TICK + 7 * 128), 1u, __ATOMIC_RELAXED, __HIP_MEMORY_SCOPE_AGENT);
                __syncthreads();
                const unsigned extra = UNI(misc[1]) - (call - 1u) * NCOMP;
                if (extra >= 256u - NCOMP) break;
                i = NCOMP + (int)extra;
            }
            if (i < 192) {
                const int r = i % 24;
                task = 1; V = 2 * (i / 24) + (r & 1); cc = r >= 12 ? 1 : 0; di = (r >> 1) - 6 * cc;
            } else {
                task = 2; di = 0; V = (i - 192) >> 2; cc = i & 3;
            }
        }
        const int d = (me + 1 + lb + di) & 7;
        const int rb = task == 0 ? 4 * me + 2 * (d & 1) + cc : task == 1 ? 4 * d + 2 * hf + cc : 4 * me + cc;
        const int pos = tile_pos(cc, V);
        const int T = tile_of(V, task == 0 ? d & 1 : hf);
        const int jt = (task == 0 ? d >> 1 : me >> 1) * 16 + T;

        i32x4 acc[4][4];
        {
            const int a0 = task == 0 ? -BIAS : task == 1 ? BIAS : 0;
            #pragma unroll
            for (int m = 0; m < 4; ++m)
                #pragma unroll
                for (int n = 0; n < 4; ++n) acc[m][n] = (i32x4){a0, a0, a0, a0};
        }
        unsigned todo = task == 0 ? 1u << me : task == 1 ? 0xffu & ~(1u << d) : 0xffu;
        bool ok = true;
        while (todo) {
            if (wv == 0) {
                const int l = lane & 15;
                const bool direct = V < 14 || l == lp;
                const unsigned* ptr = (const unsigned*)(myws + tmo_off());
                unsigned need = call;
                if ((todo >> me) & 1u) {
                    if (lane == 8 || lane == 9) ptr = (const unsigned*)(myws + O_LRW) + jt * 2 + (lane - 8);
                    if (lane == 10) ptr = (const unsigned*)(myws + O_RDY) + me * 64 + rb;
                }
                if (task && lane < 7) {
                    const int xa = l == lp ? 192 : 160;
                    const int xn = task == 2 ? G_ALL : cc ? 2 * xa + 64 * TA : xa;
                    ptr = (const unsigned*)(myws + O_G + (b & 7) * 128) + l;
                    need = (call << 12) + (unsigned)max(xn, direct ? w_done(xa, V) : 0);
                }
                if (task && lane >= 16 && lane < 23) {
                    ptr = (const unsigned*)(myws + O_G + (b & 7) * 128) + lp;
                    need = direct ? 0u : (call << 12) + (unsigned)(G_REL + 128 * (l - lb) + 64 * (V - 13));
                }
                const unsigned long long t0 = now();
                unsigned got;
                for (;;) {
                    const unsigned val = ld_sys(ptr);
                    unsigned m = (unsigned)BALLOT(val >= need);
                    if (__builtin_amdgcn_readlane(val, 7) == call) { got = 0; break; }
                    unsigned lm = m & (m >> 16) & 0x7fu;
                    if (task == 2 && ((m | (1u << lp)) & 0x7fu) != 0x7fu) lm = 0;
                    got = links_to_ranks(lm | (((m >> 8) & 7u) == 7u ? 0x80u : 0u), me) & todo;
                    if (got) break;
                    if (now() - t0 > DEADLINE) { set_tmo(myws, call, lane); break; }
                    if (task == 1 && V >= 14) __builtin_amdgcn_s_sleep(1); else __builtin_amdgcn_s_sleep(6);
                }
                if (lane == 0) misc[0] = got;
            }
            __syncthreads();
            const unsigned got = UNI(misc[0]);
            if (!got) { ok = false; break; }
            todo &= ~got;
            unsigned klist = 0, rest = got;
            int nst = 0;
            while (rest) { klist |= (unsigned)__builtin_ctz(rest) << nst; rest &= rest - 1; nst += 4; }
            const unsigned va = wv * 2048 + lane * 16, vb = wv * 4096 + lane * 16;
            auto issue = [&](int s, int slot) {
                const int k = (klist >> (s & ~3)) & 15;
                const unsigned abase = par + P_X + (unsigned)k * (2 * MiB) + (unsigned)rb * 65536u + (s & 3) * 16384u;
                const unsigned bbase = par + (k == me ? P_WS + (unsigned)jt * 131072u : P_WR + (unsigned)(k * 16 + T) * 131072u) + (s & 3) * 32768u;
                char* la = lds + slot * STAGE + wv * 2048;
                char* lb_ = lds + slot * STAGE + 16384 + wv * 4096;
                __builtin_amdgcn_raw_ptr_buffer_load_lds(loc, (__attribute__((address_space(3))) void*)la, 16, va, abase, 0, SYS);
                __builtin_amdgcn_raw_ptr_buffer_load_lds(loc, (__attribute__((address_space(3))) void*)la, 16, va, abase, 1024, SYS);
                __builtin_amdgcn_raw_ptr_buffer_load_lds(loc, (__attribute__((address_space(3))) void*)lb_, 16, vb, bbase, 0, SYS);
                __builtin_amdgcn_raw_ptr_buffer_load_lds(loc, (__attribute__((address_space(3))) void*)lb_, 16, vb, bbase, 1024, SYS);
                __builtin_amdgcn_raw_ptr_buffer_load_lds(loc, (__attribute__((address_space(3))) void*)lb_, 16, vb, bbase, 2048, SYS);
                __builtin_amdgcn_raw_ptr_buffer_load_lds(loc, (__attribute__((address_space(3))) void*)lb_, 16, vb, bbase, 3072, SYS);
            };
            auto compute = [&](int slot) {
                const char* A = lds + slot * STAGE + wm * 4096 + lane * 16;
                const char* B = lds + slot * STAGE + 16384 + wn * 4096 + lane * 16;
                i32x4 af[2][4], bf[2][4];
                #pragma unroll
                for (int kb = 0; kb < 2; ++kb) {
                    af[kb][0] = *(const i32x4*)(A + kb * 8192);
                    #pragma unroll
                    for (int i = 0; i < 4; ++i) bf[kb][i] = *(const i32x4*)(B + kb * 16384 + i * 1024);
                    #pragma unroll
                    for (int i = 1; i < 4; ++i) af[kb][i] = *(const i32x4*)(A + kb * 8192 + i * 1024);
                }
                __builtin_amdgcn_sched_barrier(0);
                #pragma unroll
                for (int kb = 0; kb < 2; ++kb)
                    #pragma unroll
                    for (int m = 0; m < 4; ++m)
                        #pragma unroll
                        for (int n = 0; n < 4; ++n)
                            acc[m][n] = __builtin_amdgcn_mfma_i32_16x16x64_i8(af[kb][m], bf[kb][n], acc[m][n], 0, 0, 0);
                __builtin_amdgcn_sched_barrier(0);
            };
            issue(0, 0);
            issue(1, 1);
            issue(2, 2);
            int slot = 0;
            for (int s = 0; s < nst; ++s) {
                if (s == 0) asm volatile("s_waitcnt vmcnt(12) lgkmcnt(0)" ::: "memory");
                else if (s + 1 < nst) asm volatile("s_waitcnt vmcnt(6) lgkmcnt(0)" ::: "memory");
                else asm volatile("s_waitcnt vmcnt(0) lgkmcnt(0)" ::: "memory");
                __builtin_amdgcn_s_barrier();
                if (s && s + 2 < nst) issue(s + 2, slot == 0 ? 2 : slot - 1);
                compute(slot);
                slot = slot == 2 ? 0 : slot + 1;
            }
        }

        if (task == 0) {
            if (ok) {
                const unsigned dst = O_PS + (unsigned)d * (4 * MiB) + (unsigned)(pos * 8 + wv) * 16384u;
                #pragma unroll
                for (int m = 0; m < 4; ++m)
                    #pragma unroll
                    for (int r = 0; r < 4; ++r)
                        st128(mk4((unsigned)acc[m][0][r], (unsigned)acc[m][1][r], (unsigned)acc[m][2][r], (unsigned)acc[m][3][r]),
                              loc, lane * 16 + r * 1024, dst + m * 4096u, SYS);
            }
            drain();
            __syncthreads();
            if (tid == 0 && ok) st_sys((unsigned*)(myws + O_PRDY) + d * 32 + pos, call);
        } else if (task == 1) {
            const unsigned dst = par + P_OS + (unsigned)d * (4 * MiB) + (unsigned)(pos * 8 + wv) * 16384u;
            if (ok) {
                #pragma unroll
                for (int m = 0; m < 4; ++m) {
                    unsigned L[8];
                    #pragma unroll
                    for (int r = 0; r < 4; ++r) {
                        L[2 * r] = perm(acc[m][1][r], acc[m][0][r], 0x05040100u);
                        L[2 * r + 1] = perm(acc[m][3][r], acc[m][2][r], 0x05040100u);
                    }
                    st128(mk4(L[0], L[1], L[2], L[3]), loc, lane * 16, dst + m * 4096u, SYS);
                    st128(mk4(L[4], L[5], L[6], L[7]), loc, lane * 16 + 1024, dst + m * 4096u, SYS);
                }
            }
            ldsw_t exl = (ldsw_t)(lds + LDS_EX) + wv * NEX;
            if (lane < NEX) exl[lane] = 0xffffffffu;
            unsigned nex = 0;
            #pragma unroll
            for (int m = 0; m < 4; ++m)
                #pragma unroll
                for (int r = 0; r < 4; ++r) {
                    const unsigned any = (unsigned)(acc[m][0][r] | acc[m][1][r]) | (unsigned)(acc[m][2][r] | acc[m][3][r]);
                    if (BALLOT((any >> 21) != 0) != 0) {
                        #pragma unroll
                        for (int n = 0; n < 4; ++n) {
                            const unsigned hi = (unsigned)acc[m][n][r] >> 21;
                            const unsigned long long bm = BALLOT(hi != 0);
                            const unsigned at = nex + __builtin_amdgcn_mbcnt_hi((unsigned)(bm >> 32), __builtin_amdgcn_mbcnt_lo((unsigned)bm, 0u));
                            if (hi != 0 && at < NEX) exl[at] = (unsigned)(m << 21 | (r * 4 + n) << 17) | ((unsigned)lane << 11) | hi;
                            nex += __builtin_popcountll(bm);
                        }
                    }
                }
            const bool raw = nex > NEX;
            if (ok) {
                const unsigned e = exl[lane & (NEX - 1)];
                __builtin_amdgcn_raw_buffer_store_b32(e, mk(p.ws[d], WS_USED),
                    lane < NEX ? O_EX + ((((call & 1u) * 8 + me) * 32 + pos) * 8 + wv) * (NEX * 4) + lane * 4 : NOWHERE, 0, SYS);
                #pragma unroll
                for (int m = 0; m < 4; ++m) {
                    unsigned H[8];
                    #pragma unroll
                    for (int r = 0; r < 4; ++r) {
                        H[2 * r] = perm(acc[m][1][r], acc[m][0][r], 0x07060302u);
                        H[2 * r + 1] = perm(acc[m][3][r], acc[m][2][r], 0x07060302u);
                    }
                    if (raw) {
                        st128(mk4(H[0], H[1], H[2], H[3]), loc, lane * 16 + 2048, dst + m * 4096u, SYS);
                        st128(mk4(H[4], H[5], H[6], H[7]), loc, lane * 16 + 3072, dst + m * 4096u, SYS);
                    } else {
                        unsigned B[4];
                        #pragma unroll
                        for (int r = 0; r < 4; ++r) B[r] = perm(H[2 * r + 1], H[2 * r], 0x06040200u);
                        const u32x2 h4 = {bfi(0x0f0f0f0fu, B[0], B[1] << 4), bfi(0x0f0f0f0fu, B[2], B[3] << 4)};
                        const unsigned t = ((B[0] >> 4) & 0x01010101u) | ((B[1] >> 3) & 0x02020202u)
                                         | ((B[2] >> 2) & 0x04040404u) | ((B[3] >> 1) & 0x08080808u);
                        __builtin_amdgcn_raw_buffer_store_b64(h4, loc, lane * 8 + 2048 + dst + m * 4096u, 0, SYS);
                        __builtin_amdgcn_raw_buffer_store_b16((unsigned short)(t | (t >> 12)), loc, lane * 2 + 2560 + dst + m * 4096u, 0, SYS);
                    }
                }
            }
            if (lane == 0) misc[4 + wv] = raw ? 1u : 0u;
            drain();
            __syncthreads();
            if (tid == 0 && ok) {
                unsigned mask = 0;
                #pragma unroll
                for (int i = 0; i < 8; ++i) mask |= misc[4 + i] << i;
                st_sys((unsigned*)(p.ws[d] + O_ORDY) + ((call & 1u) * 8 + me) * 32 + pos, (call << 8) | mask);
            }
        } else {
            const rsrc_t out = mk(p.out, OUT_BYTES);
            const float scale = p.sx[0] * p.sw[0];
            const unsigned ovoff = (unsigned)(4 * (lane >> 4)) * (NCOL * 4) + (lane & 15) * 16;
            const unsigned dst = ((unsigned)(cc * 128 + wm * 64) * NCOL + jt * 256 + wn * 64) * 4u;
            #pragma unroll
            for (int m = 0; m < 4; ++m)
                #pragma unroll
                for (int r = 0; r < 4; ++r) {
                    f32x4 f;
                    f.x = (float)acc[m][0][r] * scale; f.y = (float)acc[m][1][r] * scale;
                    f.z = (float)acc[m][2][r] * scale; f.w = (float)acc[m][3][r] * scale;
                    u32x4 o4 = __builtin_bit_cast(u32x4, f);
                    if (!ok) o4 = mk4(0x7fc00000u, 0x7fc00000u, 0x7fc00000u, 0x7fc00000u);
                    st128(o4, out, ovoff + r * (NCOL * 4), dst + (unsigned)(m * 16) * (NCOL * 4), OUT_AUX);
                }
        }
    }
}

extern "C" void kernel_launch(int rank, int nranks, ncclComm_t comm, hipStream_t stream,
                              void* const* d_in, const int* in_sizes, int n_in,
                              void* d_out, int out_size, const int* in_dtypes, int out_dtype,
                              void* const* d_out_all, void* const* d_ws_all, size_t ws_bytes) {
    (void)comm; (void)in_sizes; (void)in_dtypes; (void)out_dtype; (void)d_out_all;
    if (nranks != NR || (unsigned)rank >= (unsigned)NR || n_in < 4 || ws_bytes < WS_USED ||
        (size_t)out_size * 4 != OUT_BYTES) {
        fprintf(stderr, "kernel_launch: unexpected problem shape\n");
        return;
    }
    Params p;
    for (int i = 0; i < NR; ++i) p.ws[i] = (char*)d_ws_all[i];
    p.x = (const char*)d_in[0];
    p.w = (const char*)d_in[1];
    p.sx = (const float*)d_in[2];
    p.sw = (const float*)d_in[3];
    p.out = (float*)d_out;
    p.rank = rank;
    p.pad = 0;
    hipLaunchKernelGGL(gemm_rs_fused, dim3(NBLK), dim3(NTHR), 0, stream, p);
    HIP_CHECK(hipGetLastError());
}
```

```cpp
#include <hip/hip_runtime.h>
#include <rccl/rccl.h>
#include <cstdio>
#include <cstdint>

#define HIP_CHECK(expr) do { hipError_t _e = (expr); if (_e != hipSuccess) { \
    fprintf(stderr, "HIP error %s at %s:%d\n", hipGetErrorString(_e), __FILE__, __LINE__); \
    return; } } while (0)

typedef int i32x4 __attribute__((ext_vector_type(4)));
typedef unsigned u32x4 __attribute__((ext_vector_type(4)));
typedef unsigned u32x2 __attribute__((ext_vector_type(2)));
typedef unsigned u32x16 __attribute__((ext_vector_type(16)));
typedef float f32x4 __attribute__((ext_vector_type(4)));
typedef __amdgpu_buffer_rsrc_t rsrc_t;
typedef volatile __attribute__((address_space(3))) unsigned* ldsw_t;
#define UNI(x) __builtin_amdgcn_readfirstlane(x)
#define BALLOT(c) __builtin_amdgcn_ballot_w64(c)

namespace {

constexpr int NR = 8;
constexpr int NCOL = 16384;
constexpr int KS = 512;
constexpr unsigned MiB = 1u << 20;

constexpr int NBLK = 256, NTHR = 512;
constexpr int NPULL = 80;
constexpr int WL = 40;
constexpr int WLO = 30;
constexpr int NLW = 7 * WL;
constexpr int NCOMP = NBLK - NPULL, NCU = NCOMP / 8;
constexpr int NP0 = 48;
static_assert(NLW <= 7 * NPULL && NCOMP >= 160 && NCOMP <= 192 && 2 * (NCOMP - NP0) >= 192 && WLO <= WL && WL <= 64, "geometry");

constexpr int RP = 4, TA = 2, VS = 8;
constexpr int TE = 2, TE2 = 4;
constexpr int HOLD0 = 120, HOLD1 = 30;
constexpr int LEAD = 160;
constexpr int RELAY_TH = 36, QBIAS = 64;
constexpr int G_OWN = 320 + 64 * 14, G_REL = 384 + 64 * 16, G_OWNQ = G_REL + 768;
constexpr int G_ALL = 4095;
constexpr int NOUT = 1024;
constexpr int DYN0 = (RP + 1) * WL;
static_assert(RP >= 2 && RP * WL <= 160 && TE <= TA && TA <= TE2 && TE2 <= VS && VS <= 12, "lists");
__host__ __device__ constexpr int tile_pos(int cc, int V) { return V < TA ? cc * TA + V : 2 * V + cc; }
__host__ __device__ constexpr int tile_of(int V, int q) {
    return V < VS ? V : V < VS + 2 ? 12 + 2 * (V - VS) + q : V < 14 ? V - 2 : 13 + 2 * (V - 14) - q;
}
__host__ __device__ constexpr int w_done(int xa, int V) { return (V < TA ? xa : 2 * xa) + 64 * (V + 1); }

constexpr unsigned O_CNT = 0;
constexpr unsigned O_TMO = 8192;
constexpr unsigned O_TICK = 12288;
constexpr unsigned O_OWN = 14336;
constexpr unsigned O_RDY = 16384;
constexpr unsigned O_LRW = 20480;
constexpr unsigned O_PROG = 24576;
constexpr unsigned O_PUSH = 26624;
constexpr unsigned O_G = 28672;
constexpr unsigned O_PRDY = 32768;
constexpr unsigned O_ORDY = 36864;
constexpr unsigned O_END = 65536;
constexpr unsigned O_EX = 512 * 1024;
constexpr unsigned O_PS = 1 * MiB;
constexpr unsigned O_PAR = 33 * MiB;
constexpr unsigned P_X = 0;
constexpr unsigned P_WS = 16 * MiB;
constexpr unsigned P_WR = 24 * MiB;
constexpr unsigned P_OS = 40 * MiB;
constexpr unsigned PAR_BYTES = 72 * MiB;
constexpr unsigned WS_USED = O_PAR + 2 * PAR_BYTES;
constexpr unsigned OUT_BYTES = 512u * NCOL * 4u;
constexpr unsigned NOWHERE = 0xf0000000u;

constexpr unsigned long long DEADLINE = 40000000ull;
constexpr int BIAS = 1 << 20;
constexpr int NEX = 16;
constexpr int SYS = 17;
constexpr int OUT_AUX = 16;

constexpr int STAGE = 49152;
constexpr int LDS_MISC = 3 * STAGE;
constexpr int LDS_EX = LDS_MISC + 64;

struct Params {
    char* my;
    char* lk[NR];
    char* ws[NR + 1];
    const char* in[2];
    const float* sx;
    const float* sw;
    float* out;
    int rank;
    int pad;
};

__device__ __forceinline__ rsrc_t mk(const void* p, unsigned bytes) {
    return __builtin_amdgcn_make_buffer_rsrc((void*)p, 0, bytes, 0x00020000);
}
#define st128(v, r, voff, soff, aux) __builtin_amdgcn_raw_buffer_store_b128(v, r, (voff) + (soff), 0, aux)
#define ld128(r, voff, soff, aux) __builtin_amdgcn_raw_buffer_load_b128(r, voff, soff, aux)
#define ld32(r, voff) __builtin_amdgcn_raw_buffer_load_b32(r, voff, 0, SYS)
__device__ __forceinline__ u32x4 mk4(unsigned a, unsigned b, unsigned c, unsigned d) { return (u32x4){a, b, c, d}; }
__device__ __forceinline__ void st1(const rsrc_t r, unsigned off, unsigned v, int lane) {
    __builtin_amdgcn_raw_buffer_store_b32(v, r, lane == 0 ? off : NOWHERE, 0, SYS);
}
__device__ __forceinline__ unsigned tmo_off() { return O_TMO + (blockIdx.x & 31) * 128; }
__device__ __forceinline__ void set_tmo(char* myws, unsigned call, int lane) {
    __builtin_amdgcn_raw_buffer_store_b32(call, mk(myws, O_END), lane < 32 ? O_TMO + lane * 128 : NOWHERE, 0, SYS);
}
__device__ __forceinline__ unsigned ld_sys(const unsigned* p) {
    return __hip_atomic_load(p, __ATOMIC_RELAXED, __HIP_MEMORY_SCOPE_SYSTEM);
}
__device__ __forceinline__ void st_sys(unsigned* p, unsigned v) {
    __hip_atomic_store(p, v, __ATOMIC_RELAXED, __HIP_MEMORY_SCOPE_SYSTEM);
}
__device__ __forceinline__ void drain() {
    asm volatile("s_waitcnt vmcnt(0)" ::: "memory");
    __builtin_amdgcn_s_waitcnt(0x0F70);
}
__device__ __forceinline__ unsigned long long now() { return __builtin_amdgcn_s_memrealtime(); }
__device__ __forceinline__ unsigned perm(unsigned hi, unsigned lo, unsigned sel) {
    return __builtin_amdgcn_perm(hi, lo, sel);
}
__device__ __forceinline__ unsigned bfi(unsigned m, unsigned a, unsigned b) { return (a & m) | (b & ~m); }
__device__ __forceinline__ unsigned links_to_ranks(unsigned m, int me) { return ((m | (m << 8)) >> (7 - me)) & 0xffu; }

__device__ __forceinline__ bool wave_wait(const unsigned* ptr, unsigned need, char* myws, unsigned call, int lane) {
    const unsigned* tmo = (const unsigned*)(myws + tmo_off());
    const unsigned long long t0 = now();
    for (;;) {
        const unsigned v = ld_sys(ptr);
        const unsigned t = ld_sys(tmo);
        if (UNI(t) == call) return false;
        if (BALLOT(v < need) == 0) return true;
        if (now() - t0 > DEADLINE) {
            set_tmo(myws, call, lane);
            return false;
        }
        __builtin_amdgcn_s_sleep(1);
    }
}

__device__ __forceinline__ void monitor(const Params& p, char* myws, unsigned call, int lane) {
    const unsigned* tmo = (const unsigned*)(myws + tmo_off());
    const unsigned* prog = (const unsigned*)(myws + O_PROG);
    const int l7 = lane & 7, me = p.rank, lp = (me & 1) ? 6 : 0;
    unsigned* ownp = (unsigned*)(p.ws[lane >> 3] + O_OWN) + me * 32 + ((me + 1 + l7) & 7);
    const int xa = l7 == lp ? 192 : 160;
    const unsigned tell = (lane >> 3) == (me ^ 1) ? w_done(160, VS + 1) : 2 * xa + 64 * TA;
    const unsigned gown = l7 == lp ? G_OWNQ : G_OWN;
    const unsigned base = call << 12;
    unsigned last = 0, nin = 4095, mine, open = 0x7f;
    bool pushed = false;
    bool planned = false, told = l7 == 7 || (lane >> 3) == me || (l7 == lp && (lane >> 3) == (me ^ 1));
    const unsigned long long t0 = now();
    for (;;) {
        asm volatile("" ::: "memory");
        const unsigned tk = ld32(mk(myws, planned ? 0u : O_END), O_TICK + l7 * 128);
        if (open == 1u << lp && pushed) {
            const unsigned v = ld_sys(prog + lp * 64 + lane);
            mine = lane >= WL ? 4095u : v >= base ? v - base : 0u;
            #pragma unroll
            for (int o = 32; o; o >>= 1) mine = min(mine, (unsigned)__shfl_xor((int)mine, o));
            if (l7 != lp) mine = 4095;
        } else {
            unsigned g[7];
            bool all = true;
            #pragma unroll
            for (int l = 0; l < 7; ++l) {
                const unsigned v = ld_sys(prog + l * 64 + lane);
                const unsigned q = ld32(mk(myws, pushed ? 0u : O_END), O_PUSH + (l * 64 + lane) * 4);
                const bool in = pushed || q >= call || lane >= WL;
                all = all && in;
                g[l] = lane >= WL ? 4095u : (v >= base && in) ? v - base : 0u;
            }
            pushed = BALLOT(!all) == 0;
            #pragma unroll
            for (int o = 32; o; o >>= 1)
                #pragma unroll
                for (int l = 0; l < 7; ++l) g[l] = min(g[l], (unsigned)__shfl_xor((int)g[l], o));
            mine = 4095;
            #pragma unroll
            for (int l = 0; l < 7; ++l) if (l7 == l) mine = g[l];
        }
        if (!planned) {
            int r = l7 < 7 && tk >= base ? (int)(tk - base) : 0;
            if (BALLOT(l7 != lp && r >= G_OWN - LEAD) != 0) {
                planned = true;
                r = -r - (l7 == lp ? QBIAS : 0);
                unsigned keep = 15, recv = 0;
                #pragma unroll
                for (int j = 1; j < 7; ++j) {
                    const int to = l7 + j >= 7 ? l7 + j - 7 : l7 + j, from = l7 - j < 0 ? l7 - j + 7 : l7 - j;
                    const unsigned two = (keep >> (2 * ((me + 1 + to) & 1))) & 3u;
                    const int rto = __shfl(r, to);
                    const int give = to != lp && two && r - rto > RELAY_TH ? (two == 3 ? 3 : 1) : 0;
                    const int got = __shfl(give, from);
                    r += 32 * ((got & 1) - (give & 1));
                    if (give) keep &= ~(1u << (2 * ((me + 1 + to) & 1) + (give >> 1)));
                    recv |= (unsigned)(got & 1) << from | (unsigned)(got >> 1) << (7 + from);
                }
                nin = gown + 32 * (__builtin_popcount(keep) + __builtin_popcount(recv & 0x7fu));
                if (lane < 7) st_sys((unsigned*)(myws + O_OWN) + ((me + 1 + lane) & 7) * 32 + 8, (call << 18) | (recv << 4) | keep);
            }
        }
        mine = mine >= nin ? (unsigned)G_ALL : min(mine, gown);
        if (mine > last) {
            last = mine;
            if (l7 < 7) st_sys((unsigned*)(myws + O_G + (lane >> 3) * 128) + l7, base + mine);
        }
        if (!told && mine >= tell) {
            told = true;
            st_sys(ownp, call);
        }
        open = (unsigned)BALLOT(mine < G_ALL) & 0x7fu;
        if (!open) return;
        if (now() - t0 > 100000) {
            if (UNI(ld_sys(tmo)) == call) return;
            if (now() - t0 > DEADLINE) { set_tmo(myws, call, lane); return; }
            __builtin_amdgcn_s_sleep(1);
        }
    }
}

struct Item { u32x4 l0, l1, p0, p1, p2, p3; u32x2 h4; unsigned h1, ex; };

__device__ __forceinline__ unsigned draw(const rsrc_t tick, bool sel) {
    int t = 1, z = 0;
    asm volatile("" : "+v"(z));
    if (sel) t = __builtin_amdgcn_raw_ptr_buffer_atomic_add_i32(t, tick, z, 0, 0);
    return (unsigned)t;
}

__device__ __forceinline__ void link_wave(const Params& p, int s, int lane, int wv) {
    const int link = s % 7, idx = s / 7;
    const char* remp = p.lk[link];
    char* myws = p.my;
    const unsigned* cntp = (const unsigned*)(myws + O_CNT) + blockIdx.x * 8 + wv;
    const int me = p.rank, hf = me & 1;
    const unsigned peer = (unsigned)(me + 1 + link) & 7u;
    const bool isq = peer == (unsigned)(me ^ 1);
    const rsrc_t loc = mk(myws, WS_USED), rem = mk(remp, WS_USED);
    const unsigned v16 = lane * 16;
    auto third = [&](int r, int l, int i) {
        const int j = i + 1 - (r & 1);
        return (unsigned)(r + 1 + j + (!isq && j >= l ? 1 : 0)) & 7u;
    };

    u32x4 a0, a1;
    const rsrc_t xs = mk(p.in[0], 4096u * KS);
    const unsigned vx = (unsigned)(lane & 15) * KS + (lane >> 4) * 16;
    auto rb_of = [&](int g) { return 4 * third((int)peer, 6 - link, g >> 5) + 2 * (peer & 1); };
    auto xoff = [&](int g) { return (rb_of(g) * 128 + (unsigned)(g & 3) * 32) * KS + (unsigned)((g & 31) >> 2) * 64; };
    const unsigned cv = ld_sys(cntp);
    a0 = ld128(xs, vx, xoff(idx), 0);
    a1 = ld128(xs, vx + 16 * KS, xoff(idx), 0);
    const unsigned call = UNI(cv) + 1u;
    const unsigned par = O_PAR + (call & 1u) * PAR_BYTES;
    const unsigned dbase = par + P_X + (unsigned)me * (2 * MiB);
    const unsigned tbase = call << 12;
    #pragma unroll 1
    for (int r = 0; r < RP - 1; ++r) {
        if (lane == 0 && r == idx % (RP - 1))
            __hip_atomic_fetch_max((unsigned*)(myws + O_TICK + link * 128), tbase + DYN0, __ATOMIC_RELAXED, __HIP_MEMORY_SCOPE_AGENT);
        const int g = idx + r * WL;
        const unsigned so = xoff(g + WL);
        const u32x4 b0 = ld128(xs, vx, so, 0);
        const u32x4 b1 = ld128(xs, vx + 16 * KS, so, 0);
        asm volatile("" ::: "memory");
        const unsigned d = dbase + rb_of(g) * 65536u + (unsigned)(g & 31) * 2048u;
        st128(a0, rem, v16, d, SYS);
        st128(a1, rem, v16 + 1024, d, SYS);
        drain();
        a0 = b0; a1 = b1;
    }
    const unsigned roff = O_RDY + (peer * 64 + lane) * 4;
    const unsigned* rdyp = (const unsigned*)(myws + roff);
    const unsigned rall = lane < 32 && (unsigned)(lane >> 2) == peer ? 0u : call;
    unsigned long long unseen;
    int dead = 0;
    {
        const unsigned rv = ld32(loc, roff);
        asm volatile("" ::: "memory");
        const int g = idx + (RP - 1) * WL;
        const unsigned d = dbase + rb_of(g) * 65536u + (unsigned)(g & 31) * 2048u;
        st128(a0, rem, v16, d, SYS);
        st128(a1, rem, v16 + 1024, d, SYS);
        st1(loc, (unsigned)((const char*)cntp - myws), call, lane);
        unseen = BALLOT(rv < rall);
    }
    const float scale = p.sx[0] * p.sw[0];
    const int xa = isq ? 192 : 160, gxb = xa + 64 * TA, gown = isq ? G_OWNQ : G_OWN;
    auto wanted = [&](int g) {
        return g < xa + 64 * TE ? ((1ull << (2 * TE)) - 1) << 32 | 0x55555555ull
             : g < 2 * xa + 64 * TE2 ? (1ull << (32 + 2 * TE2)) - 1 : ~0ull;
    };
    auto await_pieces = [&](int g) {
        const unsigned long long w = wanted(g);
        if (unseen & w) {
            if (!dead) dead = wave_wait(rdyp, (w >> lane) & 1 ? rall : 0u, myws, call, lane) ? 0 : 1;
            unseen &= ~w;
        }
    };
    await_pieces(0);
    asm volatile("" ::: "memory");
    dead = UNI(dead);

    const unsigned prog = O_PROG + (link * 64 + idx) * 4;
    const rsrc_t tick = mk(myws + O_TICK + link * 128, 4);
    auto x_off = [&](unsigned k, unsigned rb, int g) {
        return par + P_X + k * (2 * MiB) + rb * 65536u + (unsigned)(g & 31) * 2048u;
    };
    auto w_off = [&](int V, int h, unsigned& src, unsigned& dst) {
        const unsigned o = (unsigned)tile_of(V, hf) * 131072u + (unsigned)(h & 63) * 2048u;
        src = par + P_WS + (unsigned)(me >> 1) * (2 * MiB) + o;
        dst = par + P_WR + peer * (2 * MiB) + o;
    };
    auto head_off = [&](int g, unsigned& src, unsigned& dst) {
        if (g < xa) src = dst = x_off(peer, 4 * third(me, link, g >> 5) + 2 * hf, g);
        else if (g < gxb) w_off((g - xa) >> 6, g - xa, src, dst);
        else if (g < gxb + xa) src = dst = x_off(peer, 4 * third(me, link, (g - gxb) >> 5) + 2 * hf + 1, g);
        else w_off((g - 2 * xa) >> 6, g - 2 * xa, src, dst);
    };
    const int ghead = isq ? G_REL : G_OWN;
    const unsigned oflag = lane < 32 ? O_ORDY + (((call & 1u) * 8 + peer) * 32 + lane) * 4
                                     : O_PRDY + (peer * 32 + (lane - 32)) * 4;
    const unsigned tflag = lane < 9 ? O_OWN + (peer * 32 + lane) * 4 : lane == 9 ? tmo_off() : NOWHERE;
    unsigned fv = 0;
    int g = idx + RP * WL, gn;
    int nin = 0;
    int held = 1;
    unsigned plan = 0, lack = 0xff, src, cdst, ndst;
    auto read_plan = [&](unsigned tv) {
        if (!nin) {
            plan = __builtin_amdgcn_readlane(tv, 8);
            if ((plan >> 18) == (call & 0x3fffu)) nin = gown + 32 * __builtin_popcount(plan & 0x7ffu);
        }
        lack = (isq ? 0xffu & ~(3u << (me & 6)) : links_to_ranks((plan >> 4) & 0x7fu, me)) & ~(unsigned)BALLOT(lane < 8 && tv >= call);
    };
    head_off(g, src, cdst);
    {
        const rsrc_t remd = mk(remp, dead ? 0u : WS_USED);
        const u32x4 b0 = ld128(remd, v16, src, SYS);
        const u32x4 b1 = ld128(remd, v16 + 1024, src, SYS);
        const unsigned tk = draw(tick, lane == 0);
        drain();
        st1(rem, O_PUSH + ((6 - link) * 64 + idx) * 4, call, lane);
        a0 = b0; a1 = b1;
        gn = (int)(UNI(tk) - tbase);
    }
    for (;;) {
        gn = UNI(gn); dead = UNI(dead); held = UNI(held); nin = UNI(nin);
        plan = UNI(plan); cdst = UNI(cdst); lack = UNI(lack);
        unseen = ((unsigned long long)UNI((unsigned)(unseen >> 32)) << 32) | UNI((unsigned)unseen);
        await_pieces(gn);
        if (gn < ghead) head_off(gn, src, ndst);
        else {
            unsigned long long t0 = 0;
            for (;;) {
                if (dead && !nin) { plan = 0; nin = gown; }
                if (gn < gown) {
                    const int r = gn - G_REL;
                    const unsigned via = third(me, link, r >> 7);
                    src = ndst = par + P_WR + via * (2 * MiB) + (unsigned)tile_of(14 + ((r >> 6) & 1), hf) * 131072u + (unsigned)(r & 63) * 2048u;
                    if (dead || !((lack >> via) & 1u)) break;
                } else if (nin) {
                    if (gn >= nin) break;
                    unsigned m = plan & 0x7ffu;
                    for (int j = (gn - gown) >> 5; j; --j) m &= m - 1;
                    const unsigned bit = __builtin_ctz(m);
                    const unsigned via = (unsigned)(me + 1 + bit - 4) & 7u;
                    src = ndst = bit < 4 ? x_off(peer, 4 * me + bit, gn)
                                         : x_off(via, 4 * me + 2 * (peer & 1) + ((plan >> (bit + 7)) & 1u), gn);
                    if (dead || bit < 4 || !((lack >> via) & 1u)) break;
                }
                if (held) {
                    st128(a0, loc, v16, cdst, SYS);
                    st128(a1, loc, v16 + 1024, cdst, SYS);
                    drain();
                    st1(loc, prog, tbase + (unsigned)gn, lane);
                    held = 0;
                } else {
                    if (!t0) t0 = now();
                    if (now() - t0 > DEADLINE) { set_tmo(myws, call, lane); dead = 1; }
                    __builtin_amdgcn_s_sleep(2);
                }
                const unsigned tv = ld32(loc, tflag);
                drain();
                if (__builtin_amdgcn_readlane(tv, 9) == call) dead = 1;
                read_plan(tv);
            }
            if (nin && gn >= nin) break;
        }
        src = UNI(src); ndst = UNI(ndst);
        const rsrc_t remd = mk(remp, dead ? 0u : WS_USED), locd = mk(myws, dead || !held ? 0u : WS_USED);
        const u32x4 b0 = ld128(remd, v16, src, SYS);
        const u32x4 b1 = ld128(remd, v16 + 1024, src, SYS);
        const unsigned tk = draw(tick, lane == 0);
        const bool look = gn + 80 >= ghead && (!nin || lack);
        const bool peek = !look && (unseen & wanted(gn + 60)) != 0;
        fv = ld32(mk(myws, nin && !isq && gn >= nin - 48 ? O_END : 0u), oflag);
        const unsigned tn = ld32(mk(myws, look || peek ? O_END : 0u), peek ? roff : tflag);
        asm volatile("" ::: "memory");
        st128(a0, locd, v16, cdst, SYS);
        st128(a1, locd, v16 + 1024, cdst, SYS);
        drain();
        st1(mk(myws, dead ? 0u : O_END), prog, tbase + (unsigned)gn, lane);
        if (look) read_plan(tn);
        if (peek) unseen = BALLOT(tn < rall);
        a0 = b0; a1 = b1; held = 1;
        g = gn; cdst = ndst;
        gn = (int)(UNI(tk) - tbase);
    }
    {
        const rsrc_t locd = mk(myws, dead || !held ? 0u : WS_USED);
        st128(a0, locd, v16, cdst, SYS);
        st128(a1, locd, v16 + 1024, cdst, SYS);
    }
    if (isq) {
        drain();
        if (!dead) st1(loc, prog, tbase + 4095u, lane);
        return;
    }

    const int NTOT = nin + NOUT;
    const unsigned ovoff = (unsigned)(4 * (lane >> 4)) * (NCOL * 4) + (lane & 15) * 16;
    const unsigned exb = O_EX + ((call & 1u) * 8 + peer) * (256 * NEX * 4);
    unsigned rmask;
    Item cur, nxt;
    int state = 2;
    int waiting = 0;
    unsigned long long t0 = 0;
    {
        const unsigned long long m = BALLOT(lane < 32 ? (fv >> 8) >= call : fv >= call);
        rmask = (unsigned)m;
        if ((unsigned)(m >> 32) != 0xffffffffu && !dead)
            dead = wave_wait((const unsigned*)(myws + O_PRDY) + peer * 32 + (lane & 31), call, myws, call, lane) ? 0 : 1;
    }
    const unsigned pflag = lane < 32 ? oflag : lane == 32 ? tmo_off() : NOWHERE;
    for (;;) {
        g = UNI(g); gn = UNI(gn); dead = UNI(dead); state = UNI(state); rmask = UNI(rmask);
        cdst = UNI(cdst); waiting = UNI(waiting);
        const bool have = gn < NTOT;
        const unsigned o = (unsigned)(gn - nin) & 1023u;
        const bool can = have && (dead || ((rmask >> (o >> 5)) & 1u));
        {
            const bool go = can && !dead;
            const rsrc_t r0 = mk(remp, go ? WS_USED : 0u), r1 = mk(myws, go ? WS_USED : 0u);
            const unsigned so = par + P_OS + (unsigned)me * (4 * MiB) + o * 4096u;
            const unsigned ps = O_PS + peer * (4 * MiB) + o * 4096u;
            nxt.l0 = ld128(r0, v16, so, SYS);
            nxt.l1 = ld128(r0, v16 + 1024, so, SYS);
            nxt.h4 = __builtin_amdgcn_raw_buffer_load_b64(r0, lane * 8 + 2048, so, SYS);
            nxt.h1 = __builtin_amdgcn_raw_buffer_load_b16(r0, lane * 2 + 2560, so, SYS);
            nxt.ex = __builtin_amdgcn_raw_buffer_load_b32(r1, (lane & (NEX - 1)) * 4, exb + (o >> 2) * (NEX * 4), SYS);
            nxt.p0 = ld128(r1, v16, ps, SYS);
            nxt.p1 = ld128(r1, v16 + 1024, ps, SYS);
            nxt.p2 = ld128(r1, v16 + 2048, ps, SYS);
            nxt.p3 = ld128(r1, v16 + 3072, ps, SYS);
        }
        const bool poll = have && !dead && (~rmask & (7u << (o >> 5))) != 0;
        const unsigned nf = ld32(mk(myws, poll ? O_END : 0u), pflag);
        const unsigned tk = draw(tick, can && idx < WLO && lane == 0);

        u32x4 o4[4];
        if (state == 1) {
            const unsigned oc = (unsigned)(g - nin);
            const bool raw = (__builtin_amdgcn_readlane(fv, oc >> 5) >> ((oc >> 2) & 7)) & 1u;
            const unsigned L[8] = {cur.l0.x, cur.l0.y, cur.l0.z, cur.l0.w, cur.l1.x, cur.l1.y, cur.l1.z, cur.l1.w};
            u32x16 pv = {cur.p0.x, cur.p0.y, cur.p0.z, cur.p0.w, cur.p1.x, cur.p1.y, cur.p1.z, cur.p1.w,
                         cur.p2.x, cur.p2.y, cur.p2.z, cur.p2.w, cur.p3.x, cur.p3.y, cur.p3.z, cur.p3.w};
            u32x4 v[4];
            if (raw && !dead) {
                const unsigned so = par + P_OS + (unsigned)me * (4 * MiB) + oc * 4096u;
                const u32x4 ha = ld128(rem, v16 + 2048, so, SYS);
                const u32x4 hb = ld128(rem, v16 + 3072, so, SYS);
                const unsigned H[8] = {ha.x, ha.y, ha.z, ha.w, hb.x, hb.y, hb.z, hb.w};
                #pragma unroll
                for (int r = 0; r < 4; ++r) {
                    v[r].x = perm(H[2 * r], L[2 * r], 0x05040100u);
                    v[r].y = perm(H[2 * r], L[2 * r], 0x07060302u);
                    v[r].z = perm(H[2 * r + 1], L[2 * r + 1], 0x05040100u);
                    v[r].w = perm(H[2 * r + 1], L[2 * r + 1], 0x07060302u);
                }
            } else {
                unsigned em = (unsigned)BALLOT(lane < NEX && (cur.ex >> 21) == (oc & 3u));
                while (em) {
                    const unsigned e = __builtin_amdgcn_readlane(cur.ex, __builtin_ctz(em));
                    em &= em - 1;
                    pv[(e >> 17) & 15u] += (unsigned)lane == ((e >> 11) & 63u) ? e << 21 : 0u;
                }
                const unsigned t = (cur.h1 & 0x0f0fu) | ((cur.h1 & 0xf0f0u) << 12);
                unsigned B[4];
                B[0] = (cur.h4.x & 0x0f0f0f0fu) | ((t << 4) & 0x10101010u);
                B[1] = ((cur.h4.x >> 4) & 0x0f0f0f0fu) | ((t << 3) & 0x10101010u);
                B[2] = (cur.h4.y & 0x0f0f0f0fu) | ((t << 2) & 0x10101010u);
                B[3] = ((cur.h4.y >> 4) & 0x0f0f0f0fu) | ((t << 1) & 0x10101010u);
                #pragma unroll
                for (int r = 0; r < 4; ++r) {
                    v[r].x = perm(B[r], L[2 * r], 0x0c040100u);
                    v[r].y = perm(B[r], L[2 * r], 0x0c050302u);
                    v[r].z = perm(B[r], L[2 * r + 1], 0x0c060100u);
                    v[r].w = perm(B[r], L[2 * r + 1], 0x0c070302u);
                }
            }
            #pragma unroll
            for (int r = 0; r < 4; ++r) {
                f32x4 f;
                f.x = (float)(int)(v[r].x + pv[4 * r]) * scale;     f.y = (float)(int)(v[r].y + pv[4 * r + 1]) * scale;
                f.z = (float)(int)(v[r].z + pv[4 * r + 2]) * scale; f.w = (float)(int)(v[r].w + pv[4 * r + 3]) * scale;
                o4[r] = __builtin_bit_cast(u32x4, f);
                if (dead) o4[r] = mk4(0x7fc00000u, 0x7fc00000u, 0x7fc00000u, 0x7fc00000u);
            }
        }
        {
            const rsrc_t out = mk(p.out, state == 1 ? OUT_BYTES : 0u);
            #pragma unroll
            for (int r = 0; r < 4; ++r) st128(o4[r], out, ovoff + r * (NCOL * 4), cdst, OUT_AUX);
        }
        if (!have && state != 2) break;
        asm volatile("s_waitcnt vmcnt(4)" ::: "memory");
        __builtin_amdgcn_s_waitcnt(0x0F74);
        if (state == 2) {
            asm volatile("s_waitcnt vmcnt(0)" ::: "memory");
            if (!dead) st1(loc, prog, tbase + 4095u, lane);
        }
        if (!have) break;
        if (poll) {
            fv = nf;
            rmask = (unsigned)BALLOT((nf >> 8) >= call);
            if (__builtin_amdgcn_readlane(nf, 32) == call) dead = 1;
        }
        if (can) {
            cur = nxt; g = gn; state = 1;
            const unsigned n = o >> 5, w8 = (o >> 2) & 7, mf = o & 3;
            const unsigned cc = n < 2 * TA ? (n >= TA ? 1 : 0) : n & 1;
            const unsigned V = n < 2 * TA ? n - cc * TA : n >> 1;
            cdst = (((2 * (peer & 1) + cc) * 128 + (w8 >> 2) * 64 + mf * 16) * NCOL
                    + (peer >> 1) * 4096 + (unsigned)tile_of((int)V, (int)(peer & 1)) * 256 + (w8 & 3) * 64) * 4u;
            gn = idx < WLO ? (int)(UNI(tk) - tbase) : NTOT;
            waiting = 0;
        } else {
            if (!waiting) { waiting = 1; t0 = now(); }
            else {
                if (now() - t0 > DEADLINE) { set_tmo(myws, call, lane); dead = 1; }
                __builtin_amdgcn_s_sleep(4);
            }
            state = 0;
        }
    }
}

__device__ __forceinline__ unsigned prep(const Params& p, char* myws, const unsigned* cntp, int id, int lane, int wv) {
    const int me = p.rank;
    unsigned call;
    const rsrc_t loc = mk(myws, WS_USED);
    const int pid0 = id - 32;
    const char* fbase = p.ws[id < 32 ? wv : wv < 2 ? (pid0 >> 5) * 2 + wv : NR];
    const unsigned foff = id < 32 ? O_RDY + (me * 64 + id) * 4 : wv < 2 ? O_RDY + (me * 64 + 32 + (pid0 & 31)) * 4 : O_LRW + pid0 * 4;
    const char* inp = p.in[id < 32 ? 0 : 1];
    asm volatile("" : "+s"(inp), "+s"(fbase));
    {
        const int tw = (pid0 >> 1) & 15;
        if (id >= 32 && tw >= TE) {
            const unsigned long long t0 = now();
            while (now() - t0 < HOLD0 + (tw - TE) * HOLD1) __builtin_amdgcn_s_sleep(4);
        }
    }
    if (id < 32) {
        const rsrc_t xs = mk(inp, 4096u * KS);
        const unsigned voff = (unsigned)(lane & 15) * KS + (lane >> 4) * 16;
        const unsigned soff = (unsigned)(id * 128 + wv * 16) * KS;
        u32x4 v[8];
        #pragma unroll
        for (int i = 0; i < 8; ++i) v[i] = ld128(xs, voff + i * 64, soff, 0);
        call = UNI(ld_sys(cntp)) + 1u;
        const unsigned dst = O_PAR + (call & 1u) * PAR_BYTES + P_X + (unsigned)me * (2 * MiB) + (unsigned)id * 65536u + wv * 1024u;
        #pragma unroll
        for (int i = 0; i < 8; ++i) st128(v[i], loc, lane * 16, dst + i * 8192u, SYS);
    } else {
        const int pid = id - 32, jt = pid >> 1, h = pid & 1;
        const rsrc_t wsr = mk(inp, (unsigned)KS * NCOL);
        const int wn = wv & 3, kq0 = h * 4 + (wv >> 2) * 2;
        const unsigned voff = (unsigned)(16 * (lane >> 4)) * NCOL + (lane & 15) * 4;
        unsigned r[2][16];
        #pragma unroll
        for (int b = 0; b < 2; ++b)
            #pragma unroll
            for (int j = 0; j < 16; ++j)
                r[b][j] = __builtin_amdgcn_raw_buffer_load_b32(wsr, voff, (unsigned)((kq0 + b) * 64 + j) * NCOL + jt * 256 + wn * 64, 0);
        call = UNI(ld_sys(cntp)) + 1u;
        const unsigned par = O_PAR + (call & 1u) * PAR_BYTES;
        #pragma unroll
        for (int b = 0; b < 2; ++b) {
            unsigned o[4][4];
            #pragma unroll
            for (int q = 0; q < 4; ++q) {
                const unsigned t0 = perm(r[b][4 * q + 1], r[b][4 * q], 0x05010400u);
                const unsigned t1 = perm(r[b][4 * q + 1], r[b][4 * q], 0x07030602u);
                const unsigned t2 = perm(r[b][4 * q + 3], r[b][4 * q + 2], 0x05010400u);
                const unsigned t3 = perm(r[b][4 * q + 3], r[b][4 * q + 2], 0x07030602u);
                o[0][q] = perm(t2, t0, 0x05040100u);
                o[1][q] = perm(t2, t0, 0x07060302u);
                o[2][q] = perm(t3, t1, 0x05040100u);
                o[3][q] = perm(t3, t1, 0x07060302u);
            }
            const unsigned dst = par + P_WS + (unsigned)jt * 131072u + (unsigned)((kq0 + b) * 16 + wn * 4) * 1024u;
            #pragma unroll
            for (int i = 0; i < 4; ++i)
                st128(mk4(o[i][0], o[i][1], o[i][2], o[i][3]), loc, lane * 16 + i * 1024, dst, SYS);
        }
    }
    drain();
    __syncthreads();
    if (id < 32 || wv < 3) st1(mk(fbase, O_END), foff, call, lane);
    return call;
}

}

__global__ __launch_bounds__(NTHR, 2) void gemm_rs_fused(Params p) {
    __shared__ __attribute__((aligned(16))) char lds[LDS_EX + 8 * NEX * 4];
    const int tid = threadIdx.x, lane = tid & 63;
    const int wv = UNI(tid >> 6);
    const int b = blockIdx.x;
    if (__builtin_expect(b < NPULL && wv * NPULL + b < NLW, 1)) {
        link_wave(p, wv * NPULL + b, lane, wv);
        return;
    }
    const int me = p.rank;
    char* myws = p.my;
    unsigned* cntp = (unsigned*)(myws + O_CNT) + b * 8 + wv;
    if (b < NPULL) {
        if (b == NPULL - 1 && wv == 7) {
            const unsigned call = UNI(ld_sys(cntp)) + 1u;
            if (lane == 0) st_sys(cntp, call);
            monitor(p, myws, call, lane);
        }
        return;
    }

    const int cb = b - NPULL;
    unsigned call;
    if (cb < 160) call = prep(p, myws, cntp, cb, lane, wv);
    else call = UNI(ld_sys(cntp)) + 1u;
    if (lane == 0) st_sys(cntp, call);

    const int ci = (b & 7) * NCU + (cb >> 3);
    const int hf = me & 1, lp = hf ? 6 : 0, lb = 1 - hf;
    const unsigned par = O_PAR + (call & 1u) * PAR_BYTES;
    const rsrc_t loc = mk(myws, WS_USED);
    ldsw_t misc = (ldsw_t)(lds + LDS_MISC);
    const int wm = wv >> 2, wn = wv & 3;

    for (int step = 0; step < 4; ++step) {
        int task, di, cc, V;
        if (step < 2) {
            const int j = step == 0 ? ci - NP0 : (NCOMP - NP0) + (NCOMP - 1 - ci);
            if (j < 0 || j >= 192) continue;
            const int n = j / 6;
            task = 0; di = j - 6 * n;
            cc = n < 2 * TA ? (n >= TA ? 1 : 0) : n & 1;
            V = n < 2 * TA ? n - cc * TA : n >> 1;
        } else {
            int i = ci;
            if (step == 3) {
                if (tid == 0) misc[1] = __hip_atomic_fetch_add((unsigned*)(myws + O_TICK + 7 * 128), 1u, __ATOMIC_RELAXED, __HIP_MEMORY_SCOPE_AGENT);
                __syncthreads();
                const unsigned extra = UNI(misc[1]) - (call - 1u) * NCOMP;
                if (extra >= 256u - NCOMP) break;
                i = NCOMP + (int)extra;
            }
            if (i < 192) {
                const int r = i % 24;
                task = 1; V = 2 * (i / 24) + (r & 1); cc = r >= 12 ? 1 : 0; di = (r >> 1) - 6 * cc;
            } else {
                task = 2; di = 0; V = (i - 192) >> 2; cc = i & 3;
            }
        }
        const int d = (me + 1 + lb + di) & 7;
        const int rb = task == 0 ? 4 * me + 2 * (d & 1) + cc : task == 1 ? 4 * d + 2 * hf + cc : 4 * me + cc;
        const int pos = tile_pos(cc, V);
        const int T = tile_of(V, task == 0 ? d & 1 : hf);
        const int jt = (task == 0 ? d >> 1 : me >> 1) * 16 + T;

        i32x4 acc[4][4];
        {
            const int a0 = task == 0 ? -BIAS : task == 1 ? BIAS : 0;
            #pragma unroll
            for (int m = 0; m < 4; ++m)
                #pragma unroll
                for (int n = 0; n < 4; ++n) acc[m][n] = (i32x4){a0, a0, a0, a0};
        }
        unsigned todo = task == 0 ? 1u << me : task == 1 ? 0xffu & ~(1u << d) : 0xffu;
        bool ok = true;
        while (todo) {
            if (wv == 0) {
                const int l = lane & 15;
                const bool direct = V < 14 || l == lp;
                const unsigned* ptr = (const unsigned*)(myws + tmo_off());
                unsigned need = call;
                if ((todo >> me) & 1u) {
                    if (lane == 8 || lane == 9) ptr = (const unsigned*)(myws + O_LRW) + jt * 2 + (lane - 8);
                    if (lane == 10) ptr = (const unsigned*)(myws + O_RDY) + me * 64 + rb;
                }
                if (task && lane < 7) {
                    const int xa = l == lp ? 192 : 160;
                    const int xn = task == 2 ? G_ALL : cc ? 2 * xa + 64 * TA : xa;
                    ptr = (const unsigned*)(myws + O_G + (b & 7) * 128) + l;
                    need = (call << 12) + (unsigned)max(xn, direct ? w_done(xa, V) : 0);
                }
                if (task && lane >= 16 && lane < 23) {
                    ptr = (const unsigned*)(myws + O_G + (b & 7) * 128) + lp;
                    need = direct ? 0u : (call << 12) + (unsigned)(G_REL + 128 * (l - lb) + 64 * (V - 13));
                }
                const unsigned long long t0 = now();
                unsigned got;
                for (;;) {
                    const unsigned val = ld_sys(ptr);
                    unsigned m = (unsigned)BALLOT(val >= need);
                    if (__builtin_amdgcn_readlane(val, 7) == call) { got = 0; break; }
                    unsigned lm = m & (m >> 16) & 0x7fu;
                    if (task == 2 && ((m | (1u << lp)) & 0x7fu) != 0x7fu) lm = 0;
                    got = links_to_ranks(lm | (((m >> 8) & 7u) == 7u ? 0x80u : 0u), me) & todo;
                    if (got) break;
                    if (now() - t0 > DEADLINE) { set_tmo(myws, call, lane); break; }
                    if (task == 1 && V >= 14) __builtin_amdgcn_s_sleep(1); else __builtin_amdgcn_s_sleep(6);
                }
                if (lane == 0) misc[0] = got;
            }
            __syncthreads();
            const unsigned got = UNI(misc[0]);
            if (!got) { ok = false; break; }
            todo &= ~got;
            unsigned klist = 0, rest = got;
            int nst = 0;
            while (rest) { klist |= (unsigned)__builtin_ctz(rest) << nst; rest &= rest - 1; nst += 4; }
            const unsigned va = wv * 2048 + lane * 16, vb = wv * 4096 + lane * 16;
            auto issue = [&](int s, int slot) {
                const int k = (klist >> (s & ~3)) & 15;
                const unsigned abase = par + P_X + (unsigned)k * (2 * MiB) + (unsigned)rb * 65536u + (s & 3) * 16384u;
                const unsigned bbase = par + (k == me ? P_WS + (unsigned)jt * 131072u : P_WR + (unsigned)(k * 16 + T) * 131072u) + (s & 3) * 32768u;
                char* la = lds + slot * STAGE + wv * 2048;
                char* lb_ = lds + slot * STAGE + 16384 + wv * 4096;
                __builtin_amdgcn_raw_ptr_buffer_load_lds(loc, (__attribute__((address_space(3))) void*)la, 16, va, abase, 0, SYS);
                __builtin_amdgcn_raw_ptr_buffer_load_lds(loc, (__attribute__((address_space(3))) void*)la, 16, va, abase, 1024, SYS);
                __builtin_amdgcn_raw_ptr_buffer_load_lds(loc, (__attribute__((address_space(3))) void*)lb_, 16, vb, bbase, 0, SYS);
                __builtin_amdgcn_raw_ptr_buffer_load_lds(loc, (__attribute__((address_space(3))) void*)lb_, 16, vb, bbase, 1024, SYS);
                __builtin_amdgcn_raw_ptr_buffer_load_lds(loc, (__attribute__((address_space(3))) void*)lb_, 16, vb, bbase, 2048, SYS);
                __builtin_amdgcn_raw_ptr_buffer_load_lds(loc, (__attribute__((address_space(3))) void*)lb_, 16, vb, bbase, 3072, SYS);
            };
            auto compute = [&](int slot) {
                const char* A = lds + slot * STAGE + wm * 4096 + lane * 16;
                const char* B = lds + slot * STAGE + 16384 + wn * 4096 + lane * 16;
                i32x4 af[2][4], bf[2][4];
                #pragma unroll
                for (int kb = 0; kb < 2; ++kb) {
                    af[kb][0] = *(const i32x4*)(A + kb * 8192);
                    #pragma unroll
                    for (int i = 0; i < 4; ++i) bf[kb][i] = *(const i32x4*)(B + kb * 16384 + i * 1024);
                    #pragma unroll
                    for (int i = 1; i < 4; ++i) af[kb][i] = *(const i32x4*)(A + kb * 8192 + i * 1024);
                }
                __builtin_amdgcn_sched_barrier(0);
                #pragma unroll
                for (int kb = 0; kb < 2; ++kb)
                    #pragma unroll
                    for (int m = 0; m < 4; ++m)
                        #pragma unroll
                        for (int n = 0; n < 4; ++n)
                            acc[m][n] = __builtin_amdgcn_mfma_i32_16x16x64_i8(af[kb][m], bf[kb][n], acc[m][n], 0, 0, 0);
                __builtin_amdgcn_sched_barrier(0);
            };
            issue(0, 0);
            issue(1, 1);
            issue(2, 2);
            int slot = 0;
            for (int s = 0; s < nst; ++s) {
                if (s == 0) asm volatile("s_waitcnt vmcnt(12) lgkmcnt(0)" ::: "memory");
                else if (s + 1 < nst) asm volatile("s_waitcnt vmcnt(6) lgkmcnt(0)" ::: "memory");
                else asm volatile("s_waitcnt vmcnt(0) lgkmcnt(0)" ::: "memory");
                __builtin_amdgcn_s_barrier();
                if (s && s + 2 < nst) issue(s + 2, slot == 0 ? 2 : slot - 1);
                compute(slot);
                slot = slot == 2 ? 0 : slot + 1;
            }
        }

        if (task == 0) {
            if (ok) {
                const unsigned dst = O_PS + (unsigned)d * (4 * MiB) + (unsigned)(pos * 8 + wv) * 16384u;
                #pragma unroll
                for (int m = 0; m < 4; ++m)
                    #pragma unroll
                    for (int r = 0; r < 4; ++r)
                        st128(mk4((unsigned)acc[m][0][r], (unsigned)acc[m][1][r], (unsigned)acc[m][2][r], (unsigned)acc[m][3][r]),
                              loc, lane * 16 + r * 1024, dst + m * 4096u, SYS);
            }
            drain();
            __syncthreads();
            if (tid == 0 && ok) st_sys((unsigned*)(myws + O_PRDY) + d * 32 + pos, call);
        } else if (task == 1) {
            const unsigned dst = par + P_OS + (unsigned)d * (4 * MiB) + (unsigned)(pos * 8 + wv) * 16384u;
            if (ok) {
                #pragma unroll
                for (int m = 0; m < 4; ++m) {
                    unsigned L[8];
                    #pragma unroll
                    for (int r = 0; r < 4; ++r) {
                        L[2 * r] = perm(acc[m][1][r], acc[m][0][r], 0x05040100u);
                        L[2 * r + 1] = perm(acc[m][3][r], acc[m][2][r], 0x05040100u);
                    }
                    st128(mk4(L[0], L[1], L[2], L[3]), loc, lane * 16, dst + m * 4096u, SYS);
                    st128(mk4(L[4], L[5], L[6], L[7]), loc, lane * 16 + 1024, dst + m * 4096u, SYS);
                }
            }
            ldsw_t exl = (ldsw_t)(lds + LDS_EX) + wv * NEX;
            if (lane < NEX) exl[lane] = 0xffffffffu;
            unsigned nex = 0;
            #pragma unroll
            for (int m = 0; m < 4; ++m)
                #pragma unroll
                for (int r = 0; r < 4; ++r) {
                    const unsigned any = (unsigned)(acc[m][0][r] | acc[m][1][r]) | (unsigned)(acc[m][2][r] | acc[m][3][r]);
                    if (BALLOT((any >> 21) != 0) != 0) {
                        #pragma unroll
                        for (int n = 0; n < 4; ++n) {
                            const unsigned hi = (unsigned)acc[m][n][r] >> 21;
                            const unsigned long long bm = BALLOT(hi != 0);
                            const unsigned at = nex + __builtin_amdgcn_mbcnt_hi((unsigned)(bm >> 32), __builtin_amdgcn_mbcnt_lo((unsigned)bm, 0u));
                            if (hi != 0 && at < NEX) exl[at] = (unsigned)(m << 21 | (r * 4 + n) << 17) | ((unsigned)lane << 11) | hi;
                            nex += __builtin_popcountll(bm);
                        }
                    }
                }
            const bool raw = nex > NEX;
            if (ok) {
                const unsigned e = exl[lane & (NEX - 1)];
                __builtin_amdgcn_raw_buffer_store_b32(e, mk(p.ws[d], WS_USED),
                    lane < NEX ? O_EX + ((((call & 1u) * 8 + me) * 32 + pos) * 8 + wv) * (NEX * 4) + lane * 4 : NOWHERE, 0, SYS);
                #pragma unroll
                for (int m = 0; m < 4; ++m) {
                    unsigned H[8];
                    #pragma unroll
                    for (int r = 0; r < 4; ++r) {
                        H[2 * r] = perm(acc[m][1][r], acc[m][0][r], 0x07060302u);
                        H[2 * r + 1] = perm(acc[m][3][r], acc[m][2][r], 0x07060302u);
                    }
                    if (raw) {
                        st128(mk4(H[0], H[1], H[2], H[3]), loc, lane * 16 + 2048, dst + m * 4096u, SYS);
                        st128(mk4(H[4], H[5], H[6], H[7]), loc, lane * 16 + 3072, dst + m * 4096u, SYS);
                    } else {
                        unsigned B[4];
                        #pragma unroll
                        for (int r = 0; r < 4; ++r) B[r] = perm(H[2 * r + 1], H[2 * r], 0x06040200u);
                        const u32x2 h4 = {bfi(0x0f0f0f0fu, B[0], B[1] << 4), bfi(0x0f0f0f0fu, B[2], B[3] << 4)};
                        const unsigned t = ((B[0] >> 4) & 0x01010101u) | ((B[1] >> 3) & 0x02020202u)
                                         | ((B[2] >> 2) & 0x04040404u) | ((B[3] >> 1) & 0x08080808u);
                        __builtin_amdgcn_raw_buffer_store_b64(h4, loc, lane * 8 + 2048 + dst + m * 4096u, 0, SYS);
                        __builtin_amdgcn_raw_buffer_store_b16((unsigned short)(t | (t >> 12)), loc, lane * 2 + 2560 + dst + m * 4096u, 0, SYS);
                    }
                }
            }
            if (lane == 0) misc[4 + wv] = raw ? 1u : 0u;
            drain();
            __syncthreads();
            if (tid == 0 && ok) {
                unsigned mask = 0;
                #pragma unroll
                for (int i = 0; i < 8; ++i) mask |= misc[4 + i] << i;
                st_sys((unsigned*)(p.ws[d] + O_ORDY) + ((call & 1u) * 8 + me) * 32 + pos, (call << 8) | mask);
            }
        } else {
            const rsrc_t out = mk(p.out, OUT_BYTES);
            const float scale = p.sx[0] * p.sw[0];
            const unsigned ovoff = (unsigned)(4 * (lane >> 4)) * (NCOL * 4) + (lane & 15) * 16;
            const unsigned dst = ((unsigned)(cc * 128 + wm * 64) * NCOL + jt * 256 + wn * 64) * 4u;
            #pragma unroll
            for (int m = 0; m < 4; ++m)
                #pragma unroll
                for (int r = 0; r < 4; ++r) {
                    f32x4 f;
                    f.x = (float)acc[m][0][r] * scale; f.y = (float)acc[m][1][r] * scale;
                    f.z = (float)acc[m][2][r] * scale; f.w = (float)acc[m][3][r] * scale;
                    u32x4 o4 = __builtin_bit_cast(u32x4, f);
                    if (!ok) o4 = mk4(0x7fc00000u, 0x7fc00000u, 0x7fc00000u, 0x7fc00000u);
                    st128(o4, out, ovoff + r * (NCOL * 4), dst + (unsigned)(m * 16) * (NCOL * 4), OUT_AUX);
                }
        }
    }
}

extern "C" void kernel_launch(int rank, int nranks, ncclComm_t comm, hipStream_t stream,
                              void* const* d_in, const int* in_sizes, int n_in,
                              void* d_out, int out_size, const int* in_dtypes, int out_dtype,
                              void* const* d_out_all, void* const* d_ws_all, size_t ws_bytes) {
    (void)comm; (void)in_sizes; (void)in_dtypes; (void)out_dtype; (void)d_out_all;
    if (nranks != NR || (unsigned)rank >= (unsigned)NR || n_in < 4 || ws_bytes < WS_USED ||
        (size_t)out_size * 4 != OUT_BYTES) {
        fprintf(stderr, "kernel_launch: unexpected problem shape\n");
        return;
    }
    Params p;
    for (int i = 0; i < NR; ++i) p.ws[i] = (char*)d_ws_all[i];
    for (int i = 0; i < NR; ++i) p.lk[i] = p.ws[(rank + 1 + i) & 7];
    p.my = p.ws[rank];
    p.ws[NR] = p.my;
    p.in[0] = (const char*)d_in[0];
    p.in[1] = (const char*)d_in[1];
    p.sx = (const float*)d_in[2];
    p.sw = (const float*)d_in[3];
    p.out = (float*)d_out;
    p.rank = rank;
    p.pad = 0;
    hipLaunchKernelGGL(gemm_rs_fused, dim3(NBLK), dim3(NTHR), 0, stream, p);
    HIP_CHECK(hipGetLastError());
}
```
